```python
import numpy as np

kernel_name = "gpu_gemm_rs_m4096_k4096_n4096_e5m2e5m2_dqrelu"


ABSMAX_RTOL = 0.05

COMPUTE_DTYPE = "float32"
EPILOGUE = "dequant_relu"
IN_QUANT = None
OUT_QUANT = None
IN_FP8 = "e5m2e5m2"
IN_DTYPES = {"x": "e5m2", "w_mat": "e5m2",
             "scale_x": "float32", "scale_w": "float32"}
WORLD_SIZE = 8

def _snap_fp8(x, mant_bits, max_abs, min_normal):
    a = np.abs(x.astype(np.float64))
    _m, _e = np.frexp(a)
    q = np.where(a >= min_normal, np.ldexp(1.0, _e - 1 - mant_bits),
                 min_normal * 2.0 ** -mant_bits)
    snapped = np.minimum(np.round(a / q) * q, max_abs)
    return (np.sign(x) * snapped).astype(np.float32)

def setup_inputs():

    x = _snap_fp8(np.random.randn(4096, 4096).astype(np.float32), 2, 57344.0, 2.0 ** -14)
    w_mat = _snap_fp8(np.random.randn(4096, 4096).astype(np.float32), 2, 57344.0, 2.0 ** -14)
    scale_x = (0.02 * (1.0 + np.random.rand(1))).astype(np.float32)
    scale_w = (0.02 * (1.0 + np.random.rand(1))).astype(np.float32)
    return {"x": x, "w_mat": w_mat, "scale_x": scale_x, "scale_w": scale_w}

def reference(x, w_mat, scale_x, scale_w):
    acc = (x @ w_mat).astype(np.float32)
    y = (acc * (scale_x * scale_w)).astype(np.float32)
    return np.maximum(y, 0.0).astype(np.float32)
```

```cpp
#include <hip/hip_runtime.h>
#include <rccl/rccl.h>
#include <cstdio>
#include <cstdint>

#define HIP_CHECK(expr) do { hipError_t _e = (expr); if (_e != hipSuccess) { \
    fprintf(stderr, "HIP error %s at %s:%d\n", hipGetErrorString(_e), __FILE__, __LINE__); \
    return; } } while (0)

typedef unsigned u32;
typedef unsigned long long u64;
typedef unsigned u32x4 __attribute__((ext_vector_type(4)));
typedef int v8i __attribute__((ext_vector_type(8)));
typedef float v4f __attribute__((ext_vector_type(4)));
#define LDSP __attribute__((address_space(3)))

constexpr int NR = 8;
constexpr int KS = 512;
constexpr int NN = 4096;
constexpr int NTHREADS = 576;
constexpr int NWG = 256;

constexpr u32 BLK_BYTES = 17 * 1024;
constexpr u32 W_SRC = 128 * BLK_BYTES;
constexpr u32 A_SRC = 256 * 1024;
constexpr u32 A_OFF = NR * W_SRC;
constexpr u32 PAR_BYTES = NR * (W_SRC + A_SRC);
constexpr u32 FB_OFF = 2 * PAR_BYTES;
constexpr u32 FA_OFF = FB_OFF + 32 * 1024;
constexpr u32 CNT_OFF = FA_OFF + 4 * 1024;
constexpr u32 TMO_OFF = CNT_OFF + NWG * 128;
constexpr u32 DBG_OFF = TMO_OFF + 4096;
constexpr u32 WS_NEED = DBG_OFF + NWG * 256;
constexpr u64 DEADLINE = 30000000ull;

constexpr int PD = 3;
constexpr int NA = PD + 2;
constexpr u32 L_A = 0;
constexpr u32 L_B = NA * 16384;
constexpr u32 L_Q = L_B + 2 * 16384;
constexpr u32 L_BYTES = L_Q + 512;

struct Params {
  char* ws[NR];
  const unsigned char* x;
  const unsigned char* w;
  const float* sx;
  const float* sw;
  float* out;
  int rank;
  int pad;
};

template <int N> __device__ __forceinline__ void waitv() { asm volatile("s_waitcnt vmcnt(%0)" :: "i"(N) : "memory"); }
__device__ __forceinline__ void waitl() { asm volatile("s_waitcnt lgkmcnt(0)" ::: "memory"); }
__device__ __forceinline__ void bar() { asm volatile("s_barrier" ::: "memory"); }
template <int OFF> __device__ __forceinline__ void st16(const void* p, u32x4 x) {
  asm volatile("global_store_dwordx4 %0, %1, off offset:%2 sc0 sc1\n\ts_nop 1" :: "v"(p), "v"(x), "i"(OFF) : "memory");
}
template <int OFF> __device__ __forceinline__ void ld16(u32x4& x, const void* p) {
  asm volatile("global_load_dwordx4 %0, %1, off offset:%2 sc0 sc1" : "=v"(x) : "v"(p), "i"(OFF) : "memory");
}
__device__ __forceinline__ u32 ld_sys(const void* p) {
  return __hip_atomic_load((const u32*)p, __ATOMIC_RELAXED, __HIP_MEMORY_SCOPE_SYSTEM);
}
__device__ __forceinline__ u32 ld_agent(const void* p) {
  return __hip_atomic_load((const u32*)p, __ATOMIC_RELAXED, __HIP_MEMORY_SCOPE_AGENT);
}
__device__ __forceinline__ void st_agent(void* p, u32 v) {
  __hip_atomic_store((u32*)p, v, __ATOMIC_RELAXED, __HIP_MEMORY_SCOPE_AGENT);
}
__device__ __forceinline__ void put_flag(char* f, u32 call, int lane) {
  if (lane < 2) { u32x4 v = {call, call, call, call}; st16<0>(f + lane * 16, v); }
}
__device__ __forceinline__ bool give_up(char* me, u32 call, u64 t0) {
  if (ld_agent(me + TMO_OFF) == call || __builtin_amdgcn_s_memrealtime() - t0 > DEADLINE) {
    st_agent(me + TMO_OFF, call);
    return true;
  }
  return false;
}
#define STAMP(i) do {} while (0)

__device__ __forceinline__ int sched_t(int s, int rel) { return 4 * ((s + rel) & 7) + ((s >> 3) & 3); }

struct Packed { u32x4 p0, p1, p2; u32 l; };

__global__ void __launch_bounds__(NTHREADS) fused_gemm_kernel(Params P) {
  __shared__ __attribute__((aligned(16))) char smem[L_BYTES];
  LDSP char* lds = (LDSP char*)smem;
  const int lane = threadIdx.x & 63;
  const int wave = __builtin_amdgcn_readfirstlane(threadIdx.x >> 6);
  const int b = blockIdx.x, xcd = b & 7, slot = b >> 3;
  const int self = P.rank;
  char* me = P.ws[self];
  const int g = lane >> 4, c = lane & 15;

  if (slot < 8) {
    if (wave >= 2) return;
    STAMP(wave * 4 + 0);
    const int dst = (self + 1 + xcd) & 7;
    const int piece = slot * 2 + wave, j = piece >> 2, mm = piece & 3;
    u32* cnt = (u32*)(me + CNT_OFF + b * 128 + wave * 64);
    const u32 call = __builtin_amdgcn_readfirstlane(ld_agent(cnt)) + 1;
    const unsigned char* src = P.x + (size_t)(dst * 512 + mm * 128 + c) * KS + j * 128 + g * 16;
    u32x4 v[16];
#pragma unroll
    for (int i = 0; i < 16; ++i) v[i] = *(const u32x4*)(src + (i >> 1) * 16 * KS + (i & 1) * 64);
    asm volatile("" :: "v"(v[0]), "v"(v[1]), "v"(v[2]), "v"(v[3]), "v"(v[4]), "v"(v[5]), "v"(v[6]), "v"(v[7]),
                 "v"(v[8]), "v"(v[9]), "v"(v[10]), "v"(v[11]), "v"(v[12]), "v"(v[13]), "v"(v[14]), "v"(v[15]));
    STAMP(wave * 4 + 1);
    char* d = P.ws[dst] + (call & 1) * PAR_BYTES + A_OFF + self * A_SRC + piece * 16384 + lane * 16;
#pragma unroll
    for (int i = 0; i < 4; ++i) {
      st16<0>(d + i * 4096, v[i * 4 + 0]); st16<1024>(d + i * 4096, v[i * 4 + 1]);
      st16<2048>(d + i * 4096, v[i * 4 + 2]); st16<3072>(d + i * 4096, v[i * 4 + 3]);
    }
    waitv<0>();
    put_flag(P.ws[dst] + FA_OFF + ((mm * 4 + j) * 8 + self) * 32, call, lane);
    if (lane == 0) st_agent(cnt, call);
    STAMP(wave * 4 + 2);
    return;
  }
  if (slot < 12) {
    if (wave != 0 || xcd == 7) return;
    STAMP(0);
    const int dst = (self + 1 + xcd) & 7;
    const int cw = slot - 8;
    u32* cnt = (u32*)(me + CNT_OFF + b * 128);
    const u32 call = __builtin_amdgcn_readfirstlane(ld_agent(cnt)) + 1;
    {
      const int seq = (lane & 31) * 4 + cw;
      const char* f = me + FB_OFF + ((sched_t(seq, xcd) * 4 + (seq >> 5)) * 8 + self) * 32;
      const u64 t0 = __builtin_amdgcn_s_memrealtime();
      while (__builtin_amdgcn_ballot_w64(ld_sys(f) < call) != 0) {
        if (give_up(me, call, t0)) break;
        __builtin_amdgcn_s_sleep(1);
      }
    }
    STAMP(1);
    const u32 roff = (call & 1) * PAR_BYTES + self * W_SRC + lane * 16;
    const char* sbase = me + roff;
    char* dbase = P.ws[dst] + roff;
    char* fbase = P.ws[dst] + FB_OFF + self * 32;
    u32x4 X[13], Y[13];
#define BLK_OF(i) ((((i) * 4 + cw) >> 5) * 32 + sched_t((i) * 4 + cw, xcd))
#define LOAD13(R, blk) do { const char* s_ = sbase + (u32)(blk) * BLK_BYTES; \
      ld16<0>(R[0], s_); ld16<1024>(R[1], s_); ld16<2048>(R[2], s_); ld16<3072>(R[3], s_); s_ += 4096; \
      ld16<0>(R[4], s_); ld16<1024>(R[5], s_); ld16<2048>(R[6], s_); ld16<3072>(R[7], s_); s_ += 4096; \
      ld16<0>(R[8], s_); ld16<1024>(R[9], s_); ld16<2048>(R[10], s_); ld16<3072>(R[11], s_); s_ += 4096; \
      ld16<0>(R[12], s_); } while (0)
#define WAIT13(R, N) asm volatile("s_waitcnt vmcnt(" #N ")" : "+v"(R[0]), "+v"(R[1]), "+v"(R[2]), "+v"(R[3]), "+v"(R[4]), \
      "+v"(R[5]), "+v"(R[6]), "+v"(R[7]), "+v"(R[8]), "+v"(R[9]), "+v"(R[10]), "+v"(R[11]), "+v"(R[12]) :: "memory")
#define STORE13(R, blk) do { char* d_ = dbase + (u32)(blk) * BLK_BYTES; \
      const u32 hdr_ = (u32)__builtin_amdgcn_ds_bpermute((lane & 48) * 4, (int)R[12].x) & 0xff; \
      if (__builtin_amdgcn_ballot_w64(hdr_ == 255) != 0) {     \
        u32x4 h0, h1, h2, h3; const char* s_ = sbase + (u32)(blk) * BLK_BYTES + 13312; \
        asm volatile("global_load_dwordx4 %0, %4, off sc0 sc1\n\tglobal_load_dwordx4 %1, %4, off offset:1024 sc0 sc1\n\t" \
                     "global_load_dwordx4 %2, %4, off offset:2048 sc0 sc1\n\tglobal_load_dwordx4 %3, %4, off offset:3072 sc0 sc1\n\t" \
                     "s_waitcnt vmcnt(0)" : "=&v"(h0), "=&v"(h1), "=&v"(h2), "=&v"(h3) : "v"(s_) : "memory"); \
        st16<0>(d_ + 13312, h0); st16<1024>(d_ + 13312, h1); st16<2048>(d_ + 13312, h2); st16<3072>(d_ + 13312, h3); \
        waitv<0>(); } \
      waitv<26>(); st16<0>(d_, R[0]); waitv<26>(); st16<1024>(d_, R[1]); waitv<26>(); st16<2048>(d_, R[2]); waitv<26>(); st16<3072>(d_, R[3]); d_ += 4096; \
      waitv<26>(); st16<0>(d_, R[4]); waitv<26>(); st16<1024>(d_, R[5]); waitv<26>(); st16<2048>(d_, R[6]); waitv<26>(); st16<3072>(d_, R[7]); d_ += 4096; \
      waitv<26>(); st16<0>(d_, R[8]); waitv<26>(); st16<1024>(d_, R[9]); waitv<26>(); st16<2048>(d_, R[10]); waitv<26>(); st16<3072>(d_, R[11]); d_ += 4096; \
      waitv<26>(); if ((u32)(c >> 2) * 32 <= (hdr_ == 255 ? 0u : hdr_)) st16<0>(d_, R[12]); } while (0)
#define FLAG_OF(blk) (fbase + ((((blk) & 31) * 4 + ((blk) >> 5)) * 8) * 32)
    LOAD13(X, BLK_OF(0));
    WAIT13(X, 0);
    for (int i = 0; i < 32; i += 2) {
      LOAD13(Y, BLK_OF(i + 1));
      STORE13(X, BLK_OF(i));
      WAIT13(Y, 12);
      if (i > 0) put_flag(FLAG_OF(BLK_OF(i - 1)), call, lane);
      const int nx = i + 2 < 32 ? i + 2 : 31;
      LOAD13(X, BLK_OF(nx));
      STORE13(Y, BLK_OF(i + 1));
      WAIT13(X, 12);
      put_flag(FLAG_OF(BLK_OF(i)), call, lane);
    }
    waitv<0>();
    put_flag(FLAG_OF(BLK_OF(31)), call, lane);
    if (lane == 0) st_agent(cnt, call);
    STAMP(2);
    return;
  }
  if (slot < 16) return;

  const int cs = slot - 16;
  const int t = xcd * 4 + (cs & 3), mm = cs >> 2;
  u32* cnt = (u32*)(me + CNT_OFF + b * 128);
  const u32 call = __builtin_amdgcn_readfirstlane(ld_agent(cnt)) + 1;
  char* par = me + (call & 1) * PAR_BYTES;
  LDSP u32* sq = (LDSP u32*)(lds + L_Q);

  if (wave >= 4 && wave < 8) {
    const int u = wave - 4, h = u & 1, nh = u >> 1;
    if (u == 0) STAMP(0);
    {
      const unsigned char* src = P.w + (size_t)(mm * 128 + h * 64 + g * 16) * NN + t * 128 + nh * 64 + c * 4;
      u32 W[16], D[4][4];
#pragma unroll
      for (int r = 0; r < 16; ++r) W[r] = *(const u32*)(src + r * NN);
#pragma unroll
      for (int q = 0; q < 4; ++q) {
        const u32 t0 = __builtin_amdgcn_perm(W[4 * q + 1], W[4 * q], 0x05010400u), t1 = __builtin_amdgcn_perm(W[4 * q + 1], W[4 * q], 0x07030602u);
        const u32 t2 = __builtin_amdgcn_perm(W[4 * q + 3], W[4 * q + 2], 0x05010400u), t3 = __builtin_amdgcn_perm(W[4 * q + 3], W[4 * q + 2], 0x07030602u);
        D[0][q] = __builtin_amdgcn_perm(t2, t0, 0x05040100u); D[1][q] = __builtin_amdgcn_perm(t2, t0, 0x07060302u);
        D[2][q] = __builtin_amdgcn_perm(t3, t1, 0x05040100u); D[3][q] = __builtin_amdgcn_perm(t3, t1, 0x07060302u);
      }
      u32 mx = 0;
#pragma unroll
      for (int r = 0; r < 16; ++r) mx = max(mx, W[r] & 0x7fffffffu);
      mx >>= 24;
#pragma unroll
      for (int s = 1; s < 64; s <<= 1) mx = max(mx, (u32)__shfl_xor((int)mx, s));
      const u32 base = __builtin_amdgcn_readfirstlane(mx > 31 ? mx - 31 : 0);
      const u32 brep = base * 0x01010101u;
      u32 Pk[3][4], H[4];
#pragma unroll
      for (int q = 0; q < 4; ++q) {
        u32 code[4];
        H[q] = 0;
#pragma unroll
        for (int i = 0; i < 4; ++i) {
          const u32 d = (D[i][q] | 0x80808080u) - brep;
          code[i] = (d & 0x1f1f1f1fu) | (D[i][q] & 0x80808080u);
          H[q] |= ((D[i][q] ^ (code[i] + brep)) >> 5) << (2 * i);
        }
        Pk[0][q] = code[0] | ((code[3] & 0x03030303u) << 5);
        Pk[1][q] = code[1] | ((code[3] & 0x0c0c0c0cu) << 3);
        Pk[2][q] = code[2] | ((code[3] & 0x10101010u) << 1) | ((code[3] & 0x80808080u) >> 1);
      }
      char* blk = par + self * W_SRC + (mm * 32 + t) * BLK_BYTES;
      {
        char* d = blk + u * 3072 + lane * 16;
        st16<0>(d, u32x4{Pk[0][0], Pk[0][1], Pk[0][2], Pk[0][3]});
        st16<1024>(d, u32x4{Pk[1][0], Pk[1][1], Pk[1][2], Pk[1][3]});
        st16<2048>(d, u32x4{Pk[2][0], Pk[2][1], Pk[2][2], Pk[2][3]});
      }
      LDSP unsigned short* lst = (LDSP unsigned short*)(lds + L_B + u * 256);
      u32 n = 0;
#pragma unroll
      for (int q = 0; q < 4; ++q) {
        u32 hq = H[q];
        u64 m;
        while ((m = __builtin_amdgcn_ballot_w64(hq != 0)) != 0 && n <= 127) {
          if (hq != 0) {
            const u32 f = __builtin_ctz(hq) & ~1u;
            const u32 pos = n + 1 + __builtin_amdgcn_mbcnt_hi((u32)(m >> 32), __builtin_amdgcn_mbcnt_lo((u32)m, 0));
            const u32 e = (((hq >> f) & 3) << 14) | (nh * 8192 + ((f >> 1) & 3) * 2048 + h * 1024 + lane * 16 + q * 4 + (f >> 3));
            if (pos <= 127) lst[pos] = (unsigned short)e;
            hq &= ~(3u << f);
          }
          n += __builtin_popcountll(m);
        }
      }
      if (n > 127) {
        n = 255;
        st16<0>(blk + 13312 + u * 1024 + lane * 16, u32x4{H[0], H[1], H[2], H[3]});
      }
      if (lane == 0) lst[0] = (unsigned short)(n | (base << 8));
      waitl();
      if (lane < 16) st16<0>(blk + 12288 + u * 256 + lane * 16, *(LDSP u32x4*)(lds + L_B + u * 256 + lane * 16));
      waitv<0>();
    }
    bar();
    bar();
    const __amdgpu_buffer_rsrc_t rs = __builtin_amdgcn_make_buffer_rsrc(par, 0, NR * W_SRC, 0x00020000);
    const u32 vo = lane * 16 + u * 3072, vl = lane * 4 + 12288 + u * 256;
    u32 so[4];
    Packed S[4];
#define P_ISSUE(n, k) do { \
      const u32 id_ = __builtin_amdgcn_readfirstlane(sq[(n) < 32 ? (n) : 31]); \
      so[k] = (id_ & 7) * W_SRC + ((id_ >> 3) * 32 + t) * BLK_BYTES; \
      S[k].p0 = __builtin_amdgcn_raw_buffer_load_b128(rs, vo, so[k], 17); \
      S[k].p1 = __builtin_amdgcn_raw_buffer_load_b128(rs, vo + 1024, so[k], 17); \
      S[k].p2 = __builtin_amdgcn_raw_buffer_load_b128(rs, vo + 2048, so[k], 17); \
      S[k].l = __builtin_amdgcn_raw_buffer_load_b32(rs, vl, so[k], 17); } while (0)
#define P_DECODE(n, k) do { if ((n) >= PD && (n) < 32 + PD) { \
      const u32 hdr_ = __builtin_amdgcn_readfirstlane(S[k].l); \
      const u32 cnt_ = hdr_ & 0xff, br_ = ((hdr_ >> 8) & 0x7f) * 0x01010101u, lut_ = br_ + 0x90801000u; \
      u32x4 d0, d1, d2, d3; \
      _Pragma("unroll") for (int q = 0; q < 4; ++q) { \
        const u32 e0 = S[k].p0[q] & 0x60606060u, e1 = S[k].p1[q] & 0x60606060u, e2 = S[k].p2[q] & 0x60606060u; \
        d0[q] = (S[k].p0[q] ^ e0) + br_; d1[q] = (S[k].p1[q] ^ e1) + br_; d2[q] = (S[k].p2[q] ^ e2) + br_; \
        d3[q] = (e0 >> 5) + (e1 >> 3) + __builtin_amdgcn_perm(lut_, lut_, e2 >> 5); } \
      if (cnt_ == 255) { \
        const u32x4 hp = __builtin_amdgcn_raw_buffer_load_b128(rs, lane * 16 + 13312 + u * 1024, so[k], 17); \
        _Pragma("unroll") for (int q = 0; q < 4; ++q) { \
          d0[q] ^= (hp[q] & 0x03030303u) << 5; d1[q] ^= (hp[q] & 0x0c0c0c0cu) << 3; \
          d2[q] ^= (hp[q] & 0x30303030u) << 1; d3[q] ^= (hp[q] & 0xc0c0c0c0u) >> 1; } } \
      LDSP char* bs_ = lds + L_B + (((n) - PD) & 1) * 16384; \
      LDSP char* bw_ = bs_ + nh * 8192 + h * 1024 + lane * 16; \
      *(LDSP u32x4*)(bw_) = d0; *(LDSP u32x4*)(bw_ + 2048) = d1; *(LDSP u32x4*)(bw_ + 4096) = d2; *(LDSP u32x4*)(bw_ + 6144) = d3; \
      if (cnt_ != 255) { \
        const u32 ea = S[k].l & 0xffff, eb = S[k].l >> 16; \
        if ((u32)(2 * lane - 1) < cnt_) \
          __hip_atomic_fetch_xor((LDSP u32*)(bs_ + (ea & 0x3ffc)), (ea >> 14) << ((ea & 3) * 8 + 5), __ATOMIC_RELAXED, __HIP_MEMORY_SCOPE_WORKGROUP); \
        if ((u32)(2 * lane) < cnt_) \
          __hip_atomic_fetch_xor((LDSP u32*)(bs_ + (eb & 0x3ffc)), (eb >> 14) << ((eb & 3) * 8 + 5), __ATOMIC_RELAXED, __HIP_MEMORY_SCOPE_WORKGROUP); } } \
      waitl(); bar(); } while (0)
    static_assert(PD == 3, "register sets below assume PD == 3");
    for (int n = 0; n < 36; n += 4) {
      P_ISSUE(n, 0); P_DECODE(n, 1);
      P_ISSUE(n + 1, 1); P_DECODE(n + 1, 2);
      P_ISSUE(n + 2, 2); P_DECODE(n + 2, 3);
      P_ISSUE(n + 3, 3); P_DECODE(n + 3, 0);
    }
    return;
  }

  if (wave == 8) {
    bar();
    put_flag(me + FB_OFF + ((t * 4 + mm) * 8 + self) * 32, call, lane);
    STAMP(1);
    const char* f = me + (lane < 32 ? FB_OFF + t * 1024 + lane * 32 : FA_OFF + mm * 1024 + (lane - 32) * 32);
    u64 seen = 0;
    u32 done = 0;
    bool dead = false;
    const u64 t0 = __builtin_amdgcn_s_memrealtime();
    for (int n = 0; n < 37; ++n) {
      if (n < 32) {
        u32 rdy = (u32)seen & (u32)(seen >> 32) & ~done;
        while (rdy == 0) {
          u32 v = 0;
          if (!((seen >> lane) & 1)) v = ld_sys(f);
          seen |= __builtin_amdgcn_ballot_w64(v >= call);
          rdy = (u32)seen & (u32)(seen >> 32) & ~done;
          if (rdy != 0) break;
          if (dead || give_up(me, call, t0)) { dead = true; seen = ~0ull; }
          else __builtin_amdgcn_s_sleep(1);
        }
        const u32 id = __builtin_ctz(rdy);
        done |= 1u << id;
        if (lane == 0) sq[n] = id;
        waitl();
        if (n == 31) STAMP(2);
      }
      bar();
    }
    if (lane == 0) st_agent(cnt, call);
    return;
  }

  {
    const int wm = wave >> 1, wn = wave & 1;
    const float scale = P.sx[0] * P.sw[0];
    v4f acc[4][4];
#pragma unroll
    for (int i = 0; i < 4; ++i)
#pragma unroll
      for (int k = 0; k < 4; ++k) acc[i][k] = v4f{0.f, 0.f, 0.f, 0.f};
    const char* abase = par + A_OFF + mm * 16384 + wave * 4096 + lane * 16;
    const u32 lbase = (u32)(size_t)lds;
    bar();
    bar();
    int sa = 0;
    for (int n = 0; n < 36; ++n) {
      {
        const u32 id = __builtin_amdgcn_readfirstlane(sq[n < 32 ? n : 31]);
        const char* s = abase + (id & 7) * A_SRC + (id >> 3) * 65536;
        const u32 dstl = lbase + L_A + sa * 16384 + wave * 4096;
        u32 keep;
        asm volatile("s_mov_b32 %0, m0\n\ts_mov_b32 m0, %2\n\ts_nop 0\n\t"
                     "global_load_lds_dwordx4 %1, off sc0 sc1\n\t"
                     "global_load_lds_dwordx4 %1, off offset:1024 sc0 sc1\n\t"
                     "global_load_lds_dwordx4 %1, off offset:2048 sc0 sc1\n\t"
                     "global_load_lds_dwordx4 %1, off offset:3072 sc0 sc1\n\t"
                     "s_mov_b32 m0, %0" : "=&s"(keep) : "v"(s), "s"(dstl) : "memory");
      }
      if (n >= PD + 1) {
        const int sr = sa + 1 == NA ? 0 : sa + 1;
        LDSP char* ap = lds + L_A + sr * 16384 + wm * 8192 + lane * 16;
        LDSP char* bp = lds + L_B + ((n - PD - 1) & 1) * 16384 + wn * 8192 + lane * 16;
        v8i af[4], bf[4];
#pragma unroll
        for (int i = 0; i < 4; ++i) {
          const u32x4 a0 = *(LDSP u32x4*)(ap + i * 2048), a1 = *(LDSP u32x4*)(ap + i * 2048 + 1024);
          const u32x4 b0 = *(LDSP u32x4*)(bp + i * 2048), b1 = *(LDSP u32x4*)(bp + i * 2048 + 1024);
          af[i] = v8i{(int)a0.x, (int)a0.y, (int)a0.z, (int)a0.w, (int)a1.x, (int)a1.y, (int)a1.z, (int)a1.w};
          bf[i] = v8i{(int)b0.x, (int)b0.y, (int)b0.z, (int)b0.w, (int)b1.x, (int)b1.y, (int)b1.z, (int)b1.w};
        }
#pragma unroll
        for (int i = 0; i < 4; ++i)
#pragma unroll
          for (int k = 0; k < 4; ++k)
            acc[i][k] = __builtin_amdgcn_mfma_scale_f32_16x16x128_f8f6f4(af[i], bf[k], acc[i][k], 1, 1, 0, 0, 0, 0);
      }
      waitv<4 * PD>();
      waitl();
      bar();
      sa = sa + 1 == NA ? 0 : sa + 1;
    }
    if (wave == 0) STAMP(3);
    const bool bad = ld_agent(me + TMO_OFF) == call;
    float* o = P.out + (size_t)(mm * 128 + wm * 64 + g * 4) * NN + t * 128 + wn * 64 + c * 4;
#pragma unroll
    for (int i = 0; i < 4; ++i)
#pragma unroll
      for (int r = 0; r < 4; ++r) {
        v4f y;
#pragma unroll
        for (int k = 0; k < 4; ++k) {
          const float z = acc[i][k][r] * scale;
          y[k] = bad ? __builtin_nanf("") : (z < 0.f ? 0.f : z);
        }
        *(v4f*)(o + (size_t)(i * 16 + r) * NN) = y;
      }
    if (wave == 0) { waitv<0>(); STAMP(4); }
  }
}

extern "C" void kernel_launch(int rank, int nranks, ncclComm_t comm,
                              hipStream_t stream,
                              void* const* d_in, const int* in_sizes, int n_in,
                              void* d_out, int out_size,
                              const int* in_dtypes, int out_dtype,
                              void* const* d_out_all,
                              void* const* d_ws_all, size_t ws_bytes) {
  if (nranks != NR || (unsigned)rank >= (unsigned)NR || n_in < 4 || ws_bytes < WS_NEED ||
      in_sizes[0] != 4096 * KS || in_sizes[1] != KS * NN || out_size != 512 * NN) {
    fprintf(stderr, "kernel_launch: unexpected problem shape\n");
    return;
  }
  Params p;
  for (int i = 0; i < NR; ++i) p.ws[i] = (char*)d_ws_all[i];
  p.x = (const unsigned char*)d_in[0];
  p.w = (const unsigned char*)d_in[1];
  p.sx = (const float*)d_in[2];
  p.sw = (const float*)d_in[3];
  p.out = (float*)d_out;
  p.rank = rank;
  p.pad = 0;
  hipLaunchKernelGGL(fused_gemm_kernel, dim3(NWG), dim3(NTHREADS), 0, stream, p);
  HIP_CHECK(hipGetLastError());
}
```

```cpp
#include <hip/hip_runtime.h>
#include <rccl/rccl.h>
#include <cstdio>
#include <cstdint>

#define HIP_CHECK(expr) do { hipError_t _e = (expr); if (_e != hipSuccess) { \
    fprintf(stderr, "HIP error %s at %s:%d\n", hipGetErrorString(_e), __FILE__, __LINE__); \
    return; } } while (0)

typedef unsigned u32;
typedef unsigned long long u64;
typedef unsigned u32x4 __attribute__((ext_vector_type(4)));
typedef int v8i __attribute__((ext_vector_type(8)));
typedef float v4f __attribute__((ext_vector_type(4)));
#define LDSP __attribute__((address_space(3)))

constexpr int NR = 8;
constexpr int KS = 512;
constexpr int NN = 4096;
constexpr int NTHREADS = 768;
constexpr int NWG = 256;
#define NCW 8
#define WD 7

constexpr u32 BLK_BYTES = 17 * 1024;
constexpr u32 W_SRC = 128 * BLK_BYTES;
constexpr u32 A_SRC = 256 * 1024;
constexpr u32 A_OFF = NR * W_SRC;
constexpr u32 PAR_BYTES = NR * (W_SRC + A_SRC);
constexpr u32 FB_OFF = 2 * PAR_BYTES;
constexpr u32 FA_OFF = FB_OFF + 32 * 1024;
constexpr u32 CNT_OFF = FA_OFF + 4 * 1024;
constexpr u32 TMO_OFF = CNT_OFF + NWG * 256;
constexpr u32 TK_OFF = TMO_OFF + 1024;
constexpr u32 DBG_OFF = TMO_OFF + 4096;
constexpr u32 WS_NEED = DBG_OFF + NWG * 256;
constexpr u64 DEADLINE = 30000000ull;

constexpr int NBS = 3;
constexpr u32 L_A = 0;
constexpr u32 L_B = 8 * 16384;
constexpr u32 L_LST = L_B + NBS * 8192;
constexpr u32 L_CTL = L_LST + 1024;
constexpr u32 L_BYTES = L_CTL + 64;
constexpr int C_FREE = 0, C_FULL = 4, C_TICKET = 12, C_ENC = 13;

#define O_SC "sc1"
#define SCHED 0xFEBADC7986543210ull
#define PRIO 2
#define MSLEEP 0

struct Params {
  char* ws[NR];
  const unsigned char* x;
  const unsigned char* w;
  const float* sx;
  const float* sw;
  float* out;
  int rank;
  int pad;
};

template <int N> __device__ __forceinline__ void waitv() { asm volatile("s_waitcnt vmcnt(%0)" :: "i"(N) : "memory"); }
__device__ __forceinline__ void waitl() { asm volatile("s_waitcnt lgkmcnt(0)" ::: "memory"); }
__device__ __forceinline__ void bar() { asm volatile("s_barrier" ::: "memory"); }
template <int OFF> __device__ __forceinline__ void st16(const void* p, u32x4 x) {
  asm volatile("global_store_dwordx4 %0, %1, off offset:%2 sc0 sc1\n\ts_nop 1" :: "v"(p), "v"(x), "i"(OFF) : "memory");
}
template <int OFF> __device__ __forceinline__ void ld16(u32x4& x, const void* p) {
  asm volatile("global_load_dwordx4 %0, %1, off offset:%2 sc0 sc1" : "=v"(x) : "v"(p), "i"(OFF) : "memory");
}
__device__ __forceinline__ u32 ld_sys(const void* p) {
  return __hip_atomic_load((const u32*)p, __ATOMIC_RELAXED, __HIP_MEMORY_SCOPE_SYSTEM);
}
__device__ __forceinline__ u32 ld_agent(const void* p) {
  return __hip_atomic_load((const u32*)p, __ATOMIC_RELAXED, __HIP_MEMORY_SCOPE_AGENT);
}
__device__ __forceinline__ void st_agent(void* p, u32 v) {
  __hip_atomic_store((u32*)p, v, __ATOMIC_RELAXED, __HIP_MEMORY_SCOPE_AGENT);
}
__device__ __forceinline__ void put_flag(char* f, u32 call, int lane) {
  if (lane < 2) { u32x4 v = {call, call, call, call}; st16<0>(f + lane * 16, v); }
}
__device__ __forceinline__ bool give_up(char* me, u32 call, u64 t0) {
  if (ld_agent(me + TMO_OFF) == call || __builtin_amdgcn_s_memrealtime() - t0 > DEADLINE) {
    st_agent(me + TMO_OFF, call);
    return true;
  }
  return false;
}
#define STAMP(i) do {} while (0)
#define TS(v) do {} while (0)

__device__ __forceinline__ int sched_blk(int q, int rel) {
  const u32 e = (u32)(SCHED >> (4 * (q >> 3))) & 15;
  return (e >> 2) * 32 + 4 * ((q + rel) & 7) + (e & 3);
}
__device__ __forceinline__ u32 min4(u32x4 v) { return min(min(v.x, v.y), min(v.z, v.w)); }

__global__ void __launch_bounds__(NTHREADS) fused_gemm_kernel(Params P) {
  __shared__ __attribute__((aligned(16))) char smem[L_BYTES];
  LDSP char* lds = (LDSP char*)smem;
  volatile LDSP u32* ctl = (volatile LDSP u32*)(lds + L_CTL);
  const int lane = threadIdx.x & 63;
  const int wave = __builtin_amdgcn_readfirstlane(threadIdx.x >> 6);
  const int b = blockIdx.x, xcd = b & 7, slot = b >> 3;
  const int ct = slot >> 2, mm = slot & 3;
  const int self = P.rank;
  char* me = P.ws[self];
  const int g = lane >> 4, c = lane & 15;

  if (ct == 0 || ct == 2) {
    const int dst = (self + 1 + xcd) & 7;
    if (wave >= NCW / 8 || xcd == 7) return;
    u32* cnt = (u32*)(me + CNT_OFF + b * 256 + wave * 64);
    const int cw = (mm + ct * 2) * (NCW / 8) + wave;
    const u32 call = __builtin_amdgcn_readfirstlane(ld_agent(cnt)) + 1;
    const u64 t0 = __builtin_amdgcn_s_memrealtime();
    {
      const int fb = sched_blk(cw + (lane & 1) * NCW, xcd);
      const char* f = me + FB_OFF + ((fb & 31) * 8 + self) * 128 + (fb >> 5) * 32;
      __builtin_amdgcn_s_sleep(24);
      for (u32 it = 1; __builtin_amdgcn_ballot_w64(ld_sys(f) < call) != 0; ++it) {
        if ((it & 15) == 0 && give_up(me, call, t0)) break;
        __builtin_amdgcn_s_sleep(1);
      }
    }
    STAMP(wave);
    const u32 roff = (call & 1) * PAR_BYTES + self * W_SRC + lane * 16;
    const char* sbase = me + roff;
    char* dbase = P.ws[dst] + roff;
    char* fbase = P.ws[dst] + FB_OFF + self * 128;
    u32* tkp = (u32*)(me + TK_OFF + xcd * 128);
    const u32 tbase = (call - 1) * (128 - NCW) - 2 * NCW;
    u32x4 X[13], Y[13];
    u32 tkv = 0;
#define BLK_OF(q) sched_blk((q), xcd)
#define LOAD13(R, blk) do { const char* s_ = sbase + (u32)(blk) * BLK_BYTES; \
      ld16<0>(R[0], s_); ld16<1024>(R[1], s_); ld16<2048>(R[2], s_); ld16<3072>(R[3], s_); s_ += 4096; \
      ld16<0>(R[4], s_); ld16<1024>(R[5], s_); ld16<2048>(R[6], s_); ld16<3072>(R[7], s_); s_ += 4096; \
      ld16<0>(R[8], s_); ld16<1024>(R[9], s_); ld16<2048>(R[10], s_); ld16<3072>(R[11], s_); s_ += 4096; \
      ld16<0>(R[12], s_); } while (0)
#define WAIT13(R, N) asm volatile("s_waitcnt vmcnt(%14)" : "+v"(R[0]), "+v"(R[1]), "+v"(R[2]), "+v"(R[3]), "+v"(R[4]), \
      "+v"(R[5]), "+v"(R[6]), "+v"(R[7]), "+v"(R[8]), "+v"(R[9]), "+v"(R[10]), "+v"(R[11]), "+v"(R[12]), "+v"(tkv) : "i"(N) : "memory")
#define FLAG_OF(blk) (fbase + ((blk) & 31) * 1024 + ((blk) >> 5) * 32)
#define ST1(R, RN, k) do { \
      if ((k) == WD) { WAIT13(RN, WD); if (prv >= 0) put_flag(FLAG_OF(prv), call, lane); } \
      waitv<((k) < WD ? WD + 13 : WD)>(); \
      if ((k) < 12) st16<((k) & 3) * 1024>(d_ + ((k) >> 2) * 4096, R[k]); \
      else if ((u32)(c >> 2) * 32 <= (hdr_ == 127 ? 0u : max(hdr_, 32u))) st16<0>(d_ + 12288, R[12]); } while (0)
#define SEND(R, RN) { \
      if (nxt < 128) { \
        if (lane == 0) asm volatile("global_atomic_add %0, %1, %2, off sc0 sc1" : "+v"(tkv) : "v"(tkp), "v"(1u) : "memory"); \
      } \
      const int bn_ = BLK_OF(nxt < 128 ? nxt : 127);     \
      LOAD13(RN, bn_); \
      const u32 bo_ = (u32)cur * BLK_BYTES; char* d_ = dbase + bo_; \
      const u32 hdr_ = (u32)__builtin_amdgcn_ds_bpermute((lane & 48) * 4, (int)R[12].x) & 0x7f; \
      if (__builtin_amdgcn_ballot_w64(hdr_ == 127) != 0) {     \
        u32x4 h0, h1, h2, h3; const char* s_ = sbase + bo_ + 13312; \
        asm volatile("global_load_dwordx4 %0, %4, off sc0 sc1\n\tglobal_load_dwordx4 %1, %4, off offset:1024 sc0 sc1\n\t" \
                     "global_load_dwordx4 %2, %4, off offset:2048 sc0 sc1\n\tglobal_load_dwordx4 %3, %4, off offset:3072 sc0 sc1\n\t" \
                     "s_waitcnt vmcnt(0)" : "=&v"(h0), "=&v"(h1), "=&v"(h2), "=&v"(h3) : "v"(s_) : "memory"); \
        st16<0>(d_ + 13312, h0); st16<1024>(d_ + 13312, h1); st16<2048>(d_ + 13312, h2); st16<3072>(d_ + 13312, h3); } \
      ST1(R, RN, 0); ST1(R, RN, 1); ST1(R, RN, 2); ST1(R, RN, 3); ST1(R, RN, 4); ST1(R, RN, 5); ST1(R, RN, 6); \
      ST1(R, RN, 7); ST1(R, RN, 8); ST1(R, RN, 9); ST1(R, RN, 10); ST1(R, RN, 11); ST1(R, RN, 12); \
      if (prv < 0) {     \
        const char* f = me + FB_OFF + ((lane >> 1) * 8 + self) * 128 + (lane & 1) * 64; \
        for (u32 it = 1; __builtin_amdgcn_ballot_w64(min(ld_sys(f), ld_sys(f + 32)) < call) != 0; ++it) { \
          if ((it & 15) == 0 && give_up(me, call, t0)) break; \
          __builtin_amdgcn_s_sleep(1); } } \
      prv = cur; \
      if (nxt >= 128) break; \
      cur = bn_; \
      nxt = __builtin_amdgcn_readfirstlane(tkv) - tbase; }
    static_assert(WD >= 1 && WD <= 12, "window");
    int prv = -1, cur = BLK_OF(cw);
    u32 nxt = cw + NCW;
    LOAD13(X, cur);
    WAIT13(X, 0);
#pragma clang loop unroll(disable)
    for (;;) {
      SEND(X, Y)
      SEND(Y, X)
    }
    waitv<0>();
    put_flag(FLAG_OF(prv), call, lane);
    if (lane == 0) st_agent(cnt, call);
    STAMP(4 + wave);
    return;
  }

  const int t = xcd * 4 + (ct >> 1);
  const int dbl = 1 - (ct >> 2);
  u32* cnt = (u32*)(me + CNT_OFF + b * 256);

  if (wave >= 4) {
    const int sw = wave - 4;
    if (sw == 0 && lane < 16) ctl[lane] = 0;
    waitl();
    bar();
    const u32 call = __builtin_amdgcn_readfirstlane(ld_agent(cnt)) + 1;
    const char* par = me + (call & 1) * PAR_BYTES;
    const u32 lA = (u32)(size_t)lds + L_A + sw * 16384;
    const u64 t0 = __builtin_amdgcn_s_memrealtime();
    u32 m60 = 0x60606060u;
    asm volatile("" : "+s"(m60));
    u32 need = call;
    u32 prev = 0;
#pragma clang loop unroll(disable)
    for (int j = 0; j < 4; ++j) {
      const int src = sw;
      {
        const char* fa = me + FA_OFF + (mm * 8 + src) * 128 + j * 32;
        for (u32 it = 1; __builtin_amdgcn_readfirstlane(ld_sys(fa)) < need; ++it) {
          if ((it & 15) == 0 && give_up(me, call, t0)) need = 0;
          __builtin_amdgcn_s_sleep(4);
        }
        while (__builtin_amdgcn_readfirstlane(min4(*(volatile LDSP u32x4*)(ctl + C_FREE))) < prev) __builtin_amdgcn_s_sleep(2);
        const char* sa = par + A_OFF + src * A_SRC + (j * 4 + mm) * 16384 + lane * 16;
        u32 keep;
#pragma unroll
        for (int i = 0; i < 4; ++i)
          asm volatile("s_mov_b32 %0, m0\n\ts_mov_b32 m0, %2\n\ts_nop 0\n\t"
                       "global_load_lds_dwordx4 %1, off sc0 sc1\n\t"
                       "global_load_lds_dwordx4 %1, off offset:1024 sc0 sc1\n\t"
                       "global_load_lds_dwordx4 %1, off offset:2048 sc0 sc1\n\t"
                       "global_load_lds_dwordx4 %1, off offset:3072 sc0 sc1\n\t"
                       "s_mov_b32 m0, %0" : "=&s"(keep) : "v"(sa + i * 4096), "s"(lA + i * 4096) : "memory");
      }
      const char* fb = me + FB_OFF + (t * 8 + src) * 128 + j * 32;
      if (j < 3)
        for (u32 it = 1; __builtin_amdgcn_readfirstlane(ld_sys(fb)) < need; ++it) {
          if ((it & 15) == 0 && give_up(me, call, t0)) need = 0;
          __builtin_amdgcn_s_sleep(8);
        }
      const int nh0 = dbl ? 0 : (ct & 1);
      const char* blk = par + src * W_SRC + (j * 32 + t) * BLK_BYTES;
      const char* pp = blk + nh0 * 6144 + 2048 + lane * 16;
      const char* pl = blk + 12288 + nh0 * 512 + lane * 2;
      u32x4 p[2][3], p2[2][3];
      u32 l[2], lx[2], l2[2], lx2[2], tv, tk, r0, r1;
      for (;;) {
        u32 it;
        asm volatile("s_mov_b32 %14, 96\n\t"
                     "global_load_dword %12, %15, off sc0 sc1\n\ts_sleep 7\n\t"
                     "global_load_dword %13, %15, off sc0 sc1\n\ts_sleep 7\n"
                     ".Lp_%=:\n\t"
                     "global_load_dword %10, %15, off sc0 sc1\n\ts_waitcnt vmcnt(2)\n\t"
                     "v_cmp_le_u32 vcc, %19, %12\n\ts_cbranch_vccnz .Lg_%=\n\t"
                     "global_load_dword %12, %15, off sc0 sc1\n\ts_waitcnt vmcnt(2)\n\t"
                     "v_cmp_le_u32 vcc, %19, %13\n\ts_cbranch_vccnz .Lg_%=\n\t"
                     "global_load_dword %13, %15, off sc0 sc1\n\ts_waitcnt vmcnt(2)\n\t"
                     "v_cmp_le_u32 vcc, %19, %10\n\ts_cbranch_vccnz .Lg_%=\n\t"
                     "s_sub_u32 %14, %14, 1\n\ts_cbranch_scc0 .Lp_%=\n\t"
                     "s_waitcnt vmcnt(0)\n\ts_branch .Le_%=\n"
                     ".Lg_%=:\n\t"
                     "global_load_ushort %6, %17, off sc0 sc1\n\t"
                     "global_load_dwordx4 %0, %16, off offset:-2048 sc0 sc1\n\t"
                     "global_load_dwordx4 %1, %16, off offset:-1024 sc0 sc1\n\t"
                     "global_load_dwordx4 %2, %16, off sc0 sc1\n\t"
                     "global_load_ushort %8, %17, off offset:128 sc0 sc1\n\t"
                     "global_load_ushort %7, %17, off offset:256 sc0 sc1\n\t"
                     "global_load_dwordx4 %3, %16, off offset:1024 sc0 sc1\n\t"
                     "global_load_dwordx4 %4, %16, off offset:2048 sc0 sc1\n\t"
                     "global_load_dwordx4 %5, %16, off offset:3072 sc0 sc1\n\t"
                     "global_load_ushort %9, %17, off offset:384 sc0 sc1\n\t"
                     "global_load_dword %10, %18, off sc1\n\t"
                     "v_mov_b32 %11, %20\n\ts_mov_b64 exec, 1\n\t"
                     "ds_add_rtn_u32 %11, %21, %11\n\ts_mov_b64 exec, -1\n\ts_waitcnt lgkmcnt(0)\n"
                     ".Le_%=:"
                     : "=&v"(p[0][0]), "=&v"(p[0][1]), "=&v"(p[0][2]), "=&v"(p[1][0]), "=&v"(p[1][1]), "=&v"(p[1][2]),
                       "=&v"(l[0]), "=&v"(l[1]), "=&v"(lx[0]), "=&v"(lx[1]), "=&v"(tv), "=&v"(tk), "=&v"(r0), "=&v"(r1), "=&s"(it)
                     : "v"(fb), "v"(pp), "v"(pl), "v"(me + TMO_OFF), "s"(need), "s"(1 + dbl),
                       "v"((u32)(size_t)(lds + L_CTL + C_TICKET * 4)) : "memory", "scc", "vcc");
        if (it != 0xffffffffu) break;
        if (give_up(me, call, t0)) need = 0;
      }
      asm volatile("s_cmp_eq_u32 %12, 0\n\ts_cbranch_scc1 .Ls_%=\n\t"
                   "global_load_ushort %6, %11, off offset:512 sc0 sc1\n\t"
                   "global_load_ushort %8, %11, off offset:640 sc0 sc1\n\t"
                   "global_load_ushort %7, %11, off offset:768 sc0 sc1\n\t"
                   "global_load_ushort %9, %11, off offset:896 sc0 sc1\n\t"
                   "global_load_dwordx4 %0, %10, off offset:-2048 sc0 sc1\n\t"
                   "global_load_dwordx4 %1, %10, off offset:-1024 sc0 sc1\n\t"
                   "global_load_dwordx4 %2, %10, off sc0 sc1\n\t"
                   "global_load_dwordx4 %3, %10, off offset:1024 sc0 sc1\n\t"
                   "global_load_dwordx4 %4, %10, off offset:2048 sc0 sc1\n\t"
                   "global_load_dwordx4 %5, %10, off offset:3072 sc0 sc1\n"
                   ".Ls_%=:"
                   : "=&v"(p2[0][0]), "=&v"(p2[0][1]), "=&v"(p2[0][2]), "=&v"(p2[1][0]), "=&v"(p2[1][1]), "=&v"(p2[1][2]),
                     "=&v"(l2[0]), "=&v"(l2[1]), "=&v"(lx2[0]), "=&v"(lx2[1])
                   : "v"(pp + 6144), "v"(pl), "s"(dbl) : "memory", "scc");
      __builtin_amdgcn_s_setprio(PRIO);
      u32 ts = __builtin_amdgcn_readfirstlane(tk);
      asm volatile("s_waitcnt vmcnt(6)" : "+v"(p[0][0]), "+v"(p[0][1]), "+v"(p[0][2]), "+v"(l[0]), "+v"(lx[0]), "+v"(r0), "+v"(r1) :: "memory");
      TS(tsL);
#define UNPACK(h) do { \
          const u32 hdr = __builtin_amdgcn_readfirstlane(l[h]); \
          const u32 br = ((hdr >> 7) & 0x7f) * 0x01010101u, lut = br + 0x90801000u; \
          cn[h] = hdr & 0x7f; \
          _Pragma("unroll") for (int q = 0; q < 4; ++q) { \
            u32 e0 = p[h][0][q] & m60, e1 = p[h][1][q] & m60, e2 = p[h][2][q] & m60; \
            asm("" : "+v"(e0), "+v"(e1), "+v"(e2)); \
            d[h][0][q] = (p[h][0][q] ^ e0) + br; d[h][1][q] = (p[h][1][q] ^ e1) + br; d[h][2][q] = (p[h][2][q] ^ e2) + br; \
            d[h][3][q] = (e0 >> 5) + (e1 >> 3) + __builtin_amdgcn_perm(lut, lut, e2 >> 5); } \
          if (cn[h] == 127) {                             \
            u32x4 hp; \
            asm volatile("global_load_dwordx4 %0, %1, off sc0 sc1\n\ts_waitcnt vmcnt(0)" : "=&v"(hp) \
                         : "v"(blk + 13312 + (nh * 2 + h) * 1024 + lane * 16) : "memory"); \
            _Pragma("unroll") for (int q = 0; q < 4; ++q) { \
              d[h][0][q] ^= (hp[q] & 0x03030303u) << 5; d[h][1][q] ^= (hp[q] & 0x0c0c0c0cu) << 3; \
              d[h][2][q] ^= (hp[q] & 0x30303030u) << 1; d[h][3][q] ^= (hp[q] & 0xc0c0c0c0u) >> 1; } \
            cn[h] = 0; l[h] = 0; } } while (0)
#define PUT(h) do { \
          LDSP char* ub = lds + L_B + bs * 8192 + h * 4096; \
          _Pragma("unroll") for (int i = 0; i < 4; ++i) *(LDSP u32x4*)(ub + i * 1024 + lane * 16) = d[h][i]; \
          __hip_atomic_fetch_xor((LDSP u32*)(ub + (l[h] & 0xffc)), (l[h] >> 14) << ((l[h] & 3) * 8 + 5), \
                                 __ATOMIC_RELAXED, __HIP_MEMORY_SCOPE_WORKGROUP); \
          if (cn[h] >= 64) { \
            const u32 eb = lx[h]; \
            if (lane + 64 <= cn[h]) \
              __hip_atomic_fetch_xor((LDSP u32*)(ub + (eb & 0xffc)), (eb >> 14) << ((eb & 3) * 8 + 5), __ATOMIC_RELAXED, __HIP_MEMORY_SCOPE_WORKGROUP); } \
          } while (0)
      int nh = nh0;
#pragma clang loop unroll(disable)
      for (;;) {
        const u32x4 fr = *(volatile LDSP u32x4*)(ctl + C_FREE);
        const u32 bs = ts % NBS;
        u32x4 d[2][4];
        u32 cn[2];
        UNPACK(0);
        const bool room = __builtin_amdgcn_readfirstlane(min4(fr)) + NBS > ts;
        asm volatile("" ::: "memory");
        if (room) PUT(0);
        asm volatile("s_waitcnt vmcnt(0)" : "+v"(p[1][0]), "+v"(p[1][1]), "+v"(p[1][2]), "+v"(l[1]), "+v"(lx[1]), "+v"(tv),
                     "+v"(p2[0][0]), "+v"(p2[0][1]), "+v"(p2[0][2]), "+v"(p2[1][0]), "+v"(p2[1][1]), "+v"(p2[1][2]),
                     "+v"(l2[0]), "+v"(l2[1]), "+v"(lx2[0]), "+v"(lx2[1]) :: "memory");
        UNPACK(1);
        if (!room) {
          while (__builtin_amdgcn_readfirstlane(min4(*(volatile LDSP u32x4*)(ctl + C_FREE))) + NBS <= ts) __builtin_amdgcn_s_sleep(1);
          asm volatile("" ::: "memory");
          PUT(0);
        }
        PUT(1);
        ++ts;
        asm volatile("" : "+s"(ts) :: "memory");
        if (lane == 0) ctl[C_FULL + bs] = ts | (sw << 8) | (tv == call ? 0x10000u : 0u);
        if (nh >= dbl) break;
        nh = 1;
#pragma unroll
        for (int h = 0; h < 2; ++h) { l[h] = l2[h]; lx[h] = lx2[h]; p[h][0] = p2[h][0]; p[h][1] = p2[h][1]; p[h][2] = p2[h][2]; }
      }
      prev = ts;
      __builtin_amdgcn_s_setprio(0);
      TS(tsD);
      if (prev == (32u << dbl)) {
        if (lane == 0) st_agent(cnt, call);
      }
    }
    return;
  }

  const int mw = wave;
  if (ct >= 4) {
    const int u = mw, h = u & 1, te = xcd * 4 + ct - 4;
    if (u == 0) STAMP(0);
    const unsigned char* src = P.w + (size_t)(mm * 128 + h * 64 + g * 16) * NN + te * 128 + (u >> 1) * 64 + c * 4;
    u32 W[16], D[4][4];
#pragma unroll
    for (int r = 0; r < 16; ++r) W[r] = *(const u32*)(src + r * NN);
    const u32 call = __builtin_amdgcn_readfirstlane(ld_agent(cnt)) + 1;
    bar();
#pragma unroll
    for (int q = 0; q < 4; ++q) {
      const u32 t0 = __builtin_amdgcn_perm(W[4 * q + 1], W[4 * q], 0x05010400u), t1 = __builtin_amdgcn_perm(W[4 * q + 1], W[4 * q], 0x07030602u);
      const u32 t2 = __builtin_amdgcn_perm(W[4 * q + 3], W[4 * q + 2], 0x05010400u), t3 = __builtin_amdgcn_perm(W[4 * q + 3], W[4 * q + 2], 0x07030602u);
      D[0][q] = __builtin_amdgcn_perm(t2, t0, 0x05040100u); D[1][q] = __builtin_amdgcn_perm(t2, t0, 0x07060302u);
      D[2][q] = __builtin_amdgcn_perm(t3, t1, 0x05040100u); D[3][q] = __builtin_amdgcn_perm(t3, t1, 0x07060302u);
    }
    u32 mx = 0;
#pragma unroll
    for (int r = 0; r < 16; ++r) mx = max(mx, W[r] & 0x7fffffffu);
    mx >>= 24;
    u32 top = 0;
#pragma unroll
    for (int s = 64; s > 0; s >>= 1) top += __builtin_amdgcn_ballot_w64(mx >= top + s) != 0 ? s : 0;
    const u32 base = top > 31 ? top - 31 : 0;
    const u32 brep = base * 0x01010101u;
    u32 Pk[3][4], H[4];
#pragma unroll
    for (int q = 0; q < 4; ++q) {
      u32 code[4];
      H[q] = 0;
#pragma unroll
      for (int i = 0; i < 4; ++i) {
        const u32 d = (D[i][q] | 0x80808080u) - brep;
        code[i] = (d & 0x1f1f1f1fu) | (D[i][q] & 0x80808080u);
        H[q] |= ((D[i][q] ^ (code[i] + brep)) >> 5) << (2 * i);
      }
      Pk[0][q] = code[0] | ((code[3] & 0x03030303u) << 5);
      Pk[1][q] = code[1] | ((code[3] & 0x0c0c0c0cu) << 3);
      Pk[2][q] = code[2] | ((code[3] & 0x10101010u) << 1) | ((code[3] & 0x80808080u) >> 1);
    }
    char* blk = me + (call & 1) * PAR_BYTES + self * W_SRC + (mm * 32 + te) * BLK_BYTES;
    {
      char* d = blk + u * 3072 + lane * 16;
      st16<0>(d, u32x4{Pk[0][0], Pk[0][1], Pk[0][2], Pk[0][3]});
      st16<1024>(d, u32x4{Pk[1][0], Pk[1][1], Pk[1][2], Pk[1][3]});
      st16<2048>(d, u32x4{Pk[2][0], Pk[2][1], Pk[2][2], Pk[2][3]});
    }
    LDSP unsigned short* lst = (LDSP unsigned short*)(lds + L_LST + u * 256);
    ((LDSP u32*)lst)[lane] = 0;
    u32 n = 0;
#pragma unroll
    for (int q = 0; q < 4; ++q) {
      u32 hq = H[q];
      u64 m;
      while ((m = __builtin_amdgcn_ballot_w64(hq != 0)) != 0 && n <= 126) {
        if (hq != 0) {
          const u32 f = __builtin_ctz(hq) & ~1u;
          const u32 pos = n + 1 + __builtin_amdgcn_mbcnt_hi((u32)(m >> 32), __builtin_amdgcn_mbcnt_lo((u32)m, 0));
          const u32 e = (((hq >> f) & 3) << 14) | (((f >> 1) & 3) * 1024 + lane * 16 + q * 4 + (f >> 3));
          if (pos <= 127) lst[pos] = (unsigned short)e;
          hq &= ~(3u << f);
        }
        n += __builtin_popcountll(m);
      }
    }
    if (n > 126) {
      n = 127;
      st16<0>(blk + 13312 + u * 1024 + lane * 16, u32x4{H[0], H[1], H[2], H[3]});
    }
    if (lane == 0) lst[0] = (unsigned short)(n | (base << 7));
    waitl();
    if (lane < 16) st16<0>(blk + 12288 + u * 256 + lane * 16, *(LDSP u32x4*)(lds + L_LST + u * 256 + lane * 16));
    waitv<0>();
    u32 k = 0;
    if (lane == 0) k = __hip_atomic_fetch_add((LDSP u32*)(ctl + C_ENC), 1u, __ATOMIC_RELAXED, __HIP_MEMORY_SCOPE_WORKGROUP);
    if (__builtin_amdgcn_readfirstlane(k) == 3) {
      put_flag(me + FB_OFF + (te * 8 + self) * 128 + mm * 32, call, lane);
      STAMP(1);
    }
  } else if (ct == 3 && mw == 0) {
      const int dst = (self + 1 + xcd) & 7;
      STAMP(4);
      const char* src = (const char*)P.x + (size_t)(dst * 512 + mm * 128 + c) * KS + g * 16;
      u32x4 X[16], Y[16];
      u32 cv;
#define LOADX(R, j) do { const char* s_ = src + (j) * 128; _Pragma("unroll") for (int i_ = 0; i_ < 8; ++i_) { \
        asm volatile("global_load_dwordx4 %0, %2, off\n\tglobal_load_dwordx4 %1, %2, off offset:64" \
                     : "=&v"(R[2 * i_]), "=&v"(R[2 * i_ + 1]) : "v"(s_ + i_ * 16 * KS) : "memory"); } } while (0)
#define WAIT16(R, N) asm volatile("s_waitcnt vmcnt(" #N ")" : "+v"(R[0]), "+v"(R[1]), "+v"(R[2]), "+v"(R[3]), "+v"(R[4]), \
      "+v"(R[5]), "+v"(R[6]), "+v"(R[7]), "+v"(R[8]), "+v"(R[9]), "+v"(R[10]), "+v"(R[11]), "+v"(R[12]), "+v"(R[13]), \
      "+v"(R[14]), "+v"(R[15]) :: "memory")
#define STOREX(R, j, T) do { char* d_ = dx + (j) * 65536; _Pragma("unroll") for (int i_ = 0; i_ < 4; ++i_) { \
        waitv<T>(); st16<0>(d_ + i_ * 4096, R[i_ * 4]); waitv<T>(); st16<1024>(d_ + i_ * 4096, R[i_ * 4 + 1]); \
        waitv<T>(); st16<2048>(d_ + i_ * 4096, R[i_ * 4 + 2]); waitv<T>(); st16<3072>(d_ + i_ * 4096, R[i_ * 4 + 3]); } } while (0)
      asm volatile("global_load_dword %0, %1, off sc1" : "=v"(cv) : "v"(cnt) : "memory");
      LOADX(X, 0);
      LOADX(Y, 1);
      bar();
      asm volatile("s_waitcnt vmcnt(32)" : "+v"(cv) :: "memory");
      const u32 call = __builtin_amdgcn_readfirstlane(cv) + 1;
      char* dx = P.ws[dst] + (call & 1) * PAR_BYTES + A_OFF + self * A_SRC + mm * 16384 + lane * 16;
      char* fx = P.ws[dst] + FA_OFF + (mm * 8 + self) * 128;
      WAIT16(X, 16);
      STAMP(5);
      STOREX(X, 0, 63);
      WAIT16(Y, 16);
      LOADX(X, 2);
      STOREX(Y, 1, 31);
      WAIT16(X, 16);
      put_flag(fx, call, lane);
      LOADX(Y, 3);
      STOREX(X, 2, 32);
      WAIT16(Y, 16);
      put_flag(fx + 32, call, lane);
      STOREX(Y, 3, 16);
      waitv<16>();
      put_flag(fx + 64, call, lane);
      waitv<0>();
      put_flag(fx + 96, call, lane);
      STAMP(6);
  } else {
    bar();
  }
  {
    u32 scale_bits = __builtin_amdgcn_readfirstlane(__builtin_bit_cast(u32, P.sx[0] * P.sw[0]));
    float* obase = P.out + (size_t)(mm * 128 + mw * 32) * NN + t * 128 + (ct & 1) * 64;
    asm volatile("" : "+s"(scale_bits), "+s"(obase));
    v4f Pc[2][4], Qc[2][4];
#pragma unroll
    for (int i = 0; i < 2; ++i)
#pragma unroll
      for (int k = 0; k < 4; ++k) Pc[i][k] = Qc[i][k] = v4f{0.f, 0.f, 0.f, 0.f};
#define FRAG(p, hs) ({ const u32x4 lo_ = *(LDSP u32x4*)(p), hi_ = *(LDSP u32x4*)((p) + (hs)); \
      v8i{(int)lo_.x, (int)lo_.y, (int)lo_.z, (int)lo_.w, (int)hi_.x, (int)hi_.y, (int)hi_.z, (int)hi_.w}; })
#define MMA(i, k, F) asm volatile("v_mfma_f32_16x16x128_f8f6f4 %0, %1, %2, %0 cbsz:1 blgp:1" : "+v"(F##c[i][k]) : "v"(F##a[i]), "v"(F##b[k]))
#define RD(F) do { bad |= w; LDSP char* ap_ = lds + L_A + ((w >> 8) & 7) * 16384 + mw * 4096 + lane * 16; \
      LDSP char* bp_ = lds + L_B + bs * 8192 + lane * 16; \
      bs = bs + 1 == NBS ? 0 : bs + 1; \
      F##a[0] = FRAG(ap_, 1024); F##b[0] = FRAG(bp_, 4096); F##b[1] = FRAG(bp_ + 1024, 4096); F##a[1] = FRAG(ap_ + 2048, 1024); \
      F##b[2] = FRAG(bp_ + 2048, 4096); F##b[3] = FRAG(bp_ + 3072, 4096); \
      asm volatile("" ::: "memory");     \
      if (lane == 0) ctl[C_FREE + mw] = n + 1; \
      __builtin_amdgcn_sched_barrier(0); } while (0)
#define SPIN() do { while (((w = __builtin_amdgcn_readfirstlane(ctl[C_FULL + bs])) & 0xff) != n + 1) __builtin_amdgcn_s_sleep(MSLEEP); \
      asm volatile("" ::: "memory"); } while (0)
#define STEP(F, G, EXTRA) do { \
      MMA(0, 0, F); \
      __builtin_amdgcn_sched_barrier(0); \
      peek = ctl[C_FULL + bs]; \
      __builtin_amdgcn_sched_barrier(0); \
      MMA(0, 1, F); MMA(1, 0, F); MMA(1, 1, F); \
      __builtin_amdgcn_sched_barrier(0); \
      ++n; \
      w = __builtin_amdgcn_readfirstlane(peek); \
      asm volatile("" ::: "memory"); \
      const bool ok_ = (w & 0xff) == n + 1; \
      if (ok_) RD(G); \
      MMA(0, 2, F); MMA(1, 2, F); MMA(0, 3, F); MMA(1, 3, F); \
      __builtin_amdgcn_sched_barrier(0); \
      EXTRA; \
      if (!ok_) { SPIN(); RD(G); } } while (0)
    v8i Pa[2], Pb[4], Qa[2], Qb[4];
    u32 n = 0, bs = 0, w, peek, bad = 0;
    SPIN();
    RD(P);
#pragma clang loop unroll(disable)
    for (int it = 1; it < (16 << dbl); ++it) { STEP(P, Q, ); STEP(Q, P, ); }
#define EPI(A, i, r, o) do { v4f y_; _Pragma("unroll") for (int k = 0; k < 4; ++k) \
        y_[k] = __builtin_elementwise_maximum(A[i][k][r] * scale, 0.f);     \
      asm volatile("global_store_dwordx4 %0, %1, %2 " O_SC "\n\ts_nop 1" :: "v"(ov), "v"(y_), "s"((o) + ((i) * 16 + (r)) * NN) : "memory"); \
      __builtin_amdgcn_sched_barrier(0); } while (0)
#define TIE(A, i, nop) asm volatile(nop : "+v"(A[i][0]), "+v"(A[i][1]), "+v"(A[i][2]), "+v"(A[i][3]))
    STEP(P, Q, if (!dbl) { TIE(Pc, 0, "s_nop 11"); TIE(Pc, 1, "");
                 _Pragma("unroll") for (int i = 0; i < 2; ++i) _Pragma("unroll") for (int k = 0; k < 4; ++k) Qc[i][k] += Pc[i][k];
                 TIE(Qc, 0, ""); TIE(Qc, 1, "s_nop 1"); });
    if (mw == 0) STAMP(2);
    MMA(0, 0, Q); MMA(0, 1, Q); MMA(0, 2, Q); MMA(0, 3, Q); MMA(1, 0, Q); MMA(1, 1, Q);
    const float scale = __builtin_bit_cast(float, (bad & 0x10000u) != 0 ? 0x7fc00000u : scale_bits);
    const u32 ln = __builtin_amdgcn_mbcnt_hi(~0u, __builtin_amdgcn_mbcnt_lo(~0u, 0));
    const u32 ov = (ln >> 4) * (4 * NN * 4) + (ln & 15) * 16;
    TIE(Qc, 0, "s_nop 9");
    EPI(Qc, 0, 0, obase); MMA(1, 2, Q); EPI(Qc, 0, 1, obase); MMA(1, 3, Q); EPI(Qc, 0, 2, obase); EPI(Qc, 0, 3, obase);
    TIE(Qc, 1, "");
    EPI(Qc, 1, 0, obase); EPI(Qc, 1, 1, obase); EPI(Qc, 1, 2, obase); EPI(Qc, 1, 3, obase);
    if (dbl) {
      EPI(Pc, 0, 0, obase - 64); EPI(Pc, 0, 1, obase - 64); EPI(Pc, 0, 2, obase - 64); EPI(Pc, 0, 3, obase - 64);
      EPI(Pc, 1, 0, obase - 64); EPI(Pc, 1, 1, obase - 64); EPI(Pc, 1, 2, obase - 64); EPI(Pc, 1, 3, obase - 64);
    }
  }
}

extern "C" void kernel_launch(int rank, int nranks, ncclComm_t comm,
                              hipStream_t stream,
                              void* const* d_in, const int* in_sizes, int n_in,
                              void* d_out, int out_size,
                              const int* in_dtypes, int out_dtype,
                              void* const* d_out_all,
                              void* const* d_ws_all, size_t ws_bytes) {
  if (nranks != NR || (unsigned)rank >= (unsigned)NR || n_in < 4 || ws_bytes < WS_NEED ||
      in_sizes[0] != 4096 * KS || in_sizes[1] != KS * NN || out_size != 512 * NN) {
    fprintf(stderr, "kernel_launch: unexpected problem shape\n");
    return;
  }
  Params p;
  for (int i = 0; i < NR; ++i) p.ws[i] = (char*)d_ws_all[i];
  p.x = (const unsigned char*)d_in[0];
  p.w = (const unsigned char*)d_in[1];
  p.sx = (const float*)d_in[2];
  p.sw = (const float*)d_in[3];
  p.out = (float*)d_out;
  p.rank = rank;
  p.pad = 0;
  hipLaunchKernelGGL(fused_gemm_kernel, dim3(NWG), dim3(NTHREADS), 0, stream, p);
  HIP_CHECK(hipGetLastError());
}
```

```cpp
#include <hip/hip_runtime.h>
#include <rccl/rccl.h>
#include <cstdio>
#include <cstdint>

#define HIP_CHECK(expr) do { hipError_t _e = (expr); if (_e != hipSuccess) { \
    fprintf(stderr, "HIP error %s at %s:%d\n", hipGetErrorString(_e), __FILE__, __LINE__); \
    return; } } while (0)

typedef unsigned u32;
typedef unsigned long long u64;
typedef unsigned u32x2 __attribute__((ext_vector_type(2)));
typedef unsigned u32x4 __attribute__((ext_vector_type(4)));
typedef int v8i __attribute__((ext_vector_type(8)));
typedef float v4f __attribute__((ext_vector_type(4)));
#define LDSP __attribute__((address_space(3)))
#define GLB __attribute__((address_space(1)))

constexpr int NR = 8;
constexpr int KS = 512;
constexpr int NN = 4096;
constexpr int NTHREADS = 768;
constexpr int NWG = 256;
constexpr int NIT = 288;
#define WD 6
#define NPUSH 8
#define GATE 150

constexpr u32 ITEM = 8192;
constexpr u32 FIX = 5120;
constexpr u32 PK_OFF = 0;
constexpr u32 RAW_OFF = 2 * 512 * ITEM;
constexpr u32 PX_OFF = RAW_OFF + 9 * NIT * ITEM;
constexpr u32 DIR_OFF = PX_OFF + 2 * NR * 16 * ITEM;
constexpr u32 UF_OFF = DIR_OFF + 2 * NR * NIT * 32;
constexpr u32 CNT_OFF = UF_OFF + NR * NIT * 64;
constexpr u32 TMO_OFF = CNT_OFF + NWG * 256;
constexpr u32 DBG_OFF = TMO_OFF + 4096;
constexpr u32 WS_NEED = DBG_OFF + NWG * 256;
constexpr u64 DEADLINE = 30000000ull;
static_assert(NPUSH <= 16, "PX region");

constexpr int NBS = 3;
constexpr u32 L_A = 0;
constexpr u32 L_B = 8 * 16384;
constexpr int NS = 12;
constexpr u32 L_IMG = NS * ITEM;
constexpr u32 L_SCR = L_B + NBS * ITEM;
constexpr u32 L_CTL = L_SCR + 4096;
constexpr u32 L_BYTES = L_CTL + 256;
constexpr int C_FREE = 0, C_FULL = 4, C_XCNT = 11, C_TICKET = 12, C_ENC = 13;
constexpr int C_CONS = 16, C_LANDED = 24, C_DESC = 32;
constexpr u32 M_RAW = 0x80, IT_END = 0x1ff;

#define O_SC "sc1"
#define SCHED 0xFEBADC7986543210ull
#define MSLEEP 0
#define DSLEEP 1
#define SPREAD 0
#define SPREAD0 0
#define POLLW 4u
#define A_SC "sc0 sc1"

struct Params {
  char* wsx[NR];
  char* me;
  const unsigned char* x;
  const unsigned char* w;
  int rank;
  u32 dstmap;
  const float* sx;
  const float* sw;
  float* out;
};

template <int N> __device__ __forceinline__ void waitv() { asm volatile("s_waitcnt vmcnt(%0)" :: "i"(N) : "memory"); }
__device__ __forceinline__ void waitl() { asm volatile("s_waitcnt lgkmcnt(0)" ::: "memory"); }
__device__ __forceinline__ void bar() { asm volatile("s_barrier" ::: "memory"); }
__device__ __forceinline__ void drain() { __builtin_amdgcn_s_waitcnt(0x0070); }
__device__ __forceinline__ void fence() { asm volatile("" ::: "memory"); }
template <int OFF> __device__ __forceinline__ void st16(const void* p, u32x4 x) {
  asm volatile("global_store_dwordx4 %0, %1, off offset:%2 sc0 sc1\n\ts_nop 1" :: "v"(p), "v"(x), "i"(OFF) : "memory");
}
__device__ __forceinline__ void st8(const void* p, u32x2 x) {
  asm volatile("global_store_dwordx2 %0, %1, off sc0 sc1\n\ts_nop 1" :: "v"(p), "v"(x) : "memory");
}
__device__ __forceinline__ void st4(const void* p, u32 x) {
  asm volatile("global_store_dword %0, %1, off sc0 sc1\n\ts_nop 1" :: "v"(p), "v"(x) : "memory");
}
__device__ __forceinline__ u32 ld_sys(const void* p) {
  return __hip_atomic_load((const GLB u32*)p, __ATOMIC_RELAXED, __HIP_MEMORY_SCOPE_SYSTEM);
}
__device__ __forceinline__ u32 ld_agent(const void* p) {
  return __hip_atomic_load((const GLB u32*)p, __ATOMIC_RELAXED, __HIP_MEMORY_SCOPE_AGENT);
}
__device__ __forceinline__ void st_agent(void* p, u32 v) {
  __hip_atomic_store((GLB u32*)p, v, __ATOMIC_RELAXED, __HIP_MEMORY_SCOPE_AGENT);
}
__device__ __forceinline__ u32 rfl(u32 v) { return __builtin_amdgcn_readfirstlane(v); }
__device__ __forceinline__ void put_flag(char* f, u32 v, int lane) {
  if (lane < 2) st16<0>(f + lane * 16, u32x4{v, v, v, v});
}
__device__ __forceinline__ bool give_up(char* me, u32 call, u64 t0) {
  if (ld_agent(me + TMO_OFF) == call || __builtin_amdgcn_s_memrealtime() - t0 > DEADLINE) {
    st_agent(me + TMO_OFF, call);
    return true;
  }
  return false;
}
#define LDS_ADD(p, v) __hip_atomic_fetch_add((LDSP u32*)(p), (v), __ATOMIC_RELAXED, __HIP_MEMORY_SCOPE_WORKGROUP)
#define LDS_SUB(p, v) __hip_atomic_fetch_sub((LDSP u32*)(p), (v), __ATOMIC_RELAXED, __HIP_MEMORY_SCOPE_WORKGROUP)
#define LDS_OR(p, v) __hip_atomic_fetch_or((LDSP u32*)(p), (v), __ATOMIC_RELAXED, __HIP_MEMORY_SCOPE_WORKGROUP)
#define LDS_XOR(p, v) __hip_atomic_fetch_xor((LDSP u32*)(p), (v), __ATOMIC_RELAXED, __HIP_MEMORY_SCOPE_WORKGROUP)
#define DPP(old, v, ctrl, rm) (u32)__builtin_amdgcn_update_dpp((int)(old), (int)(v), ctrl, rm, 0xf, false)
#define SCAN(name, id, OP) __device__ __forceinline__ u32 name(u32 v) { \
    v = OP(v, DPP(id, v, 0x111, 0xf)); v = OP(v, DPP(id, v, 0x112, 0xf)); v = OP(v, DPP(id, v, 0x114, 0xf)); \
    v = OP(v, DPP(id, v, 0x118, 0xf)); v = OP(v, DPP(id, v, 0x142, 0xa)); v = OP(v, DPP(id, v, 0x143, 0xc)); return v; }
__device__ __forceinline__ u32 addu(u32 a, u32 b) { return a + b; }
SCAN(scan_add, 0u, addu)
SCAN(scan_max, 0u, max)
SCAN(scan_min, 0xffffffffu, min)
#define LSTAMP(i) do {} while (0)
#define STAMP(i) do {} while (0)
#define TS(v) do {} while (0)

__device__ __forceinline__ u32 sched_item(u32 q, int rel) {
  if (q < 32 - NPUSH) return 256 + NPUSH + q;
  q -= 32 - NPUSH;
  const u32 e = (u32)(SCHED >> (4 * ((q >> 4) & 15))) & 15;
  return ((e >> 2) * 32 + 4 * ((q + rel) & 7) + (e & 3)) * 2 + ((q >> 3) & 1);
}
__device__ __forceinline__ u32 lds_ticket(volatile LDSP u32* p, u32 v) {
  asm volatile("s_mov_b64 exec, 1\n\tds_add_rtn_u32 %0, %1, %0\n\ts_mov_b64 exec, -1\n\ts_waitcnt lgkmcnt(0)"
               : "+v"(v) : "v"((u32)(size_t)p) : "memory");
  return rfl(v);
}
__device__ __forceinline__ u32 min4(u32x4 v) { return min(min(v.x, v.y), min(v.z, v.w)); }
__device__ __forceinline__ u32 gather8(const u32* f) {
  const u32 r0 = __builtin_amdgcn_udot4(f[1], 0x80402010u, __builtin_amdgcn_udot4(f[0], 0x08040201u, 0, false), false);
  const u32 r1 = __builtin_amdgcn_udot4(f[3], 0x80402010u, __builtin_amdgcn_udot4(f[2], 0x08040201u, 0, false), false);
  const u32 r2 = __builtin_amdgcn_udot4(f[5], 0x80402010u, __builtin_amdgcn_udot4(f[4], 0x08040201u, 0, false), false);
  const u32 r3 = __builtin_amdgcn_udot4(f[7], 0x80402010u, __builtin_amdgcn_udot4(f[6], 0x08040201u, 0, false), false);
  return (r0 >> 7) | (r1 << 1) | (r2 << 9) | (r3 << 17);
}

__global__ void __launch_bounds__(NTHREADS) fused_gemm_kernel(Params P) {
  __shared__ __attribute__((aligned(16))) char smem[L_BYTES];
  LDSP char* lds = (LDSP char*)smem;
  volatile LDSP u32* ctl = (volatile LDSP u32*)(lds + L_CTL);
  const int lane = threadIdx.x & 63;
  const int wave = rfl(threadIdx.x >> 6);
  const int b = blockIdx.x, xcd = b & 7, slot = b >> 3;
  const int ct = slot >> 2, mm = slot & 3;
  int self = P.rank;
  u32 dstmap = P.dstmap;
  char* me = P.me;
  char* pws = P.wsx[xcd];
  const bool xjob = (wave >> 2) == 1 || (wave >= 8 && ct >= 4);
  const unsigned char* in = xjob ? P.x : P.w;
  asm volatile("" : "+s"(self), "+s"(dstmap), "+s"(me), "+s"(pws), "+s"(in));
  const int peer = (dstmap >> (4 * xcd)) & 7;
  const bool link = (ct & 5) == 0;
  u32* cnt = (u32*)(me + CNT_OFF + b * 256 + wave * 16);
  u32 call;

  const u64 tstart = __builtin_amdgcn_s_memrealtime();
  if (wave == 11) LSTAMP(0);
  if (wave == 7) { ctl[lane] = 0; waitl(); }
  if (!link && (wave < 8 || (ct >> 1) != 2)) {
    const int g = lane >> 4, c = lane & 15;
    int jw = ct < 4 ? 3 - (ct & 1) : (ct - 4) >> 1;
    const int i8 = mm + (ct < 4 ? ct >> 1 : ct & 1) * 4;
    if (wave == 4) STAMP(0);
    const int j2 = wave >> 3;
    if (xjob) {
      const int xj = j2 ? 3 : 2 - jw, hx = xj * 8 + i8;
      const GLB unsigned char* src = (const GLB unsigned char*)in + (size_t)(peer * 512 + i8 * 64 + (wave & 3) * 16 + c) * KS + xj * 128 + g * 16;
      const u32x4 d0 = *(const GLB u32x4*)src, d1 = *(const GLB u32x4*)(src + 64);
      const u32 cv = ld_agent(cnt);
      bar();
      call = rfl(cv) + 1;
      const u32 par = call & 1;
      char* d = (hx < NPUSH ? pws + PX_OFF + ((par * NR + self) * 16 + hx) * ITEM
                 : peer == self ? me + RAW_OFF + ((par ? self : 8) * NIT + 256 + hx) * ITEM
                                : me + PK_OFF + (par * 512 + 256 + peer * 32 + hx) * ITEM) + (wave & 3) * 2048 + lane * 16;
      st16<0>(d, d0); st16<1024>(d, d1);
      const bool direct = hx < NPUSH || peer == self;
      char* f = pws + (direct ? UF_OFF + (self * NIT + 256 + hx) * 64 : DIR_OFF + ((par * NR + self) * NIT + 256 + hx) * 32) + lane * 16;
      const u32 v = direct ? call : (call << 8) | M_RAW;
      waitv<0>();
      if (lds_ticket(ctl + C_XCNT - j2, 1) == 3) {
        if (lane < (direct ? 4 : 2)) st16<0>(f, u32x4{v, v, v, v});
      }
    } else {
      jw += j2;
      const int u = wave & 1, hf = (wave >> 1) & 1, blk = (jw * 32 + xcd * 4) * 2 + i8;
      const GLB unsigned char* src = (const GLB unsigned char*)in + (size_t)(jw * 128 + u * 64 + g * 16) * NN + xcd * 512 + i8 * 64 + c * 4;
      u32 W[16], D[8];
#pragma unroll
      for (int r = 0; r < 16; ++r) W[r] = *(const GLB u32*)(src + r * NN);
      const u32 cv = ld_agent(cnt);
      LDSP char* scr = lds + L_SCR + (wave - j2 * 4) * 512;
      *(LDSP u32x2*)(scr + lane * 8) = u32x2{0, 0};
      bar();
#pragma unroll
      for (int s = 0; s < 8; ++s) LDS_ADD(scr + (((W[s >> 2] >> ((s & 3) * 8)) & 0x7f) << 2), 1u);
      const u32x2 hh = *(volatile LDSP u32x2*)(scr + lane * 8);
      *(volatile LDSP u32x2*)(scr + lane * 8) = u32x2{0, 0};
      const u32 sel = 0x05010400u + hf * 0x02020202u;
#pragma unroll
      for (int q = 0; q < 4; ++q) {
        const u32 t0 = __builtin_amdgcn_perm(W[4 * q + 1], W[4 * q], sel), t2 = __builtin_amdgcn_perm(W[4 * q + 3], W[4 * q + 2], sel);
        D[q] = __builtin_amdgcn_perm(t2, t0, 0x05040100u); D[4 + q] = __builtin_amdgcn_perm(t2, t0, 0x07060302u);
      }
      call = rfl(cv) + 1;
      TS(e0);
      const u32 par = call & 1;
      char* pk = me + PK_OFF + (par * 512 + blk) * ITEM;
      {
        char* d = me + RAW_OFF + ((par ? self : 8) * NIT + blk) * ITEM + u * 4096 + hf * 2048 + lane * 16;
        st16<0>(d, u32x4{D[0], D[1], D[2], D[3]}); st16<1024>(d, u32x4{D[4], D[5], D[6], D[7]});
        u32 s0 = 0;
#pragma unroll
        for (int k = 0; k < 8; ++k) s0 |= (D[k] & 0x80808080u) >> k;
        st4(pk + 4096 + u * 512 + hf * 256 + lane * 4, s0);
      }
      u32 base;
      {
        const u32 c0 = scan_add(hh.x + hh.y) - hh.x - hh.y, c1 = c0 + hh.x;
        u32 up = (u32)__builtin_amdgcn_ds_bpermute(((lane + 8) & 63) * 4, (int)(c0 | (c1 << 16)));
        if (lane >= 56) up = 512;
        const u32 k0 = ((9 * c0 + 32 * (512 - (up & 0xffff))) << 7) | (2 * lane);
        const u32 k1 = ((9 * c1 + 32 * (512 - (up >> 16))) << 7) | (2 * lane + 1);
        base = __builtin_amdgcn_readlane(scan_min(lane < 56 ? min(k0, k1) : lane == 56 ? k0 : 0xffffffffu), 63) & 0x7f;
      }
      TS(e1);
      const u32 b8f = base * 0x01010101u + 0x8f8f8f8fu;
      u32 F1[8], F2[8], FE[8], T[8];
#pragma unroll
      for (int k = 0; k < 8; ++k) {
        const u32 t = T[k] = b8f - (D[k] & 0x7f7f7f7fu);
        FE[k] = (~t | (t << 1) | ((t << 2) & (t << 3))) & 0x80808080u;
        F1[k] = ((t << 2) | (t << 3)) & ~FE[k] & 0x80808080u;
        F2[k] = (t << 2) & ~FE[k] & 0x80808080u;
      }
      {
        u32x4 cd;
#pragma unroll
        for (int q = 0; q < 4; ++q) cd[q] = ~((T[q] & 0x0f0f0f0fu) | ((T[4 + q] << 4) & 0xf0f0f0f0u));
        st16<0>(pk + u * 2048 + hf * 1024 + lane * 16, cd);
      }
      u32 x1 = gather8(F1), x2 = gather8(F2), xe = gather8(FE);
      const u64 any = __builtin_amdgcn_ballot_w64(x1 != 0);
      int prev;
      {
        const u32 lastp = x1 ? lane * 32 + 31 - __builtin_clz(x1) : 0;
        const u32 back = lane * 32 - scan_max(DPP(0, lastp, 0x138, 0xf));
        prev = -(int)(back - 15 * ((back * 4370) >> 16));
      }
      u32 a0 = 0, a1 = 0, a2 = 0, nc = 0;
      TS(e2);
#define PUSH_NIB(gg) do { a0 = __builtin_amdgcn_alignbit(a1, a0, 4); a1 = __builtin_amdgcn_alignbit(a2, a1, 4); \
        a2 = __builtin_amdgcn_alignbit((gg), a2, 4); ++nc; } while (0)
      while (x1 != 0) {
        const u32 pos = __builtin_ctz(x1);
        const u32 gap = pos - prev, gg = min(gap, 15u);
        prev += gg;
        const u32 bit = gap < 15 ? 1u << pos : 0u, in2 = x2 & bit;
        x2 ^= in2;
        x1 ^= bit ^ in2;
        PUSH_NIB(gg);
      }
      if ((any >> lane) > 1) while (prev <= 17) { prev += 15; PUSH_NIB(15u); }
      TS(e3);
      const u32 nend = scan_add(nc), nn = __builtin_amdgcn_readlane(nend, 63);
      bool over = nn > 512 || __builtin_amdgcn_ballot_w64(nc > 24) != 0;
      if (!over) {
        const u32 sh = (8 - (nend & 7)) * 4;
        LDSP char* q = scr + (nend >> 3) * 4;
        LDS_OR(q, (u32)((((u64)a0) << 32) >> sh));
        LDS_OR(q + 4, (u32)((((u64)a1 << 32) | a0) >> sh));
        LDS_OR(q + 8, (u32)((((u64)a2 << 32) | a1) >> sh));
        LDS_OR(q + 12, (u32)(((u64)a2) >> sh));
      }
      u32 ne = 0;
      {
        LDSP unsigned short* lst = (LDSP unsigned short*)(scr + 384);
        u64 mk;
        while ((mk = __builtin_amdgcn_ballot_w64(xe != 0)) != 0 && ne <= 63) {
          if (xe != 0) {
            const u32 pos = __builtin_ctz(xe);
            const u32 wa = (pos & 4) ? T[1] : T[0], wb = (pos & 4) ? T[3] : T[2], wc = (pos & 4) ? T[5] : T[4], wd = (pos & 4) ? T[7] : T[6];
            const u32 we = (pos & 8) ? wb : wa, wf = (pos & 8) ? wd : wc, wg = (pos & 16) ? wf : we;
            const u32 x4 = (wg >> ((pos & 3) * 8 + 4)) & 15;
            const u32 at = ne + __builtin_amdgcn_mbcnt_hi((u32)(mk >> 32), __builtin_amdgcn_mbcnt_lo((u32)mk, 0));
            if (at < 64) lst[at] = (unsigned short)((x4 << 12) | (hf * 2048 + lane * 32 + pos));
            xe &= xe - 1;
          }
          ne += __builtin_popcountll(mk);
        }
      }
      over = over || ne > 63;
      if (!over) {
        const u32 ed = (ne + 1) >> 1, nd = (nn + 7) >> 3;
        const u32 ev = *(volatile LDSP u32*)(scr + 384 + (lane & 31) * 4);
        u32 nv = *(volatile LDSP u32*)(scr + 12 + lane * 4);
        const u32 vo = (lds_ticket(ctl + C_ENC + j2, ed + nd) & 0xffff) + 4;
        if (lane == (nn >> 3)) nv |= ~0u << ((nn & 7) * 4);
        if (lane < nd) st4(pk + FIX + (vo + lane) * 4, nv);
        if (lane < ed) st4(pk + FIX + (vo + nd + lane) * 4, ev);
        if (lane == 0) st4(pk + FIX + (u * 2 + hf) * 4, vo | (base << 10) | (ne << 17) | (nd << 23));
      }
      TS(e4);
      waitv<0>();
      const u32 k = lds_ticket(ctl + C_ENC + j2, over ? 0x1010000u : 0x1000000u);
      if ((k >> 24) == 3) {
        const u32 v = (call << 8) | ((over || (k & 0xff0000) != 0) ? M_RAW : ((k & 0xffff) + 4 + 31) >> 5);
        if (lane < 4) st16<0>(me + UF_OFF + (self * NIT + blk) * 64 + lane * 16, u32x4{call, call, call, call});
        while (__builtin_amdgcn_s_memrealtime() - tstart < (u64)(jw * SPREAD + (jw ? SPREAD0 : 0))) __builtin_amdgcn_s_sleep(4);
        if (lane < 2) {
          const u32 fo = DIR_OFF + ((par * NR + self) * NIT + blk) * 32 + lane * 16;
#pragma unroll
          for (int x = 0; x < NR; ++x)
            asm volatile("global_store_dwordx4 %0, %1, %2 sc0 sc1\n\ts_nop 1" :: "v"(fo), "v"(u32x4{v, v, v, v}), "s"(P.wsx[x]) : "memory");
        }
        STAMP(1);
      }
    }
  } else {
    if (link && (wave >= 10 || peer == self)) return;
    bar();
    call = rfl(ld_agent(cnt)) + 1;
  }
  if (lane == 0) st_agent(cnt, call);

  if (link) {
    if (wave == 9) {
      const u32 par = call & 1;
      const u32 q = mm + ct * 2 + lane * 8;
      const u32 nitems = (NIT - NPUSH - (mm + ct * 2) + 7) >> 3;
      const u32 itv = lane < nitems ? sched_item(q, xcd) : 0;
      const char* dir = me + DIR_OFF + ((par * NR + peer) * NIT + itv) * 32;
      u32 dv = 0;
      bool dead = false;
      while (NPUSH != 0 && __builtin_amdgcn_s_memrealtime() - tstart < GATE - 50) __builtin_amdgcn_s_sleep(1);
      for (u32 n = 0; n < nitems + 4; ++n) {
        u32 w0 = (IT_END << 7) | (M_RAW << 16), w1 = 0;
        if (n < nitems) {
          const u32 it = __builtin_amdgcn_readlane(itv, n);
          u32 e;
          for (u32 sp = 1;; ++sp) {
            e = __builtin_amdgcn_readlane(dv, n);
            if ((e >> 8) == (call & 0xffffff)) break;
            if (dead) { e = M_RAW; break; }
            if (sp > 1 && n > 0) __builtin_amdgcn_s_sleep(4);
            if (lane - n < POLLW && lane < nitems && (dv >> 8) != (call & 0xffffff)) dv = ld_sys(dir);
            if ((sp & 31) == 0) dead = give_up(me, call, tstart);
          }
          e &= (e & M_RAW) ? M_RAW : 0xf;
          const u32 so = it >= 256 ? par * 512 + self * 32 + it : (e & M_RAW) ? RAW_OFF / ITEM + (par ? peer : 8) * NIT + it : par * 512 + it;
          w0 = (it << 7) | (e << 16);
          w1 = so;
        }
        if (n >= NS) {
          volatile LDSP u32x2* cp = (volatile LDSP u32x2*)(ctl + C_CONS + (n & 3) * 2);
          for (;;) { const u32x2 cv = *cp; if (rfl(min(cv.x, cv.y)) + NS > n) break; __builtin_amdgcn_s_sleep(2); }
        }
        if (lane == 0) *(volatile LDSP u32x2*)(ctl + C_DESC + (n % NS) * 2) = u32x2{w0 | (n + 1), w1};
        if (n == 0) STAMP(2);
      }
      return;
    }
    if (wave == 8) {
      drain();
      const u32 vo = lane * 16, la = (u32)(size_t)(lds + L_CTL + C_LANDED * 4);
      u32 prog = 0, mk = 0;
      __builtin_amdgcn_s_setprio(3);
#define LOAD1(sp, lp, em) do { u32 keep_; \
          mk = DPP(prog, mk, 0x111, 0xf); \
          asm volatile("s_mov_b32 %0, m0\n\ts_mov_b32 m0, %3\n\ts_mov_b64 exec, %4\n\t" \
                       "s_waitcnt vmcnt(%8)\n\t" \
                       "global_load_lds_dwordx4 %1, %2 sc0 sc1\n\t" \
                       "s_mov_b64 exec, %7\n\tds_max_u32 %5, %6\n\ts_mov_b64 exec, -1\n\ts_mov_b32 m0, %0" \
                       : "=&s"(keep_) : "v"(vo), "s"(sp), "s"(lp), "s"(em), "v"(la), "v"(mk), "i"(1 << WD), "i"(WD - 1) : "memory"); \
          } while (0)
#define EARLY(n, w0, w1) do { const u32 e_ = (w0) >> 16; \
          if (e_ & M_RAW) break; \
          const char* sb_ = pws + (size_t)(w1) * ITEM + 4096; \
          const u32 lb_ = (u32)(size_t)lds + ((n) % NS) * ITEM + 4096, nv_ = (e_ + 15) >> 3; \
          _Pragma("clang loop unroll(disable)") for (u32 k = 0; k < nv_; ++k) { \
            u64 em = ~0ull; \
            if (k + 1 == nv_) { prog = (prog & 0xffff) | (((n) + 1) << 16); if (e_ & 7) em = (1ull << ((e_ & 7) * 8)) - 1; } \
            LOAD1(sb_ + k * 1024, lb_ + k * 1024, em); } } while (0)
#define DESC(n) ((volatile LDSP u32x2*)(ctl + C_DESC + ((n) % NS) * 2))
      u32 w0, w1;
      bool pre = false;
      for (u32 n = 0;; ++n) {
        if (!pre) {
          u32x2 dv = *DESC(n);
          if ((rfl(dv.x) & 0x7f) != n + 1) {
            waitv<0>();
            if (lane == 0) ctl[C_LANDED] = prog;
            do { dv = *DESC(n); } while ((rfl(dv.x) & 0x7f) != n + 1);
          }
          w0 = rfl(dv.x); w1 = rfl(dv.y);
          EARLY(n, w0, w1);
        }
        if (((w0 >> 7) & 0x1ff) == IT_END) break;
        const char* sb = pws + (size_t)w1 * ITEM;
        const u32 lb = (u32)(size_t)lds + (n % NS) * ITEM, um = (w0 & (M_RAW << 16)) ? 3 : 1;
        const u32x2 nx = *DESC(n + 1);
        w0 = rfl(nx.x); w1 = rfl(nx.y);
        pre = (w0 & 0x7f) == n + 2;
        if (pre) EARLY(n + 1, w0, w1);
#pragma clang loop unroll(disable)
        for (u32 k = 0; k < 2 * um + 2; ++k) {
          if ((k & um) == um) ++prog;
          LOAD1(sb + k * 1024, lb + k * 1024, ~0ull);
        }
      }
#define DRAIN(N) if (N < WD) \
        asm volatile("s_mov_b64 exec, %2\n\ts_waitcnt vmcnt(%3)\n\tds_max_u32 %0, %1\n\ts_mov_b64 exec, -1" \
                     :: "v"(la), "v"(mk), "i"(1 << N), "i"(N) : "memory");
      DRAIN(6) DRAIN(4) DRAIN(2) DRAIN(0)
      STAMP(4);
      LSTAMP(4);
      return;
    }
    drain();
    const int u = wave & 1, dw = wave;
    LDSP char* img = lds + L_IMG + wave * 4096;
    u32 m0f = 0x0f0f0f0fu, m80 = 0x80808080u;
    u32 mffc = 0xffc;
    const u32 imga = (u32)(size_t)img;
    asm volatile("" : "+s"(m0f), "+s"(m80), "+v"(mffc));
    for (u32 n = dw >> 1;; n += 4) {
      volatile LDSP u32x2* dp = (volatile LDSP u32x2*)(ctl + C_DESC + (n % NS) * 2);
      u32x2 dv;
      while ((rfl((dv = *dp).x) & 0x7f) != n + 1) __builtin_amdgcn_s_sleep(2);
      const u32 w0 = rfl(dv.x), it = (w0 >> 7) & 0x1ff, e = w0 >> 16;
      if (it == IT_END) break;
      LDSP char* sl = lds + (n % NS) * ITEM;
      u32x4 o0, o1, o2, o3;
#define LANDED(m, x) do { while ((rfl(ctl[C_LANDED]) & (m)) < (x)) __builtin_amdgcn_s_sleep(DSLEEP); fence(); } while (0)
      if (e & M_RAW) {
        LANDED(0xffffu, 2 * n + u + 1);
        LDSP char* s = sl + u * 4096 + lane * 16;
        o0 = *(LDSP u32x4*)s; o1 = *(LDSP u32x4*)(s + 1024); o2 = *(LDSP u32x4*)(s + 2048); o3 = *(LDSP u32x4*)(s + 3072);
        fence();
        if (lane == 0) ctl[C_CONS + dw] = n + 1;
      } else {
        LANDED(0xffff0000u, (n + 1) << 16);
        const u32x2 hv = *(LDSP u32x2*)(sl + FIX + u * 8);
        const u32 h0 = rfl(hv.x), h1 = rfl(hv.y);
        const u32 nd0 = h0 >> 23, nd1 = h1 >> 23;
        LDSP char* v0 = sl + FIX + (h0 & 0x1ff) * 4;
        LDSP char* v1 = sl + FIX + (h1 & 0x1ff) * 4;
        u32 nb0 = *(LDSP u32*)(v0 + lane * 4), nb1 = *(LDSP u32*)(v1 + lane * 4);
        const u32 e0 = *(LDSP unsigned short*)(v0 + (nd0 & 0x7f) * 4 + lane * 2), e1 = *(LDSP unsigned short*)(v1 + (nd1 & 0x7f) * 4 + lane * 2);
        const u32 sg0 = *(LDSP u32*)(sl + 4096 + u * 512 + lane * 4), sg1 = *(LDSP u32*)(sl + 4096 + u * 512 + 256 + lane * 4);
        {
          const u32 b0 = ((h0 >> 10) & 0x7f) * 0x01010101u, b1 = ((h1 >> 10) & 0x7f) * 0x01010101u;
          LDSP char* iz = img + lane * 16;
          *(LDSP u32x4*)iz = u32x4{b0, b0, b0, b0}; *(LDSP u32x4*)(iz + 1024) = u32x4{b0, b0, b0, b0};
          *(LDSP u32x4*)(iz + 2048) = u32x4{b1, b1, b1, b1}; *(LDSP u32x4*)(iz + 3072) = u32x4{b1, b1, b1, b1};
        }
        if (lane >= nd0) nb0 = ~0u;
        if (lane >= nd1) nb1 = ~0u;
        u32 gg[4], vv[4];
        gg[0] = (nb0 & m0f) << 3; gg[1] = (nb0 >> 1) & 0x78787878u; gg[2] = (nb1 & m0f) << 3; gg[3] = (nb1 >> 1) & 0x78787878u;
        const u32 sum = __builtin_amdgcn_sad_u8(gg[0], 0, __builtin_amdgcn_sad_u8(gg[1], 0, 0)) |
                        (__builtin_amdgcn_sad_u8(gg[2], 0, __builtin_amdgcn_sad_u8(gg[3], 0, 0)) << 16);
#pragma unroll
        for (int k = 0; k < 4; ++k) {
          vv[k] = (~(gg[k] + 0x08080808u) & m80) >> 3;
          asm("" : "+v"(gg[k]), "+v"(vv[k]));
        }
        const u32 ps = scan_add(sum) - sum;
        u32 pa = ps & 0xffff, pb = (ps >> 16) + 2048 * 8;
        if (lane < max(nd0, nd1)) {
#pragma unroll
          for (int k = 0; k < 8; ++k) {
            const int w = k & 1, by = (k >> 1) * 8;
            pa += (gg[w] >> by) & 0xff;
            pb += (gg[2 + w] >> by) & 0xff;
            LDS_SUB((LDSP char*)(size_t)(((pa >> 3) & mffc) | imga), ((vv[w] >> by) & 0xff) << (pa & 31));
            LDS_SUB((LDSP char*)(size_t)(((pb >> 3) & mffc) | imga), ((vv[2 + w] >> by) & 0xff) << (pb & 31));
          }
        }
        if (lane < ((h0 >> 17) & 63)) LDS_SUB(img + (e0 & 0xffc), ((e0 >> 12) - 8) << ((e0 & 3) * 8 + 4));
        if (lane < ((h1 >> 17) & 63)) LDS_SUB(img + (e1 & 0xffc), ((e1 >> 12) - 8) << ((e1 & 3) * 8 + 4));
        fence();
        LDSP char* ip = img + lane * 32;
        o0 = *(LDSP u32x4*)ip; o1 = *(LDSP u32x4*)(ip + 16); o2 = *(LDSP u32x4*)(ip + 2048); o3 = *(LDSP u32x4*)(ip + 2064);
#pragma unroll
        for (int q = 0; q < 4; ++q) {
          o0[q] += (sg0 << q) & m80; o1[q] += (sg0 << (4 + q)) & m80;
          o2[q] += (sg1 << q) & m80; o3[q] += (sg1 << (4 + q)) & m80;
        }
        asm volatile("" : "+v"(o0), "+v"(o1), "+v"(o2), "+v"(o3));
        LANDED(0xffffu, 2 * n + u + 1);
        TS(tsL);
        const u32x4 n0 = *(LDSP u32x4*)(sl + u * 2048 + lane * 16), n1 = *(LDSP u32x4*)(sl + u * 2048 + 1024 + lane * 16);
        fence();
        if (lane == 0) ctl[C_CONS + dw] = n + 1;
#pragma unroll
        for (int q = 0; q < 4; ++q) {
          o0[q] += n0[q] & m0f; o1[q] += (n0[q] >> 4) & m0f;
          o2[q] += n1[q] & m0f; o3[q] += (n1[q] >> 4) & m0f;
        }
      }
      char* d = me + RAW_OFF + (peer * NIT + it) * ITEM + u * 4096 + lane * 16;
      st16<0>(d, o0); st16<1024>(d, o1); st16<2048>(d, o2); st16<3072>(d, o3);
      TS(tsD);
      waitv<0>();
      put_flag(me + UF_OFF + (peer * NIT + it) * 64 + u * 32, call, lane);
    }
    if (dw == 5) LSTAMP(13);
    return;
  }

  const int t = xcd * 4 + (ct >> 1);
  const int dbl = 1 - (ct >> 2);

  if (wave >= 4) {
    drain();
    const int src = wave - 4;
    const char* raw = me + RAW_OFF + (src != self || (call & 1) ? src : 8) * NIT * ITEM + lane * 16;
    const char* uf = me + UF_OFF + src * NIT * 64;
    const char* px = me + PX_OFF + ((call & 1) * NR + src) * 16 * ITEM + lane * 16;
    const u32 lA = (u32)(size_t)lds + L_A + src * 16384;
    const u64 t0 = tstart;
    u32 need = call;
    u32 prev = 0;
#pragma clang loop unroll(disable)
    for (int j = 0; j < 4; ++j) {
      {
        const char* fa = uf + (128 + j * 4 + mm) * 128 + (lane & 3) * 32;
        for (u32 it = 1; __builtin_amdgcn_ballot_w64(ld_sys(fa) < need) != 0; ++it) {
          if ((it & 15) == 0 && give_up(me, call, t0)) need = 0;
          __builtin_amdgcn_s_sleep(8);
        }
        while (rfl(min4(*(volatile LDSP u32x4*)(ctl + C_FREE))) < prev) __builtin_amdgcn_s_sleep(2);
        const int hx = (j * 4 + mm) * 2;
        const char* sa0 = hx < NPUSH ? px + hx * ITEM : raw + (256 + hx) * ITEM;
        const char* sa1 = hx + 1 < NPUSH ? px + (hx + 1) * ITEM : raw + (257 + hx) * ITEM;
        u32 keep;
#pragma unroll
        for (int i = 0; i < 4; ++i)
          asm volatile("s_mov_b32 %0, m0\n\ts_mov_b32 m0, %2\n\ts_nop 0\n\t"
                       "global_load_lds_dwordx4 %1, off " A_SC "\n\t"
                       "global_load_lds_dwordx4 %1, off offset:1024 " A_SC "\n\t"
                       "global_load_lds_dwordx4 %1, off offset:2048 " A_SC "\n\t"
                       "global_load_lds_dwordx4 %1, off offset:3072 " A_SC "\n\t"
                       "s_mov_b32 m0, %0" : "=&s"(keep) : "v"((i < 2 ? sa0 : sa1) + (i & 1) * 4096), "s"(lA + i * 4096) : "memory");
      }
      const int nh0 = dbl ? 0 : (ct & 1);
      const char* fb = uf + (j * 32 + t) * 128 + nh0 * 64 + (lane & (1 + 2 * dbl)) * 32;
      if (j < 3)
        for (u32 it = 1; __builtin_amdgcn_ballot_w64(ld_sys(fb) < need) != 0; ++it) {
          if ((it & 15) == 0 && give_up(me, call, t0)) need = 0;
          __builtin_amdgcn_s_sleep(8);
        }
      const char* pp = raw + ((j * 32 + t) * 2 + nh0) * ITEM + 4096;
      u32x4 p[8], p2[8];
      u32 tv, tk, r0, r1;
      for (;;) {
        u32 it;
        asm volatile("s_mov_b32 %12, 96\n\t"
                     "global_load_dword %10, %13, off sc0 sc1\n\ts_sleep 6\n\t"
                     "global_load_dword %11, %13, off sc0 sc1\n\ts_sleep 6\n"
                     ".Lp_%=:\n\t"
                     "global_load_dword %8, %13, off sc0 sc1\n\ts_waitcnt vmcnt(2)\n\t"
                     "v_cmp_gt_u32 vcc, %16, %10\n\ts_cbranch_vccz .Lg_%=\n\t"
                     "global_load_dword %10, %13, off sc0 sc1\n\ts_waitcnt vmcnt(2)\n\t"
                     "v_cmp_gt_u32 vcc, %16, %11\n\ts_cbranch_vccz .Lg_%=\n\t"
                     "global_load_dword %11, %13, off sc0 sc1\n\ts_waitcnt vmcnt(2)\n\t"
                     "v_cmp_gt_u32 vcc, %16, %8\n\ts_cbranch_vccz .Lg_%=\n\t"
                     "s_sub_u32 %12, %12, 1\n\ts_cbranch_scc0 .Lp_%=\n\t"
                     "s_waitcnt vmcnt(0)\n\ts_branch .Le_%=\n"
                     ".Lg_%=:\n\t"
                     "global_load_dwordx4 %0, %14, off offset:-4096 sc0 sc1\n\t"
                     "global_load_dwordx4 %1, %14, off offset:-3072 sc0 sc1\n\t"
                     "global_load_dwordx4 %2, %14, off offset:-2048 sc0 sc1\n\t"
                     "global_load_dwordx4 %3, %14, off offset:-1024 sc0 sc1\n\t"
                     "global_load_dwordx4 %4, %14, off sc0 sc1\n\t"
                     "global_load_dwordx4 %5, %14, off offset:1024 sc0 sc1\n\t"
                     "global_load_dwordx4 %6, %14, off offset:2048 sc0 sc1\n\t"
                     "global_load_dwordx4 %7, %14, off offset:3072 sc0 sc1\n\t"
                     "global_load_dword %8, %15, off sc1\n\t"
                     "v_mov_b32 %9, %17\n\ts_mov_b64 exec, 1\n\t"
                     "ds_add_rtn_u32 %9, %18, %9\n\ts_mov_b64 exec, -1\n\ts_waitcnt lgkmcnt(0)\n"
                     ".Le_%=:"
                     : "=&v"(p[0]), "=&v"(p[1]), "=&v"(p[2]), "=&v"(p[3]), "=&v"(p[4]), "=&v"(p[5]), "=&v"(p[6]), "=&v"(p[7]),
                       "=&v"(tv), "=&v"(tk), "=&v"(r0), "=&v"(r1), "=&s"(it)
                     : "v"(fb), "v"(pp), "v"(me + TMO_OFF), "s"(need), "s"(1 + dbl),
                       "v"((u32)(size_t)(lds + L_CTL + C_TICKET * 4)) : "memory", "scc", "vcc");
        if (it != 0xffffffffu) break;
        if (give_up(me, call, t0)) need = 0;
      }
      asm volatile("s_cmp_eq_u32 %9, 0\n\ts_cbranch_scc1 .Ls_%=\n\t"
                   "global_load_dwordx4 %0, %8, off offset:-4096 sc0 sc1\n\t"
                   "global_load_dwordx4 %1, %8, off offset:-3072 sc0 sc1\n\t"
                   "global_load_dwordx4 %2, %8, off offset:-2048 sc0 sc1\n\t"
                   "global_load_dwordx4 %3, %8, off offset:-1024 sc0 sc1\n\t"
                   "global_load_dwordx4 %4, %8, off sc0 sc1\n\t"
                   "global_load_dwordx4 %5, %8, off offset:1024 sc0 sc1\n\t"
                   "global_load_dwordx4 %6, %8, off offset:2048 sc0 sc1\n\t"
                   "global_load_dwordx4 %7, %8, off offset:3072 sc0 sc1\n"
                   ".Ls_%=:"
                   : "=&v"(p2[0]), "=&v"(p2[1]), "=&v"(p2[2]), "=&v"(p2[3]), "=&v"(p2[4]), "=&v"(p2[5]), "=&v"(p2[6]), "=&v"(p2[7])
                   : "v"(pp + 8192), "s"(dbl) : "memory", "scc");
      u32 ts = rfl(tk);
#define ROOM() while (rfl(min4(*(volatile LDSP u32x4*)(ctl + C_FREE))) + NBS <= ts) __builtin_amdgcn_s_sleep(1)
#define PUT(R) do { const u32 bs = ts % NBS; LDSP char* ub = lds + L_B + bs * 8192 + lane * 16; \
        _Pragma("unroll") for (int i = 0; i < 8; ++i) *(LDSP u32x4*)(ub + i * 1024) = R[i]; \
        ++ts; \
        asm volatile("" : "+s"(ts) :: "memory"); \
        if (lane == 0) ctl[C_FULL + bs] = ts | (src << 8) | (tv == call ? 0x10000u : 0u); } while (0)
      ROOM();
      asm volatile("s_waitcnt vmcnt(0)" : "+v"(p[0]), "+v"(p[1]), "+v"(p[2]), "+v"(p[3]), "+v"(p[4]), "+v"(p[5]), "+v"(p[6]), "+v"(p[7]),
                   "+v"(p2[0]), "+v"(p2[1]), "+v"(p2[2]), "+v"(p2[3]), "+v"(p2[4]), "+v"(p2[5]), "+v"(p2[6]), "+v"(p2[7]),
                   "+v"(tv), "+v"(r0), "+v"(r1) :: "memory");
      TS(tsL);
      PUT(p);
      if (dbl) { ROOM(); fence(); PUT(p2); }
      prev = ts;
      TS(tsD);
    }
    return;
  }

  const int mw = wave;
  {
    u32 scale_bits = __builtin_amdgcn_readfirstlane(__builtin_bit_cast(u32, P.sx[0] * P.sw[0]));
    float* obase = P.out + (size_t)(mm * 128 + mw * 32) * NN + t * 128 + (ct & 1) * 64;
    asm volatile("" : "+s"(scale_bits), "+s"(obase));
    drain();
    v4f Pc[2][4], Qc[2][4];
#pragma unroll
    for (int i = 0; i < 2; ++i)
#pragma unroll
      for (int k = 0; k < 4; ++k) Pc[i][k] = Qc[i][k] = v4f{0.f, 0.f, 0.f, 0.f};
#define FRAG(p, hs) ({ const u32x4 lo_ = *(LDSP u32x4*)(p), hi_ = *(LDSP u32x4*)((p) + (hs)); \
      v8i{(int)lo_.x, (int)lo_.y, (int)lo_.z, (int)lo_.w, (int)hi_.x, (int)hi_.y, (int)hi_.z, (int)hi_.w}; })
#define MMA(i, k, F) asm volatile("v_mfma_f32_16x16x128_f8f6f4 %0, %1, %2, %0 cbsz:1 blgp:1" : "+v"(F##c[i][k]) : "v"(F##a[i]), "v"(F##b[k]))
#define RD(F) do { bad |= w; LDSP char* ap_ = lds + L_A + ((w >> 8) & 7) * 16384 + mw * 4096 + lane * 16; \
      LDSP char* bp_ = lds + L_B + bs * 8192 + lane * 16; \
      bs = bs + 1 == NBS ? 0 : bs + 1; \
      F##a[0] = FRAG(ap_, 1024); F##b[0] = FRAG(bp_, 4096); F##b[1] = FRAG(bp_ + 1024, 4096); F##a[1] = FRAG(ap_ + 2048, 1024); \
      F##b[2] = FRAG(bp_ + 2048, 4096); F##b[3] = FRAG(bp_ + 3072, 4096); \
      asm volatile("" ::: "memory");     \
      if (lane == 0) ctl[C_FREE + mw] = n + 1; \
      __builtin_amdgcn_sched_barrier(0); } while (0)
#define SPIN() do { while (((w = __builtin_amdgcn_readfirstlane(ctl[C_FULL + bs])) & 0xff) != n + 1) __builtin_amdgcn_s_sleep(MSLEEP); \
      asm volatile("" ::: "memory"); } while (0)
#define STEP(F, G, EXTRA) do { \
      MMA(0, 0, F); \
      __builtin_amdgcn_sched_barrier(0); \
      peek = ctl[C_FULL + bs]; \
      __builtin_amdgcn_sched_barrier(0); \
      MMA(0, 1, F); MMA(1, 0, F); MMA(1, 1, F); \
      __builtin_amdgcn_sched_barrier(0); \
      ++n; \
      w = __builtin_amdgcn_readfirstlane(peek); \
      asm volatile("" ::: "memory"); \
      const bool ok_ = (w & 0xff) == n + 1; \
      if (ok_) RD(G); \
      MMA(0, 2, F); MMA(1, 2, F); MMA(0, 3, F); MMA(1, 3, F); \
      __builtin_amdgcn_sched_barrier(0); \
      EXTRA; \
      if (!ok_) { SPIN(); RD(G); } } while (0)
    v8i Pa[2], Pb[4], Qa[2], Qb[4];
    u32 n = 0, bs = 0, w, peek, bad = 0;
    SPIN();
    RD(P);
#pragma clang loop unroll(disable)
    for (int it = 1; it < (16 << dbl); ++it) { STEP(P, Q, ); STEP(Q, P, ); }
#define EPI(A, i, r, o) do { v4f y_; _Pragma("unroll") for (int k = 0; k < 4; ++k) \
        y_[k] = __builtin_elementwise_maximum(A[i][k][r] * scale, 0.f);     \
      asm volatile("global_store_dwordx4 %0, %1, %2 " O_SC "\n\ts_nop 1" :: "v"(ov), "v"(y_), "s"((o) + ((i) * 16 + (r)) * NN) : "memory"); \
      __builtin_amdgcn_sched_barrier(0); } while (0)
#define TIE(A, i, nop) asm volatile(nop : "+v"(A[i][0]), "+v"(A[i][1]), "+v"(A[i][2]), "+v"(A[i][3]))
    STEP(P, Q, if (!dbl) { TIE(Pc, 0, "s_nop 11"); TIE(Pc, 1, "");
                 _Pragma("unroll") for (int i = 0; i < 2; ++i) _Pragma("unroll") for (int k = 0; k < 4; ++k) Qc[i][k] += Pc[i][k];
                 TIE(Qc, 0, ""); TIE(Qc, 1, "s_nop 1"); });
    if (mw == 0) STAMP(2);
    MMA(0, 0, Q); MMA(0, 1, Q); MMA(0, 2, Q); MMA(0, 3, Q); MMA(1, 0, Q); MMA(1, 1, Q);
    const float scale = __builtin_bit_cast(float, (bad & 0x10000u) != 0 ? 0x7fc00000u : scale_bits);
    const u32 ln = __builtin_amdgcn_mbcnt_hi(~0u, __builtin_amdgcn_mbcnt_lo(~0u, 0));
    const u32 ov = (ln >> 4) * (4 * NN * 4) + (ln & 15) * 16;
    TIE(Qc, 0, "s_nop 9");
    EPI(Qc, 0, 0, obase); MMA(1, 2, Q); EPI(Qc, 0, 1, obase); MMA(1, 3, Q); EPI(Qc, 0, 2, obase); EPI(Qc, 0, 3, obase);
    TIE(Qc, 1, "");
    EPI(Qc, 1, 0, obase); EPI(Qc, 1, 1, obase); EPI(Qc, 1, 2, obase); EPI(Qc, 1, 3, obase);
    if (dbl) {
      EPI(Pc, 0, 0, obase - 64); EPI(Pc, 0, 1, obase - 64); EPI(Pc, 0, 2, obase - 64); EPI(Pc, 0, 3, obase - 64);
      EPI(Pc, 1, 0, obase - 64); EPI(Pc, 1, 1, obase - 64); EPI(Pc, 1, 2, obase - 64); EPI(Pc, 1, 3, obase - 64);
    }
  }
}

extern "C" void kernel_launch(int rank, int nranks, ncclComm_t comm,
                              hipStream_t stream,
                              void* const* d_in, const int* in_sizes, int n_in,
                              void* d_out, int out_size,
                              const int* in_dtypes, int out_dtype,
                              void* const* d_out_all,
                              void* const* d_ws_all, size_t ws_bytes) {
  if (nranks != NR || (unsigned)rank >= (unsigned)NR || n_in < 4 || ws_bytes < WS_NEED ||
      in_sizes[0] != 4096 * KS || in_sizes[1] != KS * NN || out_size != 512 * NN) {
    fprintf(stderr, "kernel_launch: unexpected problem shape\n");
    return;
  }
  Params p;
  p.me = (char*)d_ws_all[rank];
  p.x = (const unsigned char*)d_in[0];
  p.w = (const unsigned char*)d_in[1];
  p.sx = (const float*)d_in[2];
  p.sw = (const float*)d_in[3];
  p.out = (float*)d_out;
  p.rank = rank;
  static const int grp[2][4] = {{0, 2, 3, 6}, {1, 4, 5, 7}};
  int gi = 0, gp = 0;
  for (int a = 0; a < 2; ++a) for (int i = 0; i < 4; ++i) if (grp[a][i] == rank) { gi = a; gp = i; }
  p.dstmap = 0;
  for (int x = 0; x < 8; ++x) {
    p.dstmap |= (u32)(x < 4 ? grp[gi][(gp + 1 + x) & 3] : grp[1 - gi][(gp + x) & 3]) << (4 * x);
    p.wsx[x] = (char*)d_ws_all[(p.dstmap >> (4 * x)) & 7];
  }
  hipLaunchKernelGGL(fused_gemm_kernel, dim3(NWG), dim3(NTHREADS), 0, stream, p);
  HIP_CHECK(hipGetLastError());
}
```

```cpp
#include <hip/hip_runtime.h>
#include <rccl/rccl.h>
#include <cstdio>
#include <cstdint>

#define HIP_CHECK(expr) do { hipError_t _e = (expr); if (_e != hipSuccess) { \
    fprintf(stderr, "HIP error %s at %s:%d\n", hipGetErrorString(_e), __FILE__, __LINE__); \
    return; } } while (0)

typedef unsigned u32;
typedef unsigned long long u64;
typedef unsigned u32x2 __attribute__((ext_vector_type(2)));
typedef unsigned u32x4 __attribute__((ext_vector_type(4)));
typedef int v8i __attribute__((ext_vector_type(8)));
typedef float v4f __attribute__((ext_vector_type(4)));
#define LDSP __attribute__((address_space(3)))
#define GLB __attribute__((address_space(1)))

constexpr int NR = 8;
constexpr int KS = 512;
constexpr int NN = 4096;
constexpr int NTHREADS = 768;
constexpr int NWG = 256;
constexpr int NIT = 288;
#define WD 6
#define NPUSH 12
#define GATE 100

constexpr u32 ITEM = 8192;
constexpr u32 FIX = 5120;
constexpr u32 PK_OFF = 0;
constexpr u32 RAW_OFF = 2 * 512 * ITEM;
constexpr u32 PX_OFF = RAW_OFF + 9 * NIT * ITEM;
constexpr u32 DIR_OFF = PX_OFF + 2 * NR * 16 * ITEM;
constexpr u32 UF_OFF = DIR_OFF + 2 * NR * NIT * 128;
constexpr u32 CNT_OFF = UF_OFF + NR * NIT * 512;
constexpr u32 TMO_OFF = CNT_OFF + NWG * 12 * 128;
constexpr u32 BAL_OFF = TMO_OFF + NWG * 128;
constexpr u32 DBG_OFF = BAL_OFF + NR * 256;
constexpr u32 WS_NEED = DBG_OFF + NWG * 256;
constexpr u64 DEADLINE = 30000000ull;
static_assert(NPUSH <= 16 && NPUSH >= 0, "PX region");

constexpr int NBS = 3;
constexpr u32 L_A = 0;
constexpr u32 L_B = 8 * 16384;
#define NGRP 4
constexpr int NS = NGRP == 5 ? 10 : 12;
constexpr u32 L_IMG = NS * ITEM;
constexpr u32 L_SCR = L_B + NBS * ITEM;
constexpr u32 L_CTL = L_SCR + 4096;
constexpr u32 L_DUMP = L_CTL + 256;
constexpr u32 L_BYTES = L_DUMP + 256;
constexpr int C_FREE = 0, C_FULL = 4, C_TICKET = 12, C_ENC = 13;
constexpr int C_CONS = 16, C_LANDED = 28, C_WIN = 29, C_DESC = 32;
constexpr u32 M_RAW = 0x80, IT_END = 0x1ff;

#define O_SC "sc1"
#define SCHED 0xFEDCBA9876543210ull
#define MSLEEP 0
#define DSLEEP 1
#define PUSH1 8
#define PUSHD 150
#define ROT 3
#define EA 2
#define DPRIO 0
#define FASTJ 3
#define BALM 0x8020080ull
#define TITEM 77
#define GFAST 19
#define GSLOW 15
#define WFAST 8
#define REORD 1
#define POLLW 4u
#define TOUCH 6
#define B_SC "sc0 sc1"
#define F_SC "sc0 sc1"
#define PSLEEP "6"
#define A_SC "sc0 sc1"

struct Params {
  struct { char* ws; const unsigned char* x; } px[NR];
  char* me;
  const unsigned char* w;
  int rank;
  u32 dstmap;
  float* out;
  const float* sx;
  const float* sw;
};

template <int N> __device__ __forceinline__ void waitv() { asm volatile("s_waitcnt vmcnt(%0)" :: "i"(N) : "memory"); }
__device__ __forceinline__ void waitl() { asm volatile("s_waitcnt lgkmcnt(0)" ::: "memory"); }
__device__ __forceinline__ void bar() { asm volatile("s_barrier" ::: "memory"); }
__device__ __forceinline__ void drain() { __builtin_amdgcn_s_waitcnt(0x0070); }
__device__ __forceinline__ void fence() { asm volatile("" ::: "memory"); }
template <int OFF> __device__ __forceinline__ void st16(const void* p, u32x4 x) {
  asm volatile("global_store_dwordx4 %0, %1, off offset:%2 sc0 sc1\n\ts_nop 1" :: "v"(p), "v"(x), "i"(OFF) : "memory");
}
__device__ __forceinline__ void st8(const void* p, u32x2 x) {
  asm volatile("global_store_dwordx2 %0, %1, off sc0 sc1\n\ts_nop 1" :: "v"(p), "v"(x) : "memory");
}
__device__ __forceinline__ void st4(const void* p, u32 x) {
  asm volatile("global_store_dword %0, %1, off sc0 sc1\n\ts_nop 1" :: "v"(p), "v"(x) : "memory");
}
__device__ __forceinline__ u32 ld_sys(const void* p) {
  return __hip_atomic_load((const GLB u32*)p, __ATOMIC_RELAXED, __HIP_MEMORY_SCOPE_SYSTEM);
}
__device__ __forceinline__ u32 ld_agent(const void* p) {
  return __hip_atomic_load((const GLB u32*)p, __ATOMIC_RELAXED, __HIP_MEMORY_SCOPE_AGENT);
}
__device__ __forceinline__ void st_agent(void* p, u32 v) {
  __hip_atomic_store((GLB u32*)p, v, __ATOMIC_RELAXED, __HIP_MEMORY_SCOPE_AGENT);
}
__device__ __forceinline__ void touch(const void* p, u32 dump) {
  u32 keep;
  asm volatile("s_mov_b32 %0, m0\n\ts_mov_b32 m0, %2\n\ts_nop 0\n\tglobal_load_lds_dword %1, off sc0 sc1\n\ts_mov_b32 m0, %0"
               : "=&s"(keep) : "v"(p), "s"(dump) : "memory");
}
__device__ __forceinline__ u32 rfl(u32 v) { return __builtin_amdgcn_readfirstlane(v); }
__device__ __forceinline__ void put_flag(char* f, u32 v, int lane) {
  if (lane < 2) st16<0>(f + lane * 16, u32x4{v, v, v, v});
}
__device__ __forceinline__ bool give_up(char* me, u32 call, u64 t0) {
  if (rfl(ld_agent(me + TMO_OFF + blockIdx.x * 128)) == call) return true;
  if (__builtin_amdgcn_s_memrealtime() - t0 <= DEADLINE) return false;
  for (int i = threadIdx.x & 63; i < NWG; i += 64) st_agent(me + TMO_OFF + i * 128, call);
  waitv<0>();
  return true;
}
#define LDS_ADD(p, v) __hip_atomic_fetch_add((LDSP u32*)(p), (v), __ATOMIC_RELAXED, __HIP_MEMORY_SCOPE_WORKGROUP)
#define LDS_SUB(p, v) __hip_atomic_fetch_sub((LDSP u32*)(p), (v), __ATOMIC_RELAXED, __HIP_MEMORY_SCOPE_WORKGROUP)
#define LDS_OR(p, v) __hip_atomic_fetch_or((LDSP u32*)(p), (v), __ATOMIC_RELAXED, __HIP_MEMORY_SCOPE_WORKGROUP)
#define LDS_XOR(p, v) __hip_atomic_fetch_xor((LDSP u32*)(p), (v), __ATOMIC_RELAXED, __HIP_MEMORY_SCOPE_WORKGROUP)
#define DPP(old, v, ctrl, rm) (u32)__builtin_amdgcn_update_dpp((int)(old), (int)(v), ctrl, rm, 0xf, false)
#define SCAN(name, id, OP) __device__ __forceinline__ u32 name(u32 v) { \
    v = OP(v, DPP(id, v, 0x111, 0xf)); v = OP(v, DPP(id, v, 0x112, 0xf)); v = OP(v, DPP(id, v, 0x114, 0xf)); \
    v = OP(v, DPP(id, v, 0x118, 0xf)); v = OP(v, DPP(id, v, 0x142, 0xa)); v = OP(v, DPP(id, v, 0x143, 0xc)); return v; }
__device__ __forceinline__ u32 addu(u32 a, u32 b) { return a + b; }
SCAN(scan_add, 0u, addu)
SCAN(scan_max, 0u, max)
SCAN(scan_min, 0xffffffffu, min)
#define LSTAMP(i) do {} while (0)
#define STAMP(i) do {} while (0)
#define TS(v) do {} while (0)

__device__ __forceinline__ u32 sched_item(u32 q, int rel) {
  if (q < 32 - NPUSH) return 256 + (q < 8 ? 16 + q : q < 24 - NPUSH ? NPUSH - 8 + q : NPUSH + q);
  q -= 32 - NPUSH;
  if (ROT == 1) q ^= (rel & 1) << 3;
  if (ROT == 2 && q >= 224) q = 224 + ((q + rel * 8) & 31);
  if (ROT == 3) q = (q & ~31u) | ((q + rel * 8) & 31);
  if (ROT == 4 && q >= 192) q = (q & ~31u) | ((q + rel * 8) & 31);
  const u32 e = (u32)(SCHED >> (4 * ((q >> 4) & 15))) & 15;
  return ((e >> 2) * 32 + 4 * ((q + rel) & 7) + (e & 3)) * 2 + ((q >> 3) & 1);
}
__device__ __forceinline__ u32 lds_ticket(volatile LDSP u32* p, u32 v) {
  asm volatile("s_mov_b64 exec, 1\n\tds_add_rtn_u32 %0, %1, %0\n\ts_mov_b64 exec, -1\n\ts_waitcnt lgkmcnt(0)"
               : "+v"(v) : "v"((u32)(size_t)p) : "memory");
  return rfl(v);
}
__device__ __forceinline__ u32 min4(u32x4 v) { return min(min(v.x, v.y), min(v.z, v.w)); }
__device__ __forceinline__ u32 gather8(const u32* f) {
  const u32 r0 = __builtin_amdgcn_udot4(f[1], 0x80402010u, __builtin_amdgcn_udot4(f[0], 0x08040201u, 0, false), false);
  const u32 r1 = __builtin_amdgcn_udot4(f[3], 0x80402010u, __builtin_amdgcn_udot4(f[2], 0x08040201u, 0, false), false);
  const u32 r2 = __builtin_amdgcn_udot4(f[5], 0x80402010u, __builtin_amdgcn_udot4(f[4], 0x08040201u, 0, false), false);
  const u32 r3 = __builtin_amdgcn_udot4(f[7], 0x80402010u, __builtin_amdgcn_udot4(f[6], 0x08040201u, 0, false), false);
  return (r0 >> 7) | (r1 << 1) | (r2 << 9) | (r3 << 17);
}

__global__ void __launch_bounds__(NTHREADS) fused_gemm_kernel(Params P) {
  __shared__ __attribute__((aligned(16))) char smem[L_BYTES];
  LDSP char* lds = (LDSP char*)smem;
  volatile LDSP u32* ctl = (volatile LDSP u32*)(lds + L_CTL);
  const int lane = threadIdx.x & 63;
  const int wave = rfl(threadIdx.x >> 6);
  const int b = blockIdx.x, xcd = b & 7, slot = b >> 3;
  const int ct = slot >> 2, mm = slot & 3;
  typedef u32 u32x8 __attribute__((ext_vector_type(8)));
  u32x4 a4;
  u32x8 a8;
  u32 a1;
  asm volatile("s_load_dwordx4 %0, %3, %4\n\ts_load_dwordx8 %1, %3, 0x80\n\ts_load_dword %2, %3, %5"
               : "=&s"(a4), "=&s"(a8), "=&s"(a1) : "s"(__builtin_amdgcn_kernarg_segment_ptr()), "s"(xcd * 16), "s"((xcd * 16) ^ 64) : "memory");
  const bool link = (ct & 5) == 0;
  const int i8 = mm + (ct < 4 ? ct >> 1 : ct & 1) * 4;
  const bool xjob = !link & ((wave == 4) | ((wave == 5) & (ct >= 6)));
  const int xj = wave == 5 ? 1 : ct < 4 ? 0 : ct >> 1;
  u32 cno = CNT_OFF + (b * 12 + wave) * 128;
  u32 xv0 = (i8 * 64 + (lane & 15)) * KS + xj * 128 + (lane >> 4) * 16, xv1 = xv0 + 16 * KS, xv2 = xv0 + 32 * KS, xv3 = xv0 + 48 * KS;
  asm volatile("s_waitcnt lgkmcnt(0)" : "+s"(a4), "+s"(a8), "+s"(a1), "+v"(cno), "+v"(xv0), "+v"(xv1), "+v"(xv2), "+v"(xv3) :: "memory");
  char* const pws = (char*)(a4.x | ((u64)a4.y << 32));
  char* const me = (char*)(a8[0] | ((u64)a8[1] << 32));
  const int self = a8[4];
  const int peer = (a8[5] >> (4 * xcd)) & 7;
  u32* cnt = (u32*)(me + CNT_OFF + (b * 12 + wave) * 128);
  const u32 dump = (u32)(size_t)lds + L_DUMP;
  u32 call;
  const u64 tstart = __builtin_amdgcn_s_memrealtime();

  if (__builtin_expect(xjob, 1)) {
    const int hx = xj * 8 + i8;
    u32x4 d[8];
    u32 cv;
    asm volatile("global_load_dword %8, %11, %10 sc1\n\t"
                 "global_load_dwordx4 %0, %12, %9\n\tglobal_load_dwordx4 %1, %12, %9 offset:64\n\t"
                 "global_load_dwordx4 %2, %13, %9\n\tglobal_load_dwordx4 %3, %13, %9 offset:64\n\t"
                 "global_load_dwordx4 %4, %14, %9\n\tglobal_load_dwordx4 %5, %14, %9 offset:64\n\t"
                 "global_load_dwordx4 %6, %15, %9\n\tglobal_load_dwordx4 %7, %15, %9 offset:64"
                 : "=&v"(d[0]), "=&v"(d[1]), "=&v"(d[2]), "=&v"(d[3]), "=&v"(d[4]), "=&v"(d[5]), "=&v"(d[6]), "=&v"(d[7]), "=&v"(cv)
                 : "s"(a4.z | ((u64)a4.w << 32)), "s"(me), "v"(cno), "v"(xv0), "v"(xv1), "v"(xv2), "v"(xv3) : "memory");
    const bool push = hx < NPUSH, direct = push || peer == self;
    const u32 o0 = push ? PX_OFF + (self * 16 + hx) * ITEM : direct ? RAW_OFF + (8 * NIT + 256 + hx) * ITEM : PK_OFF + (256 + peer * 32 + hx) * ITEM;
    const u32 o1 = push ? NR * 16 * ITEM : direct ? (self - 8) * NIT * ITEM : 512 * ITEM;
    const u32 f0 = direct ? UF_OFF + (self * NIT + 256 + hx) * 512 : DIR_OFF + (self * NIT + 256 + hx) * 128;
    const u32 f1 = direct ? 0 : NR * NIT * 128;
    const char* db = push ? pws : me;
    const u64 fm = direct ? 0xffff : 3;
    asm volatile("s_waitcnt vmcnt(8)" : "+v"(cv) :: "memory");
    call = rfl(cv) + 1;
    const u32 par = call & 1;
    const u32 dv = o0 + par * o1 + 4096 + lane * 16, fv = f0 + par * f1 + (lane >> 2) * 128 + (lane & 3) * 16;
    const u32 v = direct ? call : (call << 8) | M_RAW;
    if (push && hx >= PUSH1) while (__builtin_amdgcn_s_memrealtime() - tstart < PUSHD) __builtin_amdgcn_s_sleep(1);
    asm volatile("s_waitcnt vmcnt(7)\n\tglobal_store_dwordx4 %8, %0, %9 offset:-4096 sc0 sc1\n\t"
                 "s_waitcnt vmcnt(7)\n\tglobal_store_dwordx4 %8, %1, %9 offset:-3072 sc0 sc1\n\t"
                 "s_waitcnt vmcnt(7)\n\tglobal_store_dwordx4 %8, %2, %9 offset:-2048 sc0 sc1\n\t"
                 "s_waitcnt vmcnt(7)\n\tglobal_store_dwordx4 %8, %3, %9 offset:-1024 sc0 sc1\n\t"
                 "s_waitcnt vmcnt(7)\n\tglobal_store_dwordx4 %8, %4, %9 sc0 sc1\n\t"
                 "s_waitcnt vmcnt(7)\n\tglobal_store_dwordx4 %8, %5, %9 offset:1024 sc0 sc1\n\t"
                 "s_waitcnt vmcnt(7)\n\tglobal_store_dwordx4 %8, %6, %9 offset:2048 sc0 sc1\n\t"
                 "s_waitcnt vmcnt(7)\n\tglobal_store_dwordx4 %8, %7, %9 offset:3072 sc0 sc1\n\ts_nop 1"
                 : "+v"(d[0]), "+v"(d[1]), "+v"(d[2]), "+v"(d[3]), "+v"(d[4]), "+v"(d[5]), "+v"(d[6]), "+v"(d[7])
                 : "v"(dv), "s"(db) : "memory");
    asm volatile("s_mov_b64 exec, %3\n\ts_waitcnt vmcnt(0)\n\tglobal_store_dwordx4 %0, %1, %2 sc0 sc1\n\ts_mov_b64 exec, -1\n\ts_nop 0"
                 :: "v"(fv), "v"(u32x4{v, v, v, v}), "s"(pws), "s"(fm) : "memory");
    bar();
  } else if (!link && (wave < 4 || (wave >= 8 && ct < 4))) {
    {
      const int g = lane >> 4, c = lane & 15, j2 = wave >> 3, jw = ct < 4 ? 2 + j2 : (ct - 4) >> 1;
      const int u = wave & 1, hf = (wave >> 1) & 1, blk = (jw * 32 + xcd * 4) * 2 + i8;
      const GLB unsigned char* src = (const GLB unsigned char*)(a8[2] | ((u64)a8[3] << 32)) + (size_t)(jw * 128 + u * 64 + g * 16) * NN + xcd * 512 + i8 * 64 + c * 4;
      u32 W[16], D[8];
#pragma unroll
      for (int r = 0; r < 16; ++r) W[r] = *(const GLB u32*)(src + r * NN);
      const u32 cv = ld_agent(cnt);
      LDSP char* scr = lds + L_SCR + (wave - j2 * 4) * 512;
      *(LDSP u32x2*)(scr + lane * 8) = u32x2{0, 0};
#pragma unroll
      for (int s = 0; s < 8; ++s) LDS_ADD(scr + (((W[s >> 2] >> ((s & 3) * 8)) & 0x7f) << 2), 1u);
      const u32x2 hh = *(volatile LDSP u32x2*)(scr + lane * 8);
      *(volatile LDSP u32x2*)(scr + lane * 8) = u32x2{0, 0};
      const u32 sel = 0x05010400u + hf * 0x02020202u;
#pragma unroll
      for (int q = 0; q < 4; ++q) {
        const u32 t0 = __builtin_amdgcn_perm(W[4 * q + 1], W[4 * q], sel), t2 = __builtin_amdgcn_perm(W[4 * q + 3], W[4 * q + 2], sel);
        D[q] = __builtin_amdgcn_perm(t2, t0, 0x05040100u); D[4 + q] = __builtin_amdgcn_perm(t2, t0, 0x07060302u);
      }
      call = rfl(cv) + 1;
      TS(e0);
      const u32 par = call & 1;
      char* pk = me + PK_OFF + (par * 512 + blk) * ITEM;
      {
        char* d = me + RAW_OFF + ((par ? self : 8) * NIT + blk) * ITEM + u * 4096 + hf * 2048 + lane * 16;
        st16<0>(d, u32x4{D[0], D[1], D[2], D[3]}); st16<1024>(d, u32x4{D[4], D[5], D[6], D[7]});
        u32 s0 = 0;
#pragma unroll
        for (int k = 0; k < 8; ++k) s0 |= (D[k] & 0x80808080u) >> k;
        st4(pk + 4096 + u * 512 + hf * 256 + lane * 4, s0);
      }
      u32 base;
      {
        const u32 c0 = scan_add(hh.x + hh.y) - hh.x - hh.y, c1 = c0 + hh.x;
        u32 u8 = (u32)__builtin_amdgcn_ds_bpermute(((lane + 8) & 63) * 4, (int)(c0 | (c1 << 16)));
        u32 u9 = (u32)__builtin_amdgcn_ds_bpermute(((lane + 9) & 63) * 4, (int)(c0 | (c1 << 16)));
        if (lane >= 56) u8 = 0x02000200u;
        if (lane >= 55) u9 = 0x02000200u;
        const u32 cs = (c0 | (c1 << 16)) + 0x08000800u - u8 - 3 * u9;
        const u32 k0 = ((cs & 0xffff) << 7) | (2 * lane), k1 = ((cs >> 16) << 7) | (2 * lane + 1);
        base = __builtin_amdgcn_readlane(scan_min(lane < 56 ? min(k0, k1) : lane == 56 ? k0 : 0xffffffffu), 63) & 0x7f;
      }
      TS(e1);
      const u32 b8f = base * 0x01010101u + 0x8f8f8f8fu;
      u32 F1[8], F2[8], FE[8], T[8];
#pragma unroll
      for (int k = 0; k < 8; ++k) {
        const u32 t = T[k] = b8f - (D[k] & 0x7f7f7f7fu);
        const u32 t1 = t << 1, t2 = t << 2, t3 = t << 3, wr = ((t & 0x0f0f0f0fu) + 0x02020202u) << 3;
        const u32 h = t & ~t1 & 0x80808080u, k1 = h & ~t2 & t3, k2 = h & t2 & ~t3, km = ~t & t1 & t2 & t3 & 0x80808080u;
        F2[k] = (wr & k1) | (~wr & k2);
        F1[k] = k1 | (wr & km) | (~wr & k2);
        FE[k] = ((h & ~t2 & ~t3) | F1[k]) ^ 0x80808080u;
      }
      {
        u32x4 cd;
#pragma unroll
        for (int q = 0; q < 4; ++q) cd[q] = ~((T[q] & 0x0f0f0f0fu) | ((T[4 + q] << 4) & 0xf0f0f0f0u));
        st16<0>(pk + u * 2048 + hf * 1024 + lane * 16, cd);
      }
      u32 x1 = gather8(F1), x2 = gather8(F2), xe = gather8(FE);
      const u64 any = __builtin_amdgcn_ballot_w64(x1 != 0);
      int prev;
      {
        const u32 lastp = x1 ? lane * 32 + 31 - __builtin_clz(x1) : 0;
        const u32 back = lane * 32 - scan_max(DPP(0, lastp, 0x138, 0xf));
        prev = -(int)(back - 15 * ((back * 4370) >> 16));
      }
      u32 a0 = 0, a1 = 0, a2 = 0, nc = 0;
      TS(e2);
#define PUSH_NIB(gg) do { a0 = __builtin_amdgcn_alignbit(a1, a0, 4); a1 = __builtin_amdgcn_alignbit(a2, a1, 4); \
        a2 = __builtin_amdgcn_alignbit((gg), a2, 4); ++nc; } while (0)
      while (x1 != 0) {
        const u32 pos = __builtin_ctz(x1);
        const u32 gap = pos - prev, gg = min(gap, 15u);
        prev += gg;
        const u32 bit = gap < 15 ? 1u << pos : 0u, in2 = x2 & bit;
        x2 ^= in2;
        x1 ^= bit ^ in2;
        PUSH_NIB(gg);
      }
      if ((any >> lane) > 1) while (prev <= 17) { prev += 15; PUSH_NIB(15u); }
      TS(e3);
      const u32 nend = scan_add(nc), nn = __builtin_amdgcn_readlane(nend, 63);
      bool over = nn > 512 || __builtin_amdgcn_ballot_w64(nc > 24) != 0;
      if (!over) {
        const u32 sh = (8 - (nend & 7)) * 4;
        LDSP char* q = scr + (nend >> 3) * 4;
        LDS_OR(q, (u32)((((u64)a0) << 32) >> sh));
        LDS_OR(q + 4, (u32)((((u64)a1 << 32) | a0) >> sh));
        LDS_OR(q + 8, (u32)((((u64)a2 << 32) | a1) >> sh));
        LDS_OR(q + 12, (u32)(((u64)a2) >> sh));
      }
      u32 ne = 0;
      {
        LDSP unsigned short* lst = (LDSP unsigned short*)(scr + 384);
        u64 mk;
        while ((mk = __builtin_amdgcn_ballot_w64(xe != 0)) != 0 && ne <= 63) {
          if (xe != 0) {
            const u32 pos = __builtin_ctz(xe);
            const u32 wa = (pos & 4) ? T[1] : T[0], wb = (pos & 4) ? T[3] : T[2], wc = (pos & 4) ? T[5] : T[4], wd = (pos & 4) ? T[7] : T[6];
            const u32 we = (pos & 8) ? wb : wa, wf = (pos & 8) ? wd : wc, wg = (pos & 16) ? wf : we;
            const u32 x4 = (wg >> ((pos & 3) * 8 + 4)) & 15;
            const u32 at = ne + __builtin_amdgcn_mbcnt_hi((u32)(mk >> 32), __builtin_amdgcn_mbcnt_lo((u32)mk, 0));
            if (at < 64) lst[at] = (unsigned short)((x4 << 12) | (hf * 2048 + lane * 32 + pos));
            xe &= xe - 1;
          }
          ne += __builtin_popcountll(mk);
        }
      }
      over = over || ne > 63;
      bar();
      if (!over) {
        const u32 ed = (ne + 1) >> 1, nd = (nn + 7) >> 3;
        const u32 ev = *(volatile LDSP u32*)(scr + 384 + (lane & 31) * 4);
        u32 nv = *(volatile LDSP u32*)(scr + 12 + lane * 4);
        const u32 vo = (lds_ticket(ctl + C_ENC + j2, ed + nd) & 0xffff) + 4;
        if (lane == (nn >> 3)) nv |= ~0u << ((nn & 7) * 4);
        if (lane < nd) st4(pk + FIX + (vo + lane) * 4, nv);
        if (lane < ed) st4(pk + FIX + (vo + nd + lane) * 4, ev);
        if (lane == 0) st4(pk + FIX + (u * 2 + hf) * 4, vo | (base << 10) | (ne << 17) | (nd << 23));
      }
      TS(e4);
      u64 pw[NR];
      asm volatile("s_load_dwordx2 %0, %8, 0x0\n\ts_load_dwordx2 %1, %8, 0x10\n\ts_load_dwordx2 %2, %8, 0x20\n\ts_load_dwordx2 %3, %8, 0x30\n\t"
                   "s_load_dwordx2 %4, %8, 0x40\n\ts_load_dwordx2 %5, %8, 0x50\n\ts_load_dwordx2 %6, %8, 0x60\n\ts_load_dwordx2 %7, %8, 0x70\n\t"
                   "s_waitcnt vmcnt(0) lgkmcnt(0)"
                   : "=&s"(pw[0]), "=&s"(pw[1]), "=&s"(pw[2]), "=&s"(pw[3]), "=&s"(pw[4]), "=&s"(pw[5]), "=&s"(pw[6]), "=&s"(pw[7])
                   : "s"(__builtin_amdgcn_kernarg_segment_ptr()) : "memory");
      const u32 k = lds_ticket(ctl + C_ENC + j2, over ? 0x1010000u : 0x1000000u);
      if ((k >> 24) == 3) {
        const u32 v = (call << 8) | ((over || (k & 0xff0000) != 0) ? M_RAW : ((k & 0xffff) + 4 + 31) >> 5);
        asm volatile("s_mov_b64 exec, 3\n\t"
                     "global_store_dwordx4 %0, %1, %2 sc0 sc1\n\tglobal_store_dwordx4 %0, %1, %3 sc0 sc1\n\t"
                     "global_store_dwordx4 %0, %1, %4 sc0 sc1\n\tglobal_store_dwordx4 %0, %1, %5 sc0 sc1\n\t"
                     "global_store_dwordx4 %0, %1, %6 sc0 sc1\n\tglobal_store_dwordx4 %0, %1, %7 sc0 sc1\n\t"
                     "global_store_dwordx4 %0, %1, %8 sc0 sc1\n\tglobal_store_dwordx4 %0, %1, %9 sc0 sc1\n\t"
                     "s_mov_b64 exec, -1\n\ts_nop 0"
                     :: "v"(DIR_OFF + ((par * NR + self) * NIT + blk) * 128 + lane * 16), "v"(u32x4{v, v, v, v}),
                        "s"(pw[0]), "s"(pw[1]), "s"(pw[2]), "s"(pw[3]), "s"(pw[4]), "s"(pw[5]), "s"(pw[6]), "s"(pw[7]) : "memory");
        if (lane < 16) st16<0>(me + UF_OFF + (self * NIT + blk) * 512 + (lane >> 2) * 128 + (lane & 3) * 16, u32x4{call, call, call, call});
        STAMP(1);
      }
    }
  } else {
    if (link && ((wave >= 2 * NGRP && wave < 10) || peer == self)) return;
    const u32 cv = ld_agent(cnt);
    if (wave == 7) { ctl[lane] = 0; waitl(); }
    bar();
    call = rfl(cv) + 1;
  }
  if (lane == 0 && wave != 11) st_agent(cnt, call);

  if (link) {
    if (wave == 11) {
      const u32 par = call & 1;
      const u32 q = mm + ct * 2 + lane * 8;
      const u32 nitems = (NIT - NPUSH - (mm + ct * 2) + 7) >> 3;
      const u32 itv = lane < nitems ? sched_item(q, xcd) : 0;
      const char* dir = me + DIR_OFF + ((par * NR + peer) * NIT + itv) * 128;
      u32 dv = 0, fown = 0;
      u64 todo = (1ull << nitems) - 1, ok = 0;
      while (NPUSH != 0 && __builtin_amdgcn_s_memrealtime() - tstart < GATE - 50) __builtin_amdgcn_s_sleep(1);
      u32 plan = 0;
      for (u32 n = 0; n < nitems + NS - 3; ++n) {
        u32 w0 = (IT_END << 6) | 0x80000;
        if (n < nitems) {
          const u32 first = __builtin_ctzll(todo);
          const u64 cand = todo & ((REORD && n < 8 ? 7ull : 1ull) << first);
          for (u32 sp = 1; (ok & cand) == 0; ++sp) {
            if (sp > 1 && n > 2) __builtin_amdgcn_s_sleep(4);
            if (lane - first < (n < 8 ? POLLW : 64u) && lane < nitems && !((ok >> lane) & 1)) dv = ld_sys(dir);
            ok = __builtin_amdgcn_ballot_w64((dv >> 8) == (call & 0xffffff));
            if ((sp & 31) == 0 && give_up(me, call, tstart)) { dv = (call << 8) | M_RAW; ok = ~0ull; }
          }
          const u32 ln = __builtin_ctzll(ok & cand);
          todo &= ~(1ull << ln);
          const u32 it = __builtin_amdgcn_readlane(itv, ln);
          u32 e = __builtin_amdgcn_readlane(dv, ln);
          e &= (e & M_RAW) ? M_RAW : 0xf;
          const u32 so = it >= 256 ? par * 512 + self * 32 + it : (e & M_RAW) ? RAW_OFF / ITEM + (par ? peer : 8) * NIT + it : par * 512 + it;
          w0 = (it << 6) | ((e & M_RAW) ? 0x80000 : e << 15) | (so << 20);
        }
        if (n >= NS) {
          volatile LDSP u32x2* cp = (volatile LDSP u32x2*)(ctl + C_CONS + (n % NGRP) * 2);
          for (;;) { const u32x2 cv = *cp; if (rfl(min(cv.x, cv.y)) + NS > n) break; __builtin_amdgcn_s_sleep(2); }
        }
        if (lane == 0 && n < nitems + NGRP) ctl[C_DESC + n % NS] = w0 | (n + 1);
        if (BALM != 0 && n >= NS) {
          const u32 k = n - NS + 1;
          char* bp = me + BAL_OFF + xcd * 256;
          if ((BALM >> k) & 1) {
            fown = ((u32)__builtin_amdgcn_s_memrealtime() + (nitems - k) * TITEM) << 12 | ((call & 63) << 6) | k;
            if (lane == 0) st_agent(bp + (mm + ct * 2) * 32, fown);
          } else if (((BALM  __H0__ ) >> k) & 1) {
            const u32 f = ld_agent(bp + (lane & 7) * 32);
            if (__builtin_amdgcn_ballot_w64(((f ^ fown) & 0xfff) != 0) == 0) {
              const int late = (int)__builtin_amdgcn_readlane(scan_add((u32)((int)(fown - f) >> 12)), 7) / 8;
              plan = late > 0 ? (late + GFAST / 2) / GFAST : (GSLOW / 2 - late) / GSLOW;
              if (lane == 0 && plan != 0) ctl[C_WIN] = late > 0 ? 1 : 2;
            }
          } else if (plan != 0 && --plan == 0) {
            if (lane == 0) ctl[C_WIN] = 0;
          }
        }
        if (n == 0) STAMP(2);
      }
      if (lane == 0) st_agent(cnt, call);
      return;
    }
    if (wave == 10) {
      if ((TOUCH & 2) && lane < 3) touch(pws + PK_OFF + ((call & 1) * 512 + (lane == 2 ? 256 + self * 32 + 16 : lane * 255)) * ITEM + (self * 8 + mm + ct * 2) * 128, dump);
      waitv<0>();
      drain();
      const u32 l0 = (u32)(size_t)lds, lw = l0 + L_CTL + C_LANDED * 4, one = lane == 0 ? 1 : 0;
      u32 prog = 0, mk = 0, wm = 0, la = lane == WD ? lw : dump + lane * 4;
      __builtin_amdgcn_s_setprio(3);
#define MODE_ "s_cmp_eq_u32 %[wm], 1\n\ts_cselect_b64 vcc, -1, 0\n\ts_cmp_eq_u32 %[wm], 2\n\t"
#define LD_(off, id) "s_waitcnt vmcnt(%[wf])\n\ts_cbranch_vccnz .Lw" #id "_%=\n\ts_waitcnt vmcnt(%[w])\n\ts_cbranch_scc0 .Lw" #id "_%=\n\t" \
                     "s_waitcnt vmcnt(%[ws])\n.Lw" #id "_%=:\n\tglobal_load_lds_dwordx4 %[va], %[sb] offset:" #off " sc0 sc1\n\t"
#define SH_ "v_mov_b32_dpp %[mk], %[mk] row_shr:1 row_mask:0xf bank_mask:0xf\n\t"
#define PUB_ "ds_max_u32 %[la], %[mk]\n\t"
#define INC_(x) "v_add_u32 %[mk], %[mk], " x "\n\t"
#define WIN_ [wm] "s"(wm), [wf] "i"(WFAST - 1), [w] "i"(WD - 1), [ws] "i"(WD - 2)
#define FOUR(src, ldst, i0) do { u32 keep_; \
          asm volatile("ds_read_b32 %[dv], %[da]\n\tds_read_b32 %[wv], %[lw] offset:4\n\ts_mov_b32 %[k], m0\n\ts_mov_b32 m0, %[ld]\n\t" MODE_ \
                       LD_(0, 0) SH_ INC_("%[i]") PUB_ LD_(1024, 1) SH_ INC_("%[o]") PUB_ LD_(2048, 2) SH_ INC_("%[i]") PUB_ LD_(3072, 3) SH_ INC_("%[o]") \
                       "s_waitcnt lgkmcnt(3)\n\t" PUB_ "s_mov_b32 m0, %[k]" \
                       : [k] "=&s"(keep_), [dv] "=&v"(dvv), [wv] "=&v"(wvv), [mk] "+v"(mk) \
                       : [va] "v"((src) + lane * 16), [sb] "s"(pws), [ld] "s"(ldst), [la] "v"(la), [da] "v"(l0 + L_CTL + (C_DESC + se) * 4), \
                         [lw] "v"(lw), [i] "v"((i0) & one), [o] "v"(one), WIN_ : "memory", "scc", "vcc"); \
          prog += 2 + 2 * (i0); } while (0)
      u64 qu = 0;
      u32 ne = 0, se = 0, sc = 0, d = 0, dvv, wvv;
      bool fin = false;
      for (u32 n = 0;; ++n) {
#pragma clang loop unroll(disable)
        while (ne <= n + EA && !fin) {
          if ((d & 63) != ne + 1) {
            if (ne > n) break;
            d = rfl(ctl[C_DESC + se]);
            if ((d & 63) != ne + 1) {
              waitv<0>();
              if (lane == 0) ctl[C_LANDED] = prog;
              do { d = rfl(ctl[C_DESC + se]); } while ((d & 63) != ne + 1);
            }
          }
          fin = ((d >> 6) & 0x1ff) == IT_END;
          qu |= (u64)((d >> 20) | ((d >> 7) & 0x1000) | (fin ? 0x2000u : 0u)) << (16 * (ne - n));
          ++ne;
          if (!(d & 0x80000)) {
            const u32 ln = (d >> 15) & 15, pn = (prog & 0xffff) | (ne << 16);
            const u64 em = (ln & 7) ? (1ull << ((ln & 7) * 8)) - 1 : ~0ull;
            u32 keep;
            asm volatile("s_mov_b32 %[k], m0\n\ts_mov_b32 m0, %[ld]\n\t" MODE_
                         LD_(0, 0) SH_ PUB_
                         "s_cmp_gt_u32 %[ln], 8\n\ts_cbranch_scc1 .L3_%=\n\ts_cmp_eq_u32 %[wm], 2\n\ts_mov_b64 exec, %[em]\n\t" LD_(1024, 1) "s_branch .L4_%=\n"
                         ".L3_%=:\n\ts_cmp_eq_u32 %[wm], 2\n\t" LD_(1024, 2) SH_ PUB_ "s_mov_b64 exec, %[em]\n\t" LD_(2048, 3)
                         "\n.L4_%=:\n\ts_mov_b64 exec, -1\n\t" SH_ "v_writelane_b32 %[mk], %[pn], 0\n\t" PUB_ "s_mov_b32 m0, %[k]"
                         : [k] "=&s"(keep), [mk] "+v"(mk)
                         : [va] "v"((d >> 20) * ITEM + 4096 + lane * 16), [sb] "s"(pws), [ld] "s"(l0 + se * ITEM + 4096), [la] "v"(la),
                           [ln] "s"(ln), [em] "s"(em), [pn] "s"(pn), WIN_ : "memory", "scc", "vcc");
            prog = pn;
          }
          se = se + 1 == NS ? 0 : se + 1;
          if (n == 0) break;
          if (ne <= n + EA) d = rfl(ctl[C_DESC + se]);
        }
        const u32 cur = (u32)qu & 0xffff;
        qu >>= 16;
        if (cur & 0x2000) break;
        u32 so = (cur & 0xfff) * ITEM, lo = l0 + sc * ITEM;
#pragma clang loop unroll(disable)
        for (u32 k = cur >> 12;; --k) {
          FOUR(so, lo, 1 - (cur >> 12));
          if (k == 0) break;
          so += 4096; lo += 4096;
        }
        d = rfl(dvv);
        wm = rfl(wvv);
        la = lane == (wm == 1 ? WFAST : wm == 2 ? WD - 1 : WD) ? lw : dump + lane * 4;
        sc = sc + 1 == NS ? 0 : sc + 1;
      }
#define DRAIN(N) if (N < WFAST) \
        asm volatile("s_mov_b64 exec, %2\n\ts_waitcnt vmcnt(%3)\n\tds_max_u32 %0, %1\n\ts_mov_b64 exec, -1" \
                     :: "v"(lw), "v"(mk), "i"(1 << N), "i"(N) : "memory");
      DRAIN(9) DRAIN(8) DRAIN(7) DRAIN(6) DRAIN(5) DRAIN(4) DRAIN(3) DRAIN(2) DRAIN(1) DRAIN(0)
      STAMP(4);
      LSTAMP(4);
      return;
    }
    if ((TOUCH & 4) && wave == 0 && lane < 4) touch(me + (lane == 3 ? UF_OFF + peer * NIT * 512 : RAW_OFF + (peer * NIT + lane * 143) * ITEM) + slot * 128, dump);
    drain();
    const int u = wave & 1, dw = wave;
    LDSP char* img = lds + L_IMG + wave * 4096;
    u32 m0f = 0x0f0f0f0fu, m80 = 0x80808080u, m3e = 0x3e3e3e3eu, m30 = 0x30303030u;
    u32 mffc = 0xffc;
    const u32 imga = (u32)(size_t)img;
    asm volatile("" : "+s"(m0f), "+s"(m80), "+s"(m3e), "+s"(m30), "+v"(mffc));
    for (u32 n = dw >> 1;; n += NGRP) {
      u32 w0;
      while (((w0 = rfl(ctl[C_DESC + n % NS])) & 63) != n + 1) __builtin_amdgcn_s_sleep(2);
      const u32 it = (w0 >> 6) & 0x1ff;
      if (it == IT_END) break;
      LDSP char* sl = lds + (n % NS) * ITEM;
      char* d = me + RAW_OFF + (peer * NIT + it) * ITEM + u * 4096 + lane * 16;
      u32x4 o0, o1, o2, o3;
#define LANDED(m, x) do { while ((rfl(ctl[C_LANDED]) & (m)) < (x)) __builtin_amdgcn_s_sleep(DSLEEP); fence(); } while (0)
      if (w0 & 0x80000) {
        LANDED(0xffffu, 4 * n + 2 * u + 2);
        LDSP char* s = sl + u * 4096 + lane * 16;
        o0 = *(LDSP u32x4*)s; o1 = *(LDSP u32x4*)(s + 1024); o2 = *(LDSP u32x4*)(s + 2048); o3 = *(LDSP u32x4*)(s + 3072);
        fence();
        if (lane == 0) ctl[C_CONS + dw] = n + 1;
        st16<0>(d, o0); st16<1024>(d, o1); st16<2048>(d, o2); st16<3072>(d, o3);
      } else {
        LANDED(0xffff0000u, (n + 1) << 16);
        __builtin_amdgcn_s_setprio(DPRIO);
        const u32x2 hv = *(LDSP u32x2*)(sl + FIX + u * 8);
        const u32 h0 = rfl(hv.x), h1 = rfl(hv.y);
        const u32 nd0 = h0 >> 23, nd1 = h1 >> 23;
        const u32 b0 = ((h0 >> 10) & 0x7f) * 0x01010101u - 0x3e3e3e3eu, b1 = ((h1 >> 10) & 0x7f) * 0x01010101u - 0x3e3e3e3eu;
        LDSP char* v0 = sl + FIX + (h0 & 0x1ff) * 4;
        LDSP char* v1 = sl + FIX + (h1 & 0x1ff) * 4;
        u32 nb0 = *(LDSP u32*)(v0 + lane * 4), nb1 = *(LDSP u32*)(v1 + lane * 4);
        const u32 e0 = *(LDSP unsigned short*)(v0 + (nd0 & 0x7f) * 4 + lane * 2), e1 = *(LDSP unsigned short*)(v1 + (nd1 & 0x7f) * 4 + lane * 2);
        const u32 sg0 = *(LDSP u32*)(sl + 4096 + u * 512 + lane * 4), sg1 = *(LDSP u32*)(sl + 4096 + u * 512 + 256 + lane * 4);
        {
          LDSP char* iz = img + lane * 16;
          *(LDSP u32x4*)iz = u32x4{b0, b0, b0, b0}; *(LDSP u32x4*)(iz + 1024) = u32x4{b0, b0, b0, b0};
          *(LDSP u32x4*)(iz + 2048) = u32x4{b1, b1, b1, b1}; *(LDSP u32x4*)(iz + 3072) = u32x4{b1, b1, b1, b1};
        }
        if (lane >= nd0) nb0 = ~0u;
        if (lane >= nd1) nb1 = ~0u;
        u32 gg[4], vv[4];
        gg[0] = (nb0 & m0f) << 3; gg[1] = (nb0 >> 1) & 0x78787878u; gg[2] = (nb1 & m0f) << 3; gg[3] = (nb1 >> 1) & 0x78787878u;
        const u32 sum = __builtin_amdgcn_sad_u8(gg[0], 0, __builtin_amdgcn_sad_u8(gg[1], 0, 0)) |
                        (__builtin_amdgcn_sad_u8(gg[2], 0, __builtin_amdgcn_sad_u8(gg[3], 0, 0)) << 16);
#pragma unroll
        for (int k = 0; k < 4; ++k) {
          vv[k] = (~(gg[k] + 0x08080808u) & m80) >> 3;
          asm("" : "+v"(gg[k]), "+v"(vv[k]));
        }
        const u32 ps = scan_add(sum) - sum;
        u32 pa = ps & 0xffff, pb = (ps >> 16) + 2048 * 8;
        if (lane < max(nd0, nd1)) {
#pragma unroll
          for (int k = 0; k < 8; ++k) {
            const int w = k & 1, by = (k >> 1) * 8;
            pa += (gg[w] >> by) & 0xff;
            pb += (gg[2 + w] >> by) & 0xff;
            LDS_SUB((LDSP char*)(size_t)(((pa >> 3) & mffc) | imga), ((vv[w] >> by) & 0xff) << (pa & 31));
            LDS_SUB((LDSP char*)(size_t)(((pb >> 3) & mffc) | imga), ((vv[2 + w] >> by) & 0xff) << (pb & 31));
          }
        }
        fence();
        LDSP char* ip = img + lane * 32;
        u32x4 a0 = *(LDSP u32x4*)ip, a1 = *(LDSP u32x4*)(ip + 16), a2 = *(LDSP u32x4*)(ip + 2048), a3 = *(LDSP u32x4*)(ip + 2064);
        fence();
        if (lane < ((h0 >> 17) & 63)) LDS_SUB(img + (e0 & 0xffc), ((e0 >> 12) - 8) << ((e0 & 3) * 8 + 4));
        if (lane < ((h1 >> 17) & 63)) LDS_SUB(img + (e1 & 0xffc), ((e1 >> 12) - 8) << ((e1 & 3) * 8 + 4));
        fence();
        o0 = *(LDSP u32x4*)ip; o1 = *(LDSP u32x4*)(ip + 16); o2 = *(LDSP u32x4*)(ip + 2048); o3 = *(LDSP u32x4*)(ip + 2064);
#pragma unroll
        for (int q = 0; q < 4; ++q) {
          o0[q] += (sg0 << q) & m80; o1[q] += (sg0 << (4 + q)) & m80;
          o2[q] += (sg1 << q) & m80; o3[q] += (sg1 << (4 + q)) & m80;
          const u32 l0 = b0 - a0[q], l1 = b0 - a1[q], l2 = b1 - a2[q], l3 = b1 - a3[q];
          a0[q] = (l0 << 1) | (l0 >> 1); a1[q] = (l1 << 1) | (l1 >> 1); a2[q] = (l2 << 1) | (l2 >> 1); a3[q] = (l3 << 1) | (l3 >> 1);
        }
        asm volatile("" : "+v"(o0), "+v"(o1), "+v"(o2), "+v"(o3), "+v"(a0), "+v"(a1), "+v"(a2), "+v"(a3));
#define CODES(o, a, x) _Pragma("unroll") for (int q = 0; q < 4; ++q) { \
          u32 c_ = ((x) & m0f) + m3e; asm("" : "+v"(c_)); o[q] += c_ + (c_ & a[q] & m30); }
        LANDED(0xffffu, 4 * n + 2 * u + 1);
        const u32x4 n0 = *(LDSP u32x4*)(sl + u * 2048 + lane * 16);
        CODES(o0, a0, n0[q]) st16<0>(d, o0);
        CODES(o1, a1, n0[q] >> 4) st16<1024>(d, o1);
        LANDED(0xffffu, 4 * n + 2 * u + 2);
        TS(tsL);
        const u32x4 n1 = *(LDSP u32x4*)(sl + u * 2048 + 1024 + lane * 16);
        CODES(o2, a2, n1[q]) st16<2048>(d, o2);
        CODES(o3, a3, n1[q] >> 4) st16<3072>(d, o3);
        if (lane == 0) ctl[C_CONS + dw] = n + 1;
      }
      TS(tsD);
      waitv<0>();
      __builtin_amdgcn_s_setprio(0);
      if (lane < 8) st16<0>(me + UF_OFF + (peer * NIT + it) * 512 + u * 32 + (lane >> 1) * 128 + (lane & 1) * 16, u32x4{call, call, call, call});
    }
    return;
  }

  const int t = xcd * 4 + (ct >> 1);
  const int dbl = 1 - (ct >> 2);

  if (wave >= 4) {
    drain();
    const int src = wave - 4;
    if (lane == 0 && wave == 11) st_agent(cnt, call);
    const char* raw = me + RAW_OFF + (src != self || (call & 1) ? src : 8) * NIT * ITEM + lane * 16;
    const char* uf = me + UF_OFF + src * NIT * 512;
    const char* px = me + PX_OFF + ((call & 1) * NR + src) * 16 * ITEM + lane * 16;
    const u32 lA = (u32)(size_t)lds + L_A + src * 16384;
    const u64 t0 = tstart;
    u32 need = call;
    u32 prev = 0;
    const char* fa = uf + ((128 + ((lane >> 2) & 3) * 4 + mm) * 2 + ((lane >> 1) & 1)) * 512 + (xcd & 3) * 128 + (lane & 1) * 32;
    u64 xmiss = 0xffff;
    u32 xf = 0;
#pragma clang loop unroll(disable)
    for (int j = 0; j < 4; ++j) {
      {
        for (u32 it = 1; ((xmiss >> (4 * j)) & 15) != 0; ++it) {
          if (it > 1) __builtin_amdgcn_s_sleep(8);
          if ((xmiss >> lane) & 1) xf = ld_sys(fa);
          xmiss = __builtin_amdgcn_ballot_w64(xf < need) & 0xffff;
          if ((it & 15) == 0 && give_up(me, call, t0)) need = 0;
        }
        while (rfl(min4(*(volatile LDSP u32x4*)(ctl + C_FREE))) < prev) __builtin_amdgcn_s_sleep(2);
        const int hx = (j * 4 + mm) * 2;
        const char* sa0 = hx < NPUSH ? px + hx * ITEM : raw + (256 + hx) * ITEM;
        const char* sa1 = hx + 1 < NPUSH ? px + (hx + 1) * ITEM : raw + (257 + hx) * ITEM;
        u32 keep;
#pragma unroll
        for (int i = 0; i < 4; ++i)
          asm volatile("s_mov_b32 %0, m0\n\ts_mov_b32 m0, %2\n\ts_nop 0\n\t"
                       "global_load_lds_dwordx4 %1, off " A_SC "\n\t"
                       "global_load_lds_dwordx4 %1, off offset:1024 " A_SC "\n\t"
                       "global_load_lds_dwordx4 %1, off offset:2048 " A_SC "\n\t"
                       "global_load_lds_dwordx4 %1, off offset:3072 " A_SC "\n\t"
                       "s_mov_b32 m0, %0" : "=&s"(keep) : "v"((i < 2 ? sa0 : sa1) + (i & 1) * 4096), "s"(lA + i * 4096) : "memory");
      }
      const int nh0 = dbl ? 0 : (ct & 1);
      const char* fb = uf + ((j * 32 + t) * 2 + nh0 + ((lane >> 1) & dbl)) * 512 + mm * 128 + (lane & 1) * 32;
      if (j < FASTJ)
        for (u32 it = 1; __builtin_amdgcn_ballot_w64(ld_sys(fb) < need) != 0; ++it) {
          if ((it & 15) == 0 && give_up(me, call, t0)) need = 0;
          __builtin_amdgcn_s_sleep(8);
        }
      const char* pp = raw + ((j * 32 + t) * 2 + nh0) * ITEM + 4096;
      if ((TOUCH & 8) && j == 3 && lane == 0) touch(pp, dump);
      u32x4 p[8], p2[8];
      u32 tv, tk, r0, r1;
      for (;;) {
        u32 it;
        asm volatile("s_mov_b32 %12, 96\n\ts_cmp_lt_u32 %19, %20\n\ts_cbranch_scc1 .Lg_%=\n\t"
                     "global_load_dword %10, %13, off " F_SC "\n\ts_sleep " PSLEEP "\n\t"
                     "global_load_dword %11, %13, off " F_SC "\n\ts_sleep " PSLEEP "\n"
                     ".Lp_%=:\n\t"
                     "global_load_dword %8, %13, off " F_SC "\n\ts_waitcnt vmcnt(2)\n\t"
                     "v_cmp_gt_u32 vcc, %16, %10\n\ts_cbranch_vccz .Lg_%=\n\t"
                     "global_load_dword %10, %13, off " F_SC "\n\ts_waitcnt vmcnt(2)\n\t"
                     "v_cmp_gt_u32 vcc, %16, %11\n\ts_cbranch_vccz .Lg_%=\n\t"
                     "global_load_dword %11, %13, off " F_SC "\n\ts_waitcnt vmcnt(2)\n\t"
                     "v_cmp_gt_u32 vcc, %16, %8\n\ts_cbranch_vccz .Lg_%=\n\t"
                     "s_sub_u32 %12, %12, 1\n\ts_cbranch_scc0 .Lp_%=\n\t"
                     "s_waitcnt vmcnt(0)\n\ts_branch .Le_%=\n"
                     ".Lg_%=:\n\t"
                     "global_load_dwordx4 %0, %14, off offset:-4096 " B_SC "\n\t"
                     "global_load_dwordx4 %1, %14, off offset:-3072 " B_SC "\n\t"
                     "global_load_dwordx4 %2, %14, off offset:-2048 " B_SC "\n\t"
                     "global_load_dwordx4 %3, %14, off offset:-1024 " B_SC "\n\t"
                     "global_load_dwordx4 %4, %14, off " B_SC "\n\t"
                     "global_load_dwordx4 %5, %14, off offset:1024 " B_SC "\n\t"
                     "global_load_dwordx4 %6, %14, off offset:2048 " B_SC "\n\t"
                     "global_load_dwordx4 %7, %14, off offset:3072 " B_SC "\n\t"
                     "global_load_dword %8, %15, off sc1\n\t"
                     "v_mov_b32 %9, %17\n\ts_mov_b64 exec, 1\n\t"
                     "ds_add_rtn_u32 %9, %18, %9\n\ts_mov_b64 exec, -1\n\ts_waitcnt lgkmcnt(0)\n"
                     ".Le_%=:"
                     : "=&v"(p[0]), "=&v"(p[1]), "=&v"(p[2]), "=&v"(p[3]), "=&v"(p[4]), "=&v"(p[5]), "=&v"(p[6]), "=&v"(p[7]),
                       "=&v"(tv), "=&v"(tk), "=&v"(r0), "=&v"(r1), "=&s"(it)
                     : "v"(fb), "v"(pp), "v"(me + TMO_OFF + b * 128), "s"(need), "s"(1 + dbl),
                       "v"((u32)(size_t)(lds + L_CTL + C_TICKET * 4)), "s"(j), "i"(FASTJ) : "memory", "scc", "vcc");
        if (it != 0xffffffffu) break;
        if (give_up(me, call, t0)) need = 0;
      }
      asm volatile("s_cmp_eq_u32 %9, 0\n\ts_cbranch_scc1 .Ls_%=\n\t"
                   "global_load_dwordx4 %0, %8, off offset:-4096 " B_SC "\n\t"
                   "global_load_dwordx4 %1, %8, off offset:-3072 " B_SC "\n\t"
                   "global_load_dwordx4 %2, %8, off offset:-2048 " B_SC "\n\t"
                   "global_load_dwordx4 %3, %8, off offset:-1024 " B_SC "\n\t"
                   "global_load_dwordx4 %4, %8, off " B_SC "\n\t"
                   "global_load_dwordx4 %5, %8, off offset:1024 " B_SC "\n\t"
                   "global_load_dwordx4 %6, %8, off offset:2048 " B_SC "\n\t"
                   "global_load_dwordx4 %7, %8, off offset:3072 " B_SC "\n"
                   ".Ls_%=:"
                   : "=&v"(p2[0]), "=&v"(p2[1]), "=&v"(p2[2]), "=&v"(p2[3]), "=&v"(p2[4]), "=&v"(p2[5]), "=&v"(p2[6]), "=&v"(p2[7])
                   : "v"(pp + 8192), "s"(dbl) : "memory", "scc");
      u32 ts = rfl(tk);
#define ROOM() while (rfl(min4(*(volatile LDSP u32x4*)(ctl + C_FREE))) + NBS <= ts) __builtin_amdgcn_s_sleep(1)
#define PUT(R) do { const u32 bs = ts % NBS; LDSP char* ub = lds + L_B + bs * 8192 + lane * 16; \
        _Pragma("unroll") for (int i = 0; i < 8; ++i) *(LDSP u32x4*)(ub + i * 1024) = R[i]; \
        ++ts; \
        asm volatile("" : "+s"(ts) :: "memory"); \
        if (lane == 0) ctl[C_FULL + bs] = ts | (src << 8) | (tv == call ? 0x10000u : 0u); } while (0)
      ROOM();
      asm volatile("s_waitcnt vmcnt(0)" : "+v"(p[0]), "+v"(p[1]), "+v"(p[2]), "+v"(p[3]), "+v"(p[4]), "+v"(p[5]), "+v"(p[6]), "+v"(p[7]),
                   "+v"(p2[0]), "+v"(p2[1]), "+v"(p2[2]), "+v"(p2[3]), "+v"(p2[4]), "+v"(p2[5]), "+v"(p2[6]), "+v"(p2[7]),
                   "+v"(tv), "+v"(r0), "+v"(r1) :: "memory");
      TS(tsL);
      PUT(p);
      if (dbl) { ROOM(); fence(); PUT(p2); }
      prev = ts;
      TS(tsD);
    }
    return;
  }

  const int mw = wave;
  {
    u32 scale_bits = __builtin_amdgcn_readfirstlane(__builtin_bit_cast(u32, P.sx[0] * P.sw[0]));
    float* obase = P.out + (size_t)(mm * 128 + mw * 32) * NN + t * 128 + (ct & 1) * 64;
    asm volatile("" : "+s"(scale_bits), "+s"(obase));
    if ((TOUCH & 16) && lane < 2) touch(obase + lane * 31 * NN, dump);
    drain();
    v4f Pc[2][4], Qc[2][4];
#pragma unroll
    for (int i = 0; i < 2; ++i)
#pragma unroll
      for (int k = 0; k < 4; ++k) Pc[i][k] = Qc[i][k] = v4f{0.f, 0.f, 0.f, 0.f};
#define FRAG(p, hs) ({ const u32x4 lo_ = *(LDSP u32x4*)(p), hi_ = *(LDSP u32x4*)((p) + (hs)); \
      v8i{(int)lo_.x, (int)lo_.y, (int)lo_.z, (int)lo_.w, (int)hi_.x, (int)hi_.y, (int)hi_.z, (int)hi_.w}; })
#define MMA(i, k, F) asm volatile("v_mfma_f32_16x16x128_f8f6f4 %0, %1, %2, %0 cbsz:1 blgp:1" : "+v"(F##c[i][k]) : "v"(F##a[i]), "v"(F##b[k]))
#define RD(F) do { bad |= w; LDSP char* ap_ = lds + L_A + ((w >> 8) & 7) * 16384 + mw * 4096 + lane * 16; \
      LDSP char* bp_ = lds + L_B + bs * 8192 + lane * 16; \
      bs = bs + 1 == NBS ? 0 : bs + 1; \
      F##a[0] = FRAG(ap_, 1024); F##b[0] = FRAG(bp_, 4096); F##b[1] = FRAG(bp_ + 1024, 4096); F##a[1] = FRAG(ap_ + 2048, 1024); \
      F##b[2] = FRAG(bp_ + 2048, 4096); F##b[3] = FRAG(bp_ + 3072, 4096); \
      asm volatile("" ::: "memory");     \
      if (lane == 0) ctl[C_FREE + mw] = n + 1; \
      __builtin_amdgcn_sched_barrier(0); } while (0)
#define SPIN() do { while (((w = __builtin_amdgcn_readfirstlane(ctl[C_FULL + bs])) & 0xff) != n + 1) __builtin_amdgcn_s_sleep(MSLEEP); \
      asm volatile("" ::: "memory"); } while (0)
#define STEP(F, G, EXTRA) do { \
      MMA(0, 0, F); \
      __builtin_amdgcn_sched_barrier(0); \
      peek = ctl[C_FULL + bs]; \
      __builtin_amdgcn_sched_barrier(0); \
      MMA(0, 1, F); MMA(1, 0, F); MMA(1, 1, F); \
      __builtin_amdgcn_sched_barrier(0); \
      ++n; \
      w = __builtin_amdgcn_readfirstlane(peek); \
      asm volatile("" ::: "memory"); \
      const bool ok_ = (w & 0xff) == n + 1; \
      if (ok_) RD(G); \
      MMA(0, 2, F); MMA(1, 2, F); MMA(0, 3, F); MMA(1, 3, F); \
      __builtin_amdgcn_sched_barrier(0); \
      EXTRA; \
      if (!ok_) { SPIN(); RD(G); } } while (0)
    v8i Pa[2], Pb[4], Qa[2], Qb[4];
    u32 n = 0, bs = 0, w, peek, bad = 0;
    SPIN();
    RD(P);
#pragma clang loop unroll(disable)
    for (int it = 1; it < (16 << dbl); ++it) { STEP(P, Q, ); STEP(Q, P, ); }
#define EPI(A, i, r, o) do { v4f y_; _Pragma("unroll") for (int k = 0; k < 4; ++k) \
        y_[k] = __builtin_elementwise_maximum(A[i][k][r] * scale, 0.f);     \
      asm volatile("global_store_dwordx4 %0, %1, %2 " O_SC "\n\ts_nop 1" :: "v"(ov), "v"(y_), "s"((o) + ((i) * 16 + (r)) * NN) : "memory"); \
      __builtin_amdgcn_sched_barrier(0); } while (0)
#define TIE(A, i, nop) asm volatile(nop : "+v"(A[i][0]), "+v"(A[i][1]), "+v"(A[i][2]), "+v"(A[i][3]))
    STEP(P, Q, if (!dbl) { TIE(Pc, 0, "s_nop 11"); TIE(Pc, 1, "");
                 _Pragma("unroll") for (int i = 0; i < 2; ++i) _Pragma("unroll") for (int k = 0; k < 4; ++k) Qc[i][k] += Pc[i][k];
                 TIE(Qc, 0, ""); TIE(Qc, 1, "s_nop 1"); });
    if (mw == 0) STAMP(2);
    MMA(0, 0, Q); MMA(0, 1, Q); MMA(0, 2, Q); MMA(0, 3, Q); MMA(1, 0, Q); MMA(1, 1, Q);
    const float scale = __builtin_bit_cast(float, (bad & 0x10000u) != 0 ? 0x7fc00000u : scale_bits);
    const u32 ln = __builtin_amdgcn_mbcnt_hi(~0u, __builtin_amdgcn_mbcnt_lo(~0u, 0));
    const u32 ov = (ln >> 4) * (4 * NN * 4) + (ln & 15) * 16;
    TIE(Qc, 0, "s_nop 9");
    EPI(Qc, 0, 0, obase); MMA(1, 2, Q); EPI(Qc, 0, 1, obase); MMA(1, 3, Q); EPI(Qc, 0, 2, obase); EPI(Qc, 0, 3, obase);
    TIE(Qc, 1, "");
    EPI(Qc, 1, 0, obase); EPI(Qc, 1, 1, obase); EPI(Qc, 1, 2, obase); EPI(Qc, 1, 3, obase);
    if (dbl) {
      EPI(Pc, 0, 0, obase - 64); EPI(Pc, 0, 1, obase - 64); EPI(Pc, 0, 2, obase - 64); EPI(Pc, 0, 3, obase - 64);
      EPI(Pc, 1, 0, obase - 64); EPI(Pc, 1, 1, obase - 64); EPI(Pc, 1, 2, obase - 64); EPI(Pc, 1, 3, obase - 64);
    }
  }
}

extern "C" void kernel_launch(int rank, int nranks, ncclComm_t comm,
                              hipStream_t stream,
                              void* const* d_in, const int* in_sizes, int n_in,
                              void* d_out, int out_size,
                              const int* in_dtypes, int out_dtype,
                              void* const* d_out_all,
                              void* const* d_ws_all, size_t ws_bytes) {
  if (nranks != NR || (unsigned)rank >= (unsigned)NR || n_in < 4 || ws_bytes < WS_NEED ||
      in_sizes[0] != 4096 * KS || in_sizes[1] != KS * NN || out_size != 512 * NN) {
    fprintf(stderr, "kernel_launch: unexpected problem shape\n");
    return;
  }
  Params p;
  p.me = (char*)d_ws_all[rank];
  p.w = (const unsigned char*)d_in[1];
  p.sx = (const float*)d_in[2];
  p.sw = (const float*)d_in[3];
  p.out = (float*)d_out;
  p.rank = rank;
  static const int grp[2][4] = {{0, 2, 3, 6}, {1, 4, 5, 7}};
  int gi = 0, gp = 0;
  for (int a = 0; a < 2; ++a) for (int i = 0; i < 4; ++i) if (grp[a][i] == rank) { gi = a; gp = i; }
  p.dstmap = 0;
  for (int x = 0; x < 8; ++x) {
    p.dstmap |= (u32)(x < 4 ? grp[gi][(gp + 1 + x) & 3] : grp[1 - gi][(gp + x) & 3]) << (4 * x);
    const int peer = (p.dstmap >> (4 * x)) & 7;
    p.px[x].ws = (char*)d_ws_all[peer];
    p.px[x].x = (const unsigned char*)d_in[0] + (size_t)peer * 512 * KS;
  }
  hipLaunchKernelGGL(fused_gemm_kernel, dim3(NWG), dim3(NTHREADS), 0, stream, p);
  HIP_CHECK(hipGetLastError());
}
```

```cpp
#include <hip/hip_runtime.h>
#include <rccl/rccl.h>
#include <cstdio>
#include <cstdint>

#define HIP_CHECK(expr) do { hipError_t _e = (expr); if (_e != hipSuccess) { \
    fprintf(stderr, "HIP error %s at %s:%d\n", hipGetErrorString(_e), __FILE__, __LINE__); \
    return; } } while (0)

typedef unsigned u32;
typedef unsigned long long u64;
typedef unsigned u32x2 __attribute__((ext_vector_type(2)));
typedef unsigned u32x4 __attribute__((ext_vector_type(4)));
typedef int v8i __attribute__((ext_vector_type(8)));
typedef float v4f __attribute__((ext_vector_type(4)));
#define LDSP __attribute__((address_space(3)))
#define GLB __attribute__((address_space(1)))

constexpr int NR = 8;
constexpr int KS = 512;
constexpr int NN = 4096;
constexpr int NTHREADS = 768;
constexpr int NWG = 256;
constexpr int NIT = 288;
#define WD 6
#define NPUSH 12
#define GATE 100

constexpr u32 ITEM = 8192;
constexpr u32 FIX = 5120;
constexpr u32 PK_OFF = 0;
constexpr u32 RAW_OFF = 2 * 512 * ITEM;
constexpr u32 PX_OFF = RAW_OFF + 9 * NIT * ITEM;
constexpr u32 RL_OFF = PX_OFF + 2 * NR * 16 * ITEM;
constexpr u32 DIR_OFF = RL_OFF + 2 * NR * 8 * ITEM;
constexpr u32 UF_OFF = DIR_OFF + 2 * NR * NIT * 128;
constexpr u32 CNT_OFF = UF_OFF + NR * NIT * 512;
constexpr u32 TMO_OFF = CNT_OFF + NWG * 12 * 128;
constexpr u32 BAL_OFF = TMO_OFF + NWG * 128;
constexpr u32 PF_OFF = BAL_OFF + NR * 256;
constexpr u32 PLAN_OFF = PF_OFF + NR * 256;
constexpr u32 RLF_OFF = PLAN_OFF + NR * 128;
static_assert(DIR_OFF / ITEM <= 4096, "12-bit item offsets");
constexpr u32 WS_NEED = RLF_OFF + NR * 8 * 128;
constexpr u64 DEADLINE = 30000000ull;
static_assert(NPUSH <= 16 && NPUSH >= 0, "PX region");

constexpr int NBS = 3;
constexpr u32 L_A = 0;
constexpr u32 L_B = 8 * 16384;
constexpr int NGRP = 4;
constexpr int NS = 12;
constexpr u32 L_IMG = NS * ITEM;
constexpr u32 L_SCR = L_B + NBS * ITEM;
constexpr u32 L_CTL = L_SCR + 4096;
constexpr u32 L_DUMP = L_CTL + 256;
constexpr u32 L_BYTES = L_DUMP + 256;
constexpr int C_FREE = 0, C_FULL = 4, C_TICKET = 12, C_ENC = 13;
constexpr int C_CONS = 16, C_LANDED = 28, C_WIN = 29, C_NIT = 30, C_DESC = 32;
constexpr u32 M_RAW = 0x80, IT_END = 0x1ff;

#define PUSH1 8
#define PUSHD 150
#define EA 2
#define LA 8
#define TITEM 77
#define WFAST 8
#define GFAST 24
#define GSLOW 19
#define K0 4
#define SP 4
#define PLANT 2000
#define DZ 10.f

struct Params {
  struct { char* ws; const unsigned char* x; } px[NR];
  char* me;
  const unsigned char* w;
  int rank;
  u32 dstmap;
  float* out;
  const float* sx;
  const float* sw;
};

template <int N> __device__ __forceinline__ void waitv() { asm volatile("s_waitcnt vmcnt(%0)" :: "i"(N) : "memory"); }
__device__ __forceinline__ void waitl() { asm volatile("s_waitcnt lgkmcnt(0)" ::: "memory"); }
__device__ __forceinline__ void bar() { asm volatile("s_barrier" ::: "memory"); }
__device__ __forceinline__ void drain() { __builtin_amdgcn_s_waitcnt(0x0070); }
__device__ __forceinline__ void fence() { asm volatile("" ::: "memory"); }
template <int OFF> __device__ __forceinline__ void st16(const void* p, u32x4 x) {
  asm volatile("global_store_dwordx4 %0, %1, off offset:%2 sc0 sc1\n\ts_nop 1" :: "v"(p), "v"(x), "i"(OFF) : "memory");
}
__device__ __forceinline__ void st4(const void* p, u32 x) {
  asm volatile("global_store_dword %0, %1, off sc0 sc1\n\ts_nop 1" :: "v"(p), "v"(x) : "memory");
}
__device__ __forceinline__ u32 ld_sys(const void* p) {
  return __hip_atomic_load((const GLB u32*)p, __ATOMIC_RELAXED, __HIP_MEMORY_SCOPE_SYSTEM);
}
__device__ __forceinline__ u32 ld_agent(const void* p) {
  return __hip_atomic_load((const GLB u32*)p, __ATOMIC_RELAXED, __HIP_MEMORY_SCOPE_AGENT);
}
__device__ __forceinline__ void st_agent(void* p, u32 v) {
  __hip_atomic_store((GLB u32*)p, v, __ATOMIC_RELAXED, __HIP_MEMORY_SCOPE_AGENT);
}
__device__ __forceinline__ void touch(const void* p, u32 dump) {
  u32 keep;
  asm volatile("s_mov_b32 %0, m0\n\ts_mov_b32 m0, %2\n\ts_nop 0\n\tglobal_load_lds_dword %1, off sc0 sc1\n\ts_mov_b32 m0, %0"
               : "=&s"(keep) : "v"(p), "s"(dump) : "memory");
}
__device__ __forceinline__ u32 rfl(u32 v) { return __builtin_amdgcn_readfirstlane(v); }
__device__ __forceinline__ void put_flag(char* f, u32 v, int lane) {
  if (lane < 2) st16<0>(f + lane * 16, u32x4{v, v, v, v});
}
__device__ __forceinline__ bool give_up(char* me, u32 call, u64 t0) {
  if (rfl(ld_agent(me + TMO_OFF + blockIdx.x * 128)) == call) return true;
  if (__builtin_amdgcn_s_memrealtime() - t0 <= DEADLINE) return false;
  for (int i = threadIdx.x & 63; i < NWG; i += 64) st_agent(me + TMO_OFF + i * 128, call);
  waitv<0>();
  return true;
}
#define LDS_ADD(p, v) __hip_atomic_fetch_add((LDSP u32*)(p), (v), __ATOMIC_RELAXED, __HIP_MEMORY_SCOPE_WORKGROUP)
#define LDS_SUB(p, v) __hip_atomic_fetch_sub((LDSP u32*)(p), (v), __ATOMIC_RELAXED, __HIP_MEMORY_SCOPE_WORKGROUP)
#define LDS_OR(p, v) __hip_atomic_fetch_or((LDSP u32*)(p), (v), __ATOMIC_RELAXED, __HIP_MEMORY_SCOPE_WORKGROUP)
#define DPP(old, v, ctrl, rm) (u32)__builtin_amdgcn_update_dpp((int)(old), (int)(v), ctrl, rm, 0xf, false)
#define SCAN(name, id, OP) __device__ __forceinline__ u32 name(u32 v) { \
    v = OP(v, DPP(id, v, 0x111, 0xf)); v = OP(v, DPP(id, v, 0x112, 0xf)); v = OP(v, DPP(id, v, 0x114, 0xf)); \
    v = OP(v, DPP(id, v, 0x118, 0xf)); v = OP(v, DPP(id, v, 0x142, 0xa)); v = OP(v, DPP(id, v, 0x143, 0xc)); return v; }
__device__ __forceinline__ u32 addu(u32 a, u32 b) { return a + b; }
SCAN(scan_add, 0u, addu)
SCAN(scan_max, 0u, max)
SCAN(scan_min, 0xffffffffu, min)

__device__ __forceinline__ u32 round_item(u32 rd, u32 col) { return ((rd >> 3) * 32 + 4 * col + ((rd >> 1) & 3)) * 2 + (rd & 1); }
__device__ __forceinline__ u32 sched_item(u32 q, u32 rel, u32 self) {
  if (q < 32 - NPUSH) return 256 + (q < 8 ? 16 + q : q < 24 - NPUSH ? NPUSH - 8 + q : NPUSH + q);
  q -= 32 - NPUSH;
  const u32 sh = 16 + self, ps = (sh & ~3u) | ((sh - rel) & 3);
  u32 r = q >> 3;
  r = r == SP ? ps : r > SP && r <= ps ? r - 1 : r;
  return round_item((r & ~3u) | ((r + rel) & 3), (q + rel) & 7);
}
__device__ __forceinline__ u32 nib_sum(u32 x) { return (((x & 0x0f0f0f0fu) + ((x >> 4) & 0x0f0f0f0fu)) * 0x01010101u) >> 24; }
__device__ __forceinline__ u32 lds_ticket(volatile LDSP u32* p, u32 v) {
  asm volatile("s_mov_b64 exec, 1\n\tds_add_rtn_u32 %0, %1, %0\n\ts_mov_b64 exec, -1\n\ts_waitcnt lgkmcnt(0)"
               : "+v"(v) : "v"((u32)(size_t)p) : "memory");
  return rfl(v);
}
__device__ __forceinline__ u32 min4(u32x4 v) { return min(min(v.x, v.y), min(v.z, v.w)); }
__device__ __forceinline__ u32 gather8(const u32* f) {
  const u32 r0 = __builtin_amdgcn_udot4(f[1], 0x80402010u, __builtin_amdgcn_udot4(f[0], 0x08040201u, 0, false), false);
  const u32 r1 = __builtin_amdgcn_udot4(f[3], 0x80402010u, __builtin_amdgcn_udot4(f[2], 0x08040201u, 0, false), false);
  const u32 r2 = __builtin_amdgcn_udot4(f[5], 0x80402010u, __builtin_amdgcn_udot4(f[4], 0x08040201u, 0, false), false);
  const u32 r3 = __builtin_amdgcn_udot4(f[7], 0x80402010u, __builtin_amdgcn_udot4(f[6], 0x08040201u, 0, false), false);
  return (r0 >> 7) | (r1 << 1) | (r2 << 9) | (r3 << 17);
}

__global__ void __launch_bounds__(NTHREADS) fused_gemm_kernel(Params P) {
  __shared__ __attribute__((aligned(16))) char smem[L_BYTES];
  LDSP char* lds = (LDSP char*)smem;
  volatile LDSP u32* ctl = (volatile LDSP u32*)(lds + L_CTL);
  const int lane = threadIdx.x & 63;
  const int wave = rfl(threadIdx.x >> 6);
  const int b = blockIdx.x, xcd = b & 7, slot = b >> 3;
  const int ct = slot >> 2, mm = slot & 3;
  typedef u32 u32x8 __attribute__((ext_vector_type(8)));
  u32x4 a4;
  u32x8 a8;
  u32 a1;
  asm volatile("s_load_dwordx4 %0, %3, %4\n\ts_load_dwordx8 %1, %3, 0x80\n\ts_load_dword %2, %3, %5"
               : "=&s"(a4), "=&s"(a8), "=&s"(a1) : "s"(__builtin_amdgcn_kernarg_segment_ptr()), "s"(xcd * 16), "s"((xcd * 16) ^ 64) : "memory");
  const bool link = (ct & 5) == 0;
  const int i8 = mm + (ct < 4 ? ct >> 1 : ct & 1) * 4;
  const bool xjob = !link & ((wave == 4) | ((wave == 5) & (ct >= 6)));
  const int xj = wave == 5 ? 1 : ct < 4 ? 0 : ct >> 1;
  u32 cno = CNT_OFF + (b * 12 + wave) * 128;
  u32 xv0 = (i8 * 64 + (lane & 15)) * KS + xj * 128 + (lane >> 4) * 16, xv1 = xv0 + 16 * KS, xv2 = xv0 + 32 * KS, xv3 = xv0 + 48 * KS;
  asm volatile("s_waitcnt lgkmcnt(0)" : "+s"(a4), "+s"(a8), "+s"(a1), "+v"(cno), "+v"(xv0), "+v"(xv1), "+v"(xv2), "+v"(xv3) :: "memory");
  char* const pws = (char*)(a4.x | ((u64)a4.y << 32));
  char* const me = (char*)(a8[0] | ((u64)a8[1] << 32));
  const int self = a8[4];
  const int peer = (a8[5] >> (4 * xcd)) & 7;
  u32* cnt = (u32*)(me + CNT_OFF + (b * 12 + wave) * 128);
  const u32 dump = (u32)(size_t)lds + L_DUMP;
  u32 call;
  const u64 tstart = __builtin_amdgcn_s_memrealtime();

  if (__builtin_expect(xjob, 1)) {
    const int hx = xj * 8 + i8;
    u32x4 d[8];
    u32 cv;
    asm volatile("global_load_dword %8, %11, %10 sc1\n\t"
                 "global_load_dwordx4 %0, %12, %9\n\tglobal_load_dwordx4 %1, %12, %9 offset:64\n\t"
                 "global_load_dwordx4 %2, %13, %9\n\tglobal_load_dwordx4 %3, %13, %9 offset:64\n\t"
                 "global_load_dwordx4 %4, %14, %9\n\tglobal_load_dwordx4 %5, %14, %9 offset:64\n\t"
                 "global_load_dwordx4 %6, %15, %9\n\tglobal_load_dwordx4 %7, %15, %9 offset:64"
                 : "=&v"(d[0]), "=&v"(d[1]), "=&v"(d[2]), "=&v"(d[3]), "=&v"(d[4]), "=&v"(d[5]), "=&v"(d[6]), "=&v"(d[7]), "=&v"(cv)
                 : "s"(a4.z | ((u64)a4.w << 32)), "s"(me), "v"(cno), "v"(xv0), "v"(xv1), "v"(xv2), "v"(xv3) : "memory");
    const bool push = hx < NPUSH, direct = push || peer == self;
    const u32 o0 = push ? PX_OFF + (self * 16 + hx) * ITEM : direct ? RAW_OFF + (8 * NIT + 256 + hx) * ITEM : PK_OFF + (256 + peer * 32 + hx) * ITEM;
    const u32 o1 = push ? NR * 16 * ITEM : direct ? (self - 8) * NIT * ITEM : 512 * ITEM;
    const u32 f0 = direct ? UF_OFF + (self * NIT + 256 + hx) * 512 : DIR_OFF + (self * NIT + 256 + hx) * 128;
    const u32 f1 = direct ? 0 : NR * NIT * 128;
    const char* db = push ? pws : me;
    const u64 fm = direct ? 0xffff : 3;
    asm volatile("s_waitcnt vmcnt(8)" : "+v"(cv) :: "memory");
    call = rfl(cv) + 1;
    const u32 par = call & 1;
    const u32 dv = o0 + par * o1 + 4096 + lane * 16, fv = f0 + par * f1 + (lane >> 2) * 128 + (lane & 3) * 16;
    const u32 v = direct ? call : (call << 8) | M_RAW;
    if (push && hx >= PUSH1) while (__builtin_amdgcn_s_memrealtime() - tstart < PUSHD) __builtin_amdgcn_s_sleep(1);
    asm volatile("s_waitcnt vmcnt(7)\n\tglobal_store_dwordx4 %8, %0, %9 offset:-4096 sc0 sc1\n\t"
                 "s_waitcnt vmcnt(7)\n\tglobal_store_dwordx4 %8, %1, %9 offset:-3072 sc0 sc1\n\t"
                 "s_waitcnt vmcnt(7)\n\tglobal_store_dwordx4 %8, %2, %9 offset:-2048 sc0 sc1\n\t"
                 "s_waitcnt vmcnt(7)\n\tglobal_store_dwordx4 %8, %3, %9 offset:-1024 sc0 sc1\n\t"
                 "s_waitcnt vmcnt(7)\n\tglobal_store_dwordx4 %8, %4, %9 sc0 sc1\n\t"
                 "s_waitcnt vmcnt(7)\n\tglobal_store_dwordx4 %8, %5, %9 offset:1024 sc0 sc1\n\t"
                 "s_waitcnt vmcnt(7)\n\tglobal_store_dwordx4 %8, %6, %9 offset:2048 sc0 sc1\n\t"
                 "s_waitcnt vmcnt(7)\n\tglobal_store_dwordx4 %8, %7, %9 offset:3072 sc0 sc1\n\ts_nop 1"
                 : "+v"(d[0]), "+v"(d[1]), "+v"(d[2]), "+v"(d[3]), "+v"(d[4]), "+v"(d[5]), "+v"(d[6]), "+v"(d[7])
                 : "v"(dv), "s"(db) : "memory");
    asm volatile("s_mov_b64 exec, %3\n\ts_waitcnt vmcnt(0)\n\tglobal_store_dwordx4 %0, %1, %2 sc0 sc1\n\ts_mov_b64 exec, -1\n\ts_nop 0"
                 :: "v"(fv), "v"(u32x4{v, v, v, v}), "s"(pws), "s"(fm) : "memory");
    bar();
  } else if (!link && (wave < 4 || (wave >= 8 && ct < 4))) {
    {
      const int g = lane >> 4, c = lane & 15, j2 = wave >> 3, jw = ct < 4 ? 2 + j2 : (ct - 4) >> 1;
      const int u = wave & 1, hf = (wave >> 1) & 1, blk = (jw * 32 + xcd * 4) * 2 + i8;
      const GLB unsigned char* src = (const GLB unsigned char*)(a8[2] | ((u64)a8[3] << 32)) + (size_t)(jw * 128 + u * 64 + g * 16) * NN + xcd * 512 + i8 * 64 + c * 4;
      u32 W[16], D[8];
#pragma unroll
      for (int r = 0; r < 16; ++r) W[r] = *(const GLB u32*)(src + r * NN);
      const u32 cv = ld_agent(cnt);
      LDSP char* scr = lds + L_SCR + (wave - j2 * 4) * 512;
      *(LDSP u32x2*)(scr + lane * 8) = u32x2{0, 0};
#pragma unroll
      for (int s = 0; s < 8; ++s) LDS_ADD(scr + (((W[s >> 2] >> ((s & 3) * 8)) & 0x7f) << 2), 1u);
      const u32x2 hh = *(volatile LDSP u32x2*)(scr + lane * 8);
      *(volatile LDSP u32x2*)(scr + lane * 8) = u32x2{0, 0};
      const u32 sel = 0x05010400u + hf * 0x02020202u;
#pragma unroll
      for (int q = 0; q < 4; ++q) {
        const u32 t0 = __builtin_amdgcn_perm(W[4 * q + 1], W[4 * q], sel), t2 = __builtin_amdgcn_perm(W[4 * q + 3], W[4 * q + 2], sel);
        D[q] = __builtin_amdgcn_perm(t2, t0, 0x05040100u); D[4 + q] = __builtin_amdgcn_perm(t2, t0, 0x07060302u);
      }
      call = rfl(cv) + 1;
      const u32 par = call & 1;
      char* pk = me + PK_OFF + (par * 512 + blk) * ITEM;
      {
        char* d = me + RAW_OFF + ((par ? self : 8) * NIT + blk) * ITEM + u * 4096 + hf * 2048 + lane * 16;
        st16<0>(d, u32x4{D[0], D[1], D[2], D[3]}); st16<1024>(d, u32x4{D[4], D[5], D[6], D[7]});
        u32 s0 = 0;
#pragma unroll
        for (int k = 0; k < 8; ++k) s0 |= (D[k] & 0x80808080u) >> k;
        st4(pk + 4096 + u * 512 + hf * 256 + lane * 4, s0);
      }
      u32 base;
      {
        const u32 c0 = scan_add(hh.x + hh.y) - hh.x - hh.y, c1 = c0 + hh.x;
        u32 u8 = (u32)__builtin_amdgcn_ds_bpermute(((lane + 8) & 63) * 4, (int)(c0 | (c1 << 16)));
        u32 u9 = (u32)__builtin_amdgcn_ds_bpermute(((lane + 9) & 63) * 4, (int)(c0 | (c1 << 16)));
        if (lane >= 56) u8 = 0x02000200u;
        if (lane >= 55) u9 = 0x02000200u;
        const u32 cs = (c0 | (c1 << 16)) + 0x08000800u - u8 - 3 * u9;
        const u32 k0 = ((cs & 0xffff) << 7) | (2 * lane), k1 = ((cs >> 16) << 7) | (2 * lane + 1);
        base = __builtin_amdgcn_readlane(scan_min(lane < 56 ? min(k0, k1) : lane == 56 ? k0 : 0xffffffffu), 63) & 0x7f;
      }
      const u32 b8f = base * 0x01010101u + 0x8f8f8f8fu;
      u32 F1[8], F2[8], FE[8], T[8];
#pragma unroll
      for (int k = 0; k < 8; ++k) {
        const u32 t = T[k] = b8f - (D[k] & 0x7f7f7f7fu);
        const u32 t1 = t << 1, t2 = t << 2, t3 = t << 3, wr = ((t & 0x0f0f0f0fu) + 0x02020202u) << 3;
        const u32 h = t & ~t1 & 0x80808080u, k1 = h & ~t2 & t3, k2 = h & t2 & ~t3, km = ~t & t1 & t2 & t3 & 0x80808080u;
        F2[k] = (wr & k1) | (~wr & k2);
        F1[k] = k1 | (wr & km) | (~wr & k2);
        FE[k] = ((h & ~t2 & ~t3) | F1[k]) ^ 0x80808080u;
      }
      {
        u32x4 cd;
#pragma unroll
        for (int q = 0; q < 4; ++q) cd[q] = ~((T[q] & 0x0f0f0f0fu) | ((T[4 + q] << 4) & 0xf0f0f0f0u));
        st16<0>(pk + u * 2048 + hf * 1024 + lane * 16, cd);
      }
      u32 x1 = gather8(F1), x2 = gather8(F2), xe = gather8(FE);
      const u64 any = __builtin_amdgcn_ballot_w64(x1 != 0);
      int prev;
      {
        const u32 lastp = x1 ? lane * 32 + 31 - __builtin_clz(x1) : 0;
        const u32 back = lane * 32 - scan_max(DPP(0, lastp, 0x138, 0xf));
        prev = -(int)(back - 15 * ((back * 4370) >> 16));
      }
      u32 a0 = 0, a1 = 0, a2 = 0, nc = 0;
#define PUSH_NIB(gg) do { a0 = __builtin_amdgcn_alignbit(a1, a0, 4); a1 = __builtin_amdgcn_alignbit(a2, a1, 4); \
        a2 = __builtin_amdgcn_alignbit((gg), a2, 4); ++nc; } while (0)
      while (x1 != 0) {
        const u32 pos = __builtin_ctz(x1);
        const u32 gap = pos - prev, gg = min(gap, 15u);
        prev += gg;
        const u32 bit = gap < 15 ? 1u << pos : 0u, in2 = x2 & bit;
        x2 ^= in2;
        x1 ^= bit ^ in2;
        PUSH_NIB(gg);
      }
      if ((any >> lane) > 1) while (prev <= 17) { prev += 15; PUSH_NIB(15u); }
      const u32 nend = scan_add(nc), nn = __builtin_amdgcn_readlane(nend, 63);
      bool over = nn > 512 || __builtin_amdgcn_ballot_w64(nc > 24) != 0;
      if (!over) {
        const u32 sh = (8 - (nend & 7)) * 4;
        LDSP char* q = scr + (nend >> 3) * 4;
        LDS_OR(q, (u32)((((u64)a0) << 32) >> sh));
        LDS_OR(q + 4, (u32)((((u64)a1 << 32) | a0) >> sh));
        LDS_OR(q + 8, (u32)((((u64)a2 << 32) | a1) >> sh));
        LDS_OR(q + 12, (u32)(((u64)a2) >> sh));
      }
      u32 ne = 0;
      {
        LDSP unsigned short* lst = (LDSP unsigned short*)(scr + 384);
        u64 mk;
        while ((mk = __builtin_amdgcn_ballot_w64(xe != 0)) != 0 && ne <= 63) {
          if (xe != 0) {
            const u32 pos = __builtin_ctz(xe);
            const u32 wa = (pos & 4) ? T[1] : T[0], wb = (pos & 4) ? T[3] : T[2], wc = (pos & 4) ? T[5] : T[4], wd = (pos & 4) ? T[7] : T[6];
            const u32 we = (pos & 8) ? wb : wa, wf = (pos & 8) ? wd : wc, wg = (pos & 16) ? wf : we;
            const u32 x4 = (wg >> ((pos & 3) * 8 + 4)) & 15;
            const u32 at = ne + __builtin_amdgcn_mbcnt_hi((u32)(mk >> 32), __builtin_amdgcn_mbcnt_lo((u32)mk, 0));
            if (at < 64) lst[at] = (unsigned short)((x4 << 12) | (hf * 2048 + lane * 32 + pos));
            xe &= xe - 1;
          }
          ne += __builtin_popcountll(mk);
        }
      }
      over = over || ne > 63;
      bar();
      if (!over) {
        const u32 ed = (ne + 1) >> 1, nd = (nn + 7) >> 3;
        const u32 ev = *(volatile LDSP u32*)(scr + 384 + (lane & 31) * 4);
        u32 nv = *(volatile LDSP u32*)(scr + 12 + lane * 4);
        const u32 vo = (lds_ticket(ctl + C_ENC + j2, ed + nd) & 0xffff) + 4;
        if (lane == (nn >> 3)) nv |= ~0u << ((nn & 7) * 4);
        if (lane < nd) st4(pk + FIX + (vo + lane) * 4, nv);
        if (lane < ed) st4(pk + FIX + (vo + nd + lane) * 4, ev);
        if (lane == 0) st4(pk + FIX + (u * 2 + hf) * 4, vo | (base << 10) | (ne << 17) | (nd << 23));
      }
      u64 pw[NR];
      asm volatile("s_load_dwordx2 %0, %8, 0x0\n\ts_load_dwordx2 %1, %8, 0x10\n\ts_load_dwordx2 %2, %8, 0x20\n\ts_load_dwordx2 %3, %8, 0x30\n\t"
                   "s_load_dwordx2 %4, %8, 0x40\n\ts_load_dwordx2 %5, %8, 0x50\n\ts_load_dwordx2 %6, %8, 0x60\n\ts_load_dwordx2 %7, %8, 0x70\n\t"
                   "s_waitcnt vmcnt(0) lgkmcnt(0)"
                   : "=&s"(pw[0]), "=&s"(pw[1]), "=&s"(pw[2]), "=&s"(pw[3]), "=&s"(pw[4]), "=&s"(pw[5]), "=&s"(pw[6]), "=&s"(pw[7])
                   : "s"(__builtin_amdgcn_kernarg_segment_ptr()) : "memory");
      const u32 k = lds_ticket(ctl + C_ENC + j2, over ? 0x1010000u : 0x1000000u);
      if ((k >> 24) == 3) {
        const u32 v = (call << 8) | ((over || (k & 0xff0000) != 0) ? M_RAW : ((k & 0xffff) + 4 + 31) >> 5);
        asm volatile("s_mov_b64 exec, 3\n\t"
                     "global_store_dwordx4 %0, %1, %2 sc0 sc1\n\tglobal_store_dwordx4 %0, %1, %3 sc0 sc1\n\t"
                     "global_store_dwordx4 %0, %1, %4 sc0 sc1\n\tglobal_store_dwordx4 %0, %1, %5 sc0 sc1\n\t"
                     "global_store_dwordx4 %0, %1, %6 sc0 sc1\n\tglobal_store_dwordx4 %0, %1, %7 sc0 sc1\n\t"
                     "global_store_dwordx4 %0, %1, %8 sc0 sc1\n\tglobal_store_dwordx4 %0, %1, %9 sc0 sc1\n\t"
                     "s_mov_b64 exec, -1\n\ts_nop 0"
                     :: "v"(DIR_OFF + ((par * NR + self) * NIT + blk) * 128 + lane * 16), "v"(u32x4{v, v, v, v}),
                        "s"(pw[0]), "s"(pw[1]), "s"(pw[2]), "s"(pw[3]), "s"(pw[4]), "s"(pw[5]), "s"(pw[6]), "s"(pw[7]) : "memory");
        if (lane < 16) st16<0>(me + UF_OFF + (self * NIT + blk) * 512 + (lane >> 2) * 128 + (lane & 3) * 16, u32x4{call, call, call, call});
      }
    }
  } else {
    if (link && ((wave >= 2 * NGRP && wave < 9) || (peer == self && !(slot == 0 && wave == 0)))) return;
    const u32 cv = ld_agent(cnt);
    if (wave == 7) { ctl[lane] = 0; waitl(); }
    bar();
    call = rfl(cv) + 1;
  }
  if (lane == 0 && wave != 11) st_agent(cnt, call);

  if (link) {
    if (peer == self) {
      const char* pf = me + PF_OFF + lane * 32;
      u32 f;
      bool all;
      do {
        __builtin_amdgcn_s_sleep(16);
        f = ld_agent(pf);
        all = __builtin_amdgcn_ballot_w64((f & 0xfff) != (((call & 63) << 6) | 1) && (lane >> 3) != xcd) == 0;
      } while (!all && __builtin_amdgcn_s_memrealtime() - tstart < PLANT);
      float v = (lane >> 3) == xcd ? 0.f : (float)((int)(f - ((u32)tstart << 12)) >> 12);
      v += __shfl_xor(v, 1); v += __shfl_xor(v, 2); v += __shfl_xor(v, 4);
      v *= 0.125f;
      float mu = v + __shfl_xor(v, 8);
      mu += __shfl_xor(mu, 16); mu += __shfl_xor(mu, 32);
      mu *= 1.f / 7;
      u32 inv = 0;
#pragma unroll
      for (int x = 0; x < NR; ++x) inv |= x << (4 * ((a8[5] >> (4 * x)) & 7));
      const int ps = lane >> 3, pr = lane & 7;
      const float fs = __shfl(v, ((inv >> (4 * ps)) & 7) * 8), fr = __shfl(v, ((inv >> (4 * pr)) & 7) * 8);
      const float ex = ps == self || fs - mu < DZ ? 0.f : fs - mu, gp = pr == self ? 0.f : fmaxf(mu - fr, 0.f);
      float gs = gp + __shfl_xor(gp, 1);
      gs += __shfl_xor(gs, 2); gs += __shfl_xor(gs, 4);
      u32 row = all && gs > 0.f ? (u32)fminf(8.f, ex * gp / gs * (8.f / TITEM) + 0.5f) << (4 * pr) : 0;
      row |= __shfl_xor(row, 1); row |= __shfl_xor(row, 2); row |= __shfl_xor(row, 4);
      row = __shfl(row, pr * 8);
      __hip_atomic_store((GLB u64*)(me + PLAN_OFF + ps * 128 + pr * 8), row | ((u64)call << 32), __ATOMIC_RELAXED, __HIP_MEMORY_SCOPE_AGENT);
      return;
    }
    if (wave == 11) {
      const u32 par = call & 1;
      const u32 cw = mm + ct * 2;
      u32 nitems = (NIT - NPUSH - cw + 7) >> 3;
      u32 itv = lane < nitems ? sched_item(cw + lane * 8, xcd, self) : 0;
      const char* dir = me + DIR_OFF + ((par * NR + peer) * NIT + itv) * 128;
      u32 dv = 0;
      u64 todo = (1ull << nitems) - 1, ok = 0;
      bool have = false;
      while (__builtin_amdgcn_s_memrealtime() - tstart < GATE - 50) __builtin_amdgcn_s_sleep(1);
      for (u32 n = 0; n < nitems + NGRP; ++n) {
        u32 w0 = (IT_END << 6) | 0x80000;
        if (n < nitems) {
          const u32 first = __builtin_ctzll(todo);
          const u64 cand = todo & (((n < 8 ? 7ull : 1ull) << first) | (0xffull << 56));
          const bool rl = (todo >> 56) != 0 && ((ok & todo) >> 56) == 0;
          for (u32 sp = 1; (ok & cand) == 0 || (rl && sp == 1); ++sp) {
            if (sp > 1 && n > 2) __builtin_amdgcn_s_sleep(4);
            if (lane - first < (n < 8 ? 4u : 64u) && ((todo & ~ok) >> lane) & 1) {
              dv = ld_sys(dir);
              if (lane >= 56) {
                const char* rf = pws + RLF_OFF + ((lane - 56) * 8 + ((itv >> 3) & 7)) * 128;
                if (min(ld_sys(rf), ld_sys(rf + 32)) < call) dv = 0;
              }
            }
            ok = __builtin_amdgcn_ballot_w64((dv >> 8) == (call & 0xffffff));
            if ((sp & 31) == 0 && give_up(me, call, tstart)) { dv = (call << 8) | M_RAW; ok = ~0ull; }
          }
          const u64 pick = ok & cand;
          const u32 ln = (pick >> 56) != 0 ? 63 - __builtin_clzll(pick) : __builtin_ctzll(pick);
          todo &= ~(1ull << ln);
          const u32 it = __builtin_amdgcn_readlane(itv, ln);
          u32 e = __builtin_amdgcn_readlane(dv, ln);
          e &= (e & M_RAW) ? M_RAW : 0xf;
          const u32 so = ln >= 56 ? RL_OFF / ITEM + (par * NR + ln - 56) * 8 + ((it >> 3) & 7)
                         : it >= 256 ? par * 512 + self * 32 + it : (e & M_RAW) ? RAW_OFF / ITEM + (par ? peer : 8) * NIT + it : par * 512 + it;
          w0 = (it << 6) | ((e & M_RAW) ? 0x80000 : e << 15) | (so << 20);
        }
        if (n >= LA) {
          volatile LDSP u32x2* cp = (volatile LDSP u32x2*)(ctl + C_CONS + (n % NGRP) * 2);
          for (;;) { const u32x2 cv = *cp; if (rfl(min(cv.x, cv.y)) + LA > n) break; __builtin_amdgcn_s_sleep(2); }
        }
        if (lane == 0) ctl[C_DESC + n % NS] = w0 | (n + 1);
        if (!have && n >= 17 && (n & 1)) {
          u64 pv;
          for (u32 sp = 1;; ++sp) {
            pv = __hip_atomic_load((const GLB u64*)(me + PLAN_OFF + xcd * 128 + (lane & 7) * 8), __ATOMIC_RELAXED, __HIP_MEMORY_SCOPE_AGENT);
            have = __builtin_amdgcn_ballot_w64((u32)(pv >> 32) != call) == 0;
            if (have || n < 27) break;
            if ((sp & 31) == 0 && give_up(me, call, tstart)) { pv = 0; have = true; break; }
          }
          if (have) {
            const u32 row = (u32)pv, mine = __builtin_amdgcn_readlane(row, peer);
            const u32 r = ((itv >> 1) & 3) * 2 + (itv & 1), col = (itv >> 3) & 7;
            const u64 skip = __builtin_amdgcn_ballot_w64((itv >> 6) == 2 && ((col - nib_sum(mine & ((1u << (4 * r)) - 1))) & 7) < ((mine >> (4 * r)) & 15));
            const u32 nt = lane >= 56 ? (row >> (4 * peer)) & 15 : 0, kk = (cw - (scan_add(nt) - nt)) & 7;
            const u64 take = __builtin_amdgcn_ballot_w64(kk < nt);
            if (lane >= 56) {
              itv = round_item(16 + peer, (nib_sum(row & ((1u << (4 * peer)) - 1)) + kk) & 7);
              dir = me + DIR_OFF + ((par * NR + lane - 56) * NIT + itv) * 128;
            }
            todo = (todo & ~skip) | take;
            nitems = n + 1 + __builtin_popcountll(todo);
            if (lane == 0) ctl[C_NIT] = nitems;
          }
        }
      }
      if (lane == 0) st_agent(cnt, call);
      return;
    }
    if (wave == 9) {
      const u32 cw = mm + ct * 2, n0 = (NIT - NPUSH - cw + 7) >> 3;
      char* bp = me + BAL_OFF + xcd * 256;
      u32 until = 4 * K0;
      for (u32 r = 0;; ++r) {
        u32 p0, p1;
        while (((p0 = rfl(ctl[C_LANDED])) & 0xffff) < until) __builtin_amdgcn_s_sleep(4);
        for (int i = 0; (p1 = rfl(ctl[C_LANDED])) == p0 && i < 12; ++i) {}
        const u32 now = (u32)__builtin_amdgcn_s_memrealtime(), nn = rfl(ctl[C_NIT]), nitems = nn ? nn : n0, hu = p1 & 0xffff;
        const u32 fown = (now + ((nitems * 23 - hu * 4 - (p1 >> 16) * 7) * TITEM) / 23) << 12 | ((call & 63) << 6) | (nn ? 2 : 1);
        if (lane == 0) st_agent(bp + cw * 32, fown);
        if (r == 0) {
          if (lane == 0) st_agent(bp + PF_OFF - BAL_OFF + cw * 32, fown);
          until = hu + 6;
          continue;
        }
        if (hu + 12 >= 4 * nitems) break;
        const u32 f = ld_agent(bp + (lane & 7) * 32);
        const bool v = ((f ^ fown) & 0xfff) == 0;
        const int cnt8 = __builtin_popcount((u32)__builtin_amdgcn_ballot_w64(v) & 0xff);
        const int late = cnt8 < 6 ? 0 : (int)__builtin_amdgcn_readlane(scan_add(v ? (u32)((int)(fown - f) >> 12) : 0u), 7) / cnt8;
        const u32 plan = min((u32)(late > 0 ? (late + GFAST / 2) / GFAST : (GSLOW / 2 - late) / GSLOW), 4u);
        until = (rfl(ctl[C_LANDED]) & 0xffff) + 4 * plan;
        if (plan != 0) {
          if (lane == 0) ctl[C_WIN] = late > 0 ? 1 : 2;
          until = min(until, 4 * (nn ? nn : n0 - 8));
          while ((rfl(ctl[C_LANDED]) & 0xffff) < until) __builtin_amdgcn_s_sleep(4);
          if (lane == 0) ctl[C_WIN] = 0;
        }
        until = min(until + 8, 4 * (nn ? nn : n0 - 8));
      }
      return;
    }
    if (wave == 10) {
      if (lane < 3) touch(pws + PK_OFF + ((call & 1) * 512 + (lane == 2 ? 256 + self * 32 + 16 : lane * 255)) * ITEM + (self * 8 + mm + ct * 2) * 128, dump);
      waitv<0>();
      drain();
      const u32 l0 = (u32)(size_t)lds, lw = l0 + L_CTL + C_LANDED * 4, one = lane == 0 ? 1 : 0;
      u32 prog = 0, mk = 0, wm = 0, la = lane == WD ? lw : dump + lane * 4;
      __builtin_amdgcn_s_setprio(3);
#define MODE_ "s_cmp_eq_u32 %[wm], 1\n\ts_cselect_b64 vcc, -1, 0\n\ts_cmp_eq_u32 %[wm], 2\n\t"
#define LD_(off, id) "s_waitcnt vmcnt(%[wf])\n\ts_cbranch_vccnz .Lw" #id "_%=\n\ts_waitcnt vmcnt(%[w])\n\ts_cbranch_scc0 .Lw" #id "_%=\n\t" \
                     "s_waitcnt vmcnt(%[ws])\n.Lw" #id "_%=:\n\tglobal_load_lds_dwordx4 %[va], %[sb] offset:" #off " sc0 sc1\n\t"
#define SH_ "v_mov_b32_dpp %[mk], %[mk] row_shr:1 row_mask:0xf bank_mask:0xf\n\t"
#define PUB_ "ds_max_u32 %[la], %[mk]\n\t"
#define INC_(x) "v_add_u32 %[mk], %[mk], " x "\n\t"
#define WIN_ [wm] "s"(wm), [wf] "i"(WFAST - 1), [w] "i"(WD - 1), [ws] "i"(WD - 2)
#define FOUR(src, ldst, i0) do { u32 keep_; \
          asm volatile("ds_read_b32 %[dv], %[da]\n\tds_read_b32 %[wv], %[lw] offset:4\n\ts_mov_b32 %[k], m0\n\ts_mov_b32 m0, %[ld]\n\t" MODE_ \
                       LD_(0, 0) SH_ INC_("%[i]") PUB_ LD_(1024, 1) SH_ INC_("%[o]") PUB_ LD_(2048, 2) SH_ INC_("%[i]") PUB_ LD_(3072, 3) SH_ INC_("%[o]") \
                       "s_waitcnt lgkmcnt(3)\n\t" PUB_ "s_mov_b32 m0, %[k]" \
                       : [k] "=&s"(keep_), [dv] "=&v"(dvv), [wv] "=&v"(wvv), [mk] "+v"(mk) \
                       : [va] "v"((src) + lane * 16), [sb] "s"(pws), [ld] "s"(ldst), [la] "v"(la), [da] "v"(l0 + L_CTL + (C_DESC + se) * 4), \
                         [lw] "v"(lw), [i] "v"((i0) & one), [o] "v"(one), WIN_ : "memory", "scc", "vcc"); \
          prog += 2 + 2 * (i0); } while (0)
      u64 qu = 0;
      u32 ne = 0, se = 0, sc = 0, d = 0, dvv, wvv;
      bool fin = false;
      for (u32 n = 0;; ++n) {
#pragma clang loop unroll(disable)
        while (ne <= n + EA && !fin) {
          if ((d & 63) != ne + 1) {
            if (ne > n) break;
            d = rfl(ctl[C_DESC + se]);
            if ((d & 63) != ne + 1) {
              waitv<0>();
              if (lane == 0) ctl[C_LANDED] = prog;
              do { d = rfl(ctl[C_DESC + se]); } while ((d & 63) != ne + 1);
            }
          }
          fin = ((d >> 6) & 0x1ff) == IT_END;
          qu |= (u64)((d >> 20) | ((d >> 7) & 0x1000) | (fin ? 0x2000u : 0u)) << (16 * (ne - n));
          ++ne;
          if (!(d & 0x80000)) {
            const u32 ln = (d >> 15) & 15, pn = (prog & 0xffff) | (ne << 16);
            const u64 em = (ln & 7) ? (1ull << ((ln & 7) * 8)) - 1 : ~0ull;
            u32 keep;
            asm volatile("s_mov_b32 %[k], m0\n\ts_mov_b32 m0, %[ld]\n\t" MODE_
                         LD_(0, 0) SH_ PUB_
                         "s_cmp_gt_u32 %[ln], 8\n\ts_cbranch_scc1 .L3_%=\n\ts_cmp_eq_u32 %[wm], 2\n\ts_mov_b64 exec, %[em]\n\t" LD_(1024, 1) "s_branch .L4_%=\n"
                         ".L3_%=:\n\ts_cmp_eq_u32 %[wm], 2\n\t" LD_(1024, 2) SH_ PUB_ "s_mov_b64 exec, %[em]\n\t" LD_(2048, 3)
                         "\n.L4_%=:\n\ts_mov_b64 exec, -1\n\t" SH_ "v_writelane_b32 %[mk], %[pn], 0\n\t" PUB_ "s_mov_b32 m0, %[k]"
                         : [k] "=&s"(keep), [mk] "+v"(mk)
                         : [va] "v"((d >> 20) * ITEM + 4096 + lane * 16), [sb] "s"(pws), [ld] "s"(l0 + se * ITEM + 4096), [la] "v"(la),
                           [ln] "s"(ln), [em] "s"(em), [pn] "s"(pn), WIN_ : "memory", "scc", "vcc");
            prog = pn;
          }
          se = se + 1 == NS ? 0 : se + 1;
          if (n == 0) break;
          if (ne <= n + EA) d = rfl(ctl[C_DESC + se]);
        }
        const u32 cur = (u32)qu & 0xffff;
        qu >>= 16;
        if (cur & 0x2000) break;
        u32 so = (cur & 0xfff) * ITEM, lo = l0 + sc * ITEM;
#pragma clang loop unroll(disable)
        for (u32 k = cur >> 12;; --k) {
          FOUR(so, lo, 1 - (cur >> 12));
          if (k == 0) break;
          so += 4096; lo += 4096;
        }
        d = rfl(dvv);
        wm = rfl(wvv);
        la = lane == (wm == 1 ? WFAST : wm == 2 ? WD - 1 : WD) ? lw : dump + lane * 4;
        sc = sc + 1 == NS ? 0 : sc + 1;
      }
#define DRAIN(N) if (N < WFAST) \
        asm volatile("s_mov_b64 exec, %2\n\ts_waitcnt vmcnt(%3)\n\tds_max_u32 %0, %1\n\ts_mov_b64 exec, -1" \
                     :: "v"(lw), "v"(mk), "i"(1 << N), "i"(N) : "memory");
      DRAIN(9) DRAIN(8) DRAIN(7) DRAIN(6) DRAIN(5) DRAIN(4) DRAIN(3) DRAIN(2) DRAIN(1) DRAIN(0)
      return;
    }
    if (wave == 0 && lane < 4) touch(me + (lane == 3 ? UF_OFF + peer * NIT * 512 : RAW_OFF + (peer * NIT + lane * 143) * ITEM) + slot * 128, dump);
    drain();
    const int u = wave & 1, dw = wave;
    LDSP char* img = lds + L_IMG + wave * 4096;
    u32 m0f = 0x0f0f0f0fu, m80 = 0x80808080u, m3e = 0x3e3e3e3eu, m30 = 0x30303030u;
    u32 mffc = 0xffc;
    const u32 imga = (u32)(size_t)img;
    asm volatile("" : "+s"(m0f), "+s"(m80), "+s"(m3e), "+s"(m30), "+v"(mffc));
    for (u32 n = dw >> 1;; n += NGRP) {
      u32 w0;
      while (((w0 = rfl(ctl[C_DESC + n % NS])) & 63) != n + 1) __builtin_amdgcn_s_sleep(2);
      const u32 it = (w0 >> 6) & 0x1ff;
      if (it == IT_END) break;
      LDSP char* sl = lds + (n % NS) * ITEM;
      const u32 rlo = (w0 >> 20) - RL_OFF / ITEM;
      const u32 src = rlo < 128 ? (rlo >> 3) & 7 : peer;
      const bool keep = (it >> 6) == 2 && ((it >> 1) & 3) * 2 + (it & 1) == self;
      char* d = me + RAW_OFF + (src * NIT + it) * ITEM + u * 4096 + lane * 16;
      u32x4 o0, o1, o2, o3;
#define LANDED(m, x) do { while ((rfl(ctl[C_LANDED]) & (m)) < (x)) __builtin_amdgcn_s_sleep(1); fence(); } while (0)
      if (w0 & 0x80000) {
        LANDED(0xffffu, 4 * n + 2 * u + 2);
        LDSP char* s = sl + u * 4096 + lane * 16;
        o0 = *(LDSP u32x4*)s; o1 = *(LDSP u32x4*)(s + 1024); o2 = *(LDSP u32x4*)(s + 2048); o3 = *(LDSP u32x4*)(s + 3072);
        st16<0>(d, o0); st16<1024>(d, o1); st16<2048>(d, o2); st16<3072>(d, o3);
      } else {
        LANDED(0xffff0000u, (n + 1) << 16);
        const u32x2 hv = *(LDSP u32x2*)(sl + FIX + u * 8);
        const u32 h0 = rfl(hv.x), h1 = rfl(hv.y);
        const u32 nd0 = h0 >> 23, nd1 = h1 >> 23;
        const u32 b0 = ((h0 >> 10) & 0x7f) * 0x01010101u - 0x3e3e3e3eu, b1 = ((h1 >> 10) & 0x7f) * 0x01010101u - 0x3e3e3e3eu;
        LDSP char* v0 = sl + FIX + (h0 & 0x1ff) * 4;
        LDSP char* v1 = sl + FIX + (h1 & 0x1ff) * 4;
        u32 nb0 = *(LDSP u32*)(v0 + lane * 4), nb1 = *(LDSP u32*)(v1 + lane * 4);
        const u32 e0 = *(LDSP unsigned short*)(v0 + (nd0 & 0x7f) * 4 + lane * 2), e1 = *(LDSP unsigned short*)(v1 + (nd1 & 0x7f) * 4 + lane * 2);
        const u32 sg0 = *(LDSP u32*)(sl + 4096 + u * 512 + lane * 4), sg1 = *(LDSP u32*)(sl + 4096 + u * 512 + 256 + lane * 4);
        {
          LDSP char* iz = img + lane * 16;
          *(LDSP u32x4*)iz = u32x4{b0, b0, b0, b0}; *(LDSP u32x4*)(iz + 1024) = u32x4{b0, b0, b0, b0};
          *(LDSP u32x4*)(iz + 2048) = u32x4{b1, b1, b1, b1}; *(LDSP u32x4*)(iz + 3072) = u32x4{b1, b1, b1, b1};
        }
        if (lane >= nd0) nb0 = ~0u;
        if (lane >= nd1) nb1 = ~0u;
        u32 gg[4], vv[4];
        gg[0] = (nb0 & m0f) << 3; gg[1] = (nb0 >> 1) & 0x78787878u; gg[2] = (nb1 & m0f) << 3; gg[3] = (nb1 >> 1) & 0x78787878u;
        const u32 sum = __builtin_amdgcn_sad_u8(gg[0], 0, __builtin_amdgcn_sad_u8(gg[1], 0, 0)) |
                        (__builtin_amdgcn_sad_u8(gg[2], 0, __builtin_amdgcn_sad_u8(gg[3], 0, 0)) << 16);
#pragma unroll
        for (int k = 0; k < 4; ++k) {
          vv[k] = (~(gg[k] + 0x08080808u) & m80) >> 3;
          asm("" : "+v"(gg[k]), "+v"(vv[k]));
        }
        const u32 ps = scan_add(sum) - sum;
        u32 pa = ps & 0xffff, pb = (ps >> 16) + 2048 * 8;
        if (lane < max(nd0, nd1)) {
#pragma unroll
          for (int k = 0; k < 8; ++k) {
            const int w = k & 1, by = (k >> 1) * 8;
            pa += (gg[w] >> by) & 0xff;
            pb += (gg[2 + w] >> by) & 0xff;
            LDS_SUB((LDSP char*)(size_t)(((pa >> 3) & mffc) | imga), ((vv[w] >> by) & 0xff) << (pa & 31));
            LDS_SUB((LDSP char*)(size_t)(((pb >> 3) & mffc) | imga), ((vv[2 + w] >> by) & 0xff) << (pb & 31));
          }
        }
        fence();
        LDSP char* ip = img + lane * 32;
        u32x4 a0 = *(LDSP u32x4*)ip, a1 = *(LDSP u32x4*)(ip + 16), a2 = *(LDSP u32x4*)(ip + 2048), a3 = *(LDSP u32x4*)(ip + 2064);
        fence();
        if (lane < ((h0 >> 17) & 63)) LDS_SUB(img + (e0 & 0xffc), ((e0 >> 12) - 8) << ((e0 & 3) * 8 + 4));
        if (lane < ((h1 >> 17) & 63)) LDS_SUB(img + (e1 & 0xffc), ((e1 >> 12) - 8) << ((e1 & 3) * 8 + 4));
        fence();
        o0 = *(LDSP u32x4*)ip; o1 = *(LDSP u32x4*)(ip + 16); o2 = *(LDSP u32x4*)(ip + 2048); o3 = *(LDSP u32x4*)(ip + 2064);
#pragma unroll
        for (int q = 0; q < 4; ++q) {
          o0[q] += (sg0 << q) & m80; o1[q] += (sg0 << (4 + q)) & m80;
          o2[q] += (sg1 << q) & m80; o3[q] += (sg1 << (4 + q)) & m80;
          const u32 l0 = b0 - a0[q], l1 = b0 - a1[q], l2 = b1 - a2[q], l3 = b1 - a3[q];
          a0[q] = (l0 << 1) | (l0 >> 1); a1[q] = (l1 << 1) | (l1 >> 1); a2[q] = (l2 << 1) | (l2 >> 1); a3[q] = (l3 << 1) | (l3 >> 1);
        }
        asm volatile("" : "+v"(o0), "+v"(o1), "+v"(o2), "+v"(o3), "+v"(a0), "+v"(a1), "+v"(a2), "+v"(a3));
#define CODES(o, a, x) _Pragma("unroll") for (int q = 0; q < 4; ++q) { \
          u32 c_ = ((x) & m0f) + m3e; asm("" : "+v"(c_)); o[q] += c_ + (c_ & a[q] & m30); }
        LANDED(0xffffu, 4 * n + 2 * u + 1);
        const u32x4 n0 = *(LDSP u32x4*)(sl + u * 2048 + lane * 16);
        CODES(o0, a0, n0[q]) st16<0>(d, o0);
        CODES(o1, a1, n0[q] >> 4) st16<1024>(d, o1);
        LANDED(0xffffu, 4 * n + 2 * u + 2);
        const u32x4 n1 = *(LDSP u32x4*)(sl + u * 2048 + 1024 + lane * 16);
        CODES(o2, a2, n1[q]) st16<2048>(d, o2);
        CODES(o3, a3, n1[q] >> 4) st16<3072>(d, o3);
      }
      if (keep) {
        LANDED(0xffffu, 4 * n + 4);
        LDSP char* s = sl + u * 4096 + lane * 16;
        char* r = me + RL_OFF + (((call & 1) * NR + peer) * 8 + ((it >> 3) & 7)) * ITEM + u * 4096 + lane * 16;
        st16<0>(r, *(LDSP u32x4*)s); st16<1024>(r, *(LDSP u32x4*)(s + 1024));
        st16<2048>(r, *(LDSP u32x4*)(s + 2048)); st16<3072>(r, *(LDSP u32x4*)(s + 3072));
      }
      if (lane == 0) ctl[C_CONS + dw] = n + 1;
      waitv<0>();
      if (lane < 8) st16<0>(me + UF_OFF + (src * NIT + it) * 512 + u * 32 + (lane >> 1) * 128 + (lane & 1) * 16, u32x4{call, call, call, call});
      if (keep) put_flag(me + RLF_OFF + (peer * 8 + ((it >> 3) & 7)) * 128 + u * 32, call, lane);
    }
    return;
  }

  const int t = xcd * 4 + (ct >> 1);
  const int dbl = 1 - (ct >> 2);

  if (wave >= 4) {
    drain();
    const int src = wave - 4;
    if (lane == 0 && wave == 11) st_agent(cnt, call);
    const char* raw = me + RAW_OFF + (src != self || (call & 1) ? src : 8) * NIT * ITEM + lane * 16;
    const char* uf = me + UF_OFF + src * NIT * 512;
    const char* px = me + PX_OFF + ((call & 1) * NR + src) * 16 * ITEM + lane * 16;
    const u32 lA = (u32)(size_t)lds + L_A + src * 16384;
    const u64 t0 = tstart;
    u32 need = call;
    u32 prev = 0;
    const char* fa = uf + ((128 + ((lane >> 2) & 3) * 4 + mm) * 2 + ((lane >> 1) & 1)) * 512 + (xcd & 3) * 128 + (lane & 1) * 32;
    u64 xmiss = 0xffff;
    u32 xf = 0;
#pragma clang loop unroll(disable)
    for (int j = 0; j < 4; ++j) {
      {
        for (u32 it = 1; ((xmiss >> (4 * j)) & 15) != 0; ++it) {
          if (it > 1) __builtin_amdgcn_s_sleep(8);
          if ((xmiss >> lane) & 1) xf = ld_sys(fa);
          xmiss = __builtin_amdgcn_ballot_w64(xf < need) & 0xffff;
          if ((it & 15) == 0 && give_up(me, call, t0)) need = 0;
        }
        while (rfl(min4(*(volatile LDSP u32x4*)(ctl + C_FREE))) < prev) __builtin_amdgcn_s_sleep(2);
        const int hx = (j * 4 + mm) * 2;
        const char* sa0 = hx < NPUSH ? px + hx * ITEM : raw + (256 + hx) * ITEM;
        const char* sa1 = hx + 1 < NPUSH ? px + (hx + 1) * ITEM : raw + (257 + hx) * ITEM;
        u32 keep;
#pragma unroll
        for (int i = 0; i < 4; ++i)
          asm volatile("s_mov_b32 %0, m0\n\ts_mov_b32 m0, %2\n\ts_nop 0\n\t"
                       "global_load_lds_dwordx4 %1, off sc0 sc1\n\t"
                       "global_load_lds_dwordx4 %1, off offset:1024 sc0 sc1\n\t"
                       "global_load_lds_dwordx4 %1, off offset:2048 sc0 sc1\n\t"
                       "global_load_lds_dwordx4 %1, off offset:3072 sc0 sc1\n\t"
                       "s_mov_b32 m0, %0" : "=&s"(keep) : "v"((i < 2 ? sa0 : sa1) + (i & 1) * 4096), "s"(lA + i * 4096) : "memory");
      }
      const int nh0 = dbl ? 0 : (ct & 1);
      const char* fb = uf + ((j * 32 + t) * 2 + nh0 + ((lane >> 1) & dbl)) * 512 + mm * 128 + (lane & 1) * 32;
      if (j < 3)
        for (u32 it = 1; __builtin_amdgcn_ballot_w64(ld_sys(fb) < need) != 0; ++it) {
          if ((it & 15) == 0 && give_up(me, call, t0)) need = 0;
          __builtin_amdgcn_s_sleep(8);
        }
      const char* pp = raw + ((j * 32 + t) * 2 + nh0) * ITEM + 4096;
      u32x4 p[8], p2[8];
      u32 tv, tk, r0, r1;
      for (;;) {
        u32 it;
        asm volatile("s_mov_b32 %12, 96\n\ts_cmp_lt_u32 %19, %20\n\ts_cbranch_scc1 .Lg_%=\n\t"
                     "global_load_dword %10, %13, off sc0 sc1\n\ts_sleep 6\n\t"
                     "global_load_dword %11, %13, off sc0 sc1\n\ts_sleep 6\n"
                     ".Lp_%=:\n\t"
                     "global_load_dword %8, %13, off sc0 sc1\n\ts_waitcnt vmcnt(2)\n\t"
                     "v_cmp_gt_u32 vcc, %16, %10\n\ts_cbranch_vccz .Lg_%=\n\t"
                     "global_load_dword %10, %13, off sc0 sc1\n\ts_waitcnt vmcnt(2)\n\t"
                     "v_cmp_gt_u32 vcc, %16, %11\n\ts_cbranch_vccz .Lg_%=\n\t"
                     "global_load_dword %11, %13, off sc0 sc1\n\ts_waitcnt vmcnt(2)\n\t"
                     "v_cmp_gt_u32 vcc, %16, %8\n\ts_cbranch_vccz .Lg_%=\n\t"
                     "s_sub_u32 %12, %12, 1\n\ts_cbranch_scc0 .Lp_%=\n\t"
                     "s_waitcnt vmcnt(0)\n\ts_branch .Le_%=\n"
                     ".Lg_%=:\n\t"
                     "global_load_dwordx4 %0, %14, off offset:-4096 sc0 sc1\n\t"
                     "global_load_dwordx4 %1, %14, off offset:-3072 sc0 sc1\n\t"
                     "global_load_dwordx4 %2, %14, off offset:-2048 sc0 sc1\n\t"
                     "global_load_dwordx4 %3, %14, off offset:-1024 sc0 sc1\n\t"
                     "global_load_dwordx4 %4, %14, off sc0 sc1\n\t"
                     "global_load_dwordx4 %5, %14, off offset:1024 sc0 sc1\n\t"
                     "global_load_dwordx4 %6, %14, off offset:2048 sc0 sc1\n\t"
                     "global_load_dwordx4 %7, %14, off offset:3072 sc0 sc1\n\t"
                     "global_load_dword %8, %15, off sc1\n\t"
                     "v_mov_b32 %9, %17\n\ts_mov_b64 exec, 1\n\t"
                     "ds_add_rtn_u32 %9, %18, %9\n\ts_mov_b64 exec, -1\n\ts_waitcnt lgkmcnt(0)\n"
                     ".Le_%=:"
                     : "=&v"(p[0]), "=&v"(p[1]), "=&v"(p[2]), "=&v"(p[3]), "=&v"(p[4]), "=&v"(p[5]), "=&v"(p[6]), "=&v"(p[7]),
                       "=&v"(tv), "=&v"(tk), "=&v"(r0), "=&v"(r1), "=&s"(it)
                     : "v"(fb), "v"(pp), "v"(me + TMO_OFF + b * 128), "s"(need), "s"(1 + dbl),
                       "v"((u32)(size_t)(lds + L_CTL + C_TICKET * 4)), "s"(j), "i"(3) : "memory", "scc", "vcc");
        if (it != 0xffffffffu) break;
        if (give_up(me, call, t0)) need = 0;
      }
      asm volatile("s_cmp_eq_u32 %9, 0\n\ts_cbranch_scc1 .Ls_%=\n\t"
                   "global_load_dwordx4 %0, %8, off offset:-4096 sc0 sc1\n\t"
                   "global_load_dwordx4 %1, %8, off offset:-3072 sc0 sc1\n\t"
                   "global_load_dwordx4 %2, %8, off offset:-2048 sc0 sc1\n\t"
                   "global_load_dwordx4 %3, %8, off offset:-1024 sc0 sc1\n\t"
                   "global_load_dwordx4 %4, %8, off sc0 sc1\n\t"
                   "global_load_dwordx4 %5, %8, off offset:1024 sc0 sc1\n\t"
                   "global_load_dwordx4 %6, %8, off offset:2048 sc0 sc1\n\t"
                   "global_load_dwordx4 %7, %8, off offset:3072 sc0 sc1\n"
                   ".Ls_%=:"
                   : "=&v"(p2[0]), "=&v"(p2[1]), "=&v"(p2[2]), "=&v"(p2[3]), "=&v"(p2[4]), "=&v"(p2[5]), "=&v"(p2[6]), "=&v"(p2[7])
                   : "v"(pp + 8192), "s"(dbl) : "memory", "scc");
      u32 ts = rfl(tk) & 0xffff, fs = 0;
      const u32 t1 = (rfl(tk) >> 16) - 1;
#define SLOT(t) ({ const u32 k_ = (t) < t1 ? (t) % 3 : ((t) - t1) % 5; k_ < 3 ? L_B / 8192 + k_ : self * 2 + k_ - 3; })
#define NEEDF(t) ((t) + 8 - ((t) >= t1 && (t) - t1 >= 3 ? 4 : 2))
      u32 sl0 = SLOT(ts), sl1 = SLOT(ts + 1), nf0 = NEEDF(ts), nf1 = NEEDF(ts + 1);
      asm volatile("" : "+s"(sl0), "+s"(sl1), "+s"(nf0), "+s"(nf1));
#define ROOM(nf) while (fs < (nf)) { fs = rfl(min4(*(volatile LDSP u32x4*)(ctl + C_FREE))) + 8; if (fs < (nf)) __builtin_amdgcn_s_sleep(1); }
#define FULL(sl) do { ++ts; \
        asm volatile("" : "+s"(ts) :: "memory"); \
        if (lane == 0) ctl[C_FULL + ((ts - 1) & 7)] = ts | (src << 8) | ((sl) << 11) | (tv == call ? 0x10000u : 0u); } while (0)
#define PUT(R, sl) do { LDSP char* ub = lds + (sl) * 8192 + lane * 16; \
        _Pragma("unroll") for (int i = 0; i < 8; ++i) *(LDSP u32x4*)(ub + i * 1024) = R[i]; \
        FULL(sl); } while (0)
      ROOM(nf0);
      {
        LDSP char* ub = lds + sl0 * 8192 + lane * 16;
#define PUT2(i, N) asm volatile("s_waitcnt vmcnt(" #N ")" : "+v"(p[i]), "+v"(p[i + 1]) :: "memory"); \
        *(LDSP u32x4*)(ub + (i) * 1024) = p[i]; *(LDSP u32x4*)(ub + (i) * 1024 + 1024) = p[i + 1];
        PUT2(0, 7) PUT2(2, 5) PUT2(4, 3) PUT2(6, 1)
        asm volatile("s_waitcnt vmcnt(0)" : "+v"(p2[0]), "+v"(p2[1]), "+v"(p2[2]), "+v"(p2[3]), "+v"(p2[4]), "+v"(p2[5]), "+v"(p2[6]), "+v"(p2[7]),
                     "+v"(tv), "+v"(r0), "+v"(r1) :: "memory");
        FULL(sl0);
      }
      if (dbl) { ROOM(nf1); fence(); PUT(p2, sl1); }
      prev = ts;
    }
    if (src == self) {
      while (rfl(min4(*(volatile LDSP u32x4*)(ctl + C_FREE))) < prev) __builtin_amdgcn_s_sleep(2);
      if (lane == 0) {
        u32 o = ctl[C_TICKET];
        while (!__hip_atomic_compare_exchange_strong((LDSP u32*)(ctl + C_TICKET), &o, o | (((o + 2) / 3 * 3 + 1) << 16),
                                                     __ATOMIC_RELAXED, __ATOMIC_RELAXED, __HIP_MEMORY_SCOPE_WORKGROUP)) {}
      }
    }
    return;
  }

  const int mw = wave;
  {
    u32 scale_bits = __builtin_amdgcn_readfirstlane(__builtin_bit_cast(u32, P.sx[0] * P.sw[0]));
    float* obase = P.out + (size_t)(mm * 128 + mw * 32) * NN + t * 128 + (ct & 1) * 64;
    asm volatile("" : "+s"(scale_bits), "+s"(obase));
    drain();
    v4f Pc[2][4], Qc[2][4];
#pragma unroll
    for (int i = 0; i < 2; ++i)
#pragma unroll
      for (int k = 0; k < 4; ++k) Pc[i][k] = Qc[i][k] = v4f{0.f, 0.f, 0.f, 0.f};
#define FRAG(p, hs) ({ const u32x4 lo_ = *(LDSP u32x4*)(p), hi_ = *(LDSP u32x4*)((p) + (hs)); \
      v8i{(int)lo_.x, (int)lo_.y, (int)lo_.z, (int)lo_.w, (int)hi_.x, (int)hi_.y, (int)hi_.z, (int)hi_.w}; })
#define MMA(i, k, F) do { F##c[i][k] = __builtin_amdgcn_mfma_scale_f32_16x16x128_f8f6f4(F##a[i], F##b[k], F##c[i][k], 1, 1, 0, 0, 0, 0); \
      __builtin_amdgcn_sched_barrier(0); } while (0)
#define SB() __builtin_amdgcn_sched_barrier(0)
#define RD(F) do { bad |= w; LDSP char* ap_ = lds + L_A + ((w >> 8) & 7) * 16384 + mw * 4096 + lane * 16; \
      bp = lds + ((w >> 11) & 31) * 8192 + lane * 16; \
      F##a[0] = FRAG(ap_, 1024); F##b[0] = FRAG(bp, 4096); SB(); F##a[1] = FRAG(ap_ + 2048, 1024); SB(); F##b[1] = FRAG(bp + 1024, 4096); \
      SB(); } while (0)
#define SPIN() do { while (((w = rfl(ctl[C_FULL + (n & 7)])) & 0xff) != n + 1) __builtin_amdgcn_s_sleep(0); \
      fence(); } while (0)
#define HEAD(F) do { \
      MMA(0, 0, F); F##b[2] = FRAG(bp + 2048, 4096); SB(); \
      MMA(1, 0, F); F##b[3] = FRAG(bp + 3072, 4096); \
      fence();     \
      ++n; \
      if (lane == 0) ctl[C_FREE + mw] = n; } while (0)
#define STEP(F, G, EXTRA) do { \
      HEAD(F); \
      peek = ctl[C_FULL + (n & 7)]; SB(); \
      MMA(0, 1, F); MMA(1, 1, F); MMA(0, 2, F); SB(); \
      w = rfl(peek); \
      fence(); \
      const bool ok_ = (w & 0xff) == n + 1; \
      if (ok_) RD(G); \
      MMA(1, 2, F); MMA(0, 3, F); MMA(1, 3, F); SB(); \
      EXTRA; \
      if (!ok_) { SPIN(); RD(G); } } while (0)
    LDSP char* bp;
    v8i Pa[2], Pb[4], Qa[2], Qb[4];
    u32 n = 0, w, peek, bad = 0;
    SPIN();
    RD(P);
#pragma clang loop unroll(disable)
    for (int it = 1; it < (16 << dbl); ++it) { STEP(P, Q, ); STEP(Q, P, ); }
#define EPI(A, i, r, o) do { v4f y_; _Pragma("unroll") for (int k = 0; k < 4; ++k) \
        y_[k] = __builtin_elementwise_maximum(A[i][k][r] * scale, 0.f);     \
      asm volatile("global_store_dwordx4 %0, %1, %2 sc1\n\ts_nop 1" :: "v"(ov), "v"(y_), "s"((o) + ((i) * 16 + (r)) * NN) : "memory"); \
      __builtin_amdgcn_sched_barrier(0); } while (0)
    STEP(P, Q, if (!dbl) { fence();
                 _Pragma("unroll") for (int i = 0; i < 2; ++i) _Pragma("unroll") for (int k = 0; k < 4; ++k) Qc[i][k] += Pc[i][k]; });
    HEAD(Q); SB();
    MMA(0, 1, Q); MMA(0, 2, Q); MMA(0, 3, Q); MMA(1, 1, Q);
    const float scale = __builtin_bit_cast(float, (bad & 0x10000u) != 0 ? 0x7fc00000u : scale_bits);
    const u32 ln = __builtin_amdgcn_mbcnt_hi(~0u, __builtin_amdgcn_mbcnt_lo(~0u, 0));
    const u32 ov = (ln >> 4) * (4 * NN * 4) + (ln & 15) * 16;
    SB();
    EPI(Qc, 0, 0, obase); MMA(1, 2, Q); EPI(Qc, 0, 1, obase); MMA(1, 3, Q); EPI(Qc, 0, 2, obase); EPI(Qc, 0, 3, obase);
    EPI(Qc, 1, 0, obase); EPI(Qc, 1, 1, obase); EPI(Qc, 1, 2, obase); EPI(Qc, 1, 3, obase);
    if (dbl) {
      EPI(Pc, 0, 0, obase - 64); EPI(Pc, 0, 1, obase - 64); EPI(Pc, 0, 2, obase - 64); EPI(Pc, 0, 3, obase - 64);
      EPI(Pc, 1, 0, obase - 64); EPI(Pc, 1, 1, obase - 64); EPI(Pc, 1, 2, obase - 64); EPI(Pc, 1, 3, obase - 64);
    }
  }
}

extern "C" void kernel_launch(int rank, int nranks, ncclComm_t comm,
                              hipStream_t stream,
                              void* const* d_in, const int* in_sizes, int n_in,
                              void* d_out, int out_size,
                              const int* in_dtypes, int out_dtype,
                              void* const* d_out_all,
                              void* const* d_ws_all, size_t ws_bytes) {
  if (nranks != NR || (unsigned)rank >= (unsigned)NR || n_in < 4 || ws_bytes < WS_NEED ||
      in_sizes[0] != 4096 * KS || in_sizes[1] != KS * NN || out_size != 512 * NN) {
    fprintf(stderr, "kernel_launch: unexpected problem shape\n");
    return;
  }
  Params p;
  p.me = (char*)d_ws_all[rank];
  p.w = (const unsigned char*)d_in[1];
  p.sx = (const float*)d_in[2];
  p.sw = (const float*)d_in[3];
  p.out = (float*)d_out;
  p.rank = rank;
  static const int grp[2][4] = {{0, 2, 3, 6}, {1, 4, 5, 7}};
  int gi = 0, gp = 0;
  for (int a = 0; a < 2; ++a) for (int i = 0; i < 4; ++i) if (grp[a][i] == rank) { gi = a; gp = i; }
  p.dstmap = 0;
  for (int x = 0; x < 8; ++x) {
    p.dstmap |= (u32)(x < 4 ? grp[gi][(gp + 1 + x) & 3] : grp[1 - gi][(gp + x) & 3]) << (4 * x);
    const int peer = (p.dstmap >> (4 * x)) & 7;
    p.px[x].ws = (char*)d_ws_all[peer];
    p.px[x].x = (const unsigned char*)d_in[0] + (size_t)peer * 512 * KS;
  }
  hipLaunchKernelGGL(fused_gemm_kernel, dim3(NWG), dim3(NTHREADS), 0, stream, p);
  HIP_CHECK(hipGetLastError());
}
```

```cpp
#include <hip/hip_runtime.h>
#include <rccl/rccl.h>
#include <cstdio>
#include <cstdint>

#define HIP_CHECK(expr) do { hipError_t _e = (expr); if (_e != hipSuccess) { \
    fprintf(stderr, "HIP error %s at %s:%d\n", hipGetErrorString(_e), __FILE__, __LINE__); \
    return; } } while (0)

typedef unsigned u32;
typedef unsigned long long u64;
typedef unsigned u32x2 __attribute__((ext_vector_type(2)));
typedef unsigned u32x4 __attribute__((ext_vector_type(4)));
typedef int v8i __attribute__((ext_vector_type(8)));
typedef float v4f __attribute__((ext_vector_type(4)));
#define LDSP __attribute__((address_space(3)))
#define GLB __attribute__((address_space(1)))

constexpr int NR = 8;
constexpr int KS = 512;
constexpr int NN = 4096;
constexpr int NTHREADS = 768;
constexpr int NWG = 256;
constexpr int NIT = 288;
#define WD 6
#define NPUSH 12
#define GATE 100

constexpr u32 ITEM = 8192;
constexpr u32 FIX = 5120;
constexpr u32 PK_OFF = 0;
constexpr u32 RAW_OFF = 2 * 512 * ITEM;
constexpr u32 PX_OFF = RAW_OFF + 9 * NIT * ITEM;
constexpr u32 RL_OFF = PX_OFF + 2 * NR * 16 * ITEM;
constexpr u32 DIR_OFF = RL_OFF + 2 * NR * 8 * ITEM;
constexpr u32 UF_OFF = DIR_OFF + 2 * NR * NIT * 128;
constexpr u32 CNT_OFF = UF_OFF + NR * NIT * 512;
constexpr u32 TMO_OFF = CNT_OFF + NWG * 12 * 128;
constexpr u32 BAL_OFF = TMO_OFF + NWG * 128;
constexpr u32 PF_OFF = BAL_OFF + NR * 256;
constexpr u32 PLAN_OFF = PF_OFF + NR * 256;
constexpr u32 RLF_OFF = PLAN_OFF + NR * 128;
static_assert(DIR_OFF / ITEM <= 4096, "12-bit item offsets");
constexpr u32 WS_NEED = RLF_OFF + NR * 8 * 128;
constexpr u64 DEADLINE = 30000000ull;
static_assert(NPUSH <= 16 && NPUSH >= 0, "PX region");

constexpr int NBS = 3;
constexpr u32 L_A = 0;
constexpr u32 L_B = 8 * 16384;
constexpr int NGRP = 4;
constexpr int NS = 12;
constexpr u32 L_IMG = NS * ITEM;
constexpr u32 L_SCR = L_B + NBS * ITEM;
constexpr u32 L_CTL = L_SCR + 4096;
constexpr u32 L_DUMP = L_CTL + 256;
constexpr u32 L_BYTES = L_DUMP + 256;
constexpr int C_FREE = 0, C_FULL = 4, C_TICKET = 12, C_ENC = 13;
constexpr int C_CONS = 16, C_LANDED = 28, C_WIN = 29, C_NIT = 30, C_DESC = 32;
constexpr u32 M_RAW = 0x80, IT_END = 0x1ff;

#define PUSH1 8
#define PUSHD 130
#define EA 2
#define LA 8
#define TITEM 77
#define WFAST 8
#define GFAST 24
#define GSLOW 19
#define K0 4
#define SP 4
#define PLANT 2000
#define DZ 10.f

struct Params {
  char* me;
  const unsigned char* x;
  const unsigned char* w;
  void* const* ws_all;
  int rank;
  u32 dstmap;
  float* out;
  const float* sx;
  const float* sw;
};

template <int N> __device__ __forceinline__ void waitv() { asm volatile("s_waitcnt vmcnt(%0)" :: "i"(N) : "memory"); }
__device__ __forceinline__ void waitl() { asm volatile("s_waitcnt lgkmcnt(0)" ::: "memory"); }
__device__ __forceinline__ void bar() { asm volatile("s_barrier" ::: "memory"); }
__device__ __forceinline__ void drain() { __builtin_amdgcn_s_waitcnt(0x0070); }
__device__ __forceinline__ void fence() { asm volatile("" ::: "memory"); }
template <int OFF> __device__ __forceinline__ void st16(const void* p, u32x4 x) {
  asm volatile("global_store_dwordx4 %0, %1, off offset:%2 sc0 sc1\n\ts_nop 1" :: "v"(p), "v"(x), "i"(OFF) : "memory");
}
__device__ __forceinline__ void st4(const void* p, u32 x) {
  asm volatile("global_store_dword %0, %1, off sc0 sc1\n\ts_nop 1" :: "v"(p), "v"(x) : "memory");
}
__device__ __forceinline__ u32 ld_sys(const void* p) {
  return __hip_atomic_load((const GLB u32*)p, __ATOMIC_RELAXED, __HIP_MEMORY_SCOPE_SYSTEM);
}
__device__ __forceinline__ u32 ld_agent(const void* p) {
  return __hip_atomic_load((const GLB u32*)p, __ATOMIC_RELAXED, __HIP_MEMORY_SCOPE_AGENT);
}
__device__ __forceinline__ void st_agent(void* p, u32 v) {
  __hip_atomic_store((GLB u32*)p, v, __ATOMIC_RELAXED, __HIP_MEMORY_SCOPE_AGENT);
}
__device__ __forceinline__ void touch(const void* p, u32 dump) {
  u32 keep;
  asm volatile("s_mov_b32 %0, m0\n\ts_mov_b32 m0, %2\n\ts_nop 0\n\tglobal_load_lds_dword %1, off sc0 sc1\n\ts_mov_b32 m0, %0"
               : "=&s"(keep) : "v"(p), "s"(dump) : "memory");
}
__device__ __forceinline__ u32 rfl(u32 v) { return __builtin_amdgcn_readfirstlane(v); }
__device__ __forceinline__ void put_flag(char* f, u32 v, int lane) {
  if (lane < 2) st16<0>(f + lane * 16, u32x4{v, v, v, v});
}
__device__ __forceinline__ bool give_up(char* me, u32 call, u64 t0) {
  if (rfl(ld_agent(me + TMO_OFF + blockIdx.x * 128)) == call) return true;
  if (__builtin_amdgcn_s_memrealtime() - t0 <= DEADLINE) return false;
  for (int i = threadIdx.x & 63; i < NWG; i += 64) st_agent(me + TMO_OFF + i * 128, call);
  waitv<0>();
  return true;
}
#define LDS_ADD(p, v) __hip_atomic_fetch_add((LDSP u32*)(p), (v), __ATOMIC_RELAXED, __HIP_MEMORY_SCOPE_WORKGROUP)
#define LDS_SUB(p, v) __hip_atomic_fetch_sub((LDSP u32*)(p), (v), __ATOMIC_RELAXED, __HIP_MEMORY_SCOPE_WORKGROUP)
#define LDS_OR(p, v) __hip_atomic_fetch_or((LDSP u32*)(p), (v), __ATOMIC_RELAXED, __HIP_MEMORY_SCOPE_WORKGROUP)
#define DPP(old, v, ctrl, rm) (u32)__builtin_amdgcn_update_dpp((int)(old), (int)(v), ctrl, rm, 0xf, false)
#define SCAN(name, id, OP) __device__ __forceinline__ u32 name(u32 v) { \
    v = OP(v, DPP(id, v, 0x111, 0xf)); v = OP(v, DPP(id, v, 0x112, 0xf)); v = OP(v, DPP(id, v, 0x114, 0xf)); \
    v = OP(v, DPP(id, v, 0x118, 0xf)); v = OP(v, DPP(id, v, 0x142, 0xa)); v = OP(v, DPP(id, v, 0x143, 0xc)); return v; }
__device__ __forceinline__ u32 addu(u32 a, u32 b) { return a + b; }
SCAN(scan_add, 0u, addu)
SCAN(scan_max, 0u, max)
SCAN(scan_min, 0xffffffffu, min)

__device__ __forceinline__ u32 round_item(u32 rd, u32 col) { return ((rd >> 3) * 32 + 4 * col + ((rd >> 1) & 3)) * 2 + (rd & 1); }
__device__ __forceinline__ u32 sched_item(u32 q, u32 rel, u32 self) {
  if (q < 32 - NPUSH) return 256 + (q < 8 ? 16 + q : q < 24 - NPUSH ? NPUSH - 8 + q : NPUSH + q);
  q -= 32 - NPUSH;
  const u32 sh = 16 + self, ps = (sh & ~3u) | ((sh - rel) & 3);
  u32 r = q >> 3;
  r = r == SP ? ps : r > SP && r <= ps ? r - 1 : r;
  return round_item((r & ~3u) | ((r + rel) & 3), (q + rel) & 7);
}
__device__ __forceinline__ u32 nib_sum(u32 x) { return (((x & 0x0f0f0f0fu) + ((x >> 4) & 0x0f0f0f0fu)) * 0x01010101u) >> 24; }
__device__ __forceinline__ u32 lds_ticket(volatile LDSP u32* p, u32 v) {
  asm volatile("s_mov_b64 exec, 1\n\tds_add_rtn_u32 %0, %1, %0\n\ts_mov_b64 exec, -1\n\ts_waitcnt lgkmcnt(0)"
               : "+v"(v) : "v"((u32)(size_t)p) : "memory");
  return rfl(v);
}
__device__ __forceinline__ u32 min4(u32x4 v) { return min(min(v.x, v.y), min(v.z, v.w)); }
__device__ __forceinline__ u32 gather8(const u32* f) {
  const u32 r0 = __builtin_amdgcn_udot4(f[1], 0x80402010u, __builtin_amdgcn_udot4(f[0], 0x08040201u, 0, false), false);
  const u32 r1 = __builtin_amdgcn_udot4(f[3], 0x80402010u, __builtin_amdgcn_udot4(f[2], 0x08040201u, 0, false), false);
  const u32 r2 = __builtin_amdgcn_udot4(f[5], 0x80402010u, __builtin_amdgcn_udot4(f[4], 0x08040201u, 0, false), false);
  const u32 r3 = __builtin_amdgcn_udot4(f[7], 0x80402010u, __builtin_amdgcn_udot4(f[6], 0x08040201u, 0, false), false);
  return (r0 >> 7) | (r1 << 1) | (r2 << 9) | (r3 << 17);
}

__global__ void __launch_bounds__(NTHREADS) fused_gemm_kernel(Params P) {
  __shared__ __attribute__((aligned(16))) char smem[L_BYTES];
  LDSP char* lds = (LDSP char*)smem;
  volatile LDSP u32* ctl = (volatile LDSP u32*)(lds + L_CTL);
  const int lane = threadIdx.x & 63;
  const int wave = rfl(threadIdx.x >> 6);
  const int b = blockIdx.x, xcd = b & 7, slot = b >> 3;
  const int ct = slot >> 2, mm = slot & 3;
  typedef u32 u32x16 __attribute__((ext_vector_type(16)));
  u32x16 ka;
  asm volatile("s_load_dwordx16 %0, %1, 0x0" : "=&s"(ka) : "s"(__builtin_amdgcn_kernarg_segment_ptr()) : "memory");
  const bool link = (ct & 5) == 0;
  const int i8 = mm + (ct < 4 ? ct >> 1 : ct & 1) * 4;
  const bool xjob = !link & ((wave == 4) | ((wave == 5) & (ct >= 6)));
  const int xj = wave == 5 ? 1 : ct < 4 ? 0 : ct >> 1;
  u32 cno = CNT_OFF + (b * 12 + wave) * 128;
  u32 xv0 = (i8 * 64 + (lane & 15)) * KS + xj * 128 + (lane >> 4) * 16, xv1 = xv0 + 16 * KS, xv2 = xv0 + 32 * KS, xv3 = xv0 + 48 * KS;
  asm volatile("s_waitcnt lgkmcnt(0)" : "+s"(ka), "+v"(cno), "+v"(xv0), "+v"(xv1), "+v"(xv2), "+v"(xv3) :: "memory");
#define KA64(i) (ka[i] | ((u64)ka[(i) + 1] << 32))
  char* const me = (char*)KA64(0);
  const int self = ka[8];
  const int peer = (ka[9] >> (4 * xcd)) & 7;
#define LOAD_PWS() char* pws; asm volatile("s_load_dwordx2 %0, %1, %2\n\ts_waitcnt lgkmcnt(0)" : "=&s"(pws) : "s"(KA64(6)), "s"(peer * 8) : "memory")
  u32* cnt = (u32*)(me + CNT_OFF + (b * 12 + wave) * 128);
  const u32 dump = (u32)(size_t)lds + L_DUMP;
  u32 call;
  const u64 tstart = __builtin_amdgcn_s_memrealtime();

  if (__builtin_expect(xjob, 1)) {
    const int hx = xj * 8 + i8;
    if (xj == 3) __builtin_amdgcn_s_sleep(15);
    u32x4 d[8];
    u32 cv;
    asm volatile("global_load_dword %8, %11, %10 sc1\n\t"
                 "global_load_dwordx4 %0, %12, %9\n\tglobal_load_dwordx4 %1, %12, %9 offset:64\n\t"
                 "global_load_dwordx4 %2, %13, %9\n\tglobal_load_dwordx4 %3, %13, %9 offset:64\n\t"
                 "global_load_dwordx4 %4, %14, %9\n\tglobal_load_dwordx4 %5, %14, %9 offset:64\n\t"
                 "global_load_dwordx4 %6, %15, %9\n\tglobal_load_dwordx4 %7, %15, %9 offset:64"
                 : "=&v"(d[0]), "=&v"(d[1]), "=&v"(d[2]), "=&v"(d[3]), "=&v"(d[4]), "=&v"(d[5]), "=&v"(d[6]), "=&v"(d[7]), "=&v"(cv)
                 : "s"(KA64(2) + peer * (512 * KS)), "s"(me), "v"(cno), "v"(xv0), "v"(xv1), "v"(xv2), "v"(xv3) : "memory");
    LOAD_PWS();
    const bool push = hx < NPUSH, direct = push || peer == self;
    const u32 o0 = push ? PX_OFF + (self * 16 + hx) * ITEM : direct ? RAW_OFF + (8 * NIT + 256 + hx) * ITEM : PK_OFF + (256 + peer * 32 + hx) * ITEM;
    const u32 o1 = push ? NR * 16 * ITEM : direct ? (self - 8) * NIT * ITEM : 512 * ITEM;
    const u32 f0 = direct ? UF_OFF + (self * NIT + 256 + hx) * 512 : DIR_OFF + (self * NIT + 256 + hx) * 128;
    const u32 f1 = direct ? 0 : NR * NIT * 128;
    const char* db = push ? pws : me;
    const u64 fm = direct ? 0xffff : 3;
    asm volatile("s_waitcnt vmcnt(8)" : "+v"(cv) :: "memory");
    call = rfl(cv) + 1;
    const u32 par = call & 1;
    const u32 dv = o0 + par * o1 + 4096 + lane * 16, fv = f0 + par * f1 + (lane >> 2) * 128 + (lane & 3) * 16;
    const u32 v = direct ? call : (call << 8) | M_RAW;
    if (push && hx >= PUSH1) while (__builtin_amdgcn_s_memrealtime() - tstart < PUSHD) __builtin_amdgcn_s_sleep(1);
    asm volatile("s_waitcnt vmcnt(7)\n\tglobal_store_dwordx4 %8, %0, %9 offset:-4096 sc0 sc1\n\t"
                 "s_waitcnt vmcnt(7)\n\tglobal_store_dwordx4 %8, %1, %9 offset:-3072 sc0 sc1\n\t"
                 "s_waitcnt vmcnt(7)\n\tglobal_store_dwordx4 %8, %2, %9 offset:-2048 sc0 sc1\n\t"
                 "s_waitcnt vmcnt(7)\n\tglobal_store_dwordx4 %8, %3, %9 offset:-1024 sc0 sc1\n\t"
                 "s_waitcnt vmcnt(7)\n\tglobal_store_dwordx4 %8, %4, %9 sc0 sc1\n\t"
                 "s_waitcnt vmcnt(7)\n\tglobal_store_dwordx4 %8, %5, %9 offset:1024 sc0 sc1\n\t"
                 "s_waitcnt vmcnt(7)\n\tglobal_store_dwordx4 %8, %6, %9 offset:2048 sc0 sc1\n\t"
                 "s_waitcnt vmcnt(7)\n\tglobal_store_dwordx4 %8, %7, %9 offset:3072 sc0 sc1\n\ts_nop 1"
                 : "+v"(d[0]), "+v"(d[1]), "+v"(d[2]), "+v"(d[3]), "+v"(d[4]), "+v"(d[5]), "+v"(d[6]), "+v"(d[7])
                 : "v"(dv), "s"(db) : "memory");
    asm volatile("s_mov_b64 exec, %3\n\ts_waitcnt vmcnt(0)\n\tglobal_store_dwordx4 %0, %1, %2 sc0 sc1\n\ts_mov_b64 exec, -1\n\ts_nop 0"
                 :: "v"(fv), "v"(u32x4{v, v, v, v}), "s"(pws), "s"(fm) : "memory");
    bar();
  } else if (!link && (wave < 4 || (wave >= 8 && ct < 4))) {
    {
      const int g = lane >> 4, c = lane & 15, j2 = wave >> 3, jw = ct < 4 ? 2 + j2 : (ct - 4) >> 1;
      const int u = wave & 1, hf = (wave >> 1) & 1, blk = (jw * 32 + xcd * 4) * 2 + i8;
      const GLB unsigned char* src = (const GLB unsigned char*)KA64(4) + (size_t)(jw * 128 + u * 64 + g * 16) * NN + xcd * 512 + i8 * 64 + c * 4;
      u32 W[16], D[8];
      if (jw == 1) __builtin_amdgcn_s_sleep(30);
      if (jw >= 2) __builtin_amdgcn_s_sleep(60);
#pragma unroll
      for (int r = 0; r < 16; ++r) W[r] = *(const GLB u32*)(src + r * NN);
      const u32 cv = ld_agent(cnt);
      LDSP char* scr = lds + L_SCR + (wave - j2 * 4) * 512;
      *(LDSP u32x2*)(scr + lane * 8) = u32x2{0, 0};
#pragma unroll
      for (int s = 0; s < 8; ++s) LDS_ADD(scr + (((W[s >> 2] >> ((s & 3) * 8)) & 0x7f) << 2), 1u);
      const u32x2 hh = *(volatile LDSP u32x2*)(scr + lane * 8);
      *(volatile LDSP u32x2*)(scr + lane * 8) = u32x2{0, 0};
      const u32 sel = 0x05010400u + hf * 0x02020202u;
#pragma unroll
      for (int q = 0; q < 4; ++q) {
        const u32 t0 = __builtin_amdgcn_perm(W[4 * q + 1], W[4 * q], sel), t2 = __builtin_amdgcn_perm(W[4 * q + 3], W[4 * q + 2], sel);
        D[q] = __builtin_amdgcn_perm(t2, t0, 0x05040100u); D[4 + q] = __builtin_amdgcn_perm(t2, t0, 0x07060302u);
      }
      call = rfl(cv) + 1;
      const u32 par = call & 1;
      char* pk = me + PK_OFF + (par * 512 + blk) * ITEM;
      {
        char* d = me + RAW_OFF + ((par ? self : 8) * NIT + blk) * ITEM + u * 4096 + hf * 2048 + lane * 16;
        st16<0>(d, u32x4{D[0], D[1], D[2], D[3]}); st16<1024>(d, u32x4{D[4], D[5], D[6], D[7]});
        u32 s0 = 0;
#pragma unroll
        for (int k = 0; k < 8; ++k) s0 |= (D[k] & 0x80808080u) >> k;
        st4(pk + 4096 + u * 512 + hf * 256 + lane * 4, s0);
      }
      u32 base;
      {
        const u32 c0 = scan_add(hh.x + hh.y) - hh.x - hh.y, c1 = c0 + hh.x;
        u32 u8 = (u32)__builtin_amdgcn_ds_bpermute(((lane + 8) & 63) * 4, (int)(c0 | (c1 << 16)));
        u32 u9 = (u32)__builtin_amdgcn_ds_bpermute(((lane + 9) & 63) * 4, (int)(c0 | (c1 << 16)));
        if (lane >= 56) u8 = 0x02000200u;
        if (lane >= 55) u9 = 0x02000200u;
        const u32 cs = (c0 | (c1 << 16)) + 0x08000800u - u8 - 3 * u9;
        const u32 k0 = ((cs & 0xffff) << 7) | (2 * lane), k1 = ((cs >> 16) << 7) | (2 * lane + 1);
        base = __builtin_amdgcn_readlane(scan_min(lane < 56 ? min(k0, k1) : lane == 56 ? k0 : 0xffffffffu), 63) & 0x7f;
      }
      const u32 b8f = base * 0x01010101u + 0x8f8f8f8fu;
      u32 F1[8], F2[8], FE[8], T[8];
#pragma unroll
      for (int k = 0; k < 8; ++k) {
        const u32 t = T[k] = b8f - (D[k] & 0x7f7f7f7fu);
        const u32 t1 = t << 1, t2 = t << 2, t3 = t << 3, wr = ((t & 0x0f0f0f0fu) + 0x02020202u) << 3;
        const u32 h = t & ~t1 & 0x80808080u, k1 = h & ~t2 & t3, k2 = h & t2 & ~t3, km = ~t & t1 & t2 & t3 & 0x80808080u;
        F2[k] = (wr & k1) | (~wr & k2);
        F1[k] = k1 | (wr & km) | (~wr & k2);
        FE[k] = ((h & ~t2 & ~t3) | F1[k]) ^ 0x80808080u;
      }
      {
        u32x4 cd;
#pragma unroll
        for (int q = 0; q < 4; ++q) cd[q] = ~((T[q] & 0x0f0f0f0fu) | ((T[4 + q] << 4) & 0xf0f0f0f0u));
        st16<0>(pk + u * 2048 + hf * 1024 + lane * 16, cd);
      }
      u32 x1 = gather8(F1), x2 = gather8(F2), xe = gather8(FE);
      const u64 any = __builtin_amdgcn_ballot_w64(x1 != 0);
      int prev;
      {
        const u32 lastp = x1 ? lane * 32 + 31 - __builtin_clz(x1) : 0;
        const u32 back = lane * 32 - scan_max(DPP(0, lastp, 0x138, 0xf));
        prev = -(int)(back - 15 * ((back * 4370) >> 16));
      }
      u32 a0 = 0, a1 = 0, a2 = 0, nc = 0;
#define PUSH_NIB(gg) do { a0 = __builtin_amdgcn_alignbit(a1, a0, 4); a1 = __builtin_amdgcn_alignbit(a2, a1, 4); \
        a2 = __builtin_amdgcn_alignbit((gg), a2, 4); ++nc; } while (0)
      while (x1 != 0) {
        const u32 pos = __builtin_ctz(x1);
        const u32 gap = pos - prev, gg = min(gap, 15u);
        prev += gg;
        const u32 bit = gap < 15 ? 1u << pos : 0u, in2 = x2 & bit;
        x2 ^= in2;
        x1 ^= bit ^ in2;
        PUSH_NIB(gg);
      }
      if ((any >> lane) > 1) while (prev <= 17) { prev += 15; PUSH_NIB(15u); }
      const u32 nend = scan_add(nc), nn = __builtin_amdgcn_readlane(nend, 63);
      bool over = nn > 512 || __builtin_amdgcn_ballot_w64(nc > 24) != 0;
      if (!over) {
        const u32 sh = (8 - (nend & 7)) * 4;
        LDSP char* q = scr + (nend >> 3) * 4;
        LDS_OR(q, (u32)((((u64)a0) << 32) >> sh));
        LDS_OR(q + 4, (u32)((((u64)a1 << 32) | a0) >> sh));
        LDS_OR(q + 8, (u32)((((u64)a2 << 32) | a1) >> sh));
        LDS_OR(q + 12, (u32)(((u64)a2) >> sh));
      }
      u32 ne = 0;
      {
        LDSP unsigned short* lst = (LDSP unsigned short*)(scr + 384);
        u64 mk;
        while ((mk = __builtin_amdgcn_ballot_w64(xe != 0)) != 0 && ne <= 63) {
          if (xe != 0) {
            const u32 pos = __builtin_ctz(xe);
            const u32 wa = (pos & 4) ? T[1] : T[0], wb = (pos & 4) ? T[3] : T[2], wc = (pos & 4) ? T[5] : T[4], wd = (pos & 4) ? T[7] : T[6];
            const u32 we = (pos & 8) ? wb : wa, wf = (pos & 8) ? wd : wc, wg = (pos & 16) ? wf : we;
            const u32 x4 = (wg >> ((pos & 3) * 8 + 4)) & 15;
            const u32 at = ne + __builtin_amdgcn_mbcnt_hi((u32)(mk >> 32), __builtin_amdgcn_mbcnt_lo((u32)mk, 0));
            if (at < 64) lst[at] = (unsigned short)((x4 << 12) | (hf * 2048 + lane * 32 + pos));
            xe &= xe - 1;
          }
          ne += __builtin_popcountll(mk);
        }
      }
      over = over || ne > 63;
      bar();
      if (!over) {
        const u32 ed = (ne + 1) >> 1, nd = (nn + 7) >> 3;
        const u32 ev = *(volatile LDSP u32*)(scr + 384 + (lane & 31) * 4);
        u32 nv = *(volatile LDSP u32*)(scr + 12 + lane * 4);
        const u32 vo = (lds_ticket(ctl + C_ENC + j2, ed + nd) & 0xffff) + 4;
        if (lane == (nn >> 3)) nv |= ~0u << ((nn & 7) * 4);
        if (lane < nd) st4(pk + FIX + (vo + lane) * 4, nv);
        if (lane < ed) st4(pk + FIX + (vo + nd + lane) * 4, ev);
        if (lane == 0) st4(pk + FIX + (u * 2 + hf) * 4, vo | (base << 10) | (ne << 17) | (nd << 23));
      }
      u32x16 pw;
      asm volatile("s_load_dwordx16 %0, %1, 0x0\n\ts_waitcnt vmcnt(0) lgkmcnt(0)" : "=&s"(pw) : "s"(KA64(6)) : "memory");
#define PW(i) "s"(pw[2 * (i)] | ((u64)pw[2 * (i) + 1] << 32))
      const u32 k = lds_ticket(ctl + C_ENC + j2, over ? 0x1010000u : 0x1000000u);
      if ((k >> 24) == 3) {
        const u32 v = (call << 8) | ((over || (k & 0xff0000) != 0) ? M_RAW : ((k & 0xffff) + 4 + 31) >> 5);
        asm volatile("s_mov_b64 exec, 3\n\t"
                     "global_store_dwordx4 %0, %1, %2 sc0 sc1\n\tglobal_store_dwordx4 %0, %1, %3 sc0 sc1\n\t"
                     "global_store_dwordx4 %0, %1, %4 sc0 sc1\n\tglobal_store_dwordx4 %0, %1, %5 sc0 sc1\n\t"
                     "global_store_dwordx4 %0, %1, %6 sc0 sc1\n\tglobal_store_dwordx4 %0, %1, %7 sc0 sc1\n\t"
                     "global_store_dwordx4 %0, %1, %8 sc0 sc1\n\tglobal_store_dwordx4 %0, %1, %9 sc0 sc1\n\t"
                     "s_mov_b64 exec, -1\n\ts_nop 0"
                     :: "v"(DIR_OFF + ((par * NR + self) * NIT + blk) * 128 + lane * 16), "v"(u32x4{v, v, v, v}),
                        PW(0), PW(1), PW(2), PW(3), PW(4), PW(5), PW(6), PW(7) : "memory");
        if (lane < 16) st16<0>(me + UF_OFF + (self * NIT + blk) * 512 + (lane >> 2) * 128 + (lane & 3) * 16, u32x4{call, call, call, call});
      }
    }
  } else {
    if (link && ((wave >= 2 * NGRP && wave < 9) || (peer == self && !(slot == 0 && wave == 0)))) return;
    const u32 cv = ld_agent(cnt);
    if (wave == 7) { ctl[lane] = 0; waitl(); }
    bar();
    call = rfl(cv) + 1;
  }
  if (lane == 0 && wave != 11) st_agent(cnt, call);

  if (link) {
    if (peer == self) {
      const char* pf = me + PF_OFF + lane * 32;
      u32 f;
      bool all;
      do {
        __builtin_amdgcn_s_sleep(16);
        f = ld_agent(pf);
        all = __builtin_amdgcn_ballot_w64((f & 0xfff) != (((call & 63) << 6) | 1) && (lane >> 3) != xcd) == 0;
      } while (!all && __builtin_amdgcn_s_memrealtime() - tstart < PLANT);
      float v = (lane >> 3) == xcd ? 0.f : (float)((int)(f - ((u32)tstart << 12)) >> 12);
      v += __shfl_xor(v, 1); v += __shfl_xor(v, 2); v += __shfl_xor(v, 4);
      v *= 0.125f;
      float mu = v + __shfl_xor(v, 8);
      mu += __shfl_xor(mu, 16); mu += __shfl_xor(mu, 32);
      mu *= 1.f / 7;
      u32 inv = 0;
#pragma unroll
      for (int x = 0; x < NR; ++x) inv |= x << (4 * ((ka[9] >> (4 * x)) & 7));
      const int ps = lane >> 3, pr = lane & 7;
      const float fs = __shfl(v, ((inv >> (4 * ps)) & 7) * 8), fr = __shfl(v, ((inv >> (4 * pr)) & 7) * 8);
      const float ex = ps == self || fs - mu < DZ ? 0.f : fs - mu, gp = pr == self ? 0.f : fmaxf(mu - fr, 0.f);
      float gs = gp + __shfl_xor(gp, 1);
      gs += __shfl_xor(gs, 2); gs += __shfl_xor(gs, 4);
      u32 row = all && gs > 0.f ? (u32)fminf(8.f, ex * gp / gs * (8.f / TITEM) + 0.5f) << (4 * pr) : 0;
      row |= __shfl_xor(row, 1); row |= __shfl_xor(row, 2); row |= __shfl_xor(row, 4);
      row = __shfl(row, pr * 8);
      __hip_atomic_store((GLB u64*)(me + PLAN_OFF + ps * 128 + pr * 8), row | ((u64)call << 32), __ATOMIC_RELAXED, __HIP_MEMORY_SCOPE_AGENT);
      return;
    }
    if (wave == 11) {
      LOAD_PWS();
      const u32 par = call & 1;
      const u32 cw = mm + ct * 2;
      u32 nitems = (NIT - NPUSH - cw + 7) >> 3;
      u32 itv = lane < nitems ? sched_item(cw + lane * 8, xcd, self) : 0;
      const char* dir = me + DIR_OFF + ((par * NR + peer) * NIT + itv) * 128;
      u32 dv = 0;
      u64 todo = (1ull << nitems) - 1, ok = 0;
      bool have = false;
      while (__builtin_amdgcn_s_memrealtime() - tstart < GATE - 50) __builtin_amdgcn_s_sleep(1);
      for (u32 n = 0; n < nitems + NGRP; ++n) {
        u32 w0 = (IT_END << 6) | 0x80000;
        if (n < nitems) {
          const u32 first = __builtin_ctzll(todo);
          const u64 cand = todo & (((n < 8 ? 7ull : 1ull) << first) | (0xffull << 56));
          const bool rl = (todo >> 56) != 0 && ((ok & todo) >> 56) == 0;
          for (u32 sp = 1; (ok & cand) == 0 || (rl && sp == 1); ++sp) {
            if (sp > 1 && n > 2) __builtin_amdgcn_s_sleep(4);
            if (lane - first < (n < 8 ? 4u : 64u) && ((todo & ~ok) >> lane) & 1) {
              dv = ld_sys(dir);
              if (lane >= 56) {
                const char* rf = pws + RLF_OFF + ((lane - 56) * 8 + ((itv >> 3) & 7)) * 128;
                if (min(ld_sys(rf), ld_sys(rf + 32)) < call) dv = 0;
              }
            }
            ok = __builtin_amdgcn_ballot_w64((dv >> 8) == (call & 0xffffff));
            if ((sp & 31) == 0 && give_up(me, call, tstart)) { dv = (call << 8) | M_RAW; ok = ~0ull; }
          }
          const u64 pick = ok & cand;
          const u32 ln = (pick >> 56) != 0 ? 63 - __builtin_clzll(pick) : __builtin_ctzll(pick);
          todo &= ~(1ull << ln);
          const u32 it = __builtin_amdgcn_readlane(itv, ln);
          u32 e = __builtin_amdgcn_readlane(dv, ln);
          e &= (e & M_RAW) ? M_RAW : 0xf;
          const u32 so = ln >= 56 ? RL_OFF / ITEM + (par * NR + ln - 56) * 8 + ((it >> 3) & 7)
                         : it >= 256 ? par * 512 + self * 32 + it : (e & M_RAW) ? RAW_OFF / ITEM + (par ? peer : 8) * NIT + it : par * 512 + it;
          w0 = (it << 6) | ((e & M_RAW) ? 0x80000 : e << 15) | (so << 20);
        }
        if (n >= LA) {
          volatile LDSP u32x2* cp = (volatile LDSP u32x2*)(ctl + C_CONS + (n % NGRP) * 2);
          for (;;) { const u32x2 cv = *cp; if (rfl(min(cv.x, cv.y)) + LA > n) break; __builtin_amdgcn_s_sleep(2); }
        }
        if (lane == 0) ctl[C_DESC + n % NS] = w0 | (n + 1);
        if (!have && n >= 17 && (n & 1)) {
          u64 pv;
          for (u32 sp = 1;; ++sp) {
            pv = __hip_atomic_load((const GLB u64*)(me + PLAN_OFF + xcd * 128 + (lane & 7) * 8), __ATOMIC_RELAXED, __HIP_MEMORY_SCOPE_AGENT);
            have = __builtin_amdgcn_ballot_w64((u32)(pv >> 32) != call) == 0;
            if (have || n < 27) break;
            if ((sp & 31) == 0 && give_up(me, call, tstart)) { pv = 0; have = true; break; }
          }
          if (have) {
            const u32 row = (u32)pv, mine = __builtin_amdgcn_readlane(row, peer);
            const u32 r = ((itv >> 1) & 3) * 2 + (itv & 1), col = (itv >> 3) & 7;
            const u64 skip = __builtin_amdgcn_ballot_w64((itv >> 6) == 2 && ((col - nib_sum(mine & ((1u << (4 * r)) - 1))) & 7) < ((mine >> (4 * r)) & 15));
            const u32 nt = lane >= 56 ? (row >> (4 * peer)) & 15 : 0, kk = (cw - (scan_add(nt) - nt)) & 7;
            const u64 take = __builtin_amdgcn_ballot_w64(kk < nt);
            if (lane >= 56) {
              itv = round_item(16 + peer, (nib_sum(row & ((1u << (4 * peer)) - 1)) + kk) & 7);
              dir = me + DIR_OFF + ((par * NR + lane - 56) * NIT + itv) * 128;
            }
            todo = (todo & ~skip) | take;
            nitems = n + 1 + __builtin_popcountll(todo);
            if (lane == 0) ctl[C_NIT] = nitems;
          }
        }
      }
      if (lane == 0) st_agent(cnt, call);
      return;
    }
    if (wave == 9) {
      const u32 cw = mm + ct * 2, n0 = (NIT - NPUSH - cw + 7) >> 3;
      char* bp = me + BAL_OFF + xcd * 256;
      u32 until = 4 * K0;
      for (u32 r = 0;; ++r) {
        u32 p0, p1;
        while (((p0 = rfl(ctl[C_LANDED])) & 0xffff) < until) __builtin_amdgcn_s_sleep(4);
        for (int i = 0; (p1 = rfl(ctl[C_LANDED])) == p0 && i < 12; ++i) {}
        const u32 now = (u32)__builtin_amdgcn_s_memrealtime(), nn = rfl(ctl[C_NIT]), nitems = nn ? nn : n0, hu = p1 & 0xffff;
        const u32 fown = (now + ((nitems * 23 - hu * 4 - (p1 >> 16) * 7) * TITEM) / 23) << 12 | ((call & 63) << 6) | (nn ? 2 : 1);
        if (lane == 0) st_agent(bp + cw * 32, fown);
        if (r == 0) {
          if (lane == 0) st_agent(bp + PF_OFF - BAL_OFF + cw * 32, fown);
          until = 64;
          continue;
        }
        if (r == 1) { until = min(hu + 6, 4 * (n0 - 8)); continue; }
        if (hu + 12 >= 4 * nitems) break;
        const u32 f = ld_agent(bp + (lane & 7) * 32);
        const bool v = ((f ^ fown) & 0xfff) == 0;
        const int cnt8 = __builtin_popcount((u32)__builtin_amdgcn_ballot_w64(v) & 0xff);
        const int late = cnt8 < 6 ? 0 : (int)__builtin_amdgcn_readlane(scan_add(v ? (u32)((int)(fown - f) >> 12) : 0u), 7) / cnt8;
        const u32 plan = min((u32)(late > 0 ? (late + GFAST / 4) / GFAST : (GSLOW / 4 - late) / GSLOW), 4u);
        until = (rfl(ctl[C_LANDED]) & 0xffff) + 4 * plan;
        if (plan != 0) {
          if (lane == 0) ctl[C_WIN] = late > 0 ? 1 : 2;
          until = min(until, 4 * (nn ? nn : n0 - 8));
          while ((rfl(ctl[C_LANDED]) & 0xffff) < until) __builtin_amdgcn_s_sleep(4);
          if (lane == 0) ctl[C_WIN] = 0;
        }
        until = min(until + 12, 4 * (nn ? nn : n0 - 8));
      }
      return;
    }
    if (wave == 10) {
      LOAD_PWS();
      if (lane < 3) touch(pws + PK_OFF + ((call & 1) * 512 + (lane == 2 ? 256 + self * 32 + 16 : lane * 255)) * ITEM + (self * 8 + mm + ct * 2) * 128, dump);
      waitv<0>();
      drain();
      const u32 l0 = (u32)(size_t)lds, lw = l0 + L_CTL + C_LANDED * 4, one = lane == 0 ? 1 : 0;
      u32 prog = 0, mk = 0, wm = 0, la = lane == WD ? lw : dump + lane * 4;
      __builtin_amdgcn_s_setprio(3);
#define MODE_ "s_cmp_eq_u32 %[wm], 1\n\ts_cselect_b64 vcc, -1, 0\n\ts_cmp_eq_u32 %[wm], 2\n\t"
#define LD_(off, id) "s_waitcnt vmcnt(%[wf])\n\ts_cbranch_vccnz .Lw" #id "_%=\n\ts_waitcnt vmcnt(%[w])\n\ts_cbranch_scc0 .Lw" #id "_%=\n\t" \
                     "s_waitcnt vmcnt(%[ws])\n.Lw" #id "_%=:\n\tglobal_load_lds_dwordx4 %[va], %[sb] offset:" #off " sc0 sc1\n\t"
#define SH_ "v_mov_b32_dpp %[mk], %[mk] row_shr:1 row_mask:0xf bank_mask:0xf\n\t"
#define PUB_ "ds_max_u32 %[la], %[mk]\n\t"
#define INC_(x) "v_add_u32 %[mk], %[mk], " x "\n\t"
#define WIN_ [wm] "s"(wm), [wf] "i"(WFAST - 1), [w] "i"(WD - 1), [ws] "i"(WD - 2)
#define FOUR(src, ldst, i0) do { u32 keep_; \
          asm volatile("ds_read_b32 %[dv], %[da]\n\tds_read_b32 %[wv], %[lw] offset:4\n\ts_mov_b32 %[k], m0\n\ts_mov_b32 m0, %[ld]\n\t" MODE_ \
                       LD_(0, 0) SH_ INC_("%[i]") PUB_ LD_(1024, 1) SH_ INC_("%[o]") PUB_ LD_(2048, 2) SH_ INC_("%[i]") PUB_ LD_(3072, 3) SH_ INC_("%[o]") \
                       "s_waitcnt lgkmcnt(3)\n\t" PUB_ "s_mov_b32 m0, %[k]" \
                       : [k] "=&s"(keep_), [dv] "=&v"(dvv), [wv] "=&v"(wvv), [mk] "+v"(mk) \
                       : [va] "v"((src) + lane * 16), [sb] "s"(pws), [ld] "s"(ldst), [la] "v"(la), [da] "v"(l0 + L_CTL + (C_DESC + se) * 4), \
                         [lw] "v"(lw), [i] "v"((i0) & one), [o] "v"(one), WIN_ : "memory", "scc", "vcc"); \
          prog += 2 + 2 * (i0); } while (0)
      u64 qu = 0;
      u32 ne = 0, se = 0, sc = 0, d = 0, dvv, wvv;
      bool fin = false;
      for (u32 n = 0;; ++n) {
#pragma clang loop unroll(disable)
        while (ne <= n + EA && !fin) {
          if ((d & 63) != ne + 1) {
            if (ne > n) break;
            d = rfl(ctl[C_DESC + se]);
            if ((d & 63) != ne + 1) {
              waitv<0>();
              if (lane == 0) ctl[C_LANDED] = prog;
              do { d = rfl(ctl[C_DESC + se]); } while ((d & 63) != ne + 1);
            }
          }
          fin = ((d >> 6) & 0x1ff) == IT_END;
          qu |= (u64)((d >> 20) | ((d >> 7) & 0x1000) | (fin ? 0x2000u : 0u)) << (16 * (ne - n));
          ++ne;
          if (!(d & 0x80000)) {
            const u32 ln = (d >> 15) & 15, pn = (prog & 0xffff) | (ne << 16);
            const u64 em = (ln & 7) ? (1ull << ((ln & 7) * 8)) - 1 : ~0ull;
            u32 keep;
            asm volatile("s_mov_b32 %[k], m0\n\ts_mov_b32 m0, %[ld]\n\t" MODE_
                         LD_(0, 0) SH_ PUB_
                         "s_cmp_gt_u32 %[ln], 8\n\ts_cbranch_scc1 .L3_%=\n\ts_cmp_eq_u32 %[wm], 2\n\ts_mov_b64 exec, %[em]\n\t" LD_(1024, 1) "s_branch .L4_%=\n"
                         ".L3_%=:\n\ts_cmp_eq_u32 %[wm], 2\n\t" LD_(1024, 2) SH_ PUB_ "s_mov_b64 exec, %[em]\n\t" LD_(2048, 3)
                         "\n.L4_%=:\n\ts_mov_b64 exec, -1\n\t" SH_ "v_writelane_b32 %[mk], %[pn], 0\n\t" PUB_ "s_mov_b32 m0, %[k]"
                         : [k] "=&s"(keep), [mk] "+v"(mk)
                         : [va] "v"((d >> 20) * ITEM + 4096 + lane * 16), [sb] "s"(pws), [ld] "s"(l0 + se * ITEM + 4096), [la] "v"(la),
                           [ln] "s"(ln), [em] "s"(em), [pn] "s"(pn), WIN_ : "memory", "scc", "vcc");
            prog = pn;
          }
          se = se + 1 == NS ? 0 : se + 1;
          if (n == 0) break;
          if (ne <= n + EA) d = rfl(ctl[C_DESC + se]);
        }
        const u32 cur = (u32)qu & 0xffff;
        qu >>= 16;
        if (cur & 0x2000) break;
        u32 so = (cur & 0xfff) * ITEM, lo = l0 + sc * ITEM;
#pragma clang loop unroll(disable)
        for (u32 k = cur >> 12;; --k) {
          FOUR(so, lo, 1 - (cur >> 12));
          if (k == 0) break;
          so += 4096; lo += 4096;
        }
        d = rfl(dvv);
        wm = rfl(wvv);
        la = lane == (wm == 1 ? WFAST : wm == 2 ? WD - 1 : WD) ? lw : dump + lane * 4;
        sc = sc + 1 == NS ? 0 : sc + 1;
      }
#define DRAIN(N) if (N < WFAST) \
        asm volatile("s_mov_b64 exec, %2\n\ts_waitcnt vmcnt(%3)\n\tds_max_u32 %0, %1\n\ts_mov_b64 exec, -1" \
                     :: "v"(lw), "v"(mk), "i"(1 << N), "i"(N) : "memory");
      DRAIN(9) DRAIN(8) DRAIN(7) DRAIN(6) DRAIN(5) DRAIN(4) DRAIN(3) DRAIN(2) DRAIN(1) DRAIN(0)
      return;
    }
    if (wave == 0 && lane < 4) touch(me + (lane == 3 ? UF_OFF + peer * NIT * 512 : RAW_OFF + (peer * NIT + lane * 143) * ITEM) + slot * 128, dump);
    drain();
    const int u = wave & 1, dw = wave;
    LDSP char* img = lds + L_IMG + wave * 4096;
    u32 m0f = 0x0f0f0f0fu, m80 = 0x80808080u, m3e = 0x3e3e3e3eu, m30 = 0x30303030u;
    u32 mffc = 0xffc;
    const u32 imga = (u32)(size_t)img;
    asm volatile("" : "+s"(m0f), "+s"(m80), "+s"(m3e), "+s"(m30), "+v"(mffc));
    for (u32 n = dw >> 1;; n += NGRP) {
      u32 w0;
      while (((w0 = rfl(ctl[C_DESC + n % NS])) & 63) != n + 1) __builtin_amdgcn_s_sleep(2);
      const u32 it = (w0 >> 6) & 0x1ff;
      if (it == IT_END) break;
      LDSP char* sl = lds + (n % NS) * ITEM;
      const u32 rlo = (w0 >> 20) - RL_OFF / ITEM;
      const u32 src = rlo < 128 ? (rlo >> 3) & 7 : peer;
      const bool keep = (it >> 6) == 2 && ((it >> 1) & 3) * 2 + (it & 1) == self;
      char* d = me + RAW_OFF + (src * NIT + it) * ITEM + u * 4096 + lane * 16;
      u32x4 o0, o1, o2, o3;
#define LANDED(m, x) do { while ((rfl(ctl[C_LANDED]) & (m)) < (x)) __builtin_amdgcn_s_sleep(1); fence(); } while (0)
      if (w0 & 0x80000) {
        LANDED(0xffffu, 4 * n + 2 * u + 2);
        LDSP char* s = sl + u * 4096 + lane * 16;
        o0 = *(LDSP u32x4*)s; o1 = *(LDSP u32x4*)(s + 1024); o2 = *(LDSP u32x4*)(s + 2048); o3 = *(LDSP u32x4*)(s + 3072);
        st16<0>(d, o0); st16<1024>(d, o1); st16<2048>(d, o2); st16<3072>(d, o3);
      } else {
        LANDED(0xffff0000u, (n + 1) << 16);
        const u32x2 hv = *(LDSP u32x2*)(sl + FIX + u * 8);
        const u32 h0 = rfl(hv.x), h1 = rfl(hv.y);
        const u32 nd0 = h0 >> 23, nd1 = h1 >> 23;
        const u32 b0 = ((h0 >> 10) & 0x7f) * 0x01010101u - 0x3e3e3e3eu, b1 = ((h1 >> 10) & 0x7f) * 0x01010101u - 0x3e3e3e3eu;
        LDSP char* v0 = sl + FIX + (h0 & 0x1ff) * 4;
        LDSP char* v1 = sl + FIX + (h1 & 0x1ff) * 4;
        u32 nb0 = *(LDSP u32*)(v0 + lane * 4), nb1 = *(LDSP u32*)(v1 + lane * 4);
        const u32 e0 = *(LDSP unsigned short*)(v0 + (nd0 & 0x7f) * 4 + lane * 2), e1 = *(LDSP unsigned short*)(v1 + (nd1 & 0x7f) * 4 + lane * 2);
        const u32 sg0 = *(LDSP u32*)(sl + 4096 + u * 512 + lane * 4), sg1 = *(LDSP u32*)(sl + 4096 + u * 512 + 256 + lane * 4);
        {
          LDSP char* iz = img + lane * 16;
          *(LDSP u32x4*)iz = u32x4{b0, b0, b0, b0}; *(LDSP u32x4*)(iz + 1024) = u32x4{b0, b0, b0, b0};
          *(LDSP u32x4*)(iz + 2048) = u32x4{b1, b1, b1, b1}; *(LDSP u32x4*)(iz + 3072) = u32x4{b1, b1, b1, b1};
        }
        if (lane >= nd0) nb0 = ~0u;
        if (lane >= nd1) nb1 = ~0u;
        u32 gg[4], vv[4];
        gg[0] = (nb0 & m0f) << 3; gg[1] = (nb0 >> 1) & 0x78787878u; gg[2] = (nb1 & m0f) << 3; gg[3] = (nb1 >> 1) & 0x78787878u;
        const u32 sum = __builtin_amdgcn_sad_u8(gg[0], 0, __builtin_amdgcn_sad_u8(gg[1], 0, 0)) |
                        (__builtin_amdgcn_sad_u8(gg[2], 0, __builtin_amdgcn_sad_u8(gg[3], 0, 0)) << 16);
#pragma unroll
        for (int k = 0; k < 4; ++k) {
          vv[k] = (~(gg[k] + 0x08080808u) & m80) >> 3;
          asm("" : "+v"(gg[k]), "+v"(vv[k]));
        }
        const u32 ps = scan_add(sum) - sum;
        u32 pa = ps & 0xffff, pb = (ps >> 16) + 2048 * 8;
        if (lane < max(nd0, nd1)) {
#pragma unroll
          for (int k = 0; k < 8; ++k) {
            const int w = k & 1, by = (k >> 1) * 8;
            pa += (gg[w] >> by) & 0xff;
            pb += (gg[2 + w] >> by) & 0xff;
            LDS_SUB((LDSP char*)(size_t)(((pa >> 3) & mffc) | imga), ((vv[w] >> by) & 0xff) << (pa & 31));
            LDS_SUB((LDSP char*)(size_t)(((pb >> 3) & mffc) | imga), ((vv[2 + w] >> by) & 0xff) << (pb & 31));
          }
        }
        fence();
        LDSP char* ip = img + lane * 32;
        u32x4 a0 = *(LDSP u32x4*)ip, a1 = *(LDSP u32x4*)(ip + 16), a2 = *(LDSP u32x4*)(ip + 2048), a3 = *(LDSP u32x4*)(ip + 2064);
        fence();
        if (lane < ((h0 >> 17) & 63)) LDS_SUB(img + (e0 & 0xffc), ((e0 >> 12) - 8) << ((e0 & 3) * 8 + 4));
        if (lane < ((h1 >> 17) & 63)) LDS_SUB(img + (e1 & 0xffc), ((e1 >> 12) - 8) << ((e1 & 3) * 8 + 4));
        fence();
        o0 = *(LDSP u32x4*)ip; o1 = *(LDSP u32x4*)(ip + 16); o2 = *(LDSP u32x4*)(ip + 2048); o3 = *(LDSP u32x4*)(ip + 2064);
#pragma unroll
        for (int q = 0; q < 4; ++q) {
          o0[q] += (sg0 << q) & m80; o1[q] += (sg0 << (4 + q)) & m80;
          o2[q] += (sg1 << q) & m80; o3[q] += (sg1 << (4 + q)) & m80;
          const u32 l0 = b0 - a0[q], l1 = b0 - a1[q], l2 = b1 - a2[q], l3 = b1 - a3[q];
          a0[q] = (l0 << 1) | (l0 >> 1); a1[q] = (l1 << 1) | (l1 >> 1); a2[q] = (l2 << 1) | (l2 >> 1); a3[q] = (l3 << 1) | (l3 >> 1);
        }
        asm volatile("" : "+v"(o0), "+v"(o1), "+v"(o2), "+v"(o3), "+v"(a0), "+v"(a1), "+v"(a2), "+v"(a3));
#define CODES(o, a, x) _Pragma("unroll") for (int q = 0; q < 4; ++q) { \
          u32 c_ = ((x) & m0f) + m3e; asm("" : "+v"(c_)); o[q] += c_ + (c_ & a[q] & m30); }
        LANDED(0xffffu, 4 * n + 2 * u + 1);
        const u32x4 n0 = *(LDSP u32x4*)(sl + u * 2048 + lane * 16);
        CODES(o0, a0, n0[q]) st16<0>(d, o0);
        CODES(o1, a1, n0[q] >> 4) st16<1024>(d, o1);
        LANDED(0xffffu, 4 * n + 2 * u + 2);
        const u32x4 n1 = *(LDSP u32x4*)(sl + u * 2048 + 1024 + lane * 16);
        CODES(o2, a2, n1[q]) st16<2048>(d, o2);
        CODES(o3, a3, n1[q] >> 4) st16<3072>(d, o3);
      }
      if (keep) {
        LANDED(0xffffu, 4 * n + 4);
        LDSP char* s = sl + u * 4096 + lane * 16;
        char* r = me + RL_OFF + (((call & 1) * NR + peer) * 8 + ((it >> 3) & 7)) * ITEM + u * 4096 + lane * 16;
        st16<0>(r, *(LDSP u32x4*)s); st16<1024>(r, *(LDSP u32x4*)(s + 1024));
        st16<2048>(r, *(LDSP u32x4*)(s + 2048)); st16<3072>(r, *(LDSP u32x4*)(s + 3072));
      }
      if (lane == 0) ctl[C_CONS + dw] = n + 1;
      waitv<0>();
      if (lane < 8) st16<0>(me + UF_OFF + (src * NIT + it) * 512 + u * 32 + (lane >> 1) * 128 + (lane & 1) * 16, u32x4{call, call, call, call});
      if (keep) put_flag(me + RLF_OFF + (peer * 8 + ((it >> 3) & 7)) * 128 + u * 32, call, lane);
    }
    return;
  }

  const int t = xcd * 4 + (ct >> 1);
  const int dbl = 1 - (ct >> 2);

  if (wave >= 4) {
    drain();
    const int src = wave - 4;
    if (lane == 0 && wave == 11) st_agent(cnt, call);
    const char* raw = me + RAW_OFF + (src != self || (call & 1) ? src : 8) * NIT * ITEM + lane * 16;
    const char* uf = me + UF_OFF + src * NIT * 512;
    const char* px = me + PX_OFF + ((call & 1) * NR + src) * 16 * ITEM + lane * 16;
    const u32 lA = (u32)(size_t)lds + L_A + src * 16384;
    const u64 t0 = tstart;
    u32 need = call;
    u32 prev = 0;
    const char* fa = uf + ((128 + ((lane >> 2) & 3) * 4 + mm) * 2 + ((lane >> 1) & 1)) * 512 + (xcd & 3) * 128 + (lane & 1) * 32;
    u64 xmiss = 0xffff;
    u32 xf = 0;
#pragma clang loop unroll(disable)
    for (int j = 0; j < 4; ++j) {
      {
        for (u32 it = 1; ((xmiss >> (4 * j)) & 15) != 0; ++it) {
          if (it > 1) __builtin_amdgcn_s_sleep(8);
          if ((xmiss >> lane) & 1) xf = ld_sys(fa);
          xmiss = __builtin_amdgcn_ballot_w64(xf < need) & 0xffff;
          if ((it & 15) == 0 && give_up(me, call, t0)) need = 0;
        }
        while (rfl(min4(*(volatile LDSP u32x4*)(ctl + C_FREE))) < prev) __builtin_amdgcn_s_sleep(2);
        const int hx = (j * 4 + mm) * 2;
        const char* sa0 = hx < NPUSH ? px + hx * ITEM : raw + (256 + hx) * ITEM;
        const char* sa1 = hx + 1 < NPUSH ? px + (hx + 1) * ITEM : raw + (257 + hx) * ITEM;
        u32 keep;
#pragma unroll
        for (int i = 0; i < 4; ++i)
          asm volatile("s_mov_b32 %0, m0\n\ts_mov_b32 m0, %2\n\ts_nop 0\n\t"
                       "global_load_lds_dwordx4 %1, off sc0 sc1\n\t"
                       "global_load_lds_dwordx4 %1, off offset:1024 sc0 sc1\n\t"
                       "global_load_lds_dwordx4 %1, off offset:2048 sc0 sc1\n\t"
                       "global_load_lds_dwordx4 %1, off offset:3072 sc0 sc1\n\t"
                       "s_mov_b32 m0, %0" : "=&s"(keep) : "v"((i < 2 ? sa0 : sa1) + (i & 1) * 4096), "s"(lA + i * 4096) : "memory");
      }
      const int nh0 = dbl ? 0 : (ct & 1);
      const char* fb = uf + ((j * 32 + t) * 2 + nh0 + ((lane >> 1) & dbl)) * 512 + mm * 128 + (lane & 1) * 32;
      if (j < 3)
        for (u32 it = 1; __builtin_amdgcn_ballot_w64(ld_sys(fb) < need) != 0; ++it) {
          if ((it & 15) == 0 && give_up(me, call, t0)) need = 0;
          __builtin_amdgcn_s_sleep(8);
        }
      const char* pp = raw + ((j * 32 + t) * 2 + nh0) * ITEM + 4096;
      u32x4 p[8], p2[8];
      u32 tv, tk, r0, r1;
      for (;;) {
        u32 it;
        asm volatile("s_mov_b32 %12, 96\n\ts_cmp_lt_u32 %19, %20\n\ts_cbranch_scc1 .Lg_%=\n\t"
                     "global_load_dword %10, %13, off sc0 sc1\n\ts_sleep 6\n\t"
                     "global_load_dword %11, %13, off sc0 sc1\n\ts_sleep 6\n"
                     ".Lp_%=:\n\t"
                     "global_load_dword %8, %13, off sc0 sc1\n\ts_waitcnt vmcnt(2)\n\t"
                     "v_cmp_gt_u32 vcc, %16, %10\n\ts_cbranch_vccz .Lg_%=\n\t"
                     "global_load_dword %10, %13, off sc0 sc1\n\ts_waitcnt vmcnt(2)\n\t"
                     "v_cmp_gt_u32 vcc, %16, %11\n\ts_cbranch_vccz .Lg_%=\n\t"
                     "global_load_dword %11, %13, off sc0 sc1\n\ts_waitcnt vmcnt(2)\n\t"
                     "v_cmp_gt_u32 vcc, %16, %8\n\ts_cbranch_vccz .Lg_%=\n\t"
                     "s_sub_u32 %12, %12, 1\n\ts_cbranch_scc0 .Lp_%=\n\t"
                     "s_waitcnt vmcnt(0)\n\ts_branch .Le_%=\n"
                     ".Lg_%=:\n\t"
                     "global_load_dwordx4 %0, %14, off offset:-4096 sc0 sc1\n\t"
                     "global_load_dwordx4 %1, %14, off offset:-3072 sc0 sc1\n\t"
                     "global_load_dwordx4 %2, %14, off offset:-2048 sc0 sc1\n\t"
                     "global_load_dwordx4 %3, %14, off offset:-1024 sc0 sc1\n\t"
                     "global_load_dwordx4 %4, %14, off sc0 sc1\n\t"
                     "global_load_dwordx4 %5, %14, off offset:1024 sc0 sc1\n\t"
                     "global_load_dwordx4 %6, %14, off offset:2048 sc0 sc1\n\t"
                     "global_load_dwordx4 %7, %14, off offset:3072 sc0 sc1\n\t"
                     "global_load_dword %8, %15, off sc1\n\t"
                     "v_mov_b32 %9, %17\n\ts_mov_b64 exec, 1\n\t"
                     "ds_add_rtn_u32 %9, %18, %9\n\ts_mov_b64 exec, -1\n\ts_waitcnt lgkmcnt(0)\n"
                     ".Le_%=:"
                     : "=&v"(p[0]), "=&v"(p[1]), "=&v"(p[2]), "=&v"(p[3]), "=&v"(p[4]), "=&v"(p[5]), "=&v"(p[6]), "=&v"(p[7]),
                       "=&v"(tv), "=&v"(tk), "=&v"(r0), "=&v"(r1), "=&s"(it)
                     : "v"(fb), "v"(pp), "v"(me + TMO_OFF + b * 128), "s"(need), "s"(1 + dbl),
                       "v"((u32)(size_t)(lds + L_CTL + C_TICKET * 4)), "s"(j), "i"(3) : "memory", "scc", "vcc");
        if (it != 0xffffffffu) break;
        if (give_up(me, call, t0)) need = 0;
      }
      asm volatile("s_cmp_eq_u32 %9, 0\n\ts_cbranch_scc1 .Ls_%=\n\t"
                   "global_load_dwordx4 %0, %8, off offset:-4096 sc0 sc1\n\t"
                   "global_load_dwordx4 %1, %8, off offset:-3072 sc0 sc1\n\t"
                   "global_load_dwordx4 %2, %8, off offset:-2048 sc0 sc1\n\t"
                   "global_load_dwordx4 %3, %8, off offset:-1024 sc0 sc1\n\t"
                   "global_load_dwordx4 %4, %8, off sc0 sc1\n\t"
                   "global_load_dwordx4 %5, %8, off offset:1024 sc0 sc1\n\t"
                   "global_load_dwordx4 %6, %8, off offset:2048 sc0 sc1\n\t"
                   "global_load_dwordx4 %7, %8, off offset:3072 sc0 sc1\n"
                   ".Ls_%=:"
                   : "=&v"(p2[0]), "=&v"(p2[1]), "=&v"(p2[2]), "=&v"(p2[3]), "=&v"(p2[4]), "=&v"(p2[5]), "=&v"(p2[6]), "=&v"(p2[7])
                   : "v"(pp + 8192), "s"(dbl) : "memory", "scc");
      u32 ts = rfl(tk) & 0xffff, fs = 0;
      const u32 t1 = (rfl(tk) >> 16) - 1;
#define SLOT(t) ({ const u32 k_ = (t) < t1 ? (t) % 3 : ((t) - t1) % 5; k_ < 3 ? L_B / 8192 + k_ : self * 2 + k_ - 3; })
#define NEEDF(t) ((t) + 8 - ((t) >= t1 && (t) - t1 >= 3 ? 4 : 2))
      u32 sl0 = SLOT(ts), sl1 = SLOT(ts + 1), nf0 = NEEDF(ts), nf1 = NEEDF(ts + 1);
      asm volatile("" : "+s"(sl0), "+s"(sl1), "+s"(nf0), "+s"(nf1));
#define ROOM(nf) while (fs < (nf)) { fs = rfl(min4(*(volatile LDSP u32x4*)(ctl + C_FREE))) + 8; if (fs < (nf)) __builtin_amdgcn_s_sleep(1); }
#define FULL(sl) do { ++ts; \
        asm volatile("" : "+s"(ts) :: "memory"); \
        if (lane == 0) ctl[C_FULL + ((ts - 1) & 7)] = ts | (src << 8) | ((sl) << 11) | (tv == call ? 0x10000u : 0u); } while (0)
#define PUT(R, sl) do { LDSP char* ub = lds + (sl) * 8192 + lane * 16; \
        _Pragma("unroll") for (int i = 0; i < 8; ++i) *(LDSP u32x4*)(ub + i * 1024) = R[i]; \
        FULL(sl); } while (0)
      ROOM(nf0);
      {
        LDSP char* ub = lds + sl0 * 8192 + lane * 16;
#define PUT2(i, N) asm volatile("s_waitcnt vmcnt(" #N ")" : "+v"(p[i]), "+v"(p[i + 1]) :: "memory"); \
        *(LDSP u32x4*)(ub + (i) * 1024) = p[i]; *(LDSP u32x4*)(ub + (i) * 1024 + 1024) = p[i + 1];
        PUT2(0, 7) PUT2(2, 5) PUT2(4, 3) PUT2(6, 1)
        asm volatile("s_waitcnt vmcnt(0)" : "+v"(p2[0]), "+v"(p2[1]), "+v"(p2[2]), "+v"(p2[3]), "+v"(p2[4]), "+v"(p2[5]), "+v"(p2[6]), "+v"(p2[7]),
                     "+v"(tv), "+v"(r0), "+v"(r1) :: "memory");
        FULL(sl0);
      }
      if (dbl) { ROOM(nf1); fence(); PUT(p2, sl1); }
      prev = ts;
    }
    if (src == self) {
      while (rfl(min4(*(volatile LDSP u32x4*)(ctl + C_FREE))) < prev) __builtin_amdgcn_s_sleep(2);
      if (lane == 0) {
        u32 o = ctl[C_TICKET];
        while (!__hip_atomic_compare_exchange_strong((LDSP u32*)(ctl + C_TICKET), &o, o | (((o + 2) / 3 * 3 + 1) << 16),
                                                     __ATOMIC_RELAXED, __ATOMIC_RELAXED, __HIP_MEMORY_SCOPE_WORKGROUP)) {}
      }
    }
    return;
  }

  const int mw = wave;
  {
    u32 scale_bits = __builtin_amdgcn_readfirstlane(__builtin_bit_cast(u32, *(const GLB float*)KA64(12) * *(const GLB float*)KA64(14)));
    float* obase = (float*)KA64(10) + (size_t)(mm * 128 + mw * 32) * NN + t * 128 + (ct & 1) * 64;
    asm volatile("" : "+s"(scale_bits), "+s"(obase));
    drain();
    v4f Pc[2][4], Qc[2][4];
#pragma unroll
    for (int i = 0; i < 2; ++i)
#pragma unroll
      for (int k = 0; k < 4; ++k) Pc[i][k] = Qc[i][k] = v4f{0.f, 0.f, 0.f, 0.f};
#define FRAG(p, hs) ({ const u32x4 lo_ = *(LDSP u32x4*)(p), hi_ = *(LDSP u32x4*)((p) + (hs)); \
      v8i{(int)lo_.x, (int)lo_.y, (int)lo_.z, (int)lo_.w, (int)hi_.x, (int)hi_.y, (int)hi_.z, (int)hi_.w}; })
#define MMA(i, k, F) do { F##c[i][k] = __builtin_amdgcn_mfma_scale_f32_16x16x128_f8f6f4(F##a[i], F##b[k], F##c[i][k], 1, 1, 0, 0, 0, 0); \
      __builtin_amdgcn_sched_barrier(0); } while (0)
#define SB() __builtin_amdgcn_sched_barrier(0)
#define RD(F) do { bad |= w; LDSP char* ap_ = lds + L_A + ((w >> 8) & 7) * 16384 + mw * 4096 + lane * 16; \
      bp = lds + ((w >> 11) & 31) * 8192 + lane * 16; \
      F##a[0] = FRAG(ap_, 1024); F##b[0] = FRAG(bp, 4096); SB(); F##a[1] = FRAG(ap_ + 2048, 1024); SB(); F##b[1] = FRAG(bp + 1024, 4096); \
      SB(); } while (0)
#define SPIN() do { while (((w = rfl(ctl[C_FULL + (n & 7)])) & 0xff) != n + 1) __builtin_amdgcn_s_sleep(0); \
      fence(); } while (0)
#define HEAD(F) do { \
      MMA(0, 0, F); F##b[2] = FRAG(bp + 2048, 4096); SB(); \
      MMA(1, 0, F); F##b[3] = FRAG(bp + 3072, 4096); \
      fence();     \
      ++n; \
      if (lane == 0) ctl[C_FREE + mw] = n; } while (0)
#define STEP(F, G, EXTRA) do { \
      HEAD(F); \
      peek = ctl[C_FULL + (n & 7)]; SB(); \
      MMA(0, 1, F); MMA(1, 1, F); MMA(0, 2, F); SB(); \
      w = rfl(peek); \
      fence(); \
      const bool ok_ = (w & 0xff) == n + 1; \
      if (ok_) RD(G); \
      MMA(1, 2, F); MMA(0, 3, F); MMA(1, 3, F); SB(); \
      EXTRA; \
      if (!ok_) { SPIN(); RD(G); } } while (0)
    LDSP char* bp;
    v8i Pa[2], Pb[4], Qa[2], Qb[4];
    u32 n = 0, w, peek, bad = 0;
    SPIN();
    RD(P);
#pragma clang loop unroll(disable)
    for (int it = 1; it < (16 << dbl); ++it) { STEP(P, Q, ); STEP(Q, P, ); }
#define EPI(A, i, r, o) do { v4f y_; _Pragma("unroll") for (int k = 0; k < 4; ++k) \
        y_[k] = __builtin_elementwise_maximum(A[i][k][r] * scale, 0.f);     \
      asm volatile("global_store_dwordx4 %0, %1, %2 sc1\n\ts_nop 1" :: "v"(ov), "v"(y_), "s"((o) + ((i) * 16 + (r)) * NN) : "memory"); \
      __builtin_amdgcn_sched_barrier(0); } while (0)
    STEP(P, Q, if (!dbl) { fence();
                 _Pragma("unroll") for (int i = 0; i < 2; ++i) _Pragma("unroll") for (int k = 0; k < 4; ++k) Qc[i][k] += Pc[i][k]; });
    HEAD(Q); SB();
    MMA(0, 1, Q); MMA(0, 2, Q); MMA(0, 3, Q); MMA(1, 1, Q);
    const float scale = __builtin_bit_cast(float, (bad & 0x10000u) != 0 ? 0x7fc00000u : scale_bits);
    const u32 ln = __builtin_amdgcn_mbcnt_hi(~0u, __builtin_amdgcn_mbcnt_lo(~0u, 0));
    const u32 ov = (ln >> 4) * (4 * NN * 4) + (ln & 15) * 16;
    SB();
    EPI(Qc, 0, 0, obase); MMA(1, 2, Q); EPI(Qc, 0, 1, obase); MMA(1, 3, Q); EPI(Qc, 0, 2, obase); EPI(Qc, 0, 3, obase);
    EPI(Qc, 1, 0, obase); EPI(Qc, 1, 1, obase); EPI(Qc, 1, 2, obase); EPI(Qc, 1, 3, obase);
    if (dbl) {
      EPI(Pc, 0, 0, obase - 64); EPI(Pc, 0, 1, obase - 64); EPI(Pc, 0, 2, obase - 64); EPI(Pc, 0, 3, obase - 64);
      EPI(Pc, 1, 0, obase - 64); EPI(Pc, 1, 1, obase - 64); EPI(Pc, 1, 2, obase - 64); EPI(Pc, 1, 3, obase - 64);
    }
  }
}

extern "C" void kernel_launch(int rank, int nranks, ncclComm_t comm,
                              hipStream_t stream,
                              void* const* d_in, const int* in_sizes, int n_in,
                              void* d_out, int out_size,
                              const int* in_dtypes, int out_dtype,
                              void* const* d_out_all,
                              void* const* d_ws_all, size_t ws_bytes) {
  if (nranks != NR || (unsigned)rank >= (unsigned)NR || n_in < 4 || ws_bytes < WS_NEED ||
      in_sizes[0] != 4096 * KS || in_sizes[1] != KS * NN || out_size != 512 * NN) {
    fprintf(stderr, "kernel_launch: unexpected problem shape\n");
    return;
  }
  Params p;
  p.me = (char*)d_ws_all[rank];
  p.ws_all = d_ws_all;
  p.x = (const unsigned char*)d_in[0];
  p.w = (const unsigned char*)d_in[1];
  p.sx = (const float*)d_in[2];
  p.sw = (const float*)d_in[3];
  p.out = (float*)d_out;
  p.rank = rank;
  static const int grp[2][4] = {{0, 2, 3, 6}, {1, 4, 5, 7}};
  int gi = 0, gp = 0;
  for (int a = 0; a < 2; ++a) for (int i = 0; i < 4; ++i) if (grp[a][i] == rank) { gi = a; gp = i; }
  p.dstmap = 0;
  for (int x = 0; x < 8; ++x) {
    p.dstmap |= (u32)(x < 4 ? grp[gi][(gp + 1 + x) & 3] : grp[1 - gi][(gp + x) & 3]) << (4 * x);
  }
  hipLaunchKernelGGL(fused_gemm_kernel, dim3(NWG), dim3(NTHREADS), 0, stream, p);
  HIP_CHECK(hipGetLastError());
}
```

```cpp
#include <hip/hip_runtime.h>
#include <rccl/rccl.h>
#include <cstdio>
#include <cstdint>

#define HIP_CHECK(expr) do { hipError_t _e = (expr); if (_e != hipSuccess) { \
    fprintf(stderr, "HIP error %s at %s:%d\n", hipGetErrorString(_e), __FILE__, __LINE__); \
    return; } } while (0)

typedef unsigned u32;
typedef unsigned long long u64;
typedef unsigned u32x2 __attribute__((ext_vector_type(2)));
typedef unsigned u32x4 __attribute__((ext_vector_type(4)));
typedef int v8i __attribute__((ext_vector_type(8)));
typedef float v4f __attribute__((ext_vector_type(4)));
#define LDSP __attribute__((address_space(3)))
#define GLB __attribute__((address_space(1)))

constexpr int NR = 8;
constexpr int KS = 512;
constexpr int NN = 4096;
constexpr int NTHREADS = 768;
constexpr int NWG = 256;
constexpr int NIT = 288;
#define WD 6
#define NPUSH 12
#define GATE 100

constexpr u32 ITEM = 8192;
constexpr u32 FIX = 5120;
constexpr u32 PK_OFF = 0;
constexpr u32 RAW_OFF = 2 * 512 * ITEM;
constexpr u32 PX_OFF = RAW_OFF + 9 * NIT * ITEM;
constexpr u32 RL_OFF = PX_OFF + 2 * NR * 16 * ITEM;
constexpr u32 DIR_OFF = RL_OFF + 2 * NR * 8 * ITEM;
constexpr u32 UF_OFF = DIR_OFF + 2 * NR * NIT * 128;
constexpr u32 CNT_OFF = UF_OFF + NR * NIT * 512;
constexpr u32 TMO_OFF = CNT_OFF + NWG * 12 * 128;
constexpr u32 BAL_OFF = TMO_OFF + NWG * 128;
constexpr u32 PF_OFF = BAL_OFF + NR * 256;
constexpr u32 PLAN_OFF = PF_OFF + NR * 256;
constexpr u32 RLF_OFF = PLAN_OFF + NR * 128;
constexpr u32 ROLE_OFF = RLF_OFF + NR * 8 * 128;
static_assert(DIR_OFF / ITEM <= 4096, "12-bit item offsets");
constexpr u32 WS_NEED = ROLE_OFF + 64 * 128;
constexpr u64 DEADLINE = 30000000ull;
static_assert(NPUSH <= 16 && NPUSH >= 0, "PX region");

constexpr int NBS = 3;
constexpr u32 L_A = 0;
constexpr u32 L_B = 8 * 16384;
constexpr int NGRP = 4;
constexpr int NS = 12;
constexpr u32 L_IMG = NS * ITEM;
constexpr u32 L_SCR = L_B + NBS * ITEM;
constexpr u32 L_CTL = L_SCR + 4096;
constexpr u32 L_DUMP = L_CTL + 256;
constexpr u32 L_BYTES = L_DUMP + 256;
constexpr int C_FREE = 0, C_FULL = 4, C_TICKET = 12, C_ENC = 13;
constexpr int C_CONS = 16, C_LANDED = 28, C_WIN = 29, C_NIT = 30, C_ROLE = 31, C_DESC = 32;
constexpr u32 M_RAW = 0x80, IT_END = 0x1ff;

#define PUSH1 8
#define PUSHD 130
#define EA 2
#define LA 8
#define TITEM 77
#define WFAST 8
#define GFAST 24
#define GSLOW 19
#define K0 4
#define SP 4
#define PLANT 2000
#define DZ 10.f

struct Params {
  char* me;
  const unsigned char* x;
  const unsigned char* w;
  void* const* ws_all;
  int rank;
  u32 dstmap;
  float* out;
  const float* sx;
  const float* sw;
};

template <int N> __device__ __forceinline__ void waitv() { asm volatile("s_waitcnt vmcnt(%0)" :: "i"(N) : "memory"); }
__device__ __forceinline__ void waitl() { asm volatile("s_waitcnt lgkmcnt(0)" ::: "memory"); }
__device__ __forceinline__ void bar() { asm volatile("s_barrier" ::: "memory"); }
__device__ __forceinline__ void drain() { __builtin_amdgcn_s_waitcnt(0x0070); }
__device__ __forceinline__ void fence() { asm volatile("" ::: "memory"); }
template <int OFF> __device__ __forceinline__ void st16(const void* p, u32x4 x) {
  asm volatile("global_store_dwordx4 %0, %1, off offset:%2 sc0 sc1\n\ts_nop 1" :: "v"(p), "v"(x), "i"(OFF) : "memory");
}
__device__ __forceinline__ void st4(const void* p, u32 x) {
  asm volatile("global_store_dword %0, %1, off sc0 sc1\n\ts_nop 1" :: "v"(p), "v"(x) : "memory");
}
__device__ __forceinline__ u32 ld_sys(const void* p) {
  return __hip_atomic_load((const GLB u32*)p, __ATOMIC_RELAXED, __HIP_MEMORY_SCOPE_SYSTEM);
}
__device__ __forceinline__ u32 ld_agent(const void* p) {
  return __hip_atomic_load((const GLB u32*)p, __ATOMIC_RELAXED, __HIP_MEMORY_SCOPE_AGENT);
}
__device__ __forceinline__ void st_agent(void* p, u32 v) {
  __hip_atomic_store((GLB u32*)p, v, __ATOMIC_RELAXED, __HIP_MEMORY_SCOPE_AGENT);
}
__device__ __forceinline__ void touch(const void* p, u32 dump) {
  u32 keep;
  asm volatile("s_mov_b32 %0, m0\n\ts_mov_b32 m0, %2\n\ts_nop 0\n\tglobal_load_lds_dword %1, off sc0 sc1\n\ts_mov_b32 m0, %0"
               : "=&s"(keep) : "v"(p), "s"(dump) : "memory");
}
__device__ __forceinline__ u32 rfl(u32 v) { return __builtin_amdgcn_readfirstlane(v); }
__device__ __forceinline__ void put_flag(char* f, u32 v, int lane) {
  if (lane < 2) st16<0>(f + lane * 16, u32x4{v, v, v, v});
}
__device__ __forceinline__ bool give_up(char* me, u32 call, u64 t0) {
  if (rfl(ld_agent(me + TMO_OFF + blockIdx.x * 128)) == call) return true;
  if (__builtin_amdgcn_s_memrealtime() - t0 <= DEADLINE) return false;
  for (int i = threadIdx.x & 63; i < NWG; i += 64) st_agent(me + TMO_OFF + i * 128, call);
  waitv<0>();
  return true;
}
#define LDS_ADD(p, v) __hip_atomic_fetch_add((LDSP u32*)(p), (v), __ATOMIC_RELAXED, __HIP_MEMORY_SCOPE_WORKGROUP)
#define LDS_SUB(p, v) __hip_atomic_fetch_sub((LDSP u32*)(p), (v), __ATOMIC_RELAXED, __HIP_MEMORY_SCOPE_WORKGROUP)
#define LDS_OR(p, v) __hip_atomic_fetch_or((LDSP u32*)(p), (v), __ATOMIC_RELAXED, __HIP_MEMORY_SCOPE_WORKGROUP)
#define DPP(old, v, ctrl, rm) (u32)__builtin_amdgcn_update_dpp((int)(old), (int)(v), ctrl, rm, 0xf, false)
#define SCAN(name, id, OP) __device__ __forceinline__ u32 name(u32 v) { \
    v = OP(v, DPP(id, v, 0x111, 0xf)); v = OP(v, DPP(id, v, 0x112, 0xf)); v = OP(v, DPP(id, v, 0x114, 0xf)); \
    v = OP(v, DPP(id, v, 0x118, 0xf)); v = OP(v, DPP(id, v, 0x142, 0xa)); v = OP(v, DPP(id, v, 0x143, 0xc)); return v; }
__device__ __forceinline__ u32 addu(u32 a, u32 b) { return a + b; }
SCAN(scan_add, 0u, addu)
SCAN(scan_max, 0u, max)
SCAN(scan_min, 0xffffffffu, min)

__device__ __forceinline__ u32 round_item(u32 rd, u32 col) { return ((rd >> 3) * 32 + 4 * col + ((rd >> 1) & 3)) * 2 + (rd & 1); }
__device__ __forceinline__ u32 sched_item(u32 q, u32 rel, u32 self) {
  if (q < 32 - NPUSH) return 256 + (q < 8 ? 16 + q : q < 24 - NPUSH ? NPUSH - 8 + q : NPUSH + q);
  q -= 32 - NPUSH;
  const u32 sh = 16 + self, ps = (sh & ~3u) | ((sh - rel) & 3);
  u32 r = q >> 3;
  r = r == SP ? ps : r > SP && r <= ps ? r - 1 : r;
  return round_item((r & ~3u) | ((r + rel) & 3), (q + rel) & 7);
}
__device__ __forceinline__ u32 nib_sum(u32 x) { return (((x & 0x0f0f0f0fu) + ((x >> 4) & 0x0f0f0f0fu)) * 0x01010101u) >> 24; }
__device__ __forceinline__ u32 lds_ticket(volatile LDSP u32* p, u32 v) {
  asm volatile("s_mov_b64 exec, 1\n\tds_add_rtn_u32 %0, %1, %0\n\ts_mov_b64 exec, -1\n\ts_waitcnt lgkmcnt(0)"
               : "+v"(v) : "v"((u32)(size_t)p) : "memory");
  return rfl(v);
}
__device__ __forceinline__ u32 min4(u32x4 v) { return min(min(v.x, v.y), min(v.z, v.w)); }
__device__ __forceinline__ u32 gather8(const u32* f) {
  const u32 r0 = __builtin_amdgcn_udot4(f[1], 0x80402010u, __builtin_amdgcn_udot4(f[0], 0x08040201u, 0, false), false);
  const u32 r1 = __builtin_amdgcn_udot4(f[3], 0x80402010u, __builtin_amdgcn_udot4(f[2], 0x08040201u, 0, false), false);
  const u32 r2 = __builtin_amdgcn_udot4(f[5], 0x80402010u, __builtin_amdgcn_udot4(f[4], 0x08040201u, 0, false), false);
  const u32 r3 = __builtin_amdgcn_udot4(f[7], 0x80402010u, __builtin_amdgcn_udot4(f[6], 0x08040201u, 0, false), false);
  return (r0 >> 7) | (r1 << 1) | (r2 << 9) | (r3 << 17);
}

__global__ void __launch_bounds__(NTHREADS) fused_gemm_kernel(Params P) {
  __shared__ __attribute__((aligned(16))) char smem[L_BYTES];
  LDSP char* lds = (LDSP char*)smem;
  volatile LDSP u32* ctl = (volatile LDSP u32*)(lds + L_CTL);
  const int lane = threadIdx.x & 63;
  const int wave = rfl(threadIdx.x >> 6);
  const int b = blockIdx.x, xcd = b & 7, slot = b >> 3;
  const int ct = slot >> 2, mm = slot & 3;
  typedef u32 u32x16 __attribute__((ext_vector_type(16)));
  u32x16 ka;
  asm volatile("s_load_dwordx16 %0, %1, 0x0" : "=&s"(ka) : "s"(__builtin_amdgcn_kernarg_segment_ptr()) : "memory");
  const bool link = (ct & 5) == 0;
  const int i8 = mm + (ct < 4 ? ct >> 1 : ct & 1) * 4;
  const bool xjob = !link & ((wave == 4) | ((wave == 5) & (ct >= 6)));
  const int xj = wave == 5 ? 1 : ct < 4 ? 0 : ct >> 1;
  u32 cno = CNT_OFF + (b * 12 + wave) * 128;
  u32 xv0 = (i8 * 64 + (lane & 15)) * KS + xj * 128 + (lane >> 4) * 16, xv1 = xv0 + 16 * KS, xv2 = xv0 + 32 * KS, xv3 = xv0 + 48 * KS;
  asm volatile("s_waitcnt lgkmcnt(0)" : "+s"(ka), "+v"(cno), "+v"(xv0), "+v"(xv1), "+v"(xv2), "+v"(xv3) :: "memory");
#define KA64(i) (ka[i] | ((u64)ka[(i) + 1] << 32))
  char* const me = (char*)KA64(0);
  const int self = ka[8];
#define LOAD_PWS() char* pws; asm volatile("s_load_dwordx2 %0, %1, %2\n\ts_waitcnt lgkmcnt(0)" : "=&s"(pws) : "s"(KA64(6)), "s"(peer * 8) : "memory")
  u32* cnt = (u32*)(me + CNT_OFF + (b * 12 + wave) * 128);
  const u32 dump = (u32)(size_t)lds + L_DUMP;
  u32 call;
  const u64 tstart = __builtin_amdgcn_s_memrealtime();

  if (__builtin_expect(xjob, 1)) {
    const int hx = xj * 8 + i8, peer = (ka[9] >> (4 * xcd)) & 7;
    if (xj == 3) __builtin_amdgcn_s_sleep(15);
    u32x4 d[8];
    u32 cv;
    asm volatile("global_load_dword %8, %11, %10 sc1\n\t"
                 "global_load_dwordx4 %0, %12, %9\n\tglobal_load_dwordx4 %1, %12, %9 offset:64\n\t"
                 "global_load_dwordx4 %2, %13, %9\n\tglobal_load_dwordx4 %3, %13, %9 offset:64\n\t"
                 "global_load_dwordx4 %4, %14, %9\n\tglobal_load_dwordx4 %5, %14, %9 offset:64\n\t"
                 "global_load_dwordx4 %6, %15, %9\n\tglobal_load_dwordx4 %7, %15, %9 offset:64"
                 : "=&v"(d[0]), "=&v"(d[1]), "=&v"(d[2]), "=&v"(d[3]), "=&v"(d[4]), "=&v"(d[5]), "=&v"(d[6]), "=&v"(d[7]), "=&v"(cv)
                 : "s"(KA64(2) + peer * (512 * KS)), "s"(me), "v"(cno), "v"(xv0), "v"(xv1), "v"(xv2), "v"(xv3) : "memory");
    LOAD_PWS();
    const bool push = hx < NPUSH, direct = push || peer == self;
    const u32 o0 = push ? PX_OFF + (self * 16 + hx) * ITEM : direct ? RAW_OFF + (8 * NIT + 256 + hx) * ITEM : PK_OFF + (256 + peer * 32 + hx) * ITEM;
    const u32 o1 = push ? NR * 16 * ITEM : direct ? (self - 8) * NIT * ITEM : 512 * ITEM;
    const u32 f0 = direct ? UF_OFF + (self * NIT + 256 + hx) * 512 : DIR_OFF + (self * NIT + 256 + hx) * 128;
    const u32 f1 = direct ? 0 : NR * NIT * 128;
    const char* db = push ? pws : me;
    const u64 fm = direct ? 0xffff : 3;
    asm volatile("s_waitcnt vmcnt(8)" : "+v"(cv) :: "memory");
    call = rfl(cv) + 1;
    const u32 par = call & 1;
    const u32 dv = o0 + par * o1 + 4096 + lane * 16, fv = f0 + par * f1 + (lane >> 2) * 128 + (lane & 3) * 16;
    const u32 v = direct ? call : (call << 8) | M_RAW;
    if (push && hx >= PUSH1) while (__builtin_amdgcn_s_memrealtime() - tstart < PUSHD) __builtin_amdgcn_s_sleep(1);
    asm volatile("s_waitcnt vmcnt(7)\n\tglobal_store_dwordx4 %8, %0, %9 offset:-4096 sc0 sc1\n\t"
                 "s_waitcnt vmcnt(7)\n\tglobal_store_dwordx4 %8, %1, %9 offset:-3072 sc0 sc1\n\t"
                 "s_waitcnt vmcnt(7)\n\tglobal_store_dwordx4 %8, %2, %9 offset:-2048 sc0 sc1\n\t"
                 "s_waitcnt vmcnt(7)\n\tglobal_store_dwordx4 %8, %3, %9 offset:-1024 sc0 sc1\n\t"
                 "s_waitcnt vmcnt(7)\n\tglobal_store_dwordx4 %8, %4, %9 sc0 sc1\n\t"
                 "s_waitcnt vmcnt(7)\n\tglobal_store_dwordx4 %8, %5, %9 offset:1024 sc0 sc1\n\t"
                 "s_waitcnt vmcnt(7)\n\tglobal_store_dwordx4 %8, %6, %9 offset:2048 sc0 sc1\n\t"
                 "s_waitcnt vmcnt(7)\n\tglobal_store_dwordx4 %8, %7, %9 offset:3072 sc0 sc1\n\ts_nop 1"
                 : "+v"(d[0]), "+v"(d[1]), "+v"(d[2]), "+v"(d[3]), "+v"(d[4]), "+v"(d[5]), "+v"(d[6]), "+v"(d[7])
                 : "v"(dv), "s"(db) : "memory");
    asm volatile("s_mov_b64 exec, %3\n\ts_waitcnt vmcnt(0)\n\tglobal_store_dwordx4 %0, %1, %2 sc0 sc1\n\ts_mov_b64 exec, -1\n\ts_nop 0"
                 :: "v"(fv), "v"(u32x4{v, v, v, v}), "s"(pws), "s"(fm) : "memory");
    bar();
  } else if (!link && (wave < 4 || (wave >= 8 && ct < 4))) {
    {
      const int g = lane >> 4, c = lane & 15, j2 = wave >> 3, jw = ct < 4 ? 2 + j2 : (ct - 4) >> 1;
      const int u = wave & 1, hf = (wave >> 1) & 1, blk = (jw * 32 + xcd * 4) * 2 + i8;
      const GLB unsigned char* src = (const GLB unsigned char*)KA64(4) + (size_t)(jw * 128 + u * 64 + g * 16) * NN + xcd * 512 + i8 * 64 + c * 4;
      u32 W[16], D[8];
      if (jw == 1) __builtin_amdgcn_s_sleep(30);
      if (jw >= 2) __builtin_amdgcn_s_sleep(60);
#pragma unroll
      for (int r = 0; r < 16; ++r) W[r] = *(const GLB u32*)(src + r * NN);
      const u32 cv = ld_agent(cnt);
      LDSP char* scr = lds + L_SCR + (wave - j2 * 4) * 512;
      *(LDSP u32x2*)(scr + lane * 8) = u32x2{0, 0};
#pragma unroll
      for (int s = 0; s < 8; ++s) LDS_ADD(scr + (((W[s >> 2] >> ((s & 3) * 8)) & 0x7f) << 2), 1u);
      const u32x2 hh = *(volatile LDSP u32x2*)(scr + lane * 8);
      *(volatile LDSP u32x2*)(scr + lane * 8) = u32x2{0, 0};
      const u32 sel = 0x05010400u + hf * 0x02020202u;
#pragma unroll
      for (int q = 0; q < 4; ++q) {
        const u32 t0 = __builtin_amdgcn_perm(W[4 * q + 1], W[4 * q], sel), t2 = __builtin_amdgcn_perm(W[4 * q + 3], W[4 * q + 2], sel);
        D[q] = __builtin_amdgcn_perm(t2, t0, 0x05040100u); D[4 + q] = __builtin_amdgcn_perm(t2, t0, 0x07060302u);
      }
      call = rfl(cv) + 1;
      const u32 par = call & 1;
      char* pk = me + PK_OFF + (par * 512 + blk) * ITEM;
      {
        char* d = me + RAW_OFF + ((par ? self : 8) * NIT + blk) * ITEM + u * 4096 + hf * 2048 + lane * 16;
        st16<0>(d, u32x4{D[0], D[1], D[2], D[3]}); st16<1024>(d, u32x4{D[4], D[5], D[6], D[7]});
        u32 s0 = 0;
#pragma unroll
        for (int k = 0; k < 8; ++k) s0 |= (D[k] & 0x80808080u) >> k;
        st4(pk + 4096 + u * 512 + hf * 256 + lane * 4, s0);
      }
      u32 base;
      {
        const u32 c0 = scan_add(hh.x + hh.y) - hh.x - hh.y, c1 = c0 + hh.x;
        u32 u8 = (u32)__builtin_amdgcn_ds_bpermute(((lane + 8) & 63) * 4, (int)(c0 | (c1 << 16)));
        u32 u9 = (u32)__builtin_amdgcn_ds_bpermute(((lane + 9) & 63) * 4, (int)(c0 | (c1 << 16)));
        if (lane >= 56) u8 = 0x02000200u;
        if (lane >= 55) u9 = 0x02000200u;
        const u32 cs = (c0 | (c1 << 16)) + 0x08000800u - u8 - 3 * u9;
        const u32 k0 = ((cs & 0xffff) << 7) | (2 * lane), k1 = ((cs >> 16) << 7) | (2 * lane + 1);
        base = __builtin_amdgcn_readlane(scan_min(lane < 56 ? min(k0, k1) : lane == 56 ? k0 : 0xffffffffu), 63) & 0x7f;
      }
      const u32 b8f = base * 0x01010101u + 0x8f8f8f8fu;
      u32 F1[8], F2[8], FE[8], T[8];
#pragma unroll
      for (int k = 0; k < 8; ++k) {
        const u32 t = T[k] = b8f - (D[k] & 0x7f7f7f7fu);
        const u32 t1 = t << 1, t2 = t << 2, t3 = t << 3, wr = ((t & 0x0f0f0f0fu) + 0x02020202u) << 3;
        const u32 h = t & ~t1 & 0x80808080u, k1 = h & ~t2 & t3, k2 = h & t2 & ~t3, km = ~t & t1 & t2 & t3 & 0x80808080u;
        F2[k] = (wr & k1) | (~wr & k2);
        F1[k] = k1 | (wr & km) | (~wr & k2);
        FE[k] = ((h & ~t2 & ~t3) | F1[k]) ^ 0x80808080u;
      }
      {
        u32x4 cd;
#pragma unroll
        for (int q = 0; q < 4; ++q) cd[q] = ~((T[q] & 0x0f0f0f0fu) | ((T[4 + q] << 4) & 0xf0f0f0f0u));
        st16<0>(pk + u * 2048 + hf * 1024 + lane * 16, cd);
      }
      u32 x1 = gather8(F1), x2 = gather8(F2), xe = gather8(FE);
      const u64 any = __builtin_amdgcn_ballot_w64(x1 != 0);
      int prev;
      {
        const u32 lastp = x1 ? lane * 32 + 31 - __builtin_clz(x1) : 0;
        const u32 back = lane * 32 - scan_max(DPP(0, lastp, 0x138, 0xf));
        prev = -(int)(back - 15 * ((back * 4370) >> 16));
      }
      u32 a0 = 0, a1 = 0, a2 = 0, nc = 0;
#define PUSH_NIB(gg) do { a0 = __builtin_amdgcn_alignbit(a1, a0, 4); a1 = __builtin_amdgcn_alignbit(a2, a1, 4); \
        a2 = __builtin_amdgcn_alignbit((gg), a2, 4); ++nc; } while (0)
      while (x1 != 0) {
        const u32 pos = __builtin_ctz(x1);
        const u32 gap = pos - prev, gg = min(gap, 15u);
        prev += gg;
        const u32 bit = gap < 15 ? 1u << pos : 0u, in2 = x2 & bit;
        x2 ^= in2;
        x1 ^= bit ^ in2;
        PUSH_NIB(gg);
      }
      if ((any >> lane) > 1) while (prev <= 17) { prev += 15; PUSH_NIB(15u); }
      const u32 nend = scan_add(nc), nn = __builtin_amdgcn_readlane(nend, 63);
      bool over = nn > 512 || __builtin_amdgcn_ballot_w64(nc > 24) != 0;
      if (!over) {
        const u32 sh = (8 - (nend & 7)) * 4;
        LDSP char* q = scr + (nend >> 3) * 4;
        LDS_OR(q, (u32)((((u64)a0) << 32) >> sh));
        LDS_OR(q + 4, (u32)((((u64)a1 << 32) | a0) >> sh));
        LDS_OR(q + 8, (u32)((((u64)a2 << 32) | a1) >> sh));
        LDS_OR(q + 12, (u32)(((u64)a2) >> sh));
      }
      u32 ne = 0;
      {
        LDSP unsigned short* lst = (LDSP unsigned short*)(scr + 384);
        u64 mk;
        while ((mk = __builtin_amdgcn_ballot_w64(xe != 0)) != 0 && ne <= 63) {
          if (xe != 0) {
            const u32 pos = __builtin_ctz(xe);
            const u32 wa = (pos & 4) ? T[1] : T[0], wb = (pos & 4) ? T[3] : T[2], wc = (pos & 4) ? T[5] : T[4], wd = (pos & 4) ? T[7] : T[6];
            const u32 we = (pos & 8) ? wb : wa, wf = (pos & 8) ? wd : wc, wg = (pos & 16) ? wf : we;
            const u32 x4 = (wg >> ((pos & 3) * 8 + 4)) & 15;
            const u32 at = ne + __builtin_amdgcn_mbcnt_hi((u32)(mk >> 32), __builtin_amdgcn_mbcnt_lo((u32)mk, 0));
            if (at < 64) lst[at] = (unsigned short)((x4 << 12) | (hf * 2048 + lane * 32 + pos));
            xe &= xe - 1;
          }
          ne += __builtin_popcountll(mk);
        }
      }
      over = over || ne > 63;
      bar();
      if (!over) {
        const u32 ed = (ne + 1) >> 1, nd = (nn + 7) >> 3;
        const u32 ev = *(volatile LDSP u32*)(scr + 384 + (lane & 31) * 4);
        u32 nv = *(volatile LDSP u32*)(scr + 12 + lane * 4);
        const u32 vo = (lds_ticket(ctl + C_ENC + j2, ed + nd) & 0xffff) + 4;
        if (lane == (nn >> 3)) nv |= ~0u << ((nn & 7) * 4);
        if (lane < nd) st4(pk + FIX + (vo + lane) * 4, nv);
        if (lane < ed) st4(pk + FIX + (vo + nd + lane) * 4, ev);
        if (lane == 0) st4(pk + FIX + (u * 2 + hf) * 4, vo | (base << 10) | (ne << 17) | (nd << 23));
      }
      u32x16 pw;
      asm volatile("s_load_dwordx16 %0, %1, 0x0\n\ts_waitcnt vmcnt(0) lgkmcnt(0)" : "=&s"(pw) : "s"(KA64(6)) : "memory");
#define PW(i) "s"(pw[2 * (i)] | ((u64)pw[2 * (i) + 1] << 32))
      const u32 k = lds_ticket(ctl + C_ENC + j2, over ? 0x1010000u : 0x1000000u);
      if ((k >> 24) == 3) {
        const u32 v = (call << 8) | ((over || (k & 0xff0000) != 0) ? M_RAW : ((k & 0xffff) + 4 + 31) >> 5);
        asm volatile("s_mov_b64 exec, 3\n\t"
                     "global_store_dwordx4 %0, %1, %2 sc0 sc1\n\tglobal_store_dwordx4 %0, %1, %3 sc0 sc1\n\t"
                     "global_store_dwordx4 %0, %1, %4 sc0 sc1\n\tglobal_store_dwordx4 %0, %1, %5 sc0 sc1\n\t"
                     "global_store_dwordx4 %0, %1, %6 sc0 sc1\n\tglobal_store_dwordx4 %0, %1, %7 sc0 sc1\n\t"
                     "global_store_dwordx4 %0, %1, %8 sc0 sc1\n\tglobal_store_dwordx4 %0, %1, %9 sc0 sc1\n\t"
                     "s_mov_b64 exec, -1\n\ts_nop 0"
                     :: "v"(DIR_OFF + ((par * NR + self) * NIT + blk) * 128 + lane * 16), "v"(u32x4{v, v, v, v}),
                        PW(0), PW(1), PW(2), PW(3), PW(4), PW(5), PW(6), PW(7) : "memory");
        if (lane < 16) st16<0>(me + UF_OFF + (self * NIT + blk) * 512 + (lane >> 2) * 128 + (lane & 3) * 16, u32x4{call, call, call, call});
      }
    }
  } else {
    if (link && wave == 8) return;
    const u32 cv = ld_agent(cnt);
    if (wave == 7) {
      u32 role = 0;
      if (link) {
        asm volatile("s_getreg_b32 %0, hwreg(HW_REG_XCC_ID)" : "=s"(role));
        role = (role & 7) * 8 + i8;
        for (int i = 0; i < 128; ++i) {
          GLB u32* rp = (GLB u32*)(me + ROLE_OFF + role * 128);
          u32 old = 0;
          if (lane == 0) old = __hip_atomic_fetch_add(rp, 1u, __ATOMIC_RELAXED, __HIP_MEMORY_SCOPE_AGENT);
          if (rfl(old) == rfl(cv)) break;
          if (lane == 0) __hip_atomic_fetch_sub(rp, 1u, __ATOMIC_RELAXED, __HIP_MEMORY_SCOPE_AGENT);
          role = (role + 1) & 63;
        }
      }
      ctl[lane] = lane == C_ROLE ? role : 0;
      waitl();
    }
    bar();
    call = rfl(cv) + 1;
  }
  if (lane == 0 && wave != 11) st_agent(cnt, call);

  if (link) {
    const u32 role = rfl(ctl[C_ROLE]), lx = role >> 3, cw = role & 7;
    const int peer = (ka[9] >> (4 * lx)) & 7;
    if (peer == self) {
      if (lane == 0 && wave == 11) st_agent(cnt, call);
      if (cw != 0 || wave != 0) return;
      const char* pf = me + PF_OFF + lane * 32;
      u32 f;
      bool all;
      do {
        __builtin_amdgcn_s_sleep(16);
        f = ld_agent(pf);
        all = __builtin_amdgcn_ballot_w64((f & 0xfff) != (((call & 63) << 6) | 1) && (lane >> 3) != lx) == 0;
      } while (!all && __builtin_amdgcn_s_memrealtime() - tstart < PLANT);
      float v = (lane >> 3) == lx ? 0.f : (float)((int)(f - ((u32)tstart << 12)) >> 12);
      v += __shfl_xor(v, 1); v += __shfl_xor(v, 2); v += __shfl_xor(v, 4);
      v *= 0.125f;
      float mu = v + __shfl_xor(v, 8);
      mu += __shfl_xor(mu, 16); mu += __shfl_xor(mu, 32);
      mu *= 1.f / 7;
      u32 inv = 0;
#pragma unroll
      for (int x = 0; x < NR; ++x) inv |= x << (4 * ((ka[9] >> (4 * x)) & 7));
      const int ps = lane >> 3, pr = lane & 7;
      const float fs = __shfl(v, ((inv >> (4 * ps)) & 7) * 8), fr = __shfl(v, ((inv >> (4 * pr)) & 7) * 8);
      const float ex = ps == self || fs - mu < DZ ? 0.f : fs - mu, gp = pr == self ? 0.f : fmaxf(mu - fr, 0.f);
      float gs = gp + __shfl_xor(gp, 1);
      gs += __shfl_xor(gs, 2); gs += __shfl_xor(gs, 4);
      u32 row = all && gs > 0.f ? (u32)fminf(8.f, ex * gp / gs * (8.f / TITEM) + 0.5f) << (4 * pr) : 0;
      row |= __shfl_xor(row, 1); row |= __shfl_xor(row, 2); row |= __shfl_xor(row, 4);
      row = __shfl(row, pr * 8);
      __hip_atomic_store((GLB u64*)(me + PLAN_OFF + ps * 128 + pr * 8), row | ((u64)call << 32), __ATOMIC_RELAXED, __HIP_MEMORY_SCOPE_AGENT);
      return;
    }
    if (wave == 11) {
      LOAD_PWS();
      const u32 par = call & 1;
      u32 nitems = (NIT - NPUSH - cw + 7) >> 3;
      u32 itv = lane < nitems ? sched_item(cw + lane * 8, lx, self) : 0;
      const char* dir = me + DIR_OFF + ((par * NR + peer) * NIT + itv) * 128;
      u32 dv = 0;
      u64 todo = (1ull << nitems) - 1, ok = 0;
      bool have = false;
      while (__builtin_amdgcn_s_memrealtime() - tstart < GATE - 50) __builtin_amdgcn_s_sleep(1);
      for (u32 n = 0; n < nitems + NGRP; ++n) {
        u32 w0 = (IT_END << 6) | 0x80000;
        if (n < nitems) {
          const u32 first = __builtin_ctzll(todo);
          const u64 cand = todo & (((n < 8 ? 7ull : 1ull) << first) | (0xffull << 56));
          const bool rl = (todo >> 56) != 0 && ((ok & todo) >> 56) == 0;
          for (u32 sp = 1; (ok & cand) == 0 || (rl && sp == 1); ++sp) {
            if (sp > 1 && n > 2) __builtin_amdgcn_s_sleep(4);
            if (lane - first < (n < 8 ? 4u : 64u) && ((todo & ~ok) >> lane) & 1) {
              dv = ld_sys(dir);
              if (lane >= 56) {
                const char* rf = pws + RLF_OFF + ((lane - 56) * 8 + ((itv >> 3) & 7)) * 128;
                if (min(ld_sys(rf), ld_sys(rf + 32)) < call) dv = 0;
              }
            }
            ok = __builtin_amdgcn_ballot_w64((dv >> 8) == (call & 0xffffff));
            if ((sp & 31) == 0 && give_up(me, call, tstart)) { dv = (call << 8) | M_RAW; ok = ~0ull; }
          }
          const u64 pick = ok & cand;
          const u32 ln = (pick >> 56) != 0 ? 63 - __builtin_clzll(pick) : __builtin_ctzll(pick);
          todo &= ~(1ull << ln);
          const u32 it = __builtin_amdgcn_readlane(itv, ln);
          u32 e = __builtin_amdgcn_readlane(dv, ln);
          e &= (e & M_RAW) ? M_RAW : 0xf;
          const u32 so = ln >= 56 ? RL_OFF / ITEM + (par * NR + ln - 56) * 8 + ((it >> 3) & 7)
                         : it >= 256 ? par * 512 + self * 32 + it : (e & M_RAW) ? RAW_OFF / ITEM + (par ? peer : 8) * NIT + it : par * 512 + it;
          w0 = (it << 6) | ((e & M_RAW) ? 0x80000 : e << 15) | (so << 20);
        }
        if (n >= LA) {
          volatile LDSP u32x2* cp = (volatile LDSP u32x2*)(ctl + C_CONS + (n % NGRP) * 2);
          for (;;) { const u32x2 cv = *cp; if (rfl(min(cv.x, cv.y)) + LA > n) break; __builtin_amdgcn_s_sleep(2); }
        }
        if (lane == 0) ctl[C_DESC + n % NS] = w0 | (n + 1);
        if (!have && n >= 17 && (n & 1)) {
          u64 pv;
          for (u32 sp = 1;; ++sp) {
            pv = __hip_atomic_load((const GLB u64*)(me + PLAN_OFF + lx * 128 + (lane & 7) * 8), __ATOMIC_RELAXED, __HIP_MEMORY_SCOPE_AGENT);
            have = __builtin_amdgcn_ballot_w64((u32)(pv >> 32) != call) == 0;
            if (have || n < 27) break;
            if ((sp & 31) == 0 && give_up(me, call, tstart)) { pv = 0; have = true; break; }
          }
          if (have) {
            const u32 row = (u32)pv, mine = __builtin_amdgcn_readlane(row, peer);
            const u32 r = ((itv >> 1) & 3) * 2 + (itv & 1), col = (itv >> 3) & 7;
            const u64 skip = __builtin_amdgcn_ballot_w64((itv >> 6) == 2 && ((col - nib_sum(mine & ((1u << (4 * r)) - 1))) & 7) < ((mine >> (4 * r)) & 15));
            const u32 nt = lane >= 56 ? (row >> (4 * peer)) & 15 : 0, kk = (cw - (scan_add(nt) - nt)) & 7;
            const u64 take = __builtin_amdgcn_ballot_w64(kk < nt);
            if (lane >= 56) {
              itv = round_item(16 + peer, (nib_sum(row & ((1u << (4 * peer)) - 1)) + kk) & 7);
              dir = me + DIR_OFF + ((par * NR + lane - 56) * NIT + itv) * 128;
            }
            todo = (todo & ~skip) | take;
            nitems = n + 1 + __builtin_popcountll(todo);
            if (lane == 0) ctl[C_NIT] = nitems;
          }
        }
      }
      if (lane == 0) st_agent(cnt, call);
      return;
    }
    if (wave == 9) {
      const u32 n0 = (NIT - NPUSH - cw + 7) >> 3;
      char* bp = me + BAL_OFF + lx * 256;
      u32 until = 4 * K0;
      for (u32 r = 0;; ++r) {
        u32 p0, p1;
        while (((p0 = rfl(ctl[C_LANDED])) & 0xffff) < until) __builtin_amdgcn_s_sleep(4);
        for (int i = 0; (p1 = rfl(ctl[C_LANDED])) == p0 && i < 12; ++i) {}
        const u32 now = (u32)__builtin_amdgcn_s_memrealtime(), nn = rfl(ctl[C_NIT]), nitems = nn ? nn : n0, hu = p1 & 0xffff;
        const u32 fown = (now + ((nitems * 23 - hu * 4 - (p1 >> 16) * 7) * TITEM) / 23) << 12 | ((call & 63) << 6) | (nn ? 2 : 1);
        if (lane == 0) st_agent(bp + cw * 32, fown);
        if (r == 0) {
          if (lane == 0) st_agent(bp + PF_OFF - BAL_OFF + cw * 32, fown);
          until = 64;
          continue;
        }
        if (r == 1) { until = min(hu + 6, 4 * (n0 - 8)); continue; }
        if (hu + 12 >= 4 * nitems) break;
        const u32 f = ld_agent(bp + (lane & 7) * 32);
        const bool v = ((f ^ fown) & 0xfff) == 0;
        const int cnt8 = __builtin_popcount((u32)__builtin_amdgcn_ballot_w64(v) & 0xff);
        const int late = cnt8 < 6 ? 0 : (int)__builtin_amdgcn_readlane(scan_add(v ? (u32)((int)(fown - f) >> 12) : 0u), 7) / cnt8;
        const u32 plan = min((u32)(late > 0 ? (late + GFAST / 4) / GFAST : (GSLOW / 4 - late) / GSLOW), 4u);
        until = (rfl(ctl[C_LANDED]) & 0xffff) + 4 * plan;
        if (plan != 0) {
          if (lane == 0) ctl[C_WIN] = late > 0 ? 1 : 2;
          until = min(until, 4 * (nn ? nn : n0 - 8));
          while ((rfl(ctl[C_LANDED]) & 0xffff) < until) __builtin_amdgcn_s_sleep(4);
          if (lane == 0) ctl[C_WIN] = 0;
        }
        until = min(until + 12, 4 * (nn ? nn : n0 - 8));
      }
      return;
    }
    if (wave == 10) {
      LOAD_PWS();
      if (lane < 3) touch(pws + PK_OFF + ((call & 1) * 512 + (lane == 2 ? 256 + self * 32 + 16 : lane * 255)) * ITEM + (self * 8 + cw) * 128, dump);
      waitv<0>();
      drain();
      const u32 l0 = (u32)(size_t)lds, lw = l0 + L_CTL + C_LANDED * 4, one = lane == 0 ? 1 : 0;
      u32 prog = 0, mk = 0, wm = 0, la = lane == WD ? lw : dump + lane * 4;
      __builtin_amdgcn_s_setprio(3);
#define MODE_ "s_cmp_eq_u32 %[wm], 1\n\ts_cselect_b64 vcc, -1, 0\n\ts_cmp_eq_u32 %[wm], 2\n\t"
#define LD_(off, id) "s_waitcnt vmcnt(%[wf])\n\ts_cbranch_vccnz .Lw" #id "_%=\n\ts_waitcnt vmcnt(%[w])\n\ts_cbranch_scc0 .Lw" #id "_%=\n\t" \
                     "s_waitcnt vmcnt(%[ws])\n.Lw" #id "_%=:\n\tglobal_load_lds_dwordx4 %[va], %[sb] offset:" #off " sc0 sc1\n\t"
#define SH_ "v_mov_b32_dpp %[mk], %[mk] row_shr:1 row_mask:0xf bank_mask:0xf\n\t"
#define PUB_ "ds_max_u32 %[la], %[mk]\n\t"
#define INC_(x) "v_add_u32 %[mk], %[mk], " x "\n\t"
#define WIN_ [wm] "s"(wm), [wf] "i"(WFAST - 1), [w] "i"(WD - 1), [ws] "i"(WD - 2)
#define FOUR(src, ldst, i0) do { u32 keep_; \
          asm volatile("ds_read_b32 %[dv], %[da]\n\tds_read_b32 %[wv], %[lw] offset:4\n\ts_mov_b32 %[k], m0\n\ts_mov_b32 m0, %[ld]\n\t" MODE_ \
                       LD_(0, 0) SH_ INC_("%[i]") PUB_ LD_(1024, 1) SH_ INC_("%[o]") PUB_ LD_(2048, 2) SH_ INC_("%[i]") PUB_ LD_(3072, 3) SH_ INC_("%[o]") \
                       "s_waitcnt lgkmcnt(3)\n\t" PUB_ "s_mov_b32 m0, %[k]" \
                       : [k] "=&s"(keep_), [dv] "=&v"(dvv), [wv] "=&v"(wvv), [mk] "+v"(mk) \
                       : [va] "v"((src) + lane * 16), [sb] "s"(pws), [ld] "s"(ldst), [la] "v"(la), [da] "v"(l0 + L_CTL + (C_DESC + se) * 4), \
                         [lw] "v"(lw), [i] "v"((i0) & one), [o] "v"(one), WIN_ : "memory", "scc", "vcc"); \
          prog += 2 + 2 * (i0); } while (0)
      u64 qu = 0;
      u32 ne = 0, se = 0, sc = 0, d = 0, dvv, wvv;
      bool fin = false;
      for (u32 n = 0;; ++n) {
#pragma clang loop unroll(disable)
        while (ne <= n + EA && !fin) {
          if ((d & 63) != ne + 1) {
            if (ne > n) break;
            d = rfl(ctl[C_DESC + se]);
            if ((d & 63) != ne + 1) {
              waitv<0>();
              if (lane == 0) ctl[C_LANDED] = prog;
              do { d = rfl(ctl[C_DESC + se]); } while ((d & 63) != ne + 1);
            }
          }
          fin = ((d >> 6) & 0x1ff) == IT_END;
          qu |= (u64)((d >> 20) | ((d >> 7) & 0x1000) | (fin ? 0x2000u : 0u)) << (16 * (ne - n));
          ++ne;
          if (!(d & 0x80000)) {
            const u32 ln = (d >> 15) & 15, pn = (prog & 0xffff) | (ne << 16);
            const u64 em = (ln & 7) ? (1ull << ((ln & 7) * 8)) - 1 : ~0ull;
            u32 keep;
            asm volatile("s_mov_b32 %[k], m0\n\ts_mov_b32 m0, %[ld]\n\t" MODE_
                         LD_(0, 0) SH_ PUB_
                         "s_cmp_gt_u32 %[ln], 8\n\ts_cbranch_scc1 .L3_%=\n\ts_cmp_eq_u32 %[wm], 2\n\ts_mov_b64 exec, %[em]\n\t" LD_(1024, 1) "s_branch .L4_%=\n"
                         ".L3_%=:\n\ts_cmp_eq_u32 %[wm], 2\n\t" LD_(1024, 2) SH_ PUB_ "s_mov_b64 exec, %[em]\n\t" LD_(2048, 3)
                         "\n.L4_%=:\n\ts_mov_b64 exec, -1\n\t" SH_ "v_writelane_b32 %[mk], %[pn], 0\n\t" PUB_ "s_mov_b32 m0, %[k]"
                         : [k] "=&s"(keep), [mk] "+v"(mk)
                         : [va] "v"((d >> 20) * ITEM + 4096 + lane * 16), [sb] "s"(pws), [ld] "s"(l0 + se * ITEM + 4096), [la] "v"(la),
                           [ln] "s"(ln), [em] "s"(em), [pn] "s"(pn), WIN_ : "memory", "scc", "vcc");
            prog = pn;
          }
          se = se + 1 == NS ? 0 : se + 1;
          if (n == 0) break;
          if (ne <= n + EA) d = rfl(ctl[C_DESC + se]);
        }
        const u32 cur = (u32)qu & 0xffff;
        qu >>= 16;
        if (cur & 0x2000) break;
        u32 so = (cur & 0xfff) * ITEM, lo = l0 + sc * ITEM;
#pragma clang loop unroll(disable)
        for (u32 k = cur >> 12;; --k) {
          FOUR(so, lo, 1 - (cur >> 12));
          if (k == 0) break;
          so += 4096; lo += 4096;
        }
        d = rfl(dvv);
        wm = rfl(wvv);
        la = lane == (wm == 1 ? WFAST : wm == 2 ? WD - 1 : WD) ? lw : dump + lane * 4;
        sc = sc + 1 == NS ? 0 : sc + 1;
      }
#define DRAIN(N) if (N < WFAST) \
        asm volatile("s_mov_b64 exec, %2\n\ts_waitcnt vmcnt(%3)\n\tds_max_u32 %0, %1\n\ts_mov_b64 exec, -1" \
                     :: "v"(lw), "v"(mk), "i"(1 << N), "i"(N) : "memory");
      DRAIN(9) DRAIN(8) DRAIN(7) DRAIN(6) DRAIN(5) DRAIN(4) DRAIN(3) DRAIN(2) DRAIN(1) DRAIN(0)
      return;
    }
    if (wave == 0 && lane < 4) touch(me + (lane == 3 ? UF_OFF + peer * NIT * 512 : RAW_OFF + (peer * NIT + lane * 143) * ITEM) + cw * 128, dump);
    drain();
    const int u = wave & 1, dw = wave;
    LDSP char* img = lds + L_IMG + wave * 4096;
    u32 m0f = 0x0f0f0f0fu, m80 = 0x80808080u, m3e = 0x3e3e3e3eu, m30 = 0x30303030u;
    u32 mffc = 0xffc;
    const u32 imga = (u32)(size_t)img;
    asm volatile("" : "+s"(m0f), "+s"(m80), "+s"(m3e), "+s"(m30), "+v"(mffc));
    for (u32 n = dw >> 1;; n += NGRP) {
      u32 w0;
      while (((w0 = rfl(ctl[C_DESC + n % NS])) & 63) != n + 1) __builtin_amdgcn_s_sleep(2);
      const u32 it = (w0 >> 6) & 0x1ff;
      if (it == IT_END) break;
      LDSP char* sl = lds + (n % NS) * ITEM;
      const u32 rlo = (w0 >> 20) - RL_OFF / ITEM;
      const u32 src = rlo < 128 ? (rlo >> 3) & 7 : peer;
      const bool keep = (it >> 6) == 2 && ((it >> 1) & 3) * 2 + (it & 1) == self;
      char* d = me + RAW_OFF + (src * NIT + it) * ITEM + u * 4096 + lane * 16;
      u32x4 o0, o1, o2, o3;
#define LANDED(m, x) do { while ((rfl(ctl[C_LANDED]) & (m)) < (x)) __builtin_amdgcn_s_sleep(1); fence(); } while (0)
      if (w0 & 0x80000) {
        LANDED(0xffffu, 4 * n + 2 * u + 2);
        LDSP char* s = sl + u * 4096 + lane * 16;
        o0 = *(LDSP u32x4*)s; o1 = *(LDSP u32x4*)(s + 1024); o2 = *(LDSP u32x4*)(s + 2048); o3 = *(LDSP u32x4*)(s + 3072);
        st16<0>(d, o0); st16<1024>(d, o1); st16<2048>(d, o2); st16<3072>(d, o3);
      } else {
        LANDED(0xffff0000u, (n + 1) << 16);
        const u32x2 hv = *(LDSP u32x2*)(sl + FIX + u * 8);
        const u32 h0 = rfl(hv.x), h1 = rfl(hv.y);
        const u32 nd0 = h0 >> 23, nd1 = h1 >> 23;
        const u32 b0 = ((h0 >> 10) & 0x7f) * 0x01010101u - 0x3e3e3e3eu, b1 = ((h1 >> 10) & 0x7f) * 0x01010101u - 0x3e3e3e3eu;
        LDSP char* v0 = sl + FIX + (h0 & 0x1ff) * 4;
        LDSP char* v1 = sl + FIX + (h1 & 0x1ff) * 4;
        u32 nb0 = *(LDSP u32*)(v0 + lane * 4), nb1 = *(LDSP u32*)(v1 + lane * 4);
        const u32 e0 = *(LDSP unsigned short*)(v0 + (nd0 & 0x7f) * 4 + lane * 2), e1 = *(LDSP unsigned short*)(v1 + (nd1 & 0x7f) * 4 + lane * 2);
        const u32 sg0 = *(LDSP u32*)(sl + 4096 + u * 512 + lane * 4), sg1 = *(LDSP u32*)(sl + 4096 + u * 512 + 256 + lane * 4);
        {
          LDSP char* iz = img + lane * 16;
          *(LDSP u32x4*)iz = u32x4{b0, b0, b0, b0}; *(LDSP u32x4*)(iz + 1024) = u32x4{b0, b0, b0, b0};
          *(LDSP u32x4*)(iz + 2048) = u32x4{b1, b1, b1, b1}; *(LDSP u32x4*)(iz + 3072) = u32x4{b1, b1, b1, b1};
        }
        if (lane >= nd0) nb0 = ~0u;
        if (lane >= nd1) nb1 = ~0u;
        u32 gg[4], vv[4];
        gg[0] = (nb0 & m0f) << 3; gg[1] = (nb0 >> 1) & 0x78787878u; gg[2] = (nb1 & m0f) << 3; gg[3] = (nb1 >> 1) & 0x78787878u;
        const u32 sum = __builtin_amdgcn_sad_u8(gg[0], 0, __builtin_amdgcn_sad_u8(gg[1], 0, 0)) |
                        (__builtin_amdgcn_sad_u8(gg[2], 0, __builtin_amdgcn_sad_u8(gg[3], 0, 0)) << 16);
#pragma unroll
        for (int k = 0; k < 4; ++k) {
          vv[k] = (~(gg[k] + 0x08080808u) & m80) >> 3;
          asm("" : "+v"(gg[k]), "+v"(vv[k]));
        }
        const u32 ps = scan_add(sum) - sum;
        u32 pa = ps & 0xffff, pb = (ps >> 16) + 2048 * 8;
        if (lane < max(nd0, nd1)) {
#pragma unroll
          for (int k = 0; k < 8; ++k) {
            const int w = k & 1, by = (k >> 1) * 8;
            pa += (gg[w] >> by) & 0xff;
            pb += (gg[2 + w] >> by) & 0xff;
            LDS_SUB((LDSP char*)(size_t)(((pa >> 3) & mffc) | imga), ((vv[w] >> by) & 0xff) << (pa & 31));
            LDS_SUB((LDSP char*)(size_t)(((pb >> 3) & mffc) | imga), ((vv[2 + w] >> by) & 0xff) << (pb & 31));
          }
        }
        fence();
        LDSP char* ip = img + lane * 32;
        u32x4 a0 = *(LDSP u32x4*)ip, a1 = *(LDSP u32x4*)(ip + 16), a2 = *(LDSP u32x4*)(ip + 2048), a3 = *(LDSP u32x4*)(ip + 2064);
        fence();
        if (lane < ((h0 >> 17) & 63)) LDS_SUB(img + (e0 & 0xffc), ((e0 >> 12) - 8) << ((e0 & 3) * 8 + 4));
        if (lane < ((h1 >> 17) & 63)) LDS_SUB(img + (e1 & 0xffc), ((e1 >> 12) - 8) << ((e1 & 3) * 8 + 4));
        fence();
        o0 = *(LDSP u32x4*)ip; o1 = *(LDSP u32x4*)(ip + 16); o2 = *(LDSP u32x4*)(ip + 2048); o3 = *(LDSP u32x4*)(ip + 2064);
#pragma unroll
        for (int q = 0; q < 4; ++q) {
          o0[q] += (sg0 << q) & m80; o1[q] += (sg0 << (4 + q)) & m80;
          o2[q] += (sg1 << q) & m80; o3[q] += (sg1 << (4 + q)) & m80;
          const u32 l0 = b0 - a0[q], l1 = b0 - a1[q], l2 = b1 - a2[q], l3 = b1 - a3[q];
          a0[q] = (l0 << 1) | (l0 >> 1); a1[q] = (l1 << 1) | (l1 >> 1); a2[q] = (l2 << 1) | (l2 >> 1); a3[q] = (l3 << 1) | (l3 >> 1);
        }
        asm volatile("" : "+v"(o0), "+v"(o1), "+v"(o2), "+v"(o3), "+v"(a0), "+v"(a1), "+v"(a2), "+v"(a3));
#define CODES(o, a, x) _Pragma("unroll") for (int q = 0; q < 4; ++q) { \
          u32 c_ = ((x) & m0f) + m3e; asm("" : "+v"(c_)); o[q] += c_ + (c_ & a[q] & m30); }
        LANDED(0xffffu, 4 * n + 2 * u + 1);
        const u32x4 n0 = *(LDSP u32x4*)(sl + u * 2048 + lane * 16);
        CODES(o0, a0, n0[q]) st16<0>(d, o0);
        CODES(o1, a1, n0[q] >> 4) st16<1024>(d, o1);
        LANDED(0xffffu, 4 * n + 2 * u + 2);
        const u32x4 n1 = *(LDSP u32x4*)(sl + u * 2048 + 1024 + lane * 16);
        CODES(o2, a2, n1[q]) st16<2048>(d, o2);
        CODES(o3, a3, n1[q] >> 4) st16<3072>(d, o3);
      }
      if (keep) {
        LANDED(0xffffu, 4 * n + 4);
        LDSP char* s = sl + u * 4096 + lane * 16;
        char* r = me + RL_OFF + (((call & 1) * NR + peer) * 8 + ((it >> 3) & 7)) * ITEM + u * 4096 + lane * 16;
        st16<0>(r, *(LDSP u32x4*)s); st16<1024>(r, *(LDSP u32x4*)(s + 1024));
        st16<2048>(r, *(LDSP u32x4*)(s + 2048)); st16<3072>(r, *(LDSP u32x4*)(s + 3072));
      }
      if (lane == 0) ctl[C_CONS + dw] = n + 1;
      waitv<0>();
      if (lane < 8) st16<0>(me + UF_OFF + (src * NIT + it) * 512 + u * 32 + (lane >> 1) * 128 + (lane & 1) * 16, u32x4{call, call, call, call});
      if (keep) put_flag(me + RLF_OFF + (peer * 8 + ((it >> 3) & 7)) * 128 + u * 32, call, lane);
    }
    return;
  }

  const int t = xcd * 4 + (ct >> 1);
  const int dbl = 1 - (ct >> 2);

  if (wave >= 4) {
    drain();
    const int src = wave - 4;
    if (lane == 0 && wave == 11) st_agent(cnt, call);
    const char* raw = me + RAW_OFF + (src != self || (call & 1) ? src : 8) * NIT * ITEM + lane * 16;
    const char* uf = me + UF_OFF + src * NIT * 512;
    const char* px = me + PX_OFF + ((call & 1) * NR + src) * 16 * ITEM + lane * 16;
    const u32 lA = (u32)(size_t)lds + L_A + src * 16384;
    const u64 t0 = tstart;
    u32 need = call;
    u32 prev = 0;
    const char* fa = uf + ((128 + ((lane >> 2) & 3) * 4 + mm) * 2 + ((lane >> 1) & 1)) * 512 + (xcd & 3) * 128 + (lane & 1) * 32;
    u64 xmiss = 0xffff;
    u32 xf = 0;
#pragma clang loop unroll(disable)
    for (int j = 0; j < 4; ++j) {
      {
        for (u32 it = 1; ((xmiss >> (4 * j)) & 15) != 0; ++it) {
          if (it > 1) __builtin_amdgcn_s_sleep(8);
          if ((xmiss >> lane) & 1) xf = ld_sys(fa);
          xmiss = __builtin_amdgcn_ballot_w64(xf < need) & 0xffff;
          if ((it & 15) == 0 && give_up(me, call, t0)) need = 0;
        }
        while (rfl(min4(*(volatile LDSP u32x4*)(ctl + C_FREE))) < prev) __builtin_amdgcn_s_sleep(2);
        const int hx = (j * 4 + mm) * 2;
        const char* sa0 = hx < NPUSH ? px + hx * ITEM : raw + (256 + hx) * ITEM;
        const char* sa1 = hx + 1 < NPUSH ? px + (hx + 1) * ITEM : raw + (257 + hx) * ITEM;
        u32 keep;
#pragma unroll
        for (int i = 0; i < 4; ++i)
          asm volatile("s_mov_b32 %0, m0\n\ts_mov_b32 m0, %2\n\ts_nop 0\n\t"
                       "global_load_lds_dwordx4 %1, off sc0 sc1\n\t"
                       "global_load_lds_dwordx4 %1, off offset:1024 sc0 sc1\n\t"
                       "global_load_lds_dwordx4 %1, off offset:2048 sc0 sc1\n\t"
                       "global_load_lds_dwordx4 %1, off offset:3072 sc0 sc1\n\t"
                       "s_mov_b32 m0, %0" : "=&s"(keep) : "v"((i < 2 ? sa0 : sa1) + (i & 1) * 4096), "s"(lA + i * 4096) : "memory");
      }
      const int nh0 = dbl ? 0 : (ct & 1);
      const char* fb = uf + ((j * 32 + t) * 2 + nh0 + ((lane >> 1) & dbl)) * 512 + mm * 128 + (lane & 1) * 32;
      if (j < 3)
        for (u32 it = 1; __builtin_amdgcn_ballot_w64(ld_sys(fb) < need) != 0; ++it) {
          if ((it & 15) == 0 && give_up(me, call, t0)) need = 0;
          __builtin_amdgcn_s_sleep(8);
        }
      const char* pp = raw + ((j * 32 + t) * 2 + nh0) * ITEM + 4096;
      u32x4 p[8], p2[8];
      u32 tv, tk, r0, r1;
      for (;;) {
        u32 it;
        asm volatile("s_mov_b32 %12, 96\n\ts_cmp_lt_u32 %19, %20\n\ts_cbranch_scc1 .Lg_%=\n\t"
                     "global_load_dword %10, %13, off sc0 sc1\n\ts_sleep 6\n\t"
                     "global_load_dword %11, %13, off sc0 sc1\n\ts_sleep 6\n"
                     ".Lp_%=:\n\t"
                     "global_load_dword %8, %13, off sc0 sc1\n\ts_waitcnt vmcnt(2)\n\t"
                     "v_cmp_gt_u32 vcc, %16, %10\n\ts_cbranch_vccz .Lg_%=\n\t"
                     "global_load_dword %10, %13, off sc0 sc1\n\ts_waitcnt vmcnt(2)\n\t"
                     "v_cmp_gt_u32 vcc, %16, %11\n\ts_cbranch_vccz .Lg_%=\n\t"
                     "global_load_dword %11, %13, off sc0 sc1\n\ts_waitcnt vmcnt(2)\n\t"
                     "v_cmp_gt_u32 vcc, %16, %8\n\ts_cbranch_vccz .Lg_%=\n\t"
                     "s_sub_u32 %12, %12, 1\n\ts_cbranch_scc0 .Lp_%=\n\t"
                     "s_waitcnt vmcnt(0)\n\ts_branch .Le_%=\n"
                     ".Lg_%=:\n\t"
                     "global_load_dwordx4 %0, %14, off offset:-4096 sc0 sc1\n\t"
                     "global_load_dwordx4 %1, %14, off offset:-3072 sc0 sc1\n\t"
                     "global_load_dwordx4 %2, %14, off offset:-2048 sc0 sc1\n\t"
                     "global_load_dwordx4 %3, %14, off offset:-1024 sc0 sc1\n\t"
                     "global_load_dwordx4 %4, %14, off sc0 sc1\n\t"
                     "global_load_dwordx4 %5, %14, off offset:1024 sc0 sc1\n\t"
                     "global_load_dwordx4 %6, %14, off offset:2048 sc0 sc1\n\t"
                     "global_load_dwordx4 %7, %14, off offset:3072 sc0 sc1\n\t"
                     "global_load_dword %8, %15, off sc1\n\t"
                     "v_mov_b32 %9, %17\n\ts_mov_b64 exec, 1\n\t"
                     "ds_add_rtn_u32 %9, %18, %9\n\ts_mov_b64 exec, -1\n\ts_waitcnt lgkmcnt(0)\n"
                     ".Le_%=:"
                     : "=&v"(p[0]), "=&v"(p[1]), "=&v"(p[2]), "=&v"(p[3]), "=&v"(p[4]), "=&v"(p[5]), "=&v"(p[6]), "=&v"(p[7]),
                       "=&v"(tv), "=&v"(tk), "=&v"(r0), "=&v"(r1), "=&s"(it)
                     : "v"(fb), "v"(pp), "v"(me + TMO_OFF + b * 128), "s"(need), "s"(1 + dbl),
                       "v"((u32)(size_t)(lds + L_CTL + C_TICKET * 4)), "s"(j), "i"(3) : "memory", "scc", "vcc");
        if (it != 0xffffffffu) break;
        if (give_up(me, call, t0)) need = 0;
      }
      asm volatile("s_cmp_eq_u32 %9, 0\n\ts_cbranch_scc1 .Ls_%=\n\t"
                   "global_load_dwordx4 %0, %8, off offset:-4096 sc0 sc1\n\t"
                   "global_load_dwordx4 %1, %8, off offset:-3072 sc0 sc1\n\t"
                   "global_load_dwordx4 %2, %8, off offset:-2048 sc0 sc1\n\t"
                   "global_load_dwordx4 %3, %8, off offset:-1024 sc0 sc1\n\t"
                   "global_load_dwordx4 %4, %8, off sc0 sc1\n\t"
                   "global_load_dwordx4 %5, %8, off offset:1024 sc0 sc1\n\t"
                   "global_load_dwordx4 %6, %8, off offset:2048 sc0 sc1\n\t"
                   "global_load_dwordx4 %7, %8, off offset:3072 sc0 sc1\n"
                   ".Ls_%=:"
                   : "=&v"(p2[0]), "=&v"(p2[1]), "=&v"(p2[2]), "=&v"(p2[3]), "=&v"(p2[4]), "=&v"(p2[5]), "=&v"(p2[6]), "=&v"(p2[7])
                   : "v"(pp + 8192), "s"(dbl) : "memory", "scc");
      u32 ts = rfl(tk) & 0xffff, fs = 0;
      const u32 t1 = (rfl(tk) >> 16) - 1;
#define SLOT(t) ({ const u32 k_ = (t) < t1 ? (t) % 3 : ((t) - t1) % 5; k_ < 3 ? L_B / 8192 + k_ : self * 2 + k_ - 3; })
#define NEEDF(t) ((t) + 8 - ((t) >= t1 && (t) - t1 >= 3 ? 4 : 2))
      u32 sl0 = SLOT(ts), sl1 = SLOT(ts + 1), nf0 = NEEDF(ts), nf1 = NEEDF(ts + 1);
      asm volatile("" : "+s"(sl0), "+s"(sl1), "+s"(nf0), "+s"(nf1));
#define ROOM(nf) while (fs < (nf)) { fs = rfl(min4(*(volatile LDSP u32x4*)(ctl + C_FREE))) + 8; if (fs < (nf)) __builtin_amdgcn_s_sleep(1); }
#define FULL(sl) do { ++ts; \
        asm volatile("" : "+s"(ts) :: "memory"); \
        if (lane == 0) ctl[C_FULL + ((ts - 1) & 7)] = ts | (src << 8) | ((sl) << 11) | (tv == call ? 0x10000u : 0u); } while (0)
#define PUT(R, sl) do { LDSP char* ub = lds + (sl) * 8192 + lane * 16; \
        _Pragma("unroll") for (int i = 0; i < 8; ++i) *(LDSP u32x4*)(ub + i * 1024) = R[i]; \
        FULL(sl); } while (0)
      ROOM(nf0);
      {
        LDSP char* ub = lds + sl0 * 8192 + lane * 16;
#define PUT2(i, N) asm volatile("s_waitcnt vmcnt(" #N ")" : "+v"(p[i]), "+v"(p[i + 1]) :: "memory"); \
        *(LDSP u32x4*)(ub + (i) * 1024) = p[i]; *(LDSP u32x4*)(ub + (i) * 1024 + 1024) = p[i + 1];
        PUT2(0, 7) PUT2(2, 5) PUT2(4, 3) PUT2(6, 1)
        asm volatile("s_waitcnt vmcnt(0)" : "+v"(p2[0]), "+v"(p2[1]), "+v"(p2[2]), "+v"(p2[3]), "+v"(p2[4]), "+v"(p2[5]), "+v"(p2[6]), "+v"(p2[7]),
                     "+v"(tv), "+v"(r0), "+v"(r1) :: "memory");
        FULL(sl0);
      }
      if (dbl) { ROOM(nf1); fence(); PUT(p2, sl1); }
      prev = ts;
    }
    if (src == self) {
      while (rfl(min4(*(volatile LDSP u32x4*)(ctl + C_FREE))) < prev) __builtin_amdgcn_s_sleep(2);
      if (lane == 0) {
        u32 o = ctl[C_TICKET];
        while (!__hip_atomic_compare_exchange_strong((LDSP u32*)(ctl + C_TICKET), &o, o | (((o + 2) / 3 * 3 + 1) << 16),
                                                     __ATOMIC_RELAXED, __ATOMIC_RELAXED, __HIP_MEMORY_SCOPE_WORKGROUP)) {}
      }
    }
    return;
  }

  const int mw = wave;
  {
    u32 scale_bits = __builtin_amdgcn_readfirstlane(__builtin_bit_cast(u32, *(const GLB float*)KA64(12) * *(const GLB float*)KA64(14)));
    float* obase = (float*)KA64(10) + (size_t)(mm * 128 + mw * 32) * NN + t * 128 + (ct & 1) * 64;
    asm volatile("" : "+s"(scale_bits), "+s"(obase));
    drain();
    v4f Pc[2][4], Qc[2][4];
#pragma unroll
    for (int i = 0; i < 2; ++i)
#pragma unroll
      for (int k = 0; k < 4; ++k) Pc[i][k] = Qc[i][k] = v4f{0.f, 0.f, 0.f, 0.f};
#define FRAG(p, hs) ({ const u32x4 lo_ = *(LDSP u32x4*)(p), hi_ = *(LDSP u32x4*)((p) + (hs)); \
      v8i{(int)lo_.x, (int)lo_.y, (int)lo_.z, (int)lo_.w, (int)hi_.x, (int)hi_.y, (int)hi_.z, (int)hi_.w}; })
#define MMA(i, k, F) do { F##c[i][k] = __builtin_amdgcn_mfma_scale_f32_16x16x128_f8f6f4(F##a[i], F##b[k], F##c[i][k], 1, 1, 0, 0, 0, 0); \
      __builtin_amdgcn_sched_barrier(0); } while (0)
#define SB() __builtin_amdgcn_sched_barrier(0)
#define RD(F) do { bad |= w; LDSP char* ap_ = lds + L_A + ((w >> 8) & 7) * 16384 + mw * 4096 + lane * 16; \
      bp = lds + ((w >> 11) & 31) * 8192 + lane * 16; \
      F##a[0] = FRAG(ap_, 1024); F##b[0] = FRAG(bp, 4096); SB(); F##a[1] = FRAG(ap_ + 2048, 1024); SB(); F##b[1] = FRAG(bp + 1024, 4096); \
      SB(); } while (0)
#define SPIN() do { while (((w = rfl(ctl[C_FULL + (n & 7)])) & 0xff) != n + 1) __builtin_amdgcn_s_sleep(0); \
      fence(); } while (0)
#define HEAD(F) do { \
      MMA(0, 0, F); F##b[2] = FRAG(bp + 2048, 4096); SB(); \
      MMA(1, 0, F); F##b[3] = FRAG(bp + 3072, 4096); \
      fence();     \
      ++n; \
      if (lane == 0) ctl[C_FREE + mw] = n; } while (0)
#define STEP(F, G, EXTRA) do { \
      HEAD(F); \
      peek = ctl[C_FULL + (n & 7)]; SB(); \
      MMA(0, 1, F); MMA(1, 1, F); MMA(0, 2, F); SB(); \
      w = rfl(peek); \
      fence(); \
      const bool ok_ = (w & 0xff) == n + 1; \
      if (ok_) RD(G); \
      MMA(1, 2, F); MMA(0, 3, F); MMA(1, 3, F); SB(); \
      EXTRA; \
      if (!ok_) { SPIN(); RD(G); } } while (0)
    LDSP char* bp;
    v8i Pa[2], Pb[4], Qa[2], Qb[4];
    u32 n = 0, w, peek, bad = 0;
    SPIN();
    RD(P);
#pragma clang loop unroll(disable)
    for (int it = 1; it < (16 << dbl); ++it) { STEP(P, Q, ); STEP(Q, P, ); }
#define EPI(A, i, r, o) do { v4f y_; _Pragma("unroll") for (int k = 0; k < 4; ++k) \
        y_[k] = __builtin_elementwise_maximum(A[i][k][r] * scale, 0.f);     \
      asm volatile("global_store_dwordx4 %0, %1, %2 sc1\n\ts_nop 1" :: "v"(ov), "v"(y_), "s"((o) + ((i) * 16 + (r)) * NN) : "memory"); \
      __builtin_amdgcn_sched_barrier(0); } while (0)
    STEP(P, Q, if (!dbl) { fence();
                 _Pragma("unroll") for (int i = 0; i < 2; ++i) _Pragma("unroll") for (int k = 0; k < 4; ++k) Qc[i][k] += Pc[i][k]; });
    HEAD(Q); SB();
    MMA(0, 1, Q); MMA(0, 2, Q); MMA(0, 3, Q); MMA(1, 1, Q);
    const float scale = __builtin_bit_cast(float, (bad & 0x10000u) != 0 ? 0x7fc00000u : scale_bits);
    const u32 ln = __builtin_amdgcn_mbcnt_hi(~0u, __builtin_amdgcn_mbcnt_lo(~0u, 0));
    const u32 ov = (ln >> 4) * (4 * NN * 4) + (ln & 15) * 16;
    SB();
    EPI(Qc, 0, 0, obase); MMA(1, 2, Q); EPI(Qc, 0, 1, obase); MMA(1, 3, Q); EPI(Qc, 0, 2, obase); EPI(Qc, 0, 3, obase);
    EPI(Qc, 1, 0, obase); EPI(Qc, 1, 1, obase); EPI(Qc, 1, 2, obase); EPI(Qc, 1, 3, obase);
    if (dbl) {
      EPI(Pc, 0, 0, obase - 64); EPI(Pc, 0, 1, obase - 64); EPI(Pc, 0, 2, obase - 64); EPI(Pc, 0, 3, obase - 64);
      EPI(Pc, 1, 0, obase - 64); EPI(Pc, 1, 1, obase - 64); EPI(Pc, 1, 2, obase - 64); EPI(Pc, 1, 3, obase - 64);
    }
  }
}

extern "C" void kernel_launch(int rank, int nranks, ncclComm_t comm,
                              hipStream_t stream,
                              void* const* d_in, const int* in_sizes, int n_in,
                              void* d_out, int out_size,
                              const int* in_dtypes, int out_dtype,
                              void* const* d_out_all,
                              void* const* d_ws_all, size_t ws_bytes) {
  if (nranks != NR || (unsigned)rank >= (unsigned)NR || n_in < 4 || ws_bytes < WS_NEED ||
      in_sizes[0] != 4096 * KS || in_sizes[1] != KS * NN || out_size != 512 * NN) {
    fprintf(stderr, "kernel_launch: unexpected problem shape\n");
    return;
  }
  Params p;
  p.me = (char*)d_ws_all[rank];
  p.ws_all = d_ws_all;
  p.x = (const unsigned char*)d_in[0];
  p.w = (const unsigned char*)d_in[1];
  p.sx = (const float*)d_in[2];
  p.sw = (const float*)d_in[3];
  p.out = (float*)d_out;
  p.rank = rank;
  static const int grp[2][4] = {{0, 2, 3, 6}, {1, 4, 5, 7}};
  int gi = 0, gp = 0;
  for (int a = 0; a < 2; ++a) for (int i = 0; i < 4; ++i) if (grp[a][i] == rank) { gi = a; gp = i; }
  p.dstmap = 0;
  for (int x = 0; x < 8; ++x) {
    p.dstmap |= (u32)(x < 4 ? grp[gi][(gp + 1 + x) & 3] : grp[1 - gi][(gp + x) & 3]) << (4 * x);
  }
  hipLaunchKernelGGL(fused_gemm_kernel, dim3(NWG), dim3(NTHREADS), 0, stream, p);
  HIP_CHECK(hipGetLastError());
}
```

```cpp
#include <hip/hip_runtime.h>
#include <rccl/rccl.h>
#include <cstdio>
#include <cstdint>

#define HIP_CHECK(expr) do { hipError_t _e = (expr); if (_e != hipSuccess) { \
    fprintf(stderr, "HIP error %s at %s:%d\n", hipGetErrorString(_e), __FILE__, __LINE__); \
    return; } } while (0)

typedef unsigned u32;
typedef unsigned long long u64;
typedef unsigned u32x2 __attribute__((ext_vector_type(2)));
typedef unsigned u32x4 __attribute__((ext_vector_type(4)));
typedef int v8i __attribute__((ext_vector_type(8)));
typedef float v4f __attribute__((ext_vector_type(4)));
#define LDSP __attribute__((address_space(3)))
#define GLB __attribute__((address_space(1)))

constexpr int NR = 8;
constexpr int KS = 512;
constexpr int NN = 4096;
constexpr int NTHREADS = 768;
constexpr int NWG = 256;
constexpr int NIT = 288;
#define WD 6
#define NPUSH 12
#define GATE 100

constexpr u32 ITEM = 8192;
constexpr u32 FIX = 5120;
constexpr u32 PK_OFF = 0;
constexpr u32 RAW_OFF = 2 * 512 * ITEM;
constexpr u32 PX_OFF = RAW_OFF + 9 * NIT * ITEM;
constexpr u32 RL_OFF = PX_OFF + 2 * NR * 16 * ITEM;
constexpr u32 DIR_OFF = RL_OFF + 2 * NR * 8 * ITEM;
constexpr u32 UF_OFF = DIR_OFF + 2 * NR * NIT * 128;
constexpr u32 CNT_OFF = UF_OFF + NR * NIT * 512;
constexpr u32 TMO_OFF = CNT_OFF + NWG * 12 * 128;
constexpr u32 BAL_OFF = TMO_OFF + NWG * 128;
constexpr u32 PF_OFF = BAL_OFF + NR * 256;
constexpr u32 PLAN_OFF = PF_OFF + NR * 256;
constexpr u32 RLF_OFF = PLAN_OFF + NR * 128;
constexpr u32 ROLE_OFF = RLF_OFF + NR * 8 * 128;
static_assert(DIR_OFF / ITEM <= 4096, "12-bit item offsets");
constexpr u32 WS_NEED = ROLE_OFF + 64 * 128;
constexpr u64 DEADLINE = 30000000ull;
static_assert(NPUSH <= 16 && NPUSH >= 0, "PX region");

constexpr int NBS = 3;
constexpr u32 L_A = 0;
constexpr u32 L_B = 8 * 16384;
constexpr int NGRP = 4;
constexpr int NS = 12;
constexpr u32 L_IMG = NS * ITEM;
constexpr u32 L_SCR = L_B + NBS * ITEM;
constexpr u32 L_CTL = L_SCR + 4096;
constexpr u32 L_DUMP = L_CTL + 256;
constexpr u32 L_BYTES = L_DUMP + 256;
constexpr int C_FREE = 0, C_FULL = 4, C_TICKET = 12, C_ENC = 13;
constexpr int C_CONS = 16, C_LANDED = 28, C_WIN = 29, C_NIT = 30, C_ROLE = 31, C_DESC = 32;
constexpr u32 M_RAW = 0x80, IT_END = 0x1ff;

#define PUSH1 8
#define PUSHD 130
#define EA 2
#define LA 8
#define TITEM 77
#define WFAST 8
#define GFAST 24
#define GSLOW 19
#define K0 4
#define SP 4
#define PLANT 2000
#define DZ 10.f

struct Params {
  char* me;
  const unsigned char* x;
  const unsigned char* w;
  void* const* ws_all;
  int rank;
  u32 dstmap;
  float* out;
  const float* sx;
  const float* sw;
};

template <int N> __device__ __forceinline__ void waitv() { asm volatile("s_waitcnt vmcnt(%0)" :: "i"(N) : "memory"); }
__device__ __forceinline__ void waitl() { asm volatile("s_waitcnt lgkmcnt(0)" ::: "memory"); }
__device__ __forceinline__ void bar() { asm volatile("s_barrier" ::: "memory"); }
__device__ __forceinline__ void drain() { __builtin_amdgcn_s_waitcnt(0x0070); }
__device__ __forceinline__ void fence() { asm volatile("" ::: "memory"); }
template <int OFF> __device__ __forceinline__ void st16(const void* p, u32x4 x) {
  asm volatile("global_store_dwordx4 %0, %1, off offset:%2 sc0 sc1\n\ts_nop 1" :: "v"(p), "v"(x), "i"(OFF) : "memory");
}
__device__ __forceinline__ void st4(const void* p, u32 x) {
  asm volatile("global_store_dword %0, %1, off sc0 sc1\n\ts_nop 1" :: "v"(p), "v"(x) : "memory");
}
__device__ __forceinline__ u32 ld_sys(const void* p) {
  return __hip_atomic_load((const GLB u32*)p, __ATOMIC_RELAXED, __HIP_MEMORY_SCOPE_SYSTEM);
}
__device__ __forceinline__ u32 ld_agent(const void* p) {
  return __hip_atomic_load((const GLB u32*)p, __ATOMIC_RELAXED, __HIP_MEMORY_SCOPE_AGENT);
}
__device__ __forceinline__ void st_agent(void* p, u32 v) {
  __hip_atomic_store((GLB u32*)p, v, __ATOMIC_RELAXED, __HIP_MEMORY_SCOPE_AGENT);
}
__device__ __forceinline__ void touch(const void* p, u32 dump) {
  u32 keep;
  asm volatile("s_mov_b32 %0, m0\n\ts_mov_b32 m0, %2\n\ts_nop 0\n\tglobal_load_lds_dword %1, off sc0 sc1\n\ts_mov_b32 m0, %0"
               : "=&s"(keep) : "v"(p), "s"(dump) : "memory");
}
__device__ __forceinline__ u32 rfl(u32 v) { return __builtin_amdgcn_readfirstlane(v); }
__device__ __forceinline__ void put_flag(char* f, u32 v, int lane) {
  if (lane < 2) st16<0>(f + lane * 16, u32x4{v, v, v, v});
}
__device__ __forceinline__ bool give_up(char* me, u32 call, u64 t0) {
  if (rfl(ld_agent(me + TMO_OFF + blockIdx.x * 128)) == call) return true;
  if (__builtin_amdgcn_s_memrealtime() - t0 <= DEADLINE) return false;
  for (int i = threadIdx.x & 63; i < NWG; i += 64) st_agent(me + TMO_OFF + i * 128, call);
  waitv<0>();
  return true;
}
#define LDS_ADD(p, v) __hip_atomic_fetch_add((LDSP u32*)(p), (v), __ATOMIC_RELAXED, __HIP_MEMORY_SCOPE_WORKGROUP)
#define LDS_SUB(p, v) __hip_atomic_fetch_sub((LDSP u32*)(p), (v), __ATOMIC_RELAXED, __HIP_MEMORY_SCOPE_WORKGROUP)
#define LDS_OR(p, v) __hip_atomic_fetch_or((LDSP u32*)(p), (v), __ATOMIC_RELAXED, __HIP_MEMORY_SCOPE_WORKGROUP)
#define DPP(old, v, ctrl, rm) (u32)__builtin_amdgcn_update_dpp((int)(old), (int)(v), ctrl, rm, 0xf, false)
#define SCAN(name, id, OP) __device__ __forceinline__ u32 name(u32 v) { \
    v = OP(v, DPP(id, v, 0x111, 0xf)); v = OP(v, DPP(id, v, 0x112, 0xf)); v = OP(v, DPP(id, v, 0x114, 0xf)); \
    v = OP(v, DPP(id, v, 0x118, 0xf)); v = OP(v, DPP(id, v, 0x142, 0xa)); v = OP(v, DPP(id, v, 0x143, 0xc)); return v; }
__device__ __forceinline__ u32 addu(u32 a, u32 b) { return a + b; }
SCAN(scan_add, 0u, addu)
SCAN(scan_max, 0u, max)
SCAN(scan_min, 0xffffffffu, min)

__device__ __forceinline__ u32 round_item(u32 rd, u32 col) { return ((rd >> 3) * 32 + 4 * col + ((rd >> 1) & 3)) * 2 + (rd & 1); }
__device__ __forceinline__ u32 sched_item(u32 q, u32 rel, u32 self) {
  if (q < 32 - NPUSH) return 256 + (q < 8 ? 16 + q : q < 24 - NPUSH ? NPUSH - 8 + q : NPUSH + q);
  q -= 32 - NPUSH;
  const u32 sh = 16 + self, ps = (sh & ~3u) | ((sh - rel) & 3);
  u32 r = q >> 3;
  r = r == SP ? ps : r > SP && r <= ps ? r - 1 : r;
  return round_item((r & ~3u) | ((r + rel) & 3), (q + rel) & 7);
}
__device__ __forceinline__ u32 nib_sum(u32 x) { return (((x & 0x0f0f0f0fu) + ((x >> 4) & 0x0f0f0f0fu)) * 0x01010101u) >> 24; }
__device__ __forceinline__ u32 lds_ticket(volatile LDSP u32* p, u32 v) {
  asm volatile("s_mov_b64 exec, 1\n\tds_add_rtn_u32 %0, %1, %0\n\ts_mov_b64 exec, -1\n\ts_waitcnt lgkmcnt(0)"
               : "+v"(v) : "v"((u32)(size_t)p) : "memory");
  return rfl(v);
}
__device__ __forceinline__ u32 min4(u32x4 v) { return min(min(v.x, v.y), min(v.z, v.w)); }
__device__ __forceinline__ u32 gather8(const u32* f) {
  const u32 r0 = __builtin_amdgcn_udot4(f[1], 0x80402010u, __builtin_amdgcn_udot4(f[0], 0x08040201u, 0, false), false);
  const u32 r1 = __builtin_amdgcn_udot4(f[3], 0x80402010u, __builtin_amdgcn_udot4(f[2], 0x08040201u, 0, false), false);
  const u32 r2 = __builtin_amdgcn_udot4(f[5], 0x80402010u, __builtin_amdgcn_udot4(f[4], 0x08040201u, 0, false), false);
  const u32 r3 = __builtin_amdgcn_udot4(f[7], 0x80402010u, __builtin_amdgcn_udot4(f[6], 0x08040201u, 0, false), false);
  return (r0 >> 7) | (r1 << 1) | (r2 << 9) | (r3 << 17);
}

__global__ void __launch_bounds__(NTHREADS) fused_gemm_kernel(Params P) {
  __shared__ __attribute__((aligned(16))) char smem[L_BYTES];
  LDSP char* lds = (LDSP char*)smem;
  volatile LDSP u32* ctl = (volatile LDSP u32*)(lds + L_CTL);
  const int lane = threadIdx.x & 63;
  const int wave = rfl(threadIdx.x >> 6);
  const int b = blockIdx.x, xcd = b & 7, slot = b >> 3;
  const int ct = slot >> 2, mm = slot & 3;
  typedef u32 u32x16 __attribute__((ext_vector_type(16)));
  u32x16 ka;
  asm volatile("s_load_dwordx16 %0, %1, 0x0" : "=&s"(ka) : "s"(__builtin_amdgcn_kernarg_segment_ptr()) : "memory");
  const bool link = (ct & 5) == 0;
  const int i8 = mm + (ct < 4 ? ct >> 1 : ct & 1) * 4;
  const bool xjob = !link & ((wave == 4) | ((wave == 5) & (ct >= 6)));
  const int xj = wave == 5 ? 1 : ct < 4 ? 0 : ct >> 1;
  u32 cno = CNT_OFF + (b * 12 + wave) * 128;
  u32 xv0 = (i8 * 64 + (lane & 15)) * KS + xj * 128 + (lane >> 4) * 16, xv1 = xv0 + 16 * KS, xv2 = xv0 + 32 * KS, xv3 = xv0 + 48 * KS;
  asm volatile("s_waitcnt lgkmcnt(0)" : "+s"(ka), "+v"(cno), "+v"(xv0), "+v"(xv1), "+v"(xv2), "+v"(xv3) :: "memory");
#define KA64(i) (ka[i] | ((u64)ka[(i) + 1] << 32))
  char* const me = (char*)KA64(0);
  const int self = ka[8];
#define LOAD_PWS() char* pws; asm volatile("s_load_dwordx2 %0, %1, %2\n\ts_waitcnt lgkmcnt(0)" : "=&s"(pws) : "s"(KA64(6)), "s"(peer * 8) : "memory")
  u32* cnt = (u32*)(me + CNT_OFF + (b * 12 + wave) * 128);
  const u32 dump = (u32)(size_t)lds + L_DUMP;
  u32 call;
  const u64 tstart = __builtin_amdgcn_s_memrealtime();

  if (__builtin_expect(xjob, 1)) {
    const int hx = xj * 8 + i8, peer = (ka[9] >> (4 * xcd)) & 7;
    if (xj == 3) __builtin_amdgcn_s_sleep(15);
    u32x4 d[8];
    u32 cv;
    asm volatile("global_load_dword %8, %11, %10 sc1\n\t"
                 "global_load_dwordx4 %0, %12, %9\n\tglobal_load_dwordx4 %1, %12, %9 offset:64\n\t"
                 "global_load_dwordx4 %2, %13, %9\n\tglobal_load_dwordx4 %3, %13, %9 offset:64\n\t"
                 "global_load_dwordx4 %4, %14, %9\n\tglobal_load_dwordx4 %5, %14, %9 offset:64\n\t"
                 "global_load_dwordx4 %6, %15, %9\n\tglobal_load_dwordx4 %7, %15, %9 offset:64"
                 : "=&v"(d[0]), "=&v"(d[1]), "=&v"(d[2]), "=&v"(d[3]), "=&v"(d[4]), "=&v"(d[5]), "=&v"(d[6]), "=&v"(d[7]), "=&v"(cv)
                 : "s"(KA64(2) + peer * (512 * KS)), "s"(me), "v"(cno), "v"(xv0), "v"(xv1), "v"(xv2), "v"(xv3) : "memory");
    LOAD_PWS();
    const bool push = hx < NPUSH, direct = push || peer == self;
    const u32 o0 = push ? PX_OFF + (self * 16 + hx) * ITEM : direct ? RAW_OFF + (8 * NIT + 256 + hx) * ITEM : PK_OFF + (256 + peer * 32 + hx) * ITEM;
    const u32 o1 = push ? NR * 16 * ITEM : direct ? (self - 8) * NIT * ITEM : 512 * ITEM;
    const u32 f0 = direct ? UF_OFF + (self * NIT + 256 + hx) * 512 : DIR_OFF + (self * NIT + 256 + hx) * 128;
    const u32 f1 = direct ? 0 : NR * NIT * 128;
    const char* db = push ? pws : me;
    const u64 fm = direct ? 0xffff : 3;
    asm volatile("s_waitcnt vmcnt(8)" : "+v"(cv) :: "memory");
    call = rfl(cv) + 1;
    const u32 par = call & 1;
    const u32 dv = o0 + par * o1 + 4096 + lane * 16, fv = f0 + par * f1 + (lane >> 2) * 128 + (lane & 3) * 16;
    const u32 v = direct ? call : (call << 8) | M_RAW;
    if (push && hx >= PUSH1) while (__builtin_amdgcn_s_memrealtime() - tstart < PUSHD) __builtin_amdgcn_s_sleep(1);
    asm volatile("s_waitcnt vmcnt(7)\n\tglobal_store_dwordx4 %8, %0, %9 offset:-4096 sc0 sc1\n\t"
                 "s_waitcnt vmcnt(7)\n\tglobal_store_dwordx4 %8, %1, %9 offset:-3072 sc0 sc1\n\t"
                 "s_waitcnt vmcnt(7)\n\tglobal_store_dwordx4 %8, %2, %9 offset:-2048 sc0 sc1\n\t"
                 "s_waitcnt vmcnt(7)\n\tglobal_store_dwordx4 %8, %3, %9 offset:-1024 sc0 sc1\n\t"
                 "s_waitcnt vmcnt(7)\n\tglobal_store_dwordx4 %8, %4, %9 sc0 sc1\n\t"
                 "s_waitcnt vmcnt(7)\n\tglobal_store_dwordx4 %8, %5, %9 offset:1024 sc0 sc1\n\t"
                 "s_waitcnt vmcnt(7)\n\tglobal_store_dwordx4 %8, %6, %9 offset:2048 sc0 sc1\n\t"
                 "s_waitcnt vmcnt(7)\n\tglobal_store_dwordx4 %8, %7, %9 offset:3072 sc0 sc1\n\ts_nop 1"
                 : "+v"(d[0]), "+v"(d[1]), "+v"(d[2]), "+v"(d[3]), "+v"(d[4]), "+v"(d[5]), "+v"(d[6]), "+v"(d[7])
                 : "v"(dv), "s"(db) : "memory");
    asm volatile("s_mov_b64 exec, %3\n\ts_waitcnt vmcnt(0)\n\tglobal_store_dwordx4 %0, %1, %2 sc0 sc1\n\ts_mov_b64 exec, -1\n\ts_nop 0"
                 :: "v"(fv), "v"(u32x4{v, v, v, v}), "s"(pws), "s"(fm) : "memory");
    bar();
  } else if (!link && (wave < 4 || (wave >= 8 && ct < 4))) {
    {
      const int g = lane >> 4, c = lane & 15, j2 = wave >> 3, jw = ct < 4 ? 2 + j2 : (ct - 4) >> 1;
      const int u = wave & 1, hf = (wave >> 1) & 1, blk = (jw * 32 + xcd * 4) * 2 + i8;
      const GLB unsigned char* src = (const GLB unsigned char*)KA64(4) + (size_t)(jw * 128 + u * 64 + g * 16) * NN + xcd * 512 + i8 * 64 + c * 4;
      u32 W[16], D[8];
      if (jw == 1) __builtin_amdgcn_s_sleep(30);
      if (jw >= 2) __builtin_amdgcn_s_sleep(60);
#pragma unroll
      for (int r = 0; r < 16; ++r) W[r] = *(const GLB u32*)(src + r * NN);
      const u32 cv = ld_agent(cnt);
      LDSP char* scr = lds + L_SCR + (wave - j2 * 4) * 512;
      *(LDSP u32x2*)(scr + lane * 8) = u32x2{0, 0};
#pragma unroll
      for (int s = 0; s < 8; ++s) LDS_ADD(scr + (((W[s >> 2] >> ((s & 3) * 8)) & 0x7f) << 2), 1u);
      const u32x2 hh = *(volatile LDSP u32x2*)(scr + lane * 8);
      *(volatile LDSP u32x2*)(scr + lane * 8) = u32x2{0, 0};
      const u32 sel = 0x05010400u + hf * 0x02020202u;
#pragma unroll
      for (int q = 0; q < 4; ++q) {
        const u32 t0 = __builtin_amdgcn_perm(W[4 * q + 1], W[4 * q], sel), t2 = __builtin_amdgcn_perm(W[4 * q + 3], W[4 * q + 2], sel);
        D[q] = __builtin_amdgcn_perm(t2, t0, 0x05040100u); D[4 + q] = __builtin_amdgcn_perm(t2, t0, 0x07060302u);
      }
      call = rfl(cv) + 1;
      const u32 par = call & 1;
      char* pk = me + PK_OFF + (par * 512 + blk) * ITEM;
      {
        char* d = me + RAW_OFF + ((par ? self : 8) * NIT + blk) * ITEM + u * 4096 + hf * 2048 + lane * 16;
        st16<0>(d, u32x4{D[0], D[1], D[2], D[3]}); st16<1024>(d, u32x4{D[4], D[5], D[6], D[7]});
        u32 s0 = 0;
#pragma unroll
        for (int k = 0; k < 8; ++k) s0 |= (D[k] & 0x80808080u) >> k;
        st4(pk + 4096 + u * 512 + hf * 256 + lane * 4, s0);
      }
      u32 base;
      {
        const u32 c0 = scan_add(hh.x + hh.y) - hh.x - hh.y, c1 = c0 + hh.x;
        u32 u8 = (u32)__builtin_amdgcn_ds_bpermute(((lane + 8) & 63) * 4, (int)(c0 | (c1 << 16)));
        u32 u9 = (u32)__builtin_amdgcn_ds_bpermute(((lane + 9) & 63) * 4, (int)(c0 | (c1 << 16)));
        if (lane >= 56) u8 = 0x02000200u;
        if (lane >= 55) u9 = 0x02000200u;
        const u32 cs = (c0 | (c1 << 16)) + 0x08000800u - u8 - 3 * u9;
        const u32 k0 = ((cs & 0xffff) << 7) | (2 * lane), k1 = ((cs >> 16) << 7) | (2 * lane + 1);
        base = __builtin_amdgcn_readlane(scan_min(lane < 56 ? min(k0, k1) : lane == 56 ? k0 : 0xffffffffu), 63) & 0x7f;
      }
      const u32 b8f = base * 0x01010101u + 0x8f8f8f8fu;
      u32 F1[8], F2[8], FE[8], T[8];
#pragma unroll
      for (int k = 0; k < 8; ++k) {
        const u32 t = T[k] = b8f - (D[k] & 0x7f7f7f7fu);
        const u32 t1 = t << 1, t2 = t << 2, t3 = t << 3, wr = ((t & 0x0f0f0f0fu) + 0x02020202u) << 3;
        const u32 h = t & ~t1 & 0x80808080u, k1 = h & ~t2 & t3, k2 = h & t2 & ~t3, km = ~t & t1 & t2 & t3 & 0x80808080u;
        F2[k] = (wr & k1) | (~wr & k2);
        F1[k] = k1 | (wr & km) | (~wr & k2);
        FE[k] = ((h & ~t2 & ~t3) | F1[k]) ^ 0x80808080u;
      }
      {
        u32x4 cd;
#pragma unroll
        for (int q = 0; q < 4; ++q) cd[q] = ~((T[q] & 0x0f0f0f0fu) | ((T[4 + q] << 4) & 0xf0f0f0f0u));
        st16<0>(pk + u * 2048 + hf * 1024 + lane * 16, cd);
      }
      u32 x1 = gather8(F1), x2 = gather8(F2), xe = gather8(FE);
      const u64 any = __builtin_amdgcn_ballot_w64(x1 != 0);
      int prev;
      {
        const u32 lastp = x1 ? lane * 32 + 31 - __builtin_clz(x1) : 0;
        const u32 back = lane * 32 - scan_max(DPP(0, lastp, 0x138, 0xf));
        prev = -(int)(back - 15 * ((back * 4370) >> 16));
      }
      u32 a0 = 0, a1 = 0, a2 = 0, nc = 0;
#define PUSH_NIB(gg) do { a0 = __builtin_amdgcn_alignbit(a1, a0, 4); a1 = __builtin_amdgcn_alignbit(a2, a1, 4); \
        a2 = __builtin_amdgcn_alignbit((gg), a2, 4); ++nc; } while (0)
      while (x1 != 0) {
        const u32 pos = __builtin_ctz(x1);
        const u32 gap = pos - prev, gg = min(gap, 15u);
        prev += gg;
        const u32 bit = gap < 15 ? 1u << pos : 0u, in2 = x2 & bit;
        x2 ^= in2;
        x1 ^= bit ^ in2;
        PUSH_NIB(gg);
      }
      if ((any >> lane) > 1) while (prev <= 17) { prev += 15; PUSH_NIB(15u); }
      const u32 nend = scan_add(nc), nn = __builtin_amdgcn_readlane(nend, 63);
      bool over = nn > 512 || __builtin_amdgcn_ballot_w64(nc > 24) != 0;
      if (!over) {
        const u32 sh = (8 - (nend & 7)) * 4;
        LDSP char* q = scr + (nend >> 3) * 4;
        LDS_OR(q, (u32)((((u64)a0) << 32) >> sh));
        LDS_OR(q + 4, (u32)((((u64)a1 << 32) | a0) >> sh));
        LDS_OR(q + 8, (u32)((((u64)a2 << 32) | a1) >> sh));
        LDS_OR(q + 12, (u32)(((u64)a2) >> sh));
      }
      u32 ne = 0;
      {
        LDSP unsigned short* lst = (LDSP unsigned short*)(scr + 384);
        u64 mk;
        while ((mk = __builtin_amdgcn_ballot_w64(xe != 0)) != 0 && ne <= 63) {
          if (xe != 0) {
            const u32 pos = __builtin_ctz(xe);
            const u32 wa = (pos & 4) ? T[1] : T[0], wb = (pos & 4) ? T[3] : T[2], wc = (pos & 4) ? T[5] : T[4], wd = (pos & 4) ? T[7] : T[6];
            const u32 we = (pos & 8) ? wb : wa, wf = (pos & 8) ? wd : wc, wg = (pos & 16) ? wf : we;
            const u32 x4 = (wg >> ((pos & 3) * 8 + 4)) & 15;
            const u32 at = ne + __builtin_amdgcn_mbcnt_hi((u32)(mk >> 32), __builtin_amdgcn_mbcnt_lo((u32)mk, 0));
            if (at < 64) lst[at] = (unsigned short)((x4 << 12) | (hf * 2048 + lane * 32 + pos));
            xe &= xe - 1;
          }
          ne += __builtin_popcountll(mk);
        }
      }
      over = over || ne > 63;
      bar();
      if (!over) {
        const u32 ed = (ne + 1) >> 1, nd = (nn + 7) >> 3;
        const u32 ev = *(volatile LDSP u32*)(scr + 384 + (lane & 31) * 4);
        u32 nv = *(volatile LDSP u32*)(scr + 12 + lane * 4);
        const u32 vo = (lds_ticket(ctl + C_ENC + j2, ed + nd) & 0xffff) + 4;
        if (lane == (nn >> 3)) nv |= ~0u << ((nn & 7) * 4);
        if (lane < nd) st4(pk + FIX + (vo + lane) * 4, nv);
        if (lane < ed) st4(pk + FIX + (vo + nd + lane) * 4, ev);
        if (lane == 0) st4(pk + FIX + (u * 2 + hf) * 4, vo | (base << 10) | (ne << 17) | (nd << 23));
      }
      u32x16 pw;
      asm volatile("s_load_dwordx16 %0, %1, 0x0\n\ts_waitcnt vmcnt(0) lgkmcnt(0)" : "=&s"(pw) : "s"(KA64(6)) : "memory");
#define PW(i) "s"(pw[2 * (i)] | ((u64)pw[2 * (i) + 1] << 32))
      const u32 k = lds_ticket(ctl + C_ENC + j2, over ? 0x1010000u : 0x1000000u);
      if ((k >> 24) == 3) {
        const u32 v = (call << 8) | ((over || (k & 0xff0000) != 0) ? M_RAW : ((k & 0xffff) + 4 + 31) >> 5);
        asm volatile("s_mov_b64 exec, 3\n\t"
                     "global_store_dwordx4 %0, %1, %2 sc0 sc1\n\tglobal_store_dwordx4 %0, %1, %3 sc0 sc1\n\t"
                     "global_store_dwordx4 %0, %1, %4 sc0 sc1\n\tglobal_store_dwordx4 %0, %1, %5 sc0 sc1\n\t"
                     "global_store_dwordx4 %0, %1, %6 sc0 sc1\n\tglobal_store_dwordx4 %0, %1, %7 sc0 sc1\n\t"
                     "global_store_dwordx4 %0, %1, %8 sc0 sc1\n\tglobal_store_dwordx4 %0, %1, %9 sc0 sc1\n\t"
                     "s_mov_b64 exec, -1\n\ts_nop 0"
                     :: "v"(DIR_OFF + ((par * NR + self) * NIT + blk) * 128 + lane * 16), "v"(u32x4{v, v, v, v}),
                        PW(0), PW(1), PW(2), PW(3), PW(4), PW(5), PW(6), PW(7) : "memory");
        if (lane < 16) st16<0>(me + UF_OFF + (self * NIT + blk) * 512 + (lane >> 2) * 128 + (lane & 3) * 16, u32x4{call, call, call, call});
      }
    }
  } else {
    if (link && wave == 8) return;
    const u32 cv = ld_agent(cnt);
    if (wave == 7) {
      u32 role = 0;
      if (link) {
        asm volatile("s_getreg_b32 %0, hwreg(HW_REG_XCC_ID)" : "=s"(role));
        role = (role & 7) * 8 + i8;
        for (int i = 0; i < 128; ++i) {
          GLB u32* rp = (GLB u32*)(me + ROLE_OFF + role * 128);
          u32 old = 0;
          if (lane == 0) old = __hip_atomic_fetch_add(rp, 1u, __ATOMIC_RELAXED, __HIP_MEMORY_SCOPE_AGENT);
          if (rfl(old) == rfl(cv)) break;
          if (lane == 0) __hip_atomic_fetch_sub(rp, 1u, __ATOMIC_RELAXED, __HIP_MEMORY_SCOPE_AGENT);
          role = (role + 1) & 63;
        }
      }
      ctl[lane] = lane == C_ROLE ? role : 0;
      waitl();
    }
    bar();
    call = rfl(cv) + 1;
  }
  if (lane == 0 && wave != 11) st_agent(cnt, call);

  if (link) {
    const u32 role = rfl(ctl[C_ROLE]), lx = role >> 3, cw = role & 7;
    const int peer = (ka[9] >> (4 * lx)) & 7;
    if (peer == self) {
      if (lane == 0 && wave == 11) st_agent(cnt, call);
      if (cw != 0 || wave != 0) return;
      const char* pf = me + PF_OFF + lane * 32;
      u32 f;
      bool all;
      do {
        __builtin_amdgcn_s_sleep(16);
        f = ld_agent(pf);
        all = __builtin_amdgcn_ballot_w64((f & 0xfff) != (((call & 63) << 6) | 1) && (lane >> 3) != lx) == 0;
      } while (!all && __builtin_amdgcn_s_memrealtime() - tstart < PLANT);
      float v = (lane >> 3) == lx ? 0.f : (float)((int)(f - ((u32)tstart << 12)) >> 12);
      v += __shfl_xor(v, 1); v += __shfl_xor(v, 2); v += __shfl_xor(v, 4);
      v *= 0.125f;
      float mu = v + __shfl_xor(v, 8);
      mu += __shfl_xor(mu, 16); mu += __shfl_xor(mu, 32);
      mu *= 1.f / 7;
      u32 inv = 0;
#pragma unroll
      for (int x = 0; x < NR; ++x) inv |= x << (4 * ((ka[9] >> (4 * x)) & 7));
      const int ps = lane >> 3, pr = lane & 7;
      const float fs = __shfl(v, ((inv >> (4 * ps)) & 7) * 8), fr = __shfl(v, ((inv >> (4 * pr)) & 7) * 8);
      const float ex = ps == self || fs - mu < DZ ? 0.f : fs - mu, gp = pr == self ? 0.f : fmaxf(mu - fr, 0.f);
      float gs = gp + __shfl_xor(gp, 1);
      gs += __shfl_xor(gs, 2); gs += __shfl_xor(gs, 4);
      float mv = gs > 0.f ? fminf((float)TITEM, ex * gp / gs) : 0.f;
#pragma clang loop unroll(disable)
      for (int i = 0; i < 4; ++i) {
        float out = mv + __shfl_xor(mv, 1), in = mv + __shfl_xor(mv, 8);
        out += __shfl_xor(out, 2); out += __shfl_xor(out, 4);
        in += __shfl_xor(in, 16); in += __shfl_xor(in, 32);
        const float ns = fs - out + __shfl(in, ps), nr = fr + in - __shfl(out, pr * 8);
        if (ps != self && pr != self && ps != pr && ns - mu > 2 * DZ && ns > nr) mv = fminf((float)TITEM, mv + (ns - nr) * 0.2f);
      }
      u32 row = all ? (u32)(mv * (8.f / TITEM) + 0.5f) << (4 * pr) : 0;
      row |= __shfl_xor(row, 1); row |= __shfl_xor(row, 2); row |= __shfl_xor(row, 4);
      row = __shfl(row, pr * 8);
      __hip_atomic_store((GLB u64*)(me + PLAN_OFF + ps * 128 + pr * 8), row | ((u64)call << 32), __ATOMIC_RELAXED, __HIP_MEMORY_SCOPE_AGENT);
      return;
    }
    if (wave == 11) {
      LOAD_PWS();
      const u32 par = call & 1;
      u32 nitems = (NIT - NPUSH - cw + 7) >> 3;
      u32 itv = lane < nitems ? sched_item(cw + lane * 8, lx, self) : 0;
      const char* dir = me + DIR_OFF + ((par * NR + peer) * NIT + itv) * 128;
      u32 dv = 0;
      u64 todo = (1ull << nitems) - 1, ok = 0;
      bool have = false;
      while (__builtin_amdgcn_s_memrealtime() - tstart < GATE - 50) __builtin_amdgcn_s_sleep(1);
      for (u32 n = 0; n < nitems + NGRP; ++n) {
        u32 w0 = (IT_END << 6) | 0x80000;
        if (n < nitems) {
          const u32 first = __builtin_ctzll(todo);
          const u64 cand = todo & (((n < 8 ? 7ull : 1ull) << first) | (0xffull << 56));
          const bool rl = (todo >> 56) != 0 && ((ok & todo) >> 56) == 0;
          for (u32 sp = 1; (ok & cand) == 0 || (rl && sp == 1); ++sp) {
            if (sp > 1 && n > 2) __builtin_amdgcn_s_sleep(4);
            if (lane - first < (n < 8 ? 4u : 64u) && ((todo & ~ok) >> lane) & 1) {
              dv = ld_sys(dir);
              if (lane >= 56) {
                const char* rf = pws + RLF_OFF + ((lane - 56) * 8 + ((itv >> 3) & 7)) * 128;
                if (min(ld_sys(rf), ld_sys(rf + 32)) < call) dv = 0;
              }
            }
            ok = __builtin_amdgcn_ballot_w64((dv >> 8) == (call & 0xffffff));
            if ((sp & 31) == 0 && give_up(me, call, tstart)) { dv = (call << 8) | M_RAW; ok = ~0ull; }
          }
          const u64 pick = ok & cand;
          const u32 ln = (pick >> 56) != 0 ? 63 - __builtin_clzll(pick) : __builtin_ctzll(pick);
          todo &= ~(1ull << ln);
          const u32 it = __builtin_amdgcn_readlane(itv, ln);
          u32 e = __builtin_amdgcn_readlane(dv, ln);
          e &= (e & M_RAW) ? M_RAW : 0xf;
          const u32 so = ln >= 56 ? RL_OFF / ITEM + (par * NR + ln - 56) * 8 + ((it >> 3) & 7)
                         : it >= 256 ? par * 512 + self * 32 + it : (e & M_RAW) ? RAW_OFF / ITEM + (par ? peer : 8) * NIT + it : par * 512 + it;
          w0 = (it << 6) | ((e & M_RAW) ? 0x80000 : e << 15) | (so << 20);
        }
        if (n >= LA) {
          volatile LDSP u32x2* cp = (volatile LDSP u32x2*)(ctl + C_CONS + (n % NGRP) * 2);
          for (;;) { const u32x2 cv = *cp; if (rfl(min(cv.x, cv.y)) + LA > n) break; __builtin_amdgcn_s_sleep(2); }
        }
        if (lane == 0) ctl[C_DESC + n % NS] = w0 | (n + 1);
        if (!have && n >= 17 && (n & 1)) {
          u64 pv;
          for (u32 sp = 1;; ++sp) {
            pv = __hip_atomic_load((const GLB u64*)(me + PLAN_OFF + lx * 128 + (lane & 7) * 8), __ATOMIC_RELAXED, __HIP_MEMORY_SCOPE_AGENT);
            have = __builtin_amdgcn_ballot_w64((u32)(pv >> 32) != call) == 0;
            if (have || n < 27) break;
            if ((sp & 31) == 0 && give_up(me, call, tstart)) { pv = 0; have = true; break; }
          }
          if (have) {
            const u32 row = (u32)pv, mine = __builtin_amdgcn_readlane(row, peer);
            const u32 r = ((itv >> 1) & 3) * 2 + (itv & 1), col = (itv >> 3) & 7;
            const u64 skip = __builtin_amdgcn_ballot_w64((itv >> 6) == 2 && ((col - nib_sum(mine & ((1u << (4 * r)) - 1))) & 7) < ((mine >> (4 * r)) & 15));
            const u32 nt = lane >= 56 ? (row >> (4 * peer)) & 15 : 0, kk = (cw - (scan_add(nt) - nt)) & 7;
            const u64 take = __builtin_amdgcn_ballot_w64(kk < nt);
            if (lane >= 56) {
              itv = round_item(16 + peer, (nib_sum(row & ((1u << (4 * peer)) - 1)) + kk) & 7);
              dir = me + DIR_OFF + ((par * NR + lane - 56) * NIT + itv) * 128;
            }
            todo = (todo & ~skip) | take;
            nitems = n + 1 + __builtin_popcountll(todo);
            if (lane == 0) ctl[C_NIT] = nitems;
          }
        }
      }
      if (lane == 0) st_agent(cnt, call);
      return;
    }
    if (wave == 9) {
      const u32 n0 = (NIT - NPUSH - cw + 7) >> 3;
      char* bp = me + BAL_OFF + lx * 256;
      u32 until = 4 * K0;
      for (u32 r = 0;; ++r) {
        u32 p0, p1;
        while (((p0 = rfl(ctl[C_LANDED])) & 0xffff) < until) __builtin_amdgcn_s_sleep(4);
        for (int i = 0; (p1 = rfl(ctl[C_LANDED])) == p0 && i < 12; ++i) {}
        const u32 now = (u32)__builtin_amdgcn_s_memrealtime(), nn = rfl(ctl[C_NIT]), nitems = nn ? nn : n0, hu = p1 & 0xffff;
        const u32 fown = (now + ((nitems * 23 - hu * 4 - (p1 >> 16) * 7) * TITEM) / 23) << 12 | ((call & 63) << 6) | (nn ? 2 : 1);
        if (lane == 0) st_agent(bp + cw * 32, fown);
        if (r == 0) {
          if (lane == 0) st_agent(bp + PF_OFF - BAL_OFF + cw * 32, fown);
          until = 64;
          continue;
        }
        if (r == 1) { until = min(hu + 6, 4 * (n0 - 8)); continue; }
        if (hu + 12 >= 4 * nitems) break;
        const u32 f = ld_agent(bp + (lane & 7) * 32);
        const bool v = ((f ^ fown) & 0xfff) == 0;
        const int cnt8 = __builtin_popcount((u32)__builtin_amdgcn_ballot_w64(v) & 0xff);
        const int late = cnt8 < 6 ? 0 : (int)__builtin_amdgcn_readlane(scan_add(v ? (u32)((int)(fown - f) >> 12) : 0u), 7) / cnt8;
        const u32 plan = min((u32)(late > 0 ? (late + GFAST / 4) / GFAST : (GSLOW / 4 - late) / GSLOW), 4u);
        until = (rfl(ctl[C_LANDED]) & 0xffff) + 4 * plan;
        if (plan != 0) {
          if (lane == 0) ctl[C_WIN] = late > 0 ? 1 : 2;
          until = min(until, 4 * (nn ? nn : n0 - 8));
          while ((rfl(ctl[C_LANDED]) & 0xffff) < until) __builtin_amdgcn_s_sleep(4);
          if (lane == 0) ctl[C_WIN] = 0;
        }
        until = min(until + 12, 4 * (nn ? nn : n0 - 8));
      }
      return;
    }
    if (wave == 10) {
      LOAD_PWS();
      if (lane < 3) touch(pws + PK_OFF + ((call & 1) * 512 + (lane == 2 ? 256 + self * 32 + 16 : lane * 255)) * ITEM + (self * 8 + cw) * 128, dump);
      waitv<0>();
      drain();
      const u32 l0 = (u32)(size_t)lds, lw = l0 + L_CTL + C_LANDED * 4, one = lane == 0 ? 1 : 0;
      u32 prog = 0, mk = 0, wm = 0, la = lane == WD ? lw : dump + lane * 4;
      __builtin_amdgcn_s_setprio(3);
#define MODE_ "s_cmp_eq_u32 %[wm], 1\n\ts_cselect_b64 vcc, -1, 0\n\ts_cmp_eq_u32 %[wm], 2\n\t"
#define LD_(off, id) "s_waitcnt vmcnt(%[wf])\n\ts_cbranch_vccnz .Lw" #id "_%=\n\ts_waitcnt vmcnt(%[w])\n\ts_cbranch_scc0 .Lw" #id "_%=\n\t" \
                     "s_waitcnt vmcnt(%[ws])\n.Lw" #id "_%=:\n\tglobal_load_lds_dwordx4 %[va], %[sb] offset:" #off " sc0 sc1\n\t"
#define SH_ "v_mov_b32_dpp %[mk], %[mk] row_shr:1 row_mask:0xf bank_mask:0xf\n\t"
#define PUB_ "ds_max_u32 %[la], %[mk]\n\t"
#define INC_(x) "v_add_u32 %[mk], %[mk], " x "\n\t"
#define WIN_ [wm] "s"(wm), [wf] "i"(WFAST - 1), [w] "i"(WD - 1), [ws] "i"(WD - 2)
#define FOUR(src, ldst, i0) do { u32 keep_; \
          asm volatile("ds_read_b32 %[dv], %[da]\n\tds_read_b32 %[wv], %[lw] offset:4\n\ts_mov_b32 %[k], m0\n\ts_mov_b32 m0, %[ld]\n\t" MODE_ \
                       LD_(0, 0) SH_ INC_("%[i]") PUB_ LD_(1024, 1) SH_ INC_("%[o]") PUB_ LD_(2048, 2) SH_ INC_("%[i]") PUB_ LD_(3072, 3) SH_ INC_("%[o]") \
                       "s_waitcnt lgkmcnt(3)\n\t" PUB_ "s_mov_b32 m0, %[k]" \
                       : [k] "=&s"(keep_), [dv] "=&v"(dvv), [wv] "=&v"(wvv), [mk] "+v"(mk) \
                       : [va] "v"((src) + lane * 16), [sb] "s"(pws), [ld] "s"(ldst), [la] "v"(la), [da] "v"(l0 + L_CTL + (C_DESC + se) * 4), \
                         [lw] "v"(lw), [i] "v"((i0) & one), [o] "v"(one), WIN_ : "memory", "scc", "vcc"); \
          prog += 2 + 2 * (i0); } while (0)
      u64 qu = 0;
      u32 ne = 0, se = 0, sc = 0, d = 0, dvv, wvv;
      bool fin = false;
      for (u32 n = 0;; ++n) {
#pragma clang loop unroll(disable)
        while (ne <= n + EA && !fin) {
          if ((d & 63) != ne + 1) {
            if (ne > n) break;
            d = rfl(ctl[C_DESC + se]);
            if ((d & 63) != ne + 1) {
              waitv<0>();
              if (lane == 0) ctl[C_LANDED] = prog;
              do { d = rfl(ctl[C_DESC + se]); } while ((d & 63) != ne + 1);
            }
          }
          fin = ((d >> 6) & 0x1ff) == IT_END;
          qu |= (u64)((d >> 20) | ((d >> 7) & 0x1000) | (fin ? 0x2000u : 0u)) << (16 * (ne - n));
          ++ne;
          if (!(d & 0x80000)) {
            const u32 ln = (d >> 15) & 15, pn = (prog & 0xffff) | (ne << 16);
            const u64 em = (ln & 7) ? (1ull << ((ln & 7) * 8)) - 1 : ~0ull;
            u32 keep;
            asm volatile("s_mov_b32 %[k], m0\n\ts_mov_b32 m0, %[ld]\n\t" MODE_
                         LD_(0, 0) SH_ PUB_
                         "s_cmp_gt_u32 %[ln], 8\n\ts_cbranch_scc1 .L3_%=\n\ts_cmp_eq_u32 %[wm], 2\n\ts_mov_b64 exec, %[em]\n\t" LD_(1024, 1) "s_branch .L4_%=\n"
                         ".L3_%=:\n\ts_cmp_eq_u32 %[wm], 2\n\t" LD_(1024, 2) SH_ PUB_ "s_mov_b64 exec, %[em]\n\t" LD_(2048, 3)
                         "\n.L4_%=:\n\ts_mov_b64 exec, -1\n\t" SH_ "v_writelane_b32 %[mk], %[pn], 0\n\t" PUB_ "s_mov_b32 m0, %[k]"
                         : [k] "=&s"(keep), [mk] "+v"(mk)
                         : [va] "v"((d >> 20) * ITEM + 4096 + lane * 16), [sb] "s"(pws), [ld] "s"(l0 + se * ITEM + 4096), [la] "v"(la),
                           [ln] "s"(ln), [em] "s"(em), [pn] "s"(pn), WIN_ : "memory", "scc", "vcc");
            prog = pn;
          }
          se = se + 1 == NS ? 0 : se + 1;
          if (n == 0) break;
          if (ne <= n + EA) d = rfl(ctl[C_DESC + se]);
        }
        const u32 cur = (u32)qu & 0xffff;
        qu >>= 16;
        if (cur & 0x2000) break;
        u32 so = (cur & 0xfff) * ITEM, lo = l0 + sc * ITEM;
#pragma clang loop unroll(disable)
        for (u32 k = cur >> 12;; --k) {
          FOUR(so, lo, 1 - (cur >> 12));
          if (k == 0) break;
          so += 4096; lo += 4096;
        }
        d = rfl(dvv);
        wm = rfl(wvv);
        la = lane == (wm == 1 ? WFAST : wm == 2 ? WD - 1 : WD) ? lw : dump + lane * 4;
        sc = sc + 1 == NS ? 0 : sc + 1;
      }
#define DRAIN(N) if (N < WFAST) \
        asm volatile("s_mov_b64 exec, %2\n\ts_waitcnt vmcnt(%3)\n\tds_max_u32 %0, %1\n\ts_mov_b64 exec, -1" \
                     :: "v"(lw), "v"(mk), "i"(1 << N), "i"(N) : "memory");
      DRAIN(9) DRAIN(8) DRAIN(7) DRAIN(6) DRAIN(5) DRAIN(4) DRAIN(3) DRAIN(2) DRAIN(1) DRAIN(0)
      return;
    }
    if (wave == 0 && lane < 4) touch(me + (lane == 3 ? UF_OFF + peer * NIT * 512 : RAW_OFF + (peer * NIT + lane * 143) * ITEM) + cw * 128, dump);
    drain();
    const int u = wave & 1, dw = wave;
    LDSP char* img = lds + L_IMG + wave * 4096;
    u32 m0f = 0x0f0f0f0fu, m80 = 0x80808080u, m3e = 0x3e3e3e3eu, m30 = 0x30303030u;
    u32 mffc = 0xffc;
    const u32 imga = (u32)(size_t)img;
    asm volatile("" : "+s"(m0f), "+s"(m80), "+s"(m3e), "+s"(m30), "+v"(mffc));
    for (u32 n = dw >> 1;; n += NGRP) {
      u32 w0;
      while (((w0 = rfl(ctl[C_DESC + n % NS])) & 63) != n + 1) __builtin_amdgcn_s_sleep(2);
      const u32 it = (w0 >> 6) & 0x1ff;
      if (it == IT_END) break;
      LDSP char* sl = lds + (n % NS) * ITEM;
      const u32 rlo = (w0 >> 20) - RL_OFF / ITEM;
      const u32 src = rlo < 128 ? (rlo >> 3) & 7 : peer;
      const bool keep = (it >> 6) == 2 && ((it >> 1) & 3) * 2 + (it & 1) == self;
      char* d = me + RAW_OFF + (src * NIT + it) * ITEM + u * 4096 + lane * 16;
      u32x4 o0, o1, o2, o3;
#define LANDED(m, x) do { while ((rfl(ctl[C_LANDED]) & (m)) < (x)) __builtin_amdgcn_s_sleep(1); fence(); } while (0)
      if (w0 & 0x80000) {
        LANDED(0xffffu, 4 * n + 2 * u + 2);
        LDSP char* s = sl + u * 4096 + lane * 16;
        o0 = *(LDSP u32x4*)s; o1 = *(LDSP u32x4*)(s + 1024); o2 = *(LDSP u32x4*)(s + 2048); o3 = *(LDSP u32x4*)(s + 3072);
        st16<0>(d, o0); st16<1024>(d, o1); st16<2048>(d, o2); st16<3072>(d, o3);
      } else {
        LANDED(0xffff0000u, (n + 1) << 16);
        const u32x2 hv = *(LDSP u32x2*)(sl + FIX + u * 8);
        const u32 h0 = rfl(hv.x), h1 = rfl(hv.y);
        const u32 nd0 = h0 >> 23, nd1 = h1 >> 23;
        const u32 b0 = ((h0 >> 10) & 0x7f) * 0x01010101u - 0x3e3e3e3eu, b1 = ((h1 >> 10) & 0x7f) * 0x01010101u - 0x3e3e3e3eu;
        LDSP char* v0 = sl + FIX + (h0 & 0x1ff) * 4;
        LDSP char* v1 = sl + FIX + (h1 & 0x1ff) * 4;
        u32 nb0 = *(LDSP u32*)(v0 + lane * 4), nb1 = *(LDSP u32*)(v1 + lane * 4);
        const u32 e0 = *(LDSP unsigned short*)(v0 + (nd0 & 0x7f) * 4 + lane * 2), e1 = *(LDSP unsigned short*)(v1 + (nd1 & 0x7f) * 4 + lane * 2);
        const u32 sg0 = *(LDSP u32*)(sl + 4096 + u * 512 + lane * 4), sg1 = *(LDSP u32*)(sl + 4096 + u * 512 + 256 + lane * 4);
        {
          LDSP char* iz = img + lane * 16;
          *(LDSP u32x4*)iz = u32x4{b0, b0, b0, b0}; *(LDSP u32x4*)(iz + 1024) = u32x4{b0, b0, b0, b0};
          *(LDSP u32x4*)(iz + 2048) = u32x4{b1, b1, b1, b1}; *(LDSP u32x4*)(iz + 3072) = u32x4{b1, b1, b1, b1};
        }
        if (lane >= nd0) nb0 = ~0u;
        if (lane >= nd1) nb1 = ~0u;
        u32 gg[4], vv[4];
        gg[0] = (nb0 & m0f) << 3; gg[1] = (nb0 >> 1) & 0x78787878u; gg[2] = (nb1 & m0f) << 3; gg[3] = (nb1 >> 1) & 0x78787878u;
        const u32 sum = __builtin_amdgcn_sad_u8(gg[0], 0, __builtin_amdgcn_sad_u8(gg[1], 0, 0)) |
                        (__builtin_amdgcn_sad_u8(gg[2], 0, __builtin_amdgcn_sad_u8(gg[3], 0, 0)) << 16);
#pragma unroll
        for (int k = 0; k < 4; ++k) {
          vv[k] = (~(gg[k] + 0x08080808u) & m80) >> 3;
          asm("" : "+v"(gg[k]), "+v"(vv[k]));
        }
        const u32 ps = scan_add(sum) - sum;
        u32 pa = ps & 0xffff, pb = (ps >> 16) + 2048 * 8;
        if (lane < max(nd0, nd1)) {
#pragma unroll
          for (int k = 0; k < 8; ++k) {
            const int w = k & 1, by = (k >> 1) * 8;
            pa += (gg[w] >> by) & 0xff;
            pb += (gg[2 + w] >> by) & 0xff;
            LDS_SUB((LDSP char*)(size_t)(((pa >> 3) & mffc) | imga), ((vv[w] >> by) & 0xff) << (pa & 31));
            LDS_SUB((LDSP char*)(size_t)(((pb >> 3) & mffc) | imga), ((vv[2 + w] >> by) & 0xff) << (pb & 31));
          }
        }
        fence();
        LDSP char* ip = img + lane * 32;
        u32x4 a0 = *(LDSP u32x4*)ip, a1 = *(LDSP u32x4*)(ip + 16), a2 = *(LDSP u32x4*)(ip + 2048), a3 = *(LDSP u32x4*)(ip + 2064);
        fence();
        if (lane < ((h0 >> 17) & 63)) LDS_SUB(img + (e0 & 0xffc), ((e0 >> 12) - 8) << ((e0 & 3) * 8 + 4));
        if (lane < ((h1 >> 17) & 63)) LDS_SUB(img + (e1 & 0xffc), ((e1 >> 12) - 8) << ((e1 & 3) * 8 + 4));
        fence();
        o0 = *(LDSP u32x4*)ip; o1 = *(LDSP u32x4*)(ip + 16); o2 = *(LDSP u32x4*)(ip + 2048); o3 = *(LDSP u32x4*)(ip + 2064);
#pragma unroll
        for (int q = 0; q < 4; ++q) {
          o0[q] += (sg0 << q) & m80; o1[q] += (sg0 << (4 + q)) & m80;
          o2[q] += (sg1 << q) & m80; o3[q] += (sg1 << (4 + q)) & m80;
          const u32 l0 = b0 - a0[q], l1 = b0 - a1[q], l2 = b1 - a2[q], l3 = b1 - a3[q];
          a0[q] = (l0 << 1) | (l0 >> 1); a1[q] = (l1 << 1) | (l1 >> 1); a2[q] = (l2 << 1) | (l2 >> 1); a3[q] = (l3 << 1) | (l3 >> 1);
        }
        asm volatile("" : "+v"(o0), "+v"(o1), "+v"(o2), "+v"(o3), "+v"(a0), "+v"(a1), "+v"(a2), "+v"(a3));
#define CODES(o, a, x) _Pragma("unroll") for (int q = 0; q < 4; ++q) { \
          u32 c_ = ((x) & m0f) + m3e; asm("" : "+v"(c_)); o[q] += c_ + (c_ & a[q] & m30); }
        LANDED(0xffffu, 4 * n + 2 * u + 1);
        const u32x4 n0 = *(LDSP u32x4*)(sl + u * 2048 + lane * 16);
        CODES(o0, a0, n0[q]) st16<0>(d, o0);
        CODES(o1, a1, n0[q] >> 4) st16<1024>(d, o1);
        LANDED(0xffffu, 4 * n + 2 * u + 2);
        const u32x4 n1 = *(LDSP u32x4*)(sl + u * 2048 + 1024 + lane * 16);
        CODES(o2, a2, n1[q]) st16<2048>(d, o2);
        CODES(o3, a3, n1[q] >> 4) st16<3072>(d, o3);
      }
      if (keep) {
        LANDED(0xffffu, 4 * n + 4);
        LDSP char* s = sl + u * 4096 + lane * 16;
        char* r = me + RL_OFF + (((call & 1) * NR + peer) * 8 + ((it >> 3) & 7)) * ITEM + u * 4096 + lane * 16;
        st16<0>(r, *(LDSP u32x4*)s); st16<1024>(r, *(LDSP u32x4*)(s + 1024));
        st16<2048>(r, *(LDSP u32x4*)(s + 2048)); st16<3072>(r, *(LDSP u32x4*)(s + 3072));
      }
      if (lane == 0) ctl[C_CONS + dw] = n + 1;
      waitv<0>();
      if (lane < 8) st16<0>(me + UF_OFF + (src * NIT + it) * 512 + u * 32 + (lane >> 1) * 128 + (lane & 1) * 16, u32x4{call, call, call, call});
      if (keep) put_flag(me + RLF_OFF + (peer * 8 + ((it >> 3) & 7)) * 128 + u * 32, call, lane);
    }
    return;
  }

  const int t = xcd * 4 + (ct >> 1);
  const int dbl = 1 - (ct >> 2);

  if (wave >= 4) {
    drain();
    const int src = wave - 4;
    if (lane == 0 && wave == 11) st_agent(cnt, call);
    const char* raw = me + RAW_OFF + (src != self || (call & 1) ? src : 8) * NIT * ITEM + lane * 16;
    const char* uf = me + UF_OFF + src * NIT * 512;
    const char* px = me + PX_OFF + ((call & 1) * NR + src) * 16 * ITEM + lane * 16;
    const u32 lA = (u32)(size_t)lds + L_A + src * 16384;
    const u64 t0 = tstart;
    u32 need = call;
    u32 prev = 0;
    const char* fa = uf + ((128 + ((lane >> 2) & 3) * 4 + mm) * 2 + ((lane >> 1) & 1)) * 512 + (xcd & 3) * 128 + (lane & 1) * 32;
    u64 xmiss = 0xffff;
    u32 xf = 0;
#pragma clang loop unroll(disable)
    for (int j = 0; j < 4; ++j) {
      {
        for (u32 it = 1; ((xmiss >> (4 * j)) & 15) != 0; ++it) {
          if (it > 1) __builtin_amdgcn_s_sleep(8);
          if ((xmiss >> lane) & 1) xf = ld_sys(fa);
          xmiss = __builtin_amdgcn_ballot_w64(xf < need) & 0xffff;
          if ((it & 15) == 0 && give_up(me, call, t0)) need = 0;
        }
        while (rfl(min4(*(volatile LDSP u32x4*)(ctl + C_FREE))) < prev) __builtin_amdgcn_s_sleep(2);
        const int hx = (j * 4 + mm) * 2;
        const char* sa0 = hx < NPUSH ? px + hx * ITEM : raw + (256 + hx) * ITEM;
        const char* sa1 = hx + 1 < NPUSH ? px + (hx + 1) * ITEM : raw + (257 + hx) * ITEM;
        u32 keep;
#pragma unroll
        for (int i = 0; i < 4; ++i)
          asm volatile("s_mov_b32 %0, m0\n\ts_mov_b32 m0, %2\n\ts_nop 0\n\t"
                       "global_load_lds_dwordx4 %1, off sc0 sc1\n\t"
                       "global_load_lds_dwordx4 %1, off offset:1024 sc0 sc1\n\t"
                       "global_load_lds_dwordx4 %1, off offset:2048 sc0 sc1\n\t"
                       "global_load_lds_dwordx4 %1, off offset:3072 sc0 sc1\n\t"
                       "s_mov_b32 m0, %0" : "=&s"(keep) : "v"((i < 2 ? sa0 : sa1) + (i & 1) * 4096), "s"(lA + i * 4096) : "memory");
      }
      const int nh0 = dbl ? 0 : (ct & 1);
      const char* fb = uf + ((j * 32 + t) * 2 + nh0 + ((lane >> 1) & dbl)) * 512 + mm * 128 + (lane & 1) * 32;
      const char* pp = raw + ((j * 32 + t) * 2 + nh0) * ITEM + 4096;
      for (u32 it = 1;; ++it) {
        u32 fv = need;
        if (lane < 4) fv = ld_sys(fb);
        if (__builtin_amdgcn_ballot_w64(fv < need) == 0) break;
        if ((it & 63) == 0 && give_up(me, call, t0)) need = 0;
        if (j < 3) __builtin_amdgcn_s_sleep(8); else __builtin_amdgcn_s_sleep(1);
      }
      u32x4 p[8], p2[8];
      u32 tv, tk = 1 + dbl;
      asm volatile("global_load_dwordx4 %0, %10, off offset:-4096 sc0 sc1\n\t"
                   "global_load_dwordx4 %1, %10, off offset:-3072 sc0 sc1\n\t"
                   "global_load_dwordx4 %2, %10, off offset:-2048 sc0 sc1\n\t"
                   "global_load_dwordx4 %3, %10, off offset:-1024 sc0 sc1\n\t"
                   "global_load_dwordx4 %4, %10, off sc0 sc1\n\t"
                   "global_load_dwordx4 %5, %10, off offset:1024 sc0 sc1\n\t"
                   "global_load_dwordx4 %6, %10, off offset:2048 sc0 sc1\n\t"
                   "global_load_dwordx4 %7, %10, off offset:3072 sc0 sc1\n\t"
                   "global_load_dword %8, %11, off sc1\n\t"
                   "s_mov_b64 exec, 1\n\tds_add_rtn_u32 %9, %12, %9\n\ts_mov_b64 exec, -1\n\ts_waitcnt lgkmcnt(0)"
                   : "=&v"(p[0]), "=&v"(p[1]), "=&v"(p[2]), "=&v"(p[3]), "=&v"(p[4]), "=&v"(p[5]), "=&v"(p[6]), "=&v"(p[7]),
                     "=&v"(tv), "+v"(tk)
                   : "v"(pp), "v"(me + TMO_OFF + b * 128), "v"((u32)(size_t)(lds + L_CTL + C_TICKET * 4)) : "memory");
      asm volatile("s_cmp_eq_u32 %9, 0\n\ts_cbranch_scc1 .Ls_%=\n\t"
                   "global_load_dwordx4 %0, %8, off offset:-4096 sc0 sc1\n\t"
                   "global_load_dwordx4 %1, %8, off offset:-3072 sc0 sc1\n\t"
                   "global_load_dwordx4 %2, %8, off offset:-2048 sc0 sc1\n\t"
                   "global_load_dwordx4 %3, %8, off offset:-1024 sc0 sc1\n\t"
                   "global_load_dwordx4 %4, %8, off sc0 sc1\n\t"
                   "global_load_dwordx4 %5, %8, off offset:1024 sc0 sc1\n\t"
                   "global_load_dwordx4 %6, %8, off offset:2048 sc0 sc1\n\t"
                   "global_load_dwordx4 %7, %8, off offset:3072 sc0 sc1\n"
                   ".Ls_%=:"
                   : "=&v"(p2[0]), "=&v"(p2[1]), "=&v"(p2[2]), "=&v"(p2[3]), "=&v"(p2[4]), "=&v"(p2[5]), "=&v"(p2[6]), "=&v"(p2[7])
                   : "v"(pp + 8192), "s"(dbl) : "memory", "scc");
      u32 ts = rfl(tk) & 0xffff, fs = 0;
      const u32 t1 = (rfl(tk) >> 16) - 1;
#define SLOT(t) ({ const u32 k_ = (t) < t1 ? (t) % 3 : ((t) - t1) % 5; k_ < 3 ? L_B / 8192 + k_ : self * 2 + k_ - 3; })
#define NEEDF(t) ((t) + 8 - ((t) >= t1 && (t) - t1 >= 3 ? 4 : 2))
      u32 sl0 = SLOT(ts), sl1 = SLOT(ts + 1), nf0 = NEEDF(ts), nf1 = NEEDF(ts + 1);
      asm volatile("" : "+s"(sl0), "+s"(sl1), "+s"(nf0), "+s"(nf1));
#define ROOM(nf) while (fs < (nf)) { fs = rfl(min4(*(volatile LDSP u32x4*)(ctl + C_FREE))) + 8; if (fs < (nf)) __builtin_amdgcn_s_sleep(1); }
#define FULL(sl) do { ++ts; \
        asm volatile("" : "+s"(ts) :: "memory"); \
        if (lane == 0) ctl[C_FULL + ((ts - 1) & 7)] = ts | (src << 8) | ((sl) << 11) | (tv == call ? 0x10000u : 0u); } while (0)
#define PUT(R, sl) do { LDSP char* ub = lds + (sl) * 8192 + lane * 16; \
        _Pragma("unroll") for (int i = 0; i < 8; ++i) *(LDSP u32x4*)(ub + i * 1024) = R[i]; \
        FULL(sl); } while (0)
      ROOM(nf0);
      {
        LDSP char* ub = lds + sl0 * 8192 + lane * 16;
#define PUT2(i, N) asm volatile("s_waitcnt vmcnt(" #N ")" : "+v"(p[i]), "+v"(p[i + 1]) :: "memory"); \
        *(LDSP u32x4*)(ub + (i) * 1024) = p[i]; *(LDSP u32x4*)(ub + (i) * 1024 + 1024) = p[i + 1];
        PUT2(0, 7) PUT2(2, 5) PUT2(4, 3) PUT2(6, 1)
        asm volatile("s_waitcnt vmcnt(0)" : "+v"(p2[0]), "+v"(p2[1]), "+v"(p2[2]), "+v"(p2[3]), "+v"(p2[4]), "+v"(p2[5]), "+v"(p2[6]), "+v"(p2[7]),
                     "+v"(tv) :: "memory");
        FULL(sl0);
      }
      if (dbl) { ROOM(nf1); fence(); PUT(p2, sl1); }
      prev = ts;
    }
    if (src == self) {
      while (rfl(min4(*(volatile LDSP u32x4*)(ctl + C_FREE))) < prev) __builtin_amdgcn_s_sleep(2);
      if (lane == 0) {
        u32 o = ctl[C_TICKET];
        while (!__hip_atomic_compare_exchange_strong((LDSP u32*)(ctl + C_TICKET), &o, o | (((o + 2) / 3 * 3 + 1) << 16),
                                                     __ATOMIC_RELAXED, __ATOMIC_RELAXED, __HIP_MEMORY_SCOPE_WORKGROUP)) {}
      }
    }
    return;
  }

  const int mw = wave;
  {
    u32 scale_bits = __builtin_amdgcn_readfirstlane(__builtin_bit_cast(u32, *(const GLB float*)KA64(12) * *(const GLB float*)KA64(14)));
    float* obase = (float*)KA64(10) + (size_t)(mm * 128 + mw * 32) * NN + t * 128 + (ct & 1) * 64;
    asm volatile("" : "+s"(scale_bits), "+s"(obase));
    drain();
    v4f Pc[2][4], Qc[2][4];
#pragma unroll
    for (int i = 0; i < 2; ++i)
#pragma unroll
      for (int k = 0; k < 4; ++k) Pc[i][k] = Qc[i][k] = v4f{0.f, 0.f, 0.f, 0.f};
#define FRAG(p, hs) ({ const u32x4 lo_ = *(LDSP u32x4*)(p), hi_ = *(LDSP u32x4*)((p) + (hs)); \
      v8i{(int)lo_.x, (int)lo_.y, (int)lo_.z, (int)lo_.w, (int)hi_.x, (int)hi_.y, (int)hi_.z, (int)hi_.w}; })
#define MMA(i, k, F) do { F##c[i][k] = __builtin_amdgcn_mfma_scale_f32_16x16x128_f8f6f4(F##a[i], F##b[k], F##c[i][k], 1, 1, 0, 0, 0, 0); \
      __builtin_amdgcn_sched_barrier(0); } while (0)
#define SB() __builtin_amdgcn_sched_barrier(0)
#define RD(F) do { bad |= w; LDSP char* ap_ = lds + L_A + ((w >> 8) & 7) * 16384 + mw * 4096 + lane * 16; \
      bp = lds + ((w >> 11) & 31) * 8192 + lane * 16; \
      F##a[0] = FRAG(ap_, 1024); F##b[0] = FRAG(bp, 4096); SB(); F##a[1] = FRAG(ap_ + 2048, 1024); SB(); F##b[1] = FRAG(bp + 1024, 4096); \
      SB(); } while (0)
#define SPIN() do { while (((w = rfl(ctl[C_FULL + (n & 7)])) & 0xff) != n + 1) __builtin_amdgcn_s_sleep(0); \
      fence(); } while (0)
#define HEAD(F) do { \
      MMA(0, 0, F); F##b[2] = FRAG(bp + 2048, 4096); SB(); \
      MMA(1, 0, F); F##b[3] = FRAG(bp + 3072, 4096); \
      fence();     \
      ++n; \
      if (lane == 0) ctl[C_FREE + mw] = n; } while (0)
#define STEP(F, G, EXTRA) do { \
      HEAD(F); \
      peek = ctl[C_FULL + (n & 7)]; SB(); \
      MMA(0, 1, F); MMA(1, 1, F); MMA(0, 2, F); SB(); \
      w = rfl(peek); \
      fence(); \
      const bool ok_ = (w & 0xff) == n + 1; \
      if (ok_) RD(G); \
      MMA(1, 2, F); MMA(0, 3, F); MMA(1, 3, F); SB(); \
      EXTRA; \
      if (!ok_) { SPIN(); RD(G); } } while (0)
    LDSP char* bp;
    v8i Pa[2], Pb[4], Qa[2], Qb[4];
    u32 n = 0, w, peek, bad = 0;
    SPIN();
    RD(P);
#pragma clang loop unroll(disable)
    for (int it = 1; it < (16 << dbl); ++it) { STEP(P, Q, ); STEP(Q, P, ); }
#define EPI(A, i, r, o) do { v4f y_; _Pragma("unroll") for (int k = 0; k < 4; ++k) \
        y_[k] = __builtin_elementwise_maximum(A[i][k][r] * scale, 0.f);     \
      asm volatile("global_store_dwordx4 %0, %1, %2 sc1\n\ts_nop 1" :: "v"(ov), "v"(y_), "s"((o) + ((i) * 16 + (r)) * NN) : "memory"); \
      __builtin_amdgcn_sched_barrier(0); } while (0)
    STEP(P, Q, if (!dbl) { fence();
                 _Pragma("unroll") for (int i = 0; i < 2; ++i) _Pragma("unroll") for (int k = 0; k < 4; ++k) Qc[i][k] += Pc[i][k]; });
    HEAD(Q); SB();
    MMA(0, 1, Q); MMA(0, 2, Q); MMA(0, 3, Q); MMA(1, 1, Q);
    const float scale = __builtin_bit_cast(float, (bad & 0x10000u) != 0 ? 0x7fc00000u : scale_bits);
    const u32 ln = __builtin_amdgcn_mbcnt_hi(~0u, __builtin_amdgcn_mbcnt_lo(~0u, 0));
    const u32 ov = (ln >> 4) * (4 * NN * 4) + (ln & 15) * 16;
    SB();
    EPI(Qc, 0, 0, obase); MMA(1, 2, Q); EPI(Qc, 0, 1, obase); MMA(1, 3, Q); EPI(Qc, 0, 2, obase); EPI(Qc, 0, 3, obase);
    EPI(Qc, 1, 0, obase); EPI(Qc, 1, 1, obase); EPI(Qc, 1, 2, obase); EPI(Qc, 1, 3, obase);
    if (dbl) {
      EPI(Pc, 0, 0, obase - 64); EPI(Pc, 0, 1, obase - 64); EPI(Pc, 0, 2, obase - 64); EPI(Pc, 0, 3, obase - 64);
      EPI(Pc, 1, 0, obase - 64); EPI(Pc, 1, 1, obase - 64); EPI(Pc, 1, 2, obase - 64); EPI(Pc, 1, 3, obase - 64);
    }
  }
}

extern "C" void kernel_launch(int rank, int nranks, ncclComm_t comm,
                              hipStream_t stream,
                              void* const* d_in, const int* in_sizes, int n_in,
                              void* d_out, int out_size,
                              const int* in_dtypes, int out_dtype,
                              void* const* d_out_all,
                              void* const* d_ws_all, size_t ws_bytes) {
  if (nranks != NR || (unsigned)rank >= (unsigned)NR || n_in < 4 || ws_bytes < WS_NEED ||
      in_sizes[0] != 4096 * KS || in_sizes[1] != KS * NN || out_size != 512 * NN) {
    fprintf(stderr, "kernel_launch: unexpected problem shape\n");
    return;
  }
  Params p;
  p.me = (char*)d_ws_all[rank];
  p.ws_all = d_ws_all;
  p.x = (const unsigned char*)d_in[0];
  p.w = (const unsigned char*)d_in[1];
  p.sx = (const float*)d_in[2];
  p.sw = (const float*)d_in[3];
  p.out = (float*)d_out;
  p.rank = rank;
  static const int grp[2][4] = {{0, 2, 3, 6}, {1, 4, 5, 7}};
  int gi = 0, gp = 0;
  for (int a = 0; a < 2; ++a) for (int i = 0; i < 4; ++i) if (grp[a][i] == rank) { gi = a; gp = i; }
  p.dstmap = 0;
  for (int x = 0; x < 8; ++x) {
    p.dstmap |= (u32)(x < 4 ? grp[gi][(gp + 1 + x) & 3] : grp[1 - gi][(gp + x) & 3]) << (4 * x);
  }
  hipLaunchKernelGGL(fused_gemm_kernel, dim3(NWG), dim3(NTHREADS), 0, stream, p);
  HIP_CHECK(hipGetLastError());
}
```

```cpp
#include <hip/hip_runtime.h>
#include <rccl/rccl.h>
#include <cstdio>
#include <cstdint>

#define HIP_CHECK(expr) do { hipError_t _e = (expr); if (_e != hipSuccess) { \
    fprintf(stderr, "HIP error %s at %s:%d\n", hipGetErrorString(_e), __FILE__, __LINE__); \
    return; } } while (0)

typedef unsigned u32;
typedef unsigned long long u64;
typedef unsigned u32x2 __attribute__((ext_vector_type(2)));
typedef unsigned u32x4 __attribute__((ext_vector_type(4)));
typedef int v8i __attribute__((ext_vector_type(8)));
typedef float v4f __attribute__((ext_vector_type(4)));
#define LDSP __attribute__((address_space(3)))
#define GLB __attribute__((address_space(1)))

constexpr int NR = 8;
constexpr int KS = 512;
constexpr int NN = 4096;
constexpr int NTHREADS = 768;
constexpr int NWG = 256;
constexpr int NIT = 288;
#define WD 6
#define NPUSH 12
#define GATE 100

constexpr u32 ITEM = 8192;
constexpr u32 FIX = 5120;
constexpr u32 PK_OFF = 0;
constexpr u32 RAW_OFF = 2 * 512 * ITEM;
constexpr u32 PX_OFF = RAW_OFF + 9 * NIT * ITEM;
constexpr u32 RL_OFF = PX_OFF + 2 * NR * 16 * ITEM;
constexpr u32 DIR_OFF = RL_OFF + 2 * NR * 8 * ITEM;
constexpr u32 UF_OFF = DIR_OFF + 2 * NR * NIT * 128;
constexpr u32 CNT_OFF = UF_OFF + NR * NIT * 512;
constexpr u32 TMO_OFF = CNT_OFF + NWG * 12 * 128;
constexpr u32 BAL_OFF = TMO_OFF + NWG * 128;
constexpr u32 PF_OFF = BAL_OFF + NR * 256;
constexpr u32 PLAN_OFF = PF_OFF + NR * 256;
constexpr u32 RLF_OFF = PLAN_OFF + NR * 128;
constexpr u32 ROLE_OFF = RLF_OFF + NR * 8 * 128;
static_assert(DIR_OFF / ITEM <= 4096, "12-bit item offsets");
constexpr u32 WS_NEED = ROLE_OFF + 64 * 128;
constexpr u64 DEADLINE = 30000000ull;
static_assert(NPUSH <= 16 && NPUSH >= 0, "PX region");

constexpr int NBS = 3;
constexpr u32 L_A = 0;
constexpr u32 L_B = 8 * 16384;
constexpr int NGRP = 4;
constexpr int NS = 12;
constexpr u32 L_IMG = NS * ITEM;
constexpr u32 L_SCR = L_B + NBS * ITEM;
constexpr u32 L_CTL = L_SCR + 4096;
constexpr u32 L_DUMP = L_CTL + 256;
constexpr u32 L_BYTES = L_DUMP + 256;
constexpr int C_FREE = 0, C_FULL = 4, C_TICKET = 12, C_ENC = 13;
constexpr int C_CONS = 16, C_LANDED = 28, C_WIN = 29, C_NIT = 30, C_ROLE = 31, C_DESC = 32;
constexpr u32 M_RAW = 0x80, IT_END = 0x1ff;

#define PUSH1 8
#define PUSHD 130
#define EA 2
#define LA 8
#define TITEM 77
#define WFAST 8
#define GFAST 24
#define GSLOW 19
#define K0 4
#define SP 4
#define PLANT 2000
#define DZ 10.f

struct Params {
  char* me;
  const unsigned char* x;
  const unsigned char* w;
  void* const* ws_all;
  int rank;
  u32 dstmap;
  float* out;
  const float* sx;
  const float* sw;
};

template <int N> __device__ __forceinline__ void waitv() { asm volatile("s_waitcnt vmcnt(%0)" :: "i"(N) : "memory"); }
__device__ __forceinline__ void waitl() { asm volatile("s_waitcnt lgkmcnt(0)" ::: "memory"); }
__device__ __forceinline__ void bar() { asm volatile("s_barrier" ::: "memory"); }
__device__ __forceinline__ void drain() { __builtin_amdgcn_s_waitcnt(0x0070); }
__device__ __forceinline__ void fence() { asm volatile("" ::: "memory"); }
template <int OFF> __device__ __forceinline__ void st16(const void* p, u32x4 x) {
  asm volatile("global_store_dwordx4 %0, %1, off offset:%2 sc0 sc1\n\ts_nop 1" :: "v"(p), "v"(x), "i"(OFF) : "memory");
}
__device__ __forceinline__ void st4(const void* p, u32 x) {
  asm volatile("global_store_dword %0, %1, off sc0 sc1\n\ts_nop 1" :: "v"(p), "v"(x) : "memory");
}
__device__ __forceinline__ u32 ld_sys(const void* p) {
  return __hip_atomic_load((const GLB u32*)p, __ATOMIC_RELAXED, __HIP_MEMORY_SCOPE_SYSTEM);
}
__device__ __forceinline__ u32 ld_agent(const void* p) {
  return __hip_atomic_load((const GLB u32*)p, __ATOMIC_RELAXED, __HIP_MEMORY_SCOPE_AGENT);
}
__device__ __forceinline__ void st_agent(void* p, u32 v) {
  __hip_atomic_store((GLB u32*)p, v, __ATOMIC_RELAXED, __HIP_MEMORY_SCOPE_AGENT);
}
__device__ __forceinline__ void touch(const void* p, u32 dump) {
  u32 keep;
  asm volatile("s_mov_b32 %0, m0\n\ts_mov_b32 m0, %2\n\ts_nop 0\n\tglobal_load_lds_dword %1, off sc0 sc1\n\ts_mov_b32 m0, %0"
               : "=&s"(keep) : "v"(p), "s"(dump) : "memory");
}
__device__ __forceinline__ u32 rfl(u32 v) { return __builtin_amdgcn_readfirstlane(v); }
__device__ __forceinline__ void put_flag(char* f, u32 v, int lane) {
  if (lane < 2) st16<0>(f + lane * 16, u32x4{v, v, v, v});
}
__device__ __forceinline__ bool give_up(char* me, u32 call, u64 t0) {
  if (rfl(ld_agent(me + TMO_OFF + blockIdx.x * 128)) == call) return true;
  if (__builtin_amdgcn_s_memrealtime() - t0 <= DEADLINE) return false;
  for (int i = threadIdx.x & 63; i < NWG; i += 64) st_agent(me + TMO_OFF + i * 128, call);
  waitv<0>();
  return true;
}
#define LDS_ADD(p, v) __hip_atomic_fetch_add((LDSP u32*)(p), (v), __ATOMIC_RELAXED, __HIP_MEMORY_SCOPE_WORKGROUP)
#define LDS_SUB(p, v) __hip_atomic_fetch_sub((LDSP u32*)(p), (v), __ATOMIC_RELAXED, __HIP_MEMORY_SCOPE_WORKGROUP)
#define LDS_OR(p, v) __hip_atomic_fetch_or((LDSP u32*)(p), (v), __ATOMIC_RELAXED, __HIP_MEMORY_SCOPE_WORKGROUP)
#define DPP(old, v, ctrl, rm) (u32)__builtin_amdgcn_update_dpp((int)(old), (int)(v), ctrl, rm, 0xf, false)
#define SCAN(name, id, OP) __device__ __forceinline__ u32 name(u32 v) { \
    v = OP(v, DPP(id, v, 0x111, 0xf)); v = OP(v, DPP(id, v, 0x112, 0xf)); v = OP(v, DPP(id, v, 0x114, 0xf)); \
    v = OP(v, DPP(id, v, 0x118, 0xf)); v = OP(v, DPP(id, v, 0x142, 0xa)); v = OP(v, DPP(id, v, 0x143, 0xc)); return v; }
__device__ __forceinline__ u32 addu(u32 a, u32 b) { return a + b; }
SCAN(scan_add, 0u, addu)
SCAN(scan_max, 0u, max)
SCAN(scan_min, 0xffffffffu, min)

__device__ __forceinline__ u32 round_item(u32 rd, u32 col) { return ((rd >> 3) * 32 + 4 * col + ((rd >> 1) & 3)) * 2 + (rd & 1); }
__device__ __forceinline__ u32 sched_item(u32 q, u32 rel, u32 self) {
  if (q < 32 - NPUSH) return 256 + (q < 8 ? 16 + q : q < 24 - NPUSH ? NPUSH - 8 + q : NPUSH + q);
  q -= 32 - NPUSH;
  const u32 sh = 16 + self, ps = (sh & ~3u) | ((sh - rel) & 3);
  u32 r = q >> 3;
  r = r == SP ? ps : r > SP && r <= ps ? r - 1 : r;
  return round_item((r & ~3u) | ((r + rel) & 3), (q + rel) & 7);
}
__device__ __forceinline__ u32 nib_sum(u32 x) { return (((x & 0x0f0f0f0fu) + ((x >> 4) & 0x0f0f0f0fu)) * 0x01010101u) >> 24; }
__device__ __forceinline__ u32 lds_ticket(volatile LDSP u32* p, u32 v) {
  asm volatile("s_mov_b64 exec, 1\n\tds_add_rtn_u32 %0, %1, %0\n\ts_mov_b64 exec, -1\n\ts_waitcnt lgkmcnt(0)"
               : "+v"(v) : "v"((u32)(size_t)p) : "memory");
  return rfl(v);
}
__device__ __forceinline__ u32 min4(u32x4 v) { return min(min(v.x, v.y), min(v.z, v.w)); }
__device__ __forceinline__ u32 gather8(const u32* f) {
  const u32 r0 = __builtin_amdgcn_udot4(f[1], 0x80402010u, __builtin_amdgcn_udot4(f[0], 0x08040201u, 0, false), false);
  const u32 r1 = __builtin_amdgcn_udot4(f[3], 0x80402010u, __builtin_amdgcn_udot4(f[2], 0x08040201u, 0, false), false);
  const u32 r2 = __builtin_amdgcn_udot4(f[5], 0x80402010u, __builtin_amdgcn_udot4(f[4], 0x08040201u, 0, false), false);
  const u32 r3 = __builtin_amdgcn_udot4(f[7], 0x80402010u, __builtin_amdgcn_udot4(f[6], 0x08040201u, 0, false), false);
  return (r0 >> 7) | (r1 << 1) | (r2 << 9) | (r3 << 17);
}

__global__ void __launch_bounds__(NTHREADS) fused_gemm_kernel(Params P) {
  __shared__ __attribute__((aligned(16))) char smem[L_BYTES];
  LDSP char* lds = (LDSP char*)smem;
  volatile LDSP u32* ctl = (volatile LDSP u32*)(lds + L_CTL);
  const int lane = threadIdx.x & 63;
  const int wave = rfl(threadIdx.x >> 6);
  const int b = blockIdx.x, xcd = b & 7, slot = b >> 3;
  const int ct = slot >> 2, mm = slot & 3;
  typedef u32 u32x16 __attribute__((ext_vector_type(16)));
  u32x16 ka;
  asm volatile("s_load_dwordx16 %0, %1, 0x0" : "=&s"(ka) : "s"(__builtin_amdgcn_kernarg_segment_ptr()) : "memory");
  if (wave == 4) __builtin_amdgcn_s_setprio(3);
  const bool link = (ct & 5) == 0;
  const int i8 = mm + (ct < 4 ? ct >> 1 : ct & 1) * 4;
  const bool xjob = !link & ((wave == 4) | ((wave == 5) & (ct >= 6)));
  const int xj = wave == 5 ? 1 : ct < 4 ? 0 : ct >> 1;
  u32 cno = CNT_OFF + (b * 12 + wave) * 128;
  u32 xv0 = (i8 * 64 + (lane & 15)) * KS + xj * 128 + (lane >> 4) * 16, xv1 = xv0 + 16 * KS, xv2 = xv0 + 32 * KS, xv3 = xv0 + 48 * KS;
  asm volatile("s_waitcnt lgkmcnt(0)" : "+s"(ka), "+v"(cno), "+v"(xv0), "+v"(xv1), "+v"(xv2), "+v"(xv3) :: "memory");
#define KA64(i) (ka[i] | ((u64)ka[(i) + 1] << 32))
  char* const me = (char*)KA64(0);
  const int self = ka[8];
#define LOAD_PWS() char* pws; asm volatile("s_load_dwordx2 %0, %1, %2\n\ts_waitcnt lgkmcnt(0)" : "=&s"(pws) : "s"(KA64(6)), "s"(peer * 8) : "memory")
  u32* cnt = (u32*)(me + CNT_OFF + (b * 12 + wave) * 128);
  const u32 dump = (u32)(size_t)lds + L_DUMP;
  u32 call;
  const u64 tstart = __builtin_amdgcn_s_memrealtime();

  if (__builtin_expect(xjob, 1)) {
    const int hx = xj * 8 + i8, peer = (ka[9] >> (4 * xcd)) & 7;
    if (xj == 3) __builtin_amdgcn_s_sleep(15);
    u32x4 d[8];
    u32 cv;
    asm volatile("global_load_dword %8, %11, %10 sc1\n\t"
                 "global_load_dwordx4 %0, %12, %9\n\tglobal_load_dwordx4 %1, %12, %9 offset:64\n\t"
                 "global_load_dwordx4 %2, %13, %9\n\tglobal_load_dwordx4 %3, %13, %9 offset:64\n\t"
                 "global_load_dwordx4 %4, %14, %9\n\tglobal_load_dwordx4 %5, %14, %9 offset:64\n\t"
                 "global_load_dwordx4 %6, %15, %9\n\tglobal_load_dwordx4 %7, %15, %9 offset:64"
                 : "=&v"(d[0]), "=&v"(d[1]), "=&v"(d[2]), "=&v"(d[3]), "=&v"(d[4]), "=&v"(d[5]), "=&v"(d[6]), "=&v"(d[7]), "=&v"(cv)
                 : "s"(KA64(2) + peer * (512 * KS)), "s"(me), "v"(cno), "v"(xv0), "v"(xv1), "v"(xv2), "v"(xv3) : "memory");
    LOAD_PWS();
    const bool push = hx < NPUSH, direct = push || peer == self;
    const u32 o0 = push ? PX_OFF + (self * 16 + hx) * ITEM : direct ? RAW_OFF + (8 * NIT + 256 + hx) * ITEM : PK_OFF + (256 + peer * 32 + hx) * ITEM;
    const u32 o1 = push ? NR * 16 * ITEM : direct ? (self - 8) * NIT * ITEM : 512 * ITEM;
    const u32 f0 = direct ? UF_OFF + (self * NIT + 256 + hx) * 512 : DIR_OFF + (self * NIT + 256 + hx) * 128;
    const u32 f1 = direct ? 0 : NR * NIT * 128;
    const char* db = push ? pws : me;
    const u64 fm = direct ? 0xffff : 3;
    asm volatile("s_waitcnt vmcnt(8)" : "+v"(cv) :: "memory");
    call = rfl(cv) + 1;
    const u32 par = call & 1;
    const u32 dv = o0 + par * o1 + 4096 + lane * 16, fv = f0 + par * f1 + (lane >> 2) * 128 + (lane & 3) * 16;
    const u32 v = direct ? call : (call << 8) | M_RAW;
    if (push && hx >= PUSH1) while (__builtin_amdgcn_s_memrealtime() - tstart < PUSHD) __builtin_amdgcn_s_sleep(1);
    asm volatile("s_waitcnt vmcnt(7)\n\tglobal_store_dwordx4 %8, %0, %9 offset:-4096 sc0 sc1\n\t"
                 "s_waitcnt vmcnt(7)\n\tglobal_store_dwordx4 %8, %1, %9 offset:-3072 sc0 sc1\n\t"
                 "s_waitcnt vmcnt(7)\n\tglobal_store_dwordx4 %8, %2, %9 offset:-2048 sc0 sc1\n\t"
                 "s_waitcnt vmcnt(7)\n\tglobal_store_dwordx4 %8, %3, %9 offset:-1024 sc0 sc1\n\t"
                 "s_waitcnt vmcnt(7)\n\tglobal_store_dwordx4 %8, %4, %9 sc0 sc1\n\t"
                 "s_waitcnt vmcnt(7)\n\tglobal_store_dwordx4 %8, %5, %9 offset:1024 sc0 sc1\n\t"
                 "s_waitcnt vmcnt(7)\n\tglobal_store_dwordx4 %8, %6, %9 offset:2048 sc0 sc1\n\t"
                 "s_waitcnt vmcnt(7)\n\tglobal_store_dwordx4 %8, %7, %9 offset:3072 sc0 sc1\n\ts_nop 1"
                 : "+v"(d[0]), "+v"(d[1]), "+v"(d[2]), "+v"(d[3]), "+v"(d[4]), "+v"(d[5]), "+v"(d[6]), "+v"(d[7])
                 : "v"(dv), "s"(db) : "memory");
    asm volatile("s_mov_b64 exec, %3\n\ts_waitcnt vmcnt(0)\n\tglobal_store_dwordx4 %0, %1, %2 sc0 sc1\n\ts_mov_b64 exec, -1\n\ts_nop 0"
                 :: "v"(fv), "v"(u32x4{v, v, v, v}), "s"(pws), "s"(fm) : "memory");
    bar();
  } else if (!link && (wave < 4 || (wave >= 8 && ct < 4))) {
    {
      const int g = lane >> 4, c = lane & 15, j2 = wave >> 3, jw = ct < 4 ? 2 + j2 : (ct - 4) >> 1;
      const int u = wave & 1, hf = (wave >> 1) & 1, blk = (jw * 32 + xcd * 4) * 2 + i8;
      const GLB unsigned char* src = (const GLB unsigned char*)KA64(4) + (size_t)(jw * 128 + u * 64 + g * 16) * NN + xcd * 512 + i8 * 64 + c * 4;
      u32 W[16], D[8];
      if (jw == 1) __builtin_amdgcn_s_sleep(30);
      if (jw >= 2) __builtin_amdgcn_s_sleep(60);
#pragma unroll
      for (int r = 0; r < 16; ++r) W[r] = *(const GLB u32*)(src + r * NN);
      const u32 cv = ld_agent(cnt);
      LDSP char* scr = lds + L_SCR + (wave - j2 * 4) * 512;
      *(LDSP u32x2*)(scr + lane * 8) = u32x2{0, 0};
#pragma unroll
      for (int s = 0; s < 8; ++s) LDS_ADD(scr + (((W[s >> 2] >> ((s & 3) * 8)) & 0x7f) << 2), 1u);
      const u32x2 hh = *(volatile LDSP u32x2*)(scr + lane * 8);
      *(volatile LDSP u32x2*)(scr + lane * 8) = u32x2{0, 0};
      const u32 sel = 0x05010400u + hf * 0x02020202u;
#pragma unroll
      for (int q = 0; q < 4; ++q) {
        const u32 t0 = __builtin_amdgcn_perm(W[4 * q + 1], W[4 * q], sel), t2 = __builtin_amdgcn_perm(W[4 * q + 3], W[4 * q + 2], sel);
        D[q] = __builtin_amdgcn_perm(t2, t0, 0x05040100u); D[4 + q] = __builtin_amdgcn_perm(t2, t0, 0x07060302u);
      }
      call = rfl(cv) + 1;
      const u32 par = call & 1;
      char* pk = me + PK_OFF + (par * 512 + blk) * ITEM;
      {
        char* d = me + RAW_OFF + ((par ? self : 8) * NIT + blk) * ITEM + u * 4096 + hf * 2048 + lane * 16;
        st16<0>(d, u32x4{D[0], D[1], D[2], D[3]}); st16<1024>(d, u32x4{D[4], D[5], D[6], D[7]});
        u32 s0 = 0;
#pragma unroll
        for (int k = 0; k < 8; ++k) s0 |= (D[k] & 0x80808080u) >> k;
        st4(pk + 4096 + u * 512 + hf * 256 + lane * 4, s0);
      }
      u32 base;
      {
        const u32 c0 = scan_add(hh.x + hh.y) - hh.x - hh.y, c1 = c0 + hh.x;
        u32 u8 = (u32)__builtin_amdgcn_ds_bpermute(((lane + 8) & 63) * 4, (int)(c0 | (c1 << 16)));
        u32 u9 = (u32)__builtin_amdgcn_ds_bpermute(((lane + 9) & 63) * 4, (int)(c0 | (c1 << 16)));
        if (lane >= 56) u8 = 0x02000200u;
        if (lane >= 55) u9 = 0x02000200u;
        const u32 cs = (c0 | (c1 << 16)) + 0x08000800u - u8 - 3 * u9;
        const u32 k0 = ((cs & 0xffff) << 7) | (2 * lane), k1 = ((cs >> 16) << 7) | (2 * lane + 1);
        base = __builtin_amdgcn_readlane(scan_min(lane < 56 ? min(k0, k1) : lane == 56 ? k0 : 0xffffffffu), 63) & 0x7f;
      }
      const u32 b8f = base * 0x01010101u + 0x8f8f8f8fu;
      u32 F1[8], F2[8], FE[8], T[8];
#pragma unroll
      for (int k = 0; k < 8; ++k) {
        const u32 t = T[k] = b8f - (D[k] & 0x7f7f7f7fu);
        const u32 t1 = t << 1, t2 = t << 2, t3 = t << 3, wr = ((t & 0x0f0f0f0fu) + 0x02020202u) << 3;
        const u32 h = t & ~t1 & 0x80808080u, k1 = h & ~t2 & t3, k2 = h & t2 & ~t3, km = ~t & t1 & t2 & t3 & 0x80808080u;
        F2[k] = (wr & k1) | (~wr & k2);
        F1[k] = k1 | (wr & km) | (~wr & k2);
        FE[k] = ((h & ~t2 & ~t3) | F1[k]) ^ 0x80808080u;
      }
      {
        u32x4 cd;
#pragma unroll
        for (int q = 0; q < 4; ++q) cd[q] = ~((T[q] & 0x0f0f0f0fu) | ((T[4 + q] << 4) & 0xf0f0f0f0u));
        st16<0>(pk + u * 2048 + hf * 1024 + lane * 16, cd);
      }
      u32 x1 = gather8(F1), x2 = gather8(F2), xe = gather8(FE);
      const u64 any = __builtin_amdgcn_ballot_w64(x1 != 0);
      int prev;
      {
        const u32 lastp = x1 ? lane * 32 + 31 - __builtin_clz(x1) : 0;
        const u32 back = lane * 32 - scan_max(DPP(0, lastp, 0x138, 0xf));
        prev = -(int)(back - 15 * ((back * 4370) >> 16));
      }
      u32 a0 = 0, a1 = 0, a2 = 0, nc = 0;
#define PUSH_NIB(gg) do { a0 = __builtin_amdgcn_alignbit(a1, a0, 4); a1 = __builtin_amdgcn_alignbit(a2, a1, 4); \
        a2 = __builtin_amdgcn_alignbit((gg), a2, 4); ++nc; } while (0)
      while (x1 != 0) {
        const u32 pos = __builtin_ctz(x1);
        const u32 gap = pos - prev, gg = min(gap, 15u);
        prev += gg;
        const u32 bit = gap < 15 ? 1u << pos : 0u, in2 = x2 & bit;
        x2 ^= in2;
        x1 ^= bit ^ in2;
        PUSH_NIB(gg);
      }
      if ((any >> lane) > 1) while (prev <= 17) { prev += 15; PUSH_NIB(15u); }
      const u32 nend = scan_add(nc), nn = __builtin_amdgcn_readlane(nend, 63);
      bool over = nn > 512 || __builtin_amdgcn_ballot_w64(nc > 24) != 0;
      if (!over) {
        const u32 sh = (8 - (nend & 7)) * 4;
        LDSP char* q = scr + (nend >> 3) * 4;
        LDS_OR(q, (u32)((((u64)a0) << 32) >> sh));
        LDS_OR(q + 4, (u32)((((u64)a1 << 32) | a0) >> sh));
        LDS_OR(q + 8, (u32)((((u64)a2 << 32) | a1) >> sh));
        LDS_OR(q + 12, (u32)(((u64)a2) >> sh));
      }
      u32 ne = 0;
      {
        LDSP unsigned short* lst = (LDSP unsigned short*)(scr + 384);
        u64 mk;
        while ((mk = __builtin_amdgcn_ballot_w64(xe != 0)) != 0 && ne <= 63) {
          if (xe != 0) {
            const u32 pos = __builtin_ctz(xe);
            const u32 wa = (pos & 4) ? T[1] : T[0], wb = (pos & 4) ? T[3] : T[2], wc = (pos & 4) ? T[5] : T[4], wd = (pos & 4) ? T[7] : T[6];
            const u32 we = (pos & 8) ? wb : wa, wf = (pos & 8) ? wd : wc, wg = (pos & 16) ? wf : we;
            const u32 x4 = (wg >> ((pos & 3) * 8 + 4)) & 15;
            const u32 at = ne + __builtin_amdgcn_mbcnt_hi((u32)(mk >> 32), __builtin_amdgcn_mbcnt_lo((u32)mk, 0));
            if (at < 64) lst[at] = (unsigned short)((x4 << 12) | (hf * 2048 + lane * 32 + pos));
            xe &= xe - 1;
          }
          ne += __builtin_popcountll(mk);
        }
      }
      over = over || ne > 63;
      bar();
      if (!over) {
        const u32 ed = (ne + 1) >> 1, nd = (nn + 7) >> 3;
        const u32 ev = *(volatile LDSP u32*)(scr + 384 + (lane & 31) * 4);
        u32 nv = *(volatile LDSP u32*)(scr + 12 + lane * 4);
        const u32 vo = (lds_ticket(ctl + C_ENC + j2, ed + nd) & 0xffff) + 4;
        if (lane == (nn >> 3)) nv |= ~0u << ((nn & 7) * 4);
        if (lane < nd) st4(pk + FIX + (vo + lane) * 4, nv);
        if (lane < ed) st4(pk + FIX + (vo + nd + lane) * 4, ev);
        if (lane == 0) st4(pk + FIX + (u * 2 + hf) * 4, vo | (base << 10) | (ne << 17) | (nd << 23));
      }
      u32x16 pw;
      asm volatile("s_load_dwordx16 %0, %1, 0x0\n\ts_waitcnt vmcnt(0) lgkmcnt(0)" : "=&s"(pw) : "s"(KA64(6)) : "memory");
#define PW(i) "s"(pw[2 * (i)] | ((u64)pw[2 * (i) + 1] << 32))
      const u32 k = lds_ticket(ctl + C_ENC + j2, over ? 0x1010000u : 0x1000000u);
      if ((k >> 24) == 3) {
        const u32 v = (call << 8) | ((over || (k & 0xff0000) != 0) ? M_RAW : ((k & 0xffff) + 4 + 31) >> 5);
        asm volatile("s_mov_b64 exec, 3\n\t"
                     "global_store_dwordx4 %0, %1, %2 sc0 sc1\n\tglobal_store_dwordx4 %0, %1, %3 sc0 sc1\n\t"
                     "global_store_dwordx4 %0, %1, %4 sc0 sc1\n\tglobal_store_dwordx4 %0, %1, %5 sc0 sc1\n\t"
                     "global_store_dwordx4 %0, %1, %6 sc0 sc1\n\tglobal_store_dwordx4 %0, %1, %7 sc0 sc1\n\t"
                     "global_store_dwordx4 %0, %1, %8 sc0 sc1\n\tglobal_store_dwordx4 %0, %1, %9 sc0 sc1\n\t"
                     "s_mov_b64 exec, -1\n\ts_nop 0"
                     :: "v"(DIR_OFF + ((par * NR + self) * NIT + blk) * 128 + lane * 16), "v"(u32x4{v, v, v, v}),
                        PW(0), PW(1), PW(2), PW(3), PW(4), PW(5), PW(6), PW(7) : "memory");
        if (lane < 16) st16<0>(me + UF_OFF + (self * NIT + blk) * 512 + (lane >> 2) * 128 + (lane & 3) * 16, u32x4{call, call, call, call});
      }
    }
  } else {
    if (link && wave == 8) return;
    const u32 cv = ld_agent(cnt);
    if (wave == 7) {
      u32 role = 0;
      if (link) {
        asm volatile("s_getreg_b32 %0, hwreg(HW_REG_XCC_ID)" : "=s"(role));
        role = (role & 7) * 8 + i8;
        for (int i = 0; i < 128; ++i) {
          GLB u32* rp = (GLB u32*)(me + ROLE_OFF + role * 128);
          u32 old = 0;
          if (lane == 0) old = __hip_atomic_fetch_add(rp, 1u, __ATOMIC_RELAXED, __HIP_MEMORY_SCOPE_AGENT);
          if (rfl(old) == rfl(cv)) break;
          if (lane == 0) __hip_atomic_fetch_sub(rp, 1u, __ATOMIC_RELAXED, __HIP_MEMORY_SCOPE_AGENT);
          role = (role + 1) & 63;
        }
      }
      ctl[lane] = lane == C_ROLE ? role : 0;
      waitl();
    }
    bar();
    call = rfl(cv) + 1;
  }
  __builtin_amdgcn_s_setprio(0);
  if (lane == 0 && wave != 11) st_agent(cnt, call);

  if (link) {
    const u32 role = rfl(ctl[C_ROLE]), lx = role >> 3, cw = role & 7;
    const int peer = (ka[9] >> (4 * lx)) & 7;
    if (peer == self) {
      if (lane == 0 && wave == 11) st_agent(cnt, call);
      if (cw != 0 || wave != 0) return;
      const char* pf = me + PF_OFF + lane * 32;
      u32 f;
      bool all;
      do {
        __builtin_amdgcn_s_sleep(16);
        f = ld_agent(pf);
        all = __builtin_amdgcn_ballot_w64((f & 0xfff) != (((call & 63) << 6) | 1) && (lane >> 3) != lx) == 0;
      } while (!all && __builtin_amdgcn_s_memrealtime() - tstart < PLANT);
      float v = (lane >> 3) == lx ? 0.f : (float)((int)(f - ((u32)tstart << 12)) >> 12);
      v += __shfl_xor(v, 1); v += __shfl_xor(v, 2); v += __shfl_xor(v, 4);
      v *= 0.125f;
      float mu = v + __shfl_xor(v, 8);
      mu += __shfl_xor(mu, 16); mu += __shfl_xor(mu, 32);
      mu *= 1.f / 7;
      u32 inv = 0;
#pragma unroll
      for (int x = 0; x < NR; ++x) inv |= x << (4 * ((ka[9] >> (4 * x)) & 7));
      const int ps = lane >> 3, pr = lane & 7;
      const float fs = __shfl(v, ((inv >> (4 * ps)) & 7) * 8), fr = __shfl(v, ((inv >> (4 * pr)) & 7) * 8);
      const float ex = ps == self || fs - mu < DZ ? 0.f : fs - mu, gp = pr == self ? 0.f : fmaxf(mu - fr, 0.f);
      float gs = gp + __shfl_xor(gp, 1);
      gs += __shfl_xor(gs, 2); gs += __shfl_xor(gs, 4);
      float mv = gs > 0.f ? fminf((float)TITEM, ex * gp / gs) : 0.f;
#pragma clang loop unroll(disable)
      for (int i = 0; i < 4; ++i) {
        float out = mv + __shfl_xor(mv, 1), in = mv + __shfl_xor(mv, 8);
        out += __shfl_xor(out, 2); out += __shfl_xor(out, 4);
        in += __shfl_xor(in, 16); in += __shfl_xor(in, 32);
        const float ns = fs - out + __shfl(in, ps), nr = fr + in - __shfl(out, pr * 8);
        if (ps != self && pr != self && ps != pr && ns - mu > 2 * DZ && ns > nr) mv = fminf((float)TITEM, mv + (ns - nr) * 0.2f);
      }
      u32 row = all ? (u32)(mv * (8.f / TITEM) + 0.5f) << (4 * pr) : 0;
      row |= __shfl_xor(row, 1); row |= __shfl_xor(row, 2); row |= __shfl_xor(row, 4);
      row = __shfl(row, pr * 8);
      __hip_atomic_store((GLB u64*)(me + PLAN_OFF + ps * 128 + pr * 8), row | ((u64)call << 32), __ATOMIC_RELAXED, __HIP_MEMORY_SCOPE_AGENT);
      return;
    }
    if (wave == 11) {
      LOAD_PWS();
      const u32 par = call & 1;
      u32 nitems = (NIT - NPUSH - cw + 7) >> 3;
      u32 itv = lane < nitems ? sched_item(cw + lane * 8, lx, self) : 0;
      const char* dir = me + DIR_OFF + ((par * NR + peer) * NIT + itv) * 128;
      u32 dv = 0;
      u64 todo = (1ull << nitems) - 1, ok = 0;
      bool have = false;
      while (__builtin_amdgcn_s_memrealtime() - tstart < GATE - 50) __builtin_amdgcn_s_sleep(1);
      for (u32 n = 0; n < nitems + NGRP; ++n) {
        u32 w0 = (IT_END << 6) | 0x80000;
        if (n < nitems) {
          const u32 first = __builtin_ctzll(todo);
          const u64 cand = todo & (((n < 8 ? 7ull : 1ull) << first) | (0xffull << 56));
          const bool rl = (todo >> 56) != 0 && ((ok & todo) >> 56) == 0;
          for (u32 sp = 1; (ok & cand) == 0 || (rl && sp == 1); ++sp) {
            if (sp > 1 && n > 2) __builtin_amdgcn_s_sleep(4);
            if (lane - first < (n < 8 ? 4u : 64u) && ((todo & ~ok) >> lane) & 1) {
              dv = ld_sys(dir);
              if (lane >= 56) {
                const char* rf = pws + RLF_OFF + ((lane - 56) * 8 + ((itv >> 3) & 7)) * 128;
                if (min(ld_sys(rf), ld_sys(rf + 32)) < call) dv = 0;
              }
            }
            ok = __builtin_amdgcn_ballot_w64((dv >> 8) == (call & 0xffffff));
            if ((sp & 31) == 0 && give_up(me, call, tstart)) { dv = (call << 8) | M_RAW; ok = ~0ull; }
          }
          const u64 pick = ok & cand;
          const u32 ln = (pick >> 56) != 0 ? 63 - __builtin_clzll(pick) : __builtin_ctzll(pick);
          todo &= ~(1ull << ln);
          const u32 it = __builtin_amdgcn_readlane(itv, ln);
          u32 e = __builtin_amdgcn_readlane(dv, ln);
          e &= (e & M_RAW) ? M_RAW : 0xf;
          const u32 so = ln >= 56 ? RL_OFF / ITEM + (par * NR + ln - 56) * 8 + ((it >> 3) & 7)
                         : it >= 256 ? par * 512 + self * 32 + it : (e & M_RAW) ? RAW_OFF / ITEM + (par ? peer : 8) * NIT + it : par * 512 + it;
          w0 = (it << 6) | ((e & M_RAW) ? 0x80000 : e << 15) | (so << 20);
        }
        if (n >= LA) {
          volatile LDSP u32x2* cp = (volatile LDSP u32x2*)(ctl + C_CONS + (n % NGRP) * 2);
          for (;;) { const u32x2 cv = *cp; if (rfl(min(cv.x, cv.y)) + LA > n) break; __builtin_amdgcn_s_sleep(2); }
        }
        if (lane == 0) ctl[C_DESC + n % NS] = w0 | (n + 1);
        if (!have && n >= 17 && (n & 1)) {
          u64 pv;
          for (u32 sp = 1;; ++sp) {
            pv = __hip_atomic_load((const GLB u64*)(me + PLAN_OFF + lx * 128 + (lane & 7) * 8), __ATOMIC_RELAXED, __HIP_MEMORY_SCOPE_AGENT);
            have = __builtin_amdgcn_ballot_w64((u32)(pv >> 32) != call) == 0;
            if (have || n < 27) break;
            if ((sp & 31) == 0 && give_up(me, call, tstart)) { pv = 0; have = true; break; }
          }
          if (have) {
            const u32 row = (u32)pv, mine = __builtin_amdgcn_readlane(row, peer);
            const u32 r = ((itv >> 1) & 3) * 2 + (itv & 1), col = (itv >> 3) & 7;
            const u64 skip = __builtin_amdgcn_ballot_w64((itv >> 6) == 2 && ((col - nib_sum(mine & ((1u << (4 * r)) - 1))) & 7) < ((mine >> (4 * r)) & 15));
            const u32 nt = lane >= 56 ? (row >> (4 * peer)) & 15 : 0, kk = (cw - (scan_add(nt) - nt)) & 7;
            const u64 take = __builtin_amdgcn_ballot_w64(kk < nt);
            if (lane >= 56) {
              itv = round_item(16 + peer, (nib_sum(row & ((1u << (4 * peer)) - 1)) + kk) & 7);
              dir = me + DIR_OFF + ((par * NR + lane - 56) * NIT + itv) * 128;
            }
            todo = (todo & ~skip) | take;
            nitems = n + 1 + __builtin_popcountll(todo);
            if (lane == 0) ctl[C_NIT] = nitems;
          }
        }
      }
      if (lane == 0) st_agent(cnt, call);
      return;
    }
    if (wave == 9) {
      const u32 n0 = (NIT - NPUSH - cw + 7) >> 3;
      char* bp = me + BAL_OFF + lx * 256;
      u32 until = 4 * K0;
      for (u32 r = 0;; ++r) {
        u32 p0, p1;
        while (((p0 = rfl(ctl[C_LANDED])) & 0xffff) < until) __builtin_amdgcn_s_sleep(4);
        for (int i = 0; (p1 = rfl(ctl[C_LANDED])) == p0 && i < 12; ++i) {}
        const u32 now = (u32)__builtin_amdgcn_s_memrealtime(), nn = rfl(ctl[C_NIT]), nitems = nn ? nn : n0, hu = p1 & 0xffff;
        const u32 fown = (now + ((nitems * 23 - hu * 4 - (p1 >> 16) * 7) * TITEM) / 23) << 12 | ((call & 63) << 6) | (nn ? 2 : 1);
        if (lane == 0) st_agent(bp + cw * 32, fown);
        if (r == 0) {
          if (lane == 0) st_agent(bp + PF_OFF - BAL_OFF + cw * 32, fown);
          until = 64;
          continue;
        }
        if (r == 1) { until = min(hu + 6, 4 * (n0 - 8)); continue; }
        if (hu + 12 >= 4 * nitems) break;
        const u32 f = ld_agent(bp + (lane & 7) * 32);
        const bool v = ((f ^ fown) & 0xfff) == 0;
        const int cnt8 = __builtin_popcount((u32)__builtin_amdgcn_ballot_w64(v) & 0xff);
        const int late = cnt8 < 6 ? 0 : (int)__builtin_amdgcn_readlane(scan_add(v ? (u32)((int)(fown - f) >> 12) : 0u), 7) / cnt8;
        const u32 plan = min((u32)(late > 0 ? (late + GFAST / 4) / GFAST : (GSLOW / 4 - late) / GSLOW), 4u);
        until = (rfl(ctl[C_LANDED]) & 0xffff) + 4 * plan;
        if (plan != 0) {
          if (lane == 0) ctl[C_WIN] = late > 0 ? 1 : 2;
          until = min(until, 4 * (nn ? nn : n0 - 8));
          while ((rfl(ctl[C_LANDED]) & 0xffff) < until) __builtin_amdgcn_s_sleep(4);
          if (lane == 0) ctl[C_WIN] = 0;
        }
        until = min(until + 12, 4 * (nn ? nn : n0 - 8));
      }
      return;
    }
    if (wave == 10) {
      LOAD_PWS();
      if (lane < 3) touch(pws + PK_OFF + ((call & 1) * 512 + (lane == 2 ? 256 + self * 32 + 16 : lane * 255)) * ITEM + (self * 8 + cw) * 128, dump);
      waitv<0>();
      drain();
      const u32 l0 = (u32)(size_t)lds, lw = l0 + L_CTL + C_LANDED * 4, one = lane == 0 ? 1 : 0;
      u32 prog = 0, mk = 0, wm = 0, la = lane == WD ? lw : dump + lane * 4;
      __builtin_amdgcn_s_setprio(3);
#define MODE_ "s_cmp_eq_u32 %[wm], 1\n\ts_cselect_b64 vcc, -1, 0\n\ts_cmp_eq_u32 %[wm], 2\n\t"
#define LD_(off, id) "s_waitcnt vmcnt(%[wf])\n\ts_cbranch_vccnz .Lw" #id "_%=\n\ts_waitcnt vmcnt(%[w])\n\ts_cbranch_scc0 .Lw" #id "_%=\n\t" \
                     "s_waitcnt vmcnt(%[ws])\n.Lw" #id "_%=:\n\tglobal_load_lds_dwordx4 %[va], %[sb] offset:" #off " sc0 sc1\n\t"
#define SH_ "v_mov_b32_dpp %[mk], %[mk] row_shr:1 row_mask:0xf bank_mask:0xf\n\t"
#define PUB_ "ds_max_u32 %[la], %[mk]\n\t"
#define INC_(x) "v_add_u32 %[mk], %[mk], " x "\n\t"
#define WIN_ [wm] "s"(wm), [wf] "i"(WFAST - 1), [w] "i"(WD - 1), [ws] "i"(WD - 2)
#define FOUR(src, ldst, i0) do { u32 keep_; \
          asm volatile("ds_read_b32 %[dv], %[da]\n\tds_read_b32 %[wv], %[lw] offset:4\n\ts_mov_b32 %[k], m0\n\ts_mov_b32 m0, %[ld]\n\t" MODE_ \
                       LD_(0, 0) SH_ INC_("%[i]") PUB_ LD_(1024, 1) SH_ INC_("%[o]") PUB_ LD_(2048, 2) SH_ INC_("%[i]") PUB_ LD_(3072, 3) SH_ INC_("%[o]") \
                       "s_waitcnt lgkmcnt(3)\n\t" PUB_ "s_mov_b32 m0, %[k]" \
                       : [k] "=&s"(keep_), [dv] "=&v"(dvv), [wv] "=&v"(wvv), [mk] "+v"(mk) \
                       : [va] "v"((src) + lane * 16), [sb] "s"(pws), [ld] "s"(ldst), [la] "v"(la), [da] "v"(l0 + L_CTL + (C_DESC + se) * 4), \
                         [lw] "v"(lw), [i] "v"((i0) & one), [o] "v"(one), WIN_ : "memory", "scc", "vcc"); \
          prog += 2 + 2 * (i0); } while (0)
      u64 qu = 0;
      u32 ne = 0, se = 0, sc = 0, d = 0, dvv, wvv;
      bool fin = false;
      for (u32 n = 0;; ++n) {
#pragma clang loop unroll(disable)
        while (ne <= n + EA && !fin) {
          if ((d & 63) != ne + 1) {
            if (ne > n) break;
            d = rfl(ctl[C_DESC + se]);
            if ((d & 63) != ne + 1) {
              waitv<0>();
              if (lane == 0) ctl[C_LANDED] = prog;
              do { d = rfl(ctl[C_DESC + se]); } while ((d & 63) != ne + 1);
            }
          }
          fin = ((d >> 6) & 0x1ff) == IT_END;
          qu |= (u64)((d >> 20) | ((d >> 7) & 0x1000) | (fin ? 0x2000u : 0u)) << (16 * (ne - n));
          ++ne;
          if (!(d & 0x80000)) {
            const u32 ln = (d >> 15) & 15, pn = (prog & 0xffff) | (ne << 16);
            const u64 em = (ln & 7) ? (1ull << ((ln & 7) * 8)) - 1 : ~0ull;
            u32 keep;
            asm volatile("s_mov_b32 %[k], m0\n\ts_mov_b32 m0, %[ld]\n\t" MODE_
                         LD_(0, 0) SH_ PUB_
                         "s_cmp_gt_u32 %[ln], 8\n\ts_cbranch_scc1 .L3_%=\n\ts_cmp_eq_u32 %[wm], 2\n\ts_mov_b64 exec, %[em]\n\t" LD_(1024, 1) "s_branch .L4_%=\n"
                         ".L3_%=:\n\ts_cmp_eq_u32 %[wm], 2\n\t" LD_(1024, 2) SH_ PUB_ "s_mov_b64 exec, %[em]\n\t" LD_(2048, 3)
                         "\n.L4_%=:\n\ts_mov_b64 exec, -1\n\t" SH_ "v_writelane_b32 %[mk], %[pn], 0\n\t" PUB_ "s_mov_b32 m0, %[k]"
                         : [k] "=&s"(keep), [mk] "+v"(mk)
                         : [va] "v"((d >> 20) * ITEM + 4096 + lane * 16), [sb] "s"(pws), [ld] "s"(l0 + se * ITEM + 4096), [la] "v"(la),
                           [ln] "s"(ln), [em] "s"(em), [pn] "s"(pn), WIN_ : "memory", "scc", "vcc");
            prog = pn;
          }
          se = se + 1 == NS ? 0 : se + 1;
          if (n == 0) break;
          if (ne <= n + EA) d = rfl(ctl[C_DESC + se]);
        }
        const u32 cur = (u32)qu & 0xffff;
        qu >>= 16;
        if (cur & 0x2000) break;
        u32 so = (cur & 0xfff) * ITEM, lo = l0 + sc * ITEM;
#pragma clang loop unroll(disable)
        for (u32 k = cur >> 12;; --k) {
          FOUR(so, lo, 1 - (cur >> 12));
          if (k == 0) break;
          so += 4096; lo += 4096;
        }
        d = rfl(dvv);
        wm = rfl(wvv);
        la = lane == (wm == 1 ? WFAST : wm == 2 ? WD - 1 : WD) ? lw : dump + lane * 4;
        sc = sc + 1 == NS ? 0 : sc + 1;
      }
#define DRAIN(N) if (N < WFAST) \
        asm volatile("s_mov_b64 exec, %2\n\ts_waitcnt vmcnt(%3)\n\tds_max_u32 %0, %1\n\ts_mov_b64 exec, -1" \
                     :: "v"(lw), "v"(mk), "i"(1 << N), "i"(N) : "memory");
      DRAIN(9) DRAIN(8) DRAIN(7) DRAIN(6) DRAIN(5) DRAIN(4) DRAIN(3) DRAIN(2) DRAIN(1) DRAIN(0)
      return;
    }
    if (wave == 0 && lane < 4) touch(me + (lane == 3 ? UF_OFF + peer * NIT * 512 : RAW_OFF + (peer * NIT + lane * 143) * ITEM) + cw * 128, dump);
    drain();
    const int u = wave & 1, dw = wave;
    LDSP char* img = lds + L_IMG + wave * 4096;
    u32 m0f = 0x0f0f0f0fu, m80 = 0x80808080u, m3e = 0x3e3e3e3eu, m30 = 0x30303030u;
    u32 mffc = 0xffc;
    const u32 imga = (u32)(size_t)img;
    asm volatile("" : "+s"(m0f), "+s"(m80), "+s"(m3e), "+s"(m30), "+v"(mffc));
    u32 pend = 0;
#define PUBLISH() do { if (pend != 0) { waitv<0>(); \
        if (lane < 8) st16<0>(me + UF_OFF + (pend - 1) * 512 + u * 32 + (lane >> 1) * 128 + (lane & 1) * 16, u32x4{call, call, call, call}); \
        pend = 0; } } while (0)
    for (u32 n = dw >> 1;; n += NGRP) {
      u32 w0 = rfl(ctl[C_DESC + n % NS]);
      if ((w0 & 0x8003f) != n + 1 || (rfl(ctl[C_LANDED]) >> 16) <= n) PUBLISH();
      while ((w0 & 63) != n + 1) { __builtin_amdgcn_s_sleep(2); w0 = rfl(ctl[C_DESC + n % NS]); }
      const u32 it = (w0 >> 6) & 0x1ff;
      if (it == IT_END) break;
      LDSP char* sl = lds + (n % NS) * ITEM;
      const u32 rlo = (w0 >> 20) - RL_OFF / ITEM;
      const u32 src = rlo < 128 ? (rlo >> 3) & 7 : peer;
      const bool keep = (it >> 6) == 2 && ((it >> 1) & 3) * 2 + (it & 1) == self;
      char* d = me + RAW_OFF + (src * NIT + it) * ITEM + u * 4096 + lane * 16;
      u32x4 o0, o1, o2, o3;
#define LANDED(m, x) do { while ((rfl(ctl[C_LANDED]) & (m)) < (x)) __builtin_amdgcn_s_sleep(1); fence(); } while (0)
      if (w0 & 0x80000) {
        LANDED(0xffffu, 4 * n + 2 * u + 2);
        LDSP char* s = sl + u * 4096 + lane * 16;
        o0 = *(LDSP u32x4*)s; o1 = *(LDSP u32x4*)(s + 1024); o2 = *(LDSP u32x4*)(s + 2048); o3 = *(LDSP u32x4*)(s + 3072);
        st16<0>(d, o0); st16<1024>(d, o1); st16<2048>(d, o2); st16<3072>(d, o3);
      } else {
        LANDED(0xffff0000u, (n + 1) << 16);
        const u32x2 hv = *(LDSP u32x2*)(sl + FIX + u * 8);
        const u32 h0 = rfl(hv.x), h1 = rfl(hv.y);
        const u32 nd0 = h0 >> 23, nd1 = h1 >> 23;
        const u32 b0 = ((h0 >> 10) & 0x7f) * 0x01010101u - 0x3e3e3e3eu, b1 = ((h1 >> 10) & 0x7f) * 0x01010101u - 0x3e3e3e3eu;
        LDSP char* v0 = sl + FIX + (h0 & 0x1ff) * 4;
        LDSP char* v1 = sl + FIX + (h1 & 0x1ff) * 4;
        u32 nb0 = *(LDSP u32*)(v0 + lane * 4), nb1 = *(LDSP u32*)(v1 + lane * 4);
        const u32 e0 = *(LDSP unsigned short*)(v0 + (nd0 & 0x7f) * 4 + lane * 2), e1 = *(LDSP unsigned short*)(v1 + (nd1 & 0x7f) * 4 + lane * 2);
        const u32 sg0 = *(LDSP u32*)(sl + 4096 + u * 512 + lane * 4), sg1 = *(LDSP u32*)(sl + 4096 + u * 512 + 256 + lane * 4);
        {
          LDSP char* iz = img + lane * 16;
          *(LDSP u32x4*)iz = u32x4{b0, b0, b0, b0}; *(LDSP u32x4*)(iz + 1024) = u32x4{b0, b0, b0, b0};
          *(LDSP u32x4*)(iz + 2048) = u32x4{b1, b1, b1, b1}; *(LDSP u32x4*)(iz + 3072) = u32x4{b1, b1, b1, b1};
        }
        if (lane >= nd0) nb0 = ~0u;
        if (lane >= nd1) nb1 = ~0u;
        u32 gg[4], vv[4];
        gg[0] = (nb0 & m0f) << 3; gg[1] = (nb0 >> 1) & 0x78787878u; gg[2] = (nb1 & m0f) << 3; gg[3] = (nb1 >> 1) & 0x78787878u;
        const u32 sum = __builtin_amdgcn_sad_u8(gg[0], 0, __builtin_amdgcn_sad_u8(gg[1], 0, 0)) |
                        (__builtin_amdgcn_sad_u8(gg[2], 0, __builtin_amdgcn_sad_u8(gg[3], 0, 0)) << 16);
#pragma unroll
        for (int k = 0; k < 4; ++k) {
          vv[k] = (~(gg[k] + 0x08080808u) & m80) >> 3;
          asm("" : "+v"(gg[k]), "+v"(vv[k]));
        }
        const u32 ps = scan_add(sum) - sum;
        u32 pa = ps & 0xffff, pb = (ps >> 16) + 2048 * 8;
        if (lane < max(nd0, nd1)) {
#pragma unroll
          for (int k = 0; k < 8; ++k) {
            const int w = k & 1, by = (k >> 1) * 8;
            pa += (gg[w] >> by) & 0xff;
            pb += (gg[2 + w] >> by) & 0xff;
            LDS_SUB((LDSP char*)(size_t)(((pa >> 3) & mffc) | imga), ((vv[w] >> by) & 0xff) << (pa & 31));
            LDS_SUB((LDSP char*)(size_t)(((pb >> 3) & mffc) | imga), ((vv[2 + w] >> by) & 0xff) << (pb & 31));
          }
        }
        fence();
        PUBLISH();
        LDSP char* ip = img + lane * 32;
        u32x4 a0 = *(LDSP u32x4*)ip, a1 = *(LDSP u32x4*)(ip + 16), a2 = *(LDSP u32x4*)(ip + 2048), a3 = *(LDSP u32x4*)(ip + 2064);
        fence();
        if (lane < ((h0 >> 17) & 63)) LDS_SUB(img + (e0 & 0xffc), ((e0 >> 12) - 8) << ((e0 & 3) * 8 + 4));
        if (lane < ((h1 >> 17) & 63)) LDS_SUB(img + (e1 & 0xffc), ((e1 >> 12) - 8) << ((e1 & 3) * 8 + 4));
        fence();
        o0 = *(LDSP u32x4*)ip; o1 = *(LDSP u32x4*)(ip + 16); o2 = *(LDSP u32x4*)(ip + 2048); o3 = *(LDSP u32x4*)(ip + 2064);
#pragma unroll
        for (int q = 0; q < 4; ++q) {
          o0[q] += (sg0 << q) & m80; o1[q] += (sg0 << (4 + q)) & m80;
          o2[q] += (sg1 << q) & m80; o3[q] += (sg1 << (4 + q)) & m80;
          const u32 l0 = b0 - a0[q], l1 = b0 - a1[q], l2 = b1 - a2[q], l3 = b1 - a3[q];
          a0[q] = (l0 << 1) | (l0 >> 1); a1[q] = (l1 << 1) | (l1 >> 1); a2[q] = (l2 << 1) | (l2 >> 1); a3[q] = (l3 << 1) | (l3 >> 1);
        }
        asm volatile("" : "+v"(o0), "+v"(o1), "+v"(o2), "+v"(o3), "+v"(a0), "+v"(a1), "+v"(a2), "+v"(a3));
#define CODES(o, a, x) _Pragma("unroll") for (int q = 0; q < 4; ++q) { \
          u32 c_ = ((x) & m0f) + m3e; asm("" : "+v"(c_)); o[q] += c_ + (c_ & a[q] & m30); }
        LANDED(0xffffu, 4 * n + 2 * u + 1);
        const u32x4 n0 = *(LDSP u32x4*)(sl + u * 2048 + lane * 16);
        CODES(o0, a0, n0[q]) st16<0>(d, o0);
        CODES(o1, a1, n0[q] >> 4) st16<1024>(d, o1);
        LANDED(0xffffu, 4 * n + 2 * u + 2);
        const u32x4 n1 = *(LDSP u32x4*)(sl + u * 2048 + 1024 + lane * 16);
        CODES(o2, a2, n1[q]) st16<2048>(d, o2);
        CODES(o3, a3, n1[q] >> 4) st16<3072>(d, o3);
      }
      if (keep) {
        LANDED(0xffffu, 4 * n + 4);
        LDSP char* s = sl + u * 4096 + lane * 16;
        char* r = me + RL_OFF + (((call & 1) * NR + peer) * 8 + ((it >> 3) & 7)) * ITEM + u * 4096 + lane * 16;
        st16<0>(r, *(LDSP u32x4*)s); st16<1024>(r, *(LDSP u32x4*)(s + 1024));
        st16<2048>(r, *(LDSP u32x4*)(s + 2048)); st16<3072>(r, *(LDSP u32x4*)(s + 3072));
      }
      if (lane == 0) ctl[C_CONS + dw] = n + 1;
      pend = src * NIT + it + 1;
      if (keep) { PUBLISH(); put_flag(me + RLF_OFF + (peer * 8 + ((it >> 3) & 7)) * 128 + u * 32, call, lane); }
    }
    return;
  }

  const int t = xcd * 4 + (ct >> 1);
  const int dbl = 1 - (ct >> 2);

  if (wave >= 4) {
    drain();
    const int src = wave - 4;
    if (lane == 0 && wave == 11) st_agent(cnt, call);
    const char* raw = me + RAW_OFF + (src != self || (call & 1) ? src : 8) * NIT * ITEM + lane * 16;
    const char* uf = me + UF_OFF + src * NIT * 512;
    const char* px = me + PX_OFF + ((call & 1) * NR + src) * 16 * ITEM + lane * 16;
    const u32 lA = (u32)(size_t)lds + L_A + src * 16384;
    const u64 t0 = tstart;
    u32 need = call;
    u32 prev = 0;
    const char* fa = uf + ((128 + ((lane >> 2) & 3) * 4 + mm) * 2 + ((lane >> 1) & 1)) * 512 + (xcd & 3) * 128 + (lane & 1) * 32;
    u64 xmiss = 0xffff;
    u32 xf = 0;
#pragma clang loop unroll(disable)
    for (int j = 0; j < 4; ++j) {
      {
        for (u32 it = 1; ((xmiss >> (4 * j)) & 15) != 0; ++it) {
          if (it > 1) __builtin_amdgcn_s_sleep(8);
          if ((xmiss >> lane) & 1) xf = ld_sys(fa);
          xmiss = __builtin_amdgcn_ballot_w64(xf < need) & 0xffff;
          if ((it & 15) == 0 && give_up(me, call, t0)) need = 0;
        }
        while (rfl(min4(*(volatile LDSP u32x4*)(ctl + C_FREE))) < prev) __builtin_amdgcn_s_sleep(2);
        const int hx = (j * 4 + mm) * 2;
        const char* sa0 = hx < NPUSH ? px + hx * ITEM : raw + (256 + hx) * ITEM;
        const char* sa1 = hx + 1 < NPUSH ? px + (hx + 1) * ITEM : raw + (257 + hx) * ITEM;
        u32 keep;
#pragma unroll
        for (int i = 0; i < 4; ++i)
          asm volatile("s_mov_b32 %0, m0\n\ts_mov_b32 m0, %2\n\ts_nop 0\n\t"
                       "global_load_lds_dwordx4 %1, off sc0 sc1\n\t"
                       "global_load_lds_dwordx4 %1, off offset:1024 sc0 sc1\n\t"
                       "global_load_lds_dwordx4 %1, off offset:2048 sc0 sc1\n\t"
                       "global_load_lds_dwordx4 %1, off offset:3072 sc0 sc1\n\t"
                       "s_mov_b32 m0, %0" : "=&s"(keep) : "v"((i < 2 ? sa0 : sa1) + (i & 1) * 4096), "s"(lA + i * 4096) : "memory");
      }
      const int nh0 = dbl ? 0 : (ct & 1);
      const char* fb = uf + ((j * 32 + t) * 2 + nh0 + ((lane >> 1) & dbl)) * 512 + mm * 128 + (lane & 1) * 32;
      const char* pp = raw + ((j * 32 + t) * 2 + nh0) * ITEM + 4096;
      for (u32 it = 1;; ++it) {
        u32 fv = need;
        if (lane < 4) fv = ld_sys(fb);
        if (__builtin_amdgcn_ballot_w64(fv < need) == 0) break;
        if ((it & 63) == 0 && give_up(me, call, t0)) need = 0;
        if (j < 3) __builtin_amdgcn_s_sleep(8); else __builtin_amdgcn_s_sleep(1);
      }
      u32x4 p[8], p2[8];
      u32 tv, tk = 1 + dbl;
      asm volatile("global_load_dwordx4 %0, %10, off offset:-4096 sc0 sc1\n\t"
                   "global_load_dwordx4 %1, %10, off offset:-3072 sc0 sc1\n\t"
                   "global_load_dwordx4 %2, %10, off offset:-2048 sc0 sc1\n\t"
                   "global_load_dwordx4 %3, %10, off offset:-1024 sc0 sc1\n\t"
                   "global_load_dwordx4 %4, %10, off sc0 sc1\n\t"
                   "global_load_dwordx4 %5, %10, off offset:1024 sc0 sc1\n\t"
                   "global_load_dwordx4 %6, %10, off offset:2048 sc0 sc1\n\t"
                   "global_load_dwordx4 %7, %10, off offset:3072 sc0 sc1\n\t"
                   "global_load_dword %8, %11, off sc1\n\t"
                   "s_mov_b64 exec, 1\n\tds_add_rtn_u32 %9, %12, %9\n\ts_mov_b64 exec, -1\n\ts_waitcnt lgkmcnt(0)"
                   : "=&v"(p[0]), "=&v"(p[1]), "=&v"(p[2]), "=&v"(p[3]), "=&v"(p[4]), "=&v"(p[5]), "=&v"(p[6]), "=&v"(p[7]),
                     "=&v"(tv), "+v"(tk)
                   : "v"(pp), "v"(me + TMO_OFF + b * 128), "v"((u32)(size_t)(lds + L_CTL + C_TICKET * 4)) : "memory");
      asm volatile("s_cmp_eq_u32 %9, 0\n\ts_cbranch_scc1 .Ls_%=\n\t"
                   "global_load_dwordx4 %0, %8, off offset:-4096 sc0 sc1\n\t"
                   "global_load_dwordx4 %1, %8, off offset:-3072 sc0 sc1\n\t"
                   "global_load_dwordx4 %2, %8, off offset:-2048 sc0 sc1\n\t"
                   "global_load_dwordx4 %3, %8, off offset:-1024 sc0 sc1\n\t"
                   "global_load_dwordx4 %4, %8, off sc0 sc1\n\t"
                   "global_load_dwordx4 %5, %8, off offset:1024 sc0 sc1\n\t"
                   "global_load_dwordx4 %6, %8, off offset:2048 sc0 sc1\n\t"
                   "global_load_dwordx4 %7, %8, off offset:3072 sc0 sc1\n"
                   ".Ls_%=:"
                   : "=&v"(p2[0]), "=&v"(p2[1]), "=&v"(p2[2]), "=&v"(p2[3]), "=&v"(p2[4]), "=&v"(p2[5]), "=&v"(p2[6]), "=&v"(p2[7])
                   : "v"(pp + 8192), "s"(dbl) : "memory", "scc");
      u32 ts = rfl(tk) & 0xffff, fs = 0;
      const u32 t1 = (rfl(tk) >> 16) - 1;
#define SLOT(t) ({ const u32 k_ = (t) < t1 ? (t) % 3 : ((t) - t1) % 5; k_ < 3 ? L_B / 8192 + k_ : self * 2 + k_ - 3; })
#define NEEDF(t) ((t) + 8 - ((t) >= t1 && (t) - t1 >= 3 ? 4 : 2))
      u32 sl0 = SLOT(ts), sl1 = SLOT(ts + 1), nf0 = NEEDF(ts), nf1 = NEEDF(ts + 1);
      asm volatile("" : "+s"(sl0), "+s"(sl1), "+s"(nf0), "+s"(nf1));
#define ROOM(nf) while (fs < (nf)) { fs = rfl(min4(*(volatile LDSP u32x4*)(ctl + C_FREE))) + 8; if (fs < (nf)) __builtin_amdgcn_s_sleep(1); }
#define FULL(sl) do { ++ts; \
        asm volatile("" : "+s"(ts) :: "memory"); \
        if (lane == 0) ctl[C_FULL + ((ts - 1) & 7)] = ts | (src << 8) | ((sl) << 11) | (tv == call ? 0x10000u : 0u); } while (0)
#define PUT(R, sl) do { LDSP char* ub = lds + (sl) * 8192 + lane * 16; \
        _Pragma("unroll") for (int i = 0; i < 8; ++i) *(LDSP u32x4*)(ub + i * 1024) = R[i]; \
        FULL(sl); } while (0)
      ROOM(nf0);
      {
        LDSP char* ub = lds + sl0 * 8192 + lane * 16;
#define PUT2(i, N) asm volatile("s_waitcnt vmcnt(" #N ")" : "+v"(p[i]), "+v"(p[i + 1]) :: "memory"); \
        *(LDSP u32x4*)(ub + (i) * 1024) = p[i]; *(LDSP u32x4*)(ub + (i) * 1024 + 1024) = p[i + 1];
        PUT2(0, 7) PUT2(2, 5) PUT2(4, 3) PUT2(6, 1)
        asm volatile("s_waitcnt vmcnt(0)" : "+v"(p2[0]), "+v"(p2[1]), "+v"(p2[2]), "+v"(p2[3]), "+v"(p2[4]), "+v"(p2[5]), "+v"(p2[6]), "+v"(p2[7]),
                     "+v"(tv) :: "memory");
        FULL(sl0);
      }
      if (dbl) { ROOM(nf1); fence(); PUT(p2, sl1); }
      prev = ts;
    }
    if (src == self) {
      while (rfl(min4(*(volatile LDSP u32x4*)(ctl + C_FREE))) < prev) __builtin_amdgcn_s_sleep(2);
      if (lane == 0) {
        u32 o = ctl[C_TICKET];
        while (!__hip_atomic_compare_exchange_strong((LDSP u32*)(ctl + C_TICKET), &o, o | (((o + 2) / 3 * 3 + 1) << 16),
                                                     __ATOMIC_RELAXED, __ATOMIC_RELAXED, __HIP_MEMORY_SCOPE_WORKGROUP)) {}
      }
    }
    return;
  }

  const int mw = wave;
  {
    u32 scale_bits = __builtin_amdgcn_readfirstlane(__builtin_bit_cast(u32, *(const GLB float*)KA64(12) * *(const GLB float*)KA64(14)));
    float* obase = (float*)KA64(10) + (size_t)(mm * 128 + mw * 32) * NN + t * 128 + (ct & 1) * 64;
    asm volatile("" : "+s"(scale_bits), "+s"(obase));
    drain();
    v4f Pc[2][4], Qc[2][4];
#pragma unroll
    for (int i = 0; i < 2; ++i)
#pragma unroll
      for (int k = 0; k < 4; ++k) Pc[i][k] = Qc[i][k] = v4f{0.f, 0.f, 0.f, 0.f};
#define FRAG(p, hs) ({ const u32x4 lo_ = *(LDSP u32x4*)(p), hi_ = *(LDSP u32x4*)((p) + (hs)); \
      v8i{(int)lo_.x, (int)lo_.y, (int)lo_.z, (int)lo_.w, (int)hi_.x, (int)hi_.y, (int)hi_.z, (int)hi_.w}; })
#define MMA(i, k, F) do { F##c[i][k] = __builtin_amdgcn_mfma_scale_f32_16x16x128_f8f6f4(F##a[i], F##b[k], F##c[i][k], 1, 1, 0, 0, 0, 0); \
      __builtin_amdgcn_sched_barrier(0); } while (0)
#define SB() __builtin_amdgcn_sched_barrier(0)
#define RD(F) do { bad |= w; LDSP char* ap_ = lds + L_A + ((w >> 8) & 7) * 16384 + mw * 4096 + lane * 16; \
      bp = lds + ((w >> 11) & 31) * 8192 + lane * 16; \
      F##a[0] = FRAG(ap_, 1024); F##b[0] = FRAG(bp, 4096); SB(); F##a[1] = FRAG(ap_ + 2048, 1024); SB(); F##b[1] = FRAG(bp + 1024, 4096); \
      SB(); } while (0)
#define SPIN() do { while (((w = rfl(ctl[C_FULL + (n & 7)])) & 0xff) != n + 1) __builtin_amdgcn_s_sleep(0); \
      fence(); } while (0)
#define HEAD(F) do { \
      MMA(0, 0, F); F##b[2] = FRAG(bp + 2048, 4096); SB(); \
      MMA(1, 0, F); F##b[3] = FRAG(bp + 3072, 4096); \
      fence();     \
      ++n; \
      if (lane == 0) ctl[C_FREE + mw] = n; } while (0)
#define STEP(F, G, EXTRA) do { \
      HEAD(F); \
      peek = ctl[C_FULL + (n & 7)]; SB(); \
      MMA(0, 1, F); MMA(1, 1, F); MMA(0, 2, F); SB(); \
      w = rfl(peek); \
      fence(); \
      const bool ok_ = (w & 0xff) == n + 1; \
      if (ok_) RD(G); \
      MMA(1, 2, F); MMA(0, 3, F); MMA(1, 3, F); SB(); \
      EXTRA; \
      if (!ok_) { SPIN(); RD(G); } } while (0)
    LDSP char* bp;
    v8i Pa[2], Pb[4], Qa[2], Qb[4];
    u32 n = 0, w, peek, bad = 0;
    SPIN();
    RD(P);
#pragma clang loop unroll(disable)
    for (int it = 1; it < (16 << dbl); ++it) { STEP(P, Q, ); STEP(Q, P, ); }
#define EPI(A, i, r, o) do { v4f y_; _Pragma("unroll") for (int k = 0; k < 4; ++k) \
        y_[k] = __builtin_elementwise_maximum(A[i][k][r] * scale, 0.f);     \
      asm volatile("global_store_dwordx4 %0, %1, %2 sc1\n\ts_nop 1" :: "v"(ov), "v"(y_), "s"((o) + ((i) * 16 + (r)) * NN) : "memory"); \
      __builtin_amdgcn_sched_barrier(0); } while (0)
    STEP(P, Q, if (!dbl) { fence();
                 _Pragma("unroll") for (int i = 0; i < 2; ++i) _Pragma("unroll") for (int k = 0; k < 4; ++k) Qc[i][k] += Pc[i][k]; });
    HEAD(Q); SB();
    MMA(0, 1, Q); MMA(0, 2, Q); MMA(0, 3, Q); MMA(1, 1, Q);
    const float scale = __builtin_bit_cast(float, (bad & 0x10000u) != 0 ? 0x7fc00000u : scale_bits);
    const u32 ln = __builtin_amdgcn_mbcnt_hi(~0u, __builtin_amdgcn_mbcnt_lo(~0u, 0));
    const u32 ov = (ln >> 4) * (4 * NN * 4) + (ln & 15) * 16;
    SB();
    EPI(Qc, 0, 0, obase); MMA(1, 2, Q); EPI(Qc, 0, 1, obase); MMA(1, 3, Q); EPI(Qc, 0, 2, obase); EPI(Qc, 0, 3, obase);
    EPI(Qc, 1, 0, obase); EPI(Qc, 1, 1, obase); EPI(Qc, 1, 2, obase); EPI(Qc, 1, 3, obase);
    if (dbl) {
      EPI(Pc, 0, 0, obase - 64); EPI(Pc, 0, 1, obase - 64); EPI(Pc, 0, 2, obase - 64); EPI(Pc, 0, 3, obase - 64);
      EPI(Pc, 1, 0, obase - 64); EPI(Pc, 1, 1, obase - 64); EPI(Pc, 1, 2, obase - 64); EPI(Pc, 1, 3, obase - 64);
    }
  }
}

extern "C" void kernel_launch(int rank, int nranks, ncclComm_t comm,
                              hipStream_t stream,
                              void* const* d_in, const int* in_sizes, int n_in,
                              void* d_out, int out_size,
                              const int* in_dtypes, int out_dtype,
                              void* const* d_out_all,
                              void* const* d_ws_all, size_t ws_bytes) {
  if (nranks != NR || (unsigned)rank >= (unsigned)NR || n_in < 4 || ws_bytes < WS_NEED ||
      in_sizes[0] != 4096 * KS || in_sizes[1] != KS * NN || out_size != 512 * NN) {
    fprintf(stderr, "kernel_launch: unexpected problem shape\n");
    return;
  }
  Params p;
  p.me = (char*)d_ws_all[rank];
  p.ws_all = d_ws_all;
  p.x = (const unsigned char*)d_in[0];
  p.w = (const unsigned char*)d_in[1];
  p.sx = (const float*)d_in[2];
  p.sw = (const float*)d_in[3];
  p.out = (float*)d_out;
  p.rank = rank;
  static const int grp[2][4] = {{0, 2, 3, 6}, {1, 4, 5, 7}};
  int gi = 0, gp = 0;
  for (int a = 0; a < 2; ++a) for (int i = 0; i < 4; ++i) if (grp[a][i] == rank) { gi = a; gp = i; }
  p.dstmap = 0;
  for (int x = 0; x < 8; ++x) {
    p.dstmap |= (u32)(x < 4 ? grp[gi][(gp + 1 + x) & 3] : grp[1 - gi][(gp + x) & 3]) << (4 * x);
  }
  hipLaunchKernelGGL(fused_gemm_kernel, dim3(NWG), dim3(NTHREADS), 0, stream, p);
  HIP_CHECK(hipGetLastError());
}
```
